```python
import numpy as np

kernel_name = "gpu_gemm_ar_m4096_k4096_n16384_f32_q8"


ABSMAX_RTOL = 0.02

COMPUTE_DTYPE = "float32"
EPILOGUE = "quant_int8"
IN_QUANT = None
OUT_QUANT = "int8"
IN_DTYPES = {"x": "float32", "w_mat": "float32"}
WORLD_SIZE = 8

def setup_inputs():

    x = np.random.randn(4096, 4096).astype(np.float32)
    w_mat = (np.random.randn(4096, 16384) / np.sqrt(4096)).astype(np.float32)
    return {"x": x, "w_mat": w_mat}

def _quant_dequant(y, amax):
    scale = amax / 127.0
    q = np.clip(np.round(y / scale), -127.0, 127.0)
    return (q * scale).astype(np.float32)
def _epilogue(y):
    return _quant_dequant(y, np.abs(y).max())

def reference(x, w_mat):
    y = (x @ w_mat).astype(np.float32)
    return _epilogue(y)
```

```cpp
#include <hip/hip_runtime.h>
#include <rccl/rccl.h>
#include <cstdio>
#include <cstdint>

#define HIP_CHECK(expr) do { hipError_t _e = (expr); if (_e != hipSuccess) { \
    fprintf(stderr, "HIP error %s at %s:%d\n", hipGetErrorString(_e), __FILE__, __LINE__); \
    return; } } while (0)

namespace {

typedef unsigned u32x4 __attribute__((ext_vector_type(4)));
typedef unsigned u32x2 __attribute__((ext_vector_type(2)));
typedef float f32x4 __attribute__((ext_vector_type(4)));
typedef float f32x2 __attribute__((ext_vector_type(2)));
typedef __bf16 bf16x8 __attribute__((ext_vector_type(8)));
typedef __bf16 bf16x2 __attribute__((ext_vector_type(2)));
#define GAS __attribute__((address_space(1)))
#define LAS __attribute__((address_space(3)))
#define CAS __attribute__((address_space(4)))

constexpr int NR = 8;
constexpr int NTOT = 16384;
constexpr int KL = 512;
constexpr int NL = 2048;
constexpr int PIECE = 32768;
constexpr int PPK = 24;
constexpr int NSEQ = 192;
constexpr int NGEMM = 128;
constexpr int NWG = 256;
constexpr int NTHR = 512;
#define PROD_G 16

constexpr size_t OFF_CNT = 0;
constexpr size_t OFF_TMO = 4096;
constexpr size_t OFF_AMAX = 8192;
constexpr size_t OFF_DONE = 16384;
constexpr size_t OFF_FLAG = 65536;
constexpr size_t OFF_DBG = 512 * 1024;
constexpr size_t OFF_DATA = 1u << 20;
constexpr size_t WS_NEED = OFF_DATA + (size_t)NR * 8 * PPK * PIECE;
constexpr unsigned long long DEADLINE = 50000000ull;
constexpr int LDS_BYTES = 2 * 65536 + 256;

struct Params {
    const float* x;
    const float* w;
    char* ws[NR];
    char* out[NR];
    int rank;
    int pad;
};

__device__ __forceinline__ unsigned ld_sys(const char* p) {
    return __hip_atomic_load((const GAS unsigned*)p, __ATOMIC_RELAXED, __HIP_MEMORY_SCOPE_SYSTEM);
}
__device__ __forceinline__ void st_sys(char* p, unsigned v) {
    __hip_atomic_store((GAS unsigned*)p, v, __ATOMIC_RELAXED, __HIP_MEMORY_SCOPE_SYSTEM);
}
__device__ __forceinline__ unsigned ld_agent(const char* p) {
    return __hip_atomic_load((const GAS unsigned*)p, __ATOMIC_RELAXED, __HIP_MEMORY_SCOPE_AGENT);
}
__device__ __forceinline__ void st_agent(char* p, unsigned v) {
    __hip_atomic_store((GAS unsigned*)p, v, __ATOMIC_RELAXED, __HIP_MEMORY_SCOPE_AGENT);
}
__device__ __forceinline__ void st16_wt(const char* base, unsigned off, u32x4 v) {
    asm volatile("global_store_dwordx4 %0, %1, %2 sc0 sc1\n\ts_nop 1" :: "v"(off), "v"(v), "s"(base) : "memory");
}
__device__ __forceinline__ void stage8k(const char* src, unsigned loff, unsigned ldst) {
    unsigned keep;
    asm volatile(
        "s_mov_b32 %0, m0\n\t"
        "s_mov_b32 m0, %4\n\t"
        "s_nop 0\n\t"
        "global_load_lds_dwordx4 %1, %3 sc0 sc1\n\t"
        "global_load_lds_dwordx4 %1, %3 offset:1024 sc0 sc1\n\t"
        "global_load_lds_dwordx4 %1, %3 offset:2048 sc0 sc1\n\t"
        "global_load_lds_dwordx4 %1, %3 offset:3072 sc0 sc1\n\t"
        "s_add_u32 m0, m0, 0x1000\n\t"
        "s_nop 0\n\t"
        "global_load_lds_dwordx4 %2, %3 sc0 sc1\n\t"
        "global_load_lds_dwordx4 %2, %3 offset:1024 sc0 sc1\n\t"
        "global_load_lds_dwordx4 %2, %3 offset:2048 sc0 sc1\n\t"
        "global_load_lds_dwordx4 %2, %3 offset:3072 sc0 sc1\n\t"
        "s_mov_b32 m0, %0"
        : "=&s"(keep) : "v"(loff), "v"(loff + 4096u), "s"(src), "s"(ldst) : "memory", "scc");
}
#define WAIT_VM0() asm volatile("s_waitcnt vmcnt(0)" ::: "memory")
#define RAW_BAR() do { asm volatile("s_waitcnt lgkmcnt(0)" ::: "memory"); __builtin_amdgcn_s_barrier(); asm volatile("" ::: "memory"); } while (0)

__device__ __forceinline__ unsigned pk_bf16(float a, float b) {
    f32x2 v = {a, b};
    bf16x2 r = __builtin_convertvector(v, bf16x2);
    return __builtin_bit_cast(unsigned, r);
}

#define STAMP(i) do {} while (0)

__device__ __forceinline__ void spin_flag(const char* flag, unsigned seq, char* tmo) {
    unsigned long long t0 = __builtin_amdgcn_s_memrealtime();
    while ((unsigned)__builtin_amdgcn_readfirstlane(ld_sys(flag)) < seq) {
        if ((unsigned)__builtin_amdgcn_readfirstlane(ld_agent(tmo)) == seq ||
            __builtin_amdgcn_s_memrealtime() - t0 > DEADLINE) {
            st_agent(tmo, seq);
            break;
        }
        __builtin_amdgcn_s_sleep(1);
    }
    asm volatile("" ::: "memory");
}

__device__ __forceinline__ void seq_decode(int q, int delta, int& kt, int& slot) {
    if (q < 8) { kt = 0; slot = 16 + q; return; }
    int qq = q - 8;
    int r = qq / 24, e = qq - r * 24;
    if (r == 7) { kt = 7; slot = (e + 2 * delta) & 15; return; }
    int t3 = e / 3, m3 = e - t3 * 3;
    if (m3 < 2) { kt = r; slot = (2 * t3 + m3 + 2 * delta) & 15; }
    else { kt = r + 1; slot = 16 + t3; }
}

__device__ __forceinline__ void load_piece(const CAS Params& p, int dst, int kt, int slot, int wid, int lane, f32x4 (&v)[8]) {
    const int kq = lane >> 4, c = lane & 15;
    if (slot < 16) {
        const GAS float* b = (const GAS float*)p.x + (size_t)(256 * slot + 32 * wid + c) * KL + 64 * kt + 8 * kq;
#pragma unroll
        for (int rb = 0; rb < 2; ++rb)
#pragma unroll
            for (int kb = 0; kb < 2; ++kb) {
                v[(rb * 2 + kb) * 2 + 0] = *(const GAS f32x4*)(b + rb * 16 * KL + kb * 32);
                v[(rb * 2 + kb) * 2 + 1] = *(const GAS f32x4*)(b + rb * 16 * KL + kb * 32 + 4);
            }
    } else {
        const int wc = wid >> 1, kb = wid & 1;
        const GAS float* b = (const GAS float*)p.w + (size_t)(64 * kt + 32 * kb + 8 * kq) * NTOT + dst * NL + 256 * (slot - 16) + 64 * wc + 4 * c;
#pragma unroll
        for (int jj = 0; jj < 8; ++jj) v[jj] = *(const GAS f32x4*)(b + (size_t)jj * NTOT);
    }
}

__device__ __forceinline__ void producer(const CAS Params& p, int c, unsigned seq, char* myws) {
    const int wid = __builtin_amdgcn_readfirstlane(threadIdx.x >> 6), lane = threadIdx.x & 63;
    const int sl = c >> 3, xcd = c & 7;
    const int dst = (p.rank + 1 + (sl & 7)) & 7;
    const int g = xcd + 8 * (sl >> 3);
    if (g >= PROD_G) return;
    const int delta = (p.rank - dst) & 7;
    char* dws = p.ws[dst];
    f32x4 v[8];
    int kt, slot;
    seq_decode(g, delta, kt, slot);
    load_piece(p, dst, kt, slot, wid, lane, v);
    for (int q = g; q < NSEQ; q += PROD_G) {
        u32x4 ch[4];
        unsigned off;
        if (slot < 16) {
#pragma unroll
            for (int i = 0; i < 4; ++i) {
                ch[i].x = pk_bf16(v[2 * i][0], v[2 * i][1]); ch[i].y = pk_bf16(v[2 * i][2], v[2 * i][3]);
                ch[i].z = pk_bf16(v[2 * i + 1][0], v[2 * i + 1][1]); ch[i].w = pk_bf16(v[2 * i + 1][2], v[2 * i + 1][3]);
            }
            off = wid * 4096 + lane * 16;
        } else {
#pragma unroll
            for (int i = 0; i < 4; ++i) {
                ch[i].x = pk_bf16(v[0][i], v[1][i]); ch[i].y = pk_bf16(v[2][i], v[3][i]);
                ch[i].z = pk_bf16(v[4][i], v[5][i]); ch[i].w = pk_bf16(v[6][i], v[7][i]);
            }
            off = (wid >> 1) * 8192 + (wid & 1) * 1024 + lane * 16;
        }
        const unsigned step = slot < 16 ? 1024u : 2048u;
        const int pidx = (p.rank * 8 + kt) * PPK + slot;
        const char* dbase = dws + OFF_DATA + (size_t)pidx * PIECE;
        char* flag = dws + OFF_FLAG + (size_t)pidx * 128;
        if (q + PROD_G < NSEQ) {
            seq_decode(q + PROD_G, delta, kt, slot);
            load_piece(p, dst, kt, slot, wid, lane, v);
        }
#pragma unroll
        for (int i = 0; i < 4; ++i) st16_wt(dbase, off + i * step, ch[i]);
        WAIT_VM0();
        __builtin_amdgcn_s_barrier();
        if (threadIdx.x == 0) st_sys(flag, seq);
    }
    STAMP(1);
}

__device__ __forceinline__ int step_group(int s, int rank, int d0) {
    if (s < 8) return rank * 8 + s;
    int t = s - 8;
    int kt = t / 7, u = t - kt * 7;
    int v = u < d0 ? u : u + 1;
    int delta = (d0 - v) & 7;
    return ((rank + delta) & 7) * 8 + kt;
}

__device__ __forceinline__ void gemm_role(const CAS Params& p, int gidx, unsigned seq, char* myws, LAS char* lds) {
    const int tid = threadIdx.x;
    const int wid = __builtin_amdgcn_readfirstlane(tid >> 6), lane = tid & 63;
    const int wr = wid >> 2, wc = wid & 3;
    const int xcd = gidx & 7, sl = gidx >> 3;
    const int ti = (xcd >> 1) * 4 + (sl >> 2), tj = (xcd & 1) * 4 + (sl & 3);
    const int d0 = ti >> 1;
    const int myslot = wid < 4 ? ti : 16 + tj;
    char* tmo = myws + OFF_TMO;
    const char* dsrc = myws + OFF_DATA + (size_t)myslot * PIECE + (wid & 3) * 8192;
    const unsigned loff = lane * 16;
    const unsigned ldsw = (unsigned)(size_t)lds + wid * 8192;
    const char* fsrc = myws + OFF_FLAG + (size_t)myslot * 128;

    f32x4 acc[8][4];
#pragma unroll
    for (int m = 0; m < 8; ++m)
#pragma unroll
        for (int i = 0; i < 4; ++i) acc[m][i] = (f32x4){0.f, 0.f, 0.f, 0.f};

#define STAGE(grp, buf) stage8k(dsrc + (size_t)(grp) * (PPK * PIECE), loff, ldsw + (buf) * 65536)

    int g0 = step_group(0, p.rank, d0);
    spin_flag(fsrc + (size_t)g0 * (PPK * 128), seq, tmo);
    STAGE(g0, 0);
    int gn = step_group(1, p.rank, d0);
    unsigned fnext = ld_sys(fsrc + (size_t)gn * (PPK * 128));
    WAIT_VM0();
    fnext = (unsigned)__builtin_amdgcn_readfirstlane(fnext);
    RAW_BAR();
    STAMP(1);

    const LAS char* abase = lds + wr * 16384 + lane * 16;
    const LAS char* bbase = lds + 32768 + wc * 8192 + lane * 16;
    for (int s = 0; s < 64; ++s) {
        const int cur = s & 1;
        const bool more = s + 1 < 64;
        bool staged = !more;
        if (more && fnext >= seq) { STAGE(gn, cur ^ 1); staged = true; }
        const int gnn = step_group(s + 2 < 64 ? s + 2 : 63, p.rank, d0);
        const LAS char* ab = abase + cur * 65536;
        const LAS char* bb = bbase + cur * 65536;
        {
            bf16x8 bf[2][4], af[2][8];
#pragma unroll
            for (int kb = 0; kb < 2; ++kb) {
#pragma unroll
                for (int i = 0; i < 4; ++i) bf[kb][i] = *(const LAS bf16x8*)(bb + (i * 2 + kb) * 1024);
#pragma unroll
                for (int m = 0; m < 8; ++m) af[kb][m] = *(const LAS bf16x8*)(ab + (m * 2 + kb) * 1024);
            }
#pragma unroll
            for (int kb = 0; kb < 2; ++kb)
#pragma unroll
                for (int m = 0; m < 8; ++m)
#pragma unroll
                    for (int i = 0; i < 4; ++i)
                        acc[m][i] = __builtin_amdgcn_mfma_f32_16x16x32_bf16(bf[kb][i], af[kb][m], acc[m][i], 0, 0, 0);
            __builtin_amdgcn_sched_group_barrier(0x100, 8, 0);
#pragma unroll
            for (int r = 0; r < 8; ++r) {
                __builtin_amdgcn_sched_group_barrier(0x8, 4, 0);
                __builtin_amdgcn_sched_group_barrier(0x100, 2, 0);
            }
            __builtin_amdgcn_sched_group_barrier(0x8, 32, 0);
        }
        if (!staged) {
            spin_flag(fsrc + (size_t)gn * (PPK * 128), seq, tmo);
            STAGE(gn, cur ^ 1);
        }
        fnext = ld_sys(fsrc + (size_t)gnn * (PPK * 128));
        gn = gnn;
        WAIT_VM0();
        fnext = (unsigned)__builtin_amdgcn_readfirstlane(fnext);
        RAW_BAR();
    }
    STAMP(2);

    float am = 0.f;
#pragma unroll
    for (int m = 0; m < 8; ++m)
#pragma unroll
        for (int i = 0; i < 4; ++i)
#pragma unroll
            for (int j = 0; j < 4; ++j) am = fmaxf(am, fabsf(acc[m][i][j]));
#pragma unroll
    for (int o = 32; o > 0; o >>= 1) am = fmaxf(am, __shfl_xor(am, o));
    LAS float* red = (LAS float*)(lds + 131072);
    if (lane == 0) red[wid] = am;
    RAW_BAR();
    if (tid < NR) {
        float a = red[0];
#pragma unroll
        for (int i = 1; i < 8; ++i) a = fmaxf(a, red[i]);
        unsigned long long gv = ((unsigned long long)seq << 32) | __float_as_uint(a);
        char* dp = p.ws[0];
#pragma unroll
        for (int i = 1; i < NR; ++i) dp = tid == i ? p.ws[i] : dp;
        __hip_atomic_store((GAS unsigned long long*)(dp + OFF_AMAX) + p.rank * NGEMM + gidx, gv, __ATOMIC_RELAXED, __HIP_MEMORY_SCOPE_SYSTEM);
    }
    {
        const GAS unsigned long long* gp = (const GAS unsigned long long*)(myws + OFF_AMAX) + tid * 2;
        unsigned long long t0 = __builtin_amdgcn_s_memrealtime();
        unsigned long long a, b;
        for (;;) {
            a = __hip_atomic_load(gp, __ATOMIC_RELAXED, __HIP_MEMORY_SCOPE_SYSTEM);
            b = __hip_atomic_load(gp + 1, __ATOMIC_RELAXED, __HIP_MEMORY_SCOPE_SYSTEM);
            bool ok = (unsigned)(a >> 32) == seq && (unsigned)(b >> 32) == seq;
            if (__all(ok)) break;
            if ((unsigned)__builtin_amdgcn_readfirstlane(ld_agent(tmo)) == seq ||
                __builtin_amdgcn_s_memrealtime() - t0 > DEADLINE) { st_agent(tmo, seq); break; }
            __builtin_amdgcn_s_sleep(2);
        }
        am = fmaxf(__uint_as_float((unsigned)a), __uint_as_float((unsigned)b));
#pragma unroll
        for (int o = 32; o > 0; o >>= 1) am = fmaxf(am, __shfl_xor(am, o));
        if (lane == 0) red[8 + wid] = am;
        WAIT_VM0();
        RAW_BAR();
#pragma unroll
        for (int i = 0; i < 8; ++i) am = fmaxf(am, red[8 + i]);
    }
    const bool bad = ld_agent(tmo) == seq;
    STAMP(3);

    const float inv = am > 0.f ? 127.0f / am : 0.f;
    {
        const int fr = lane & 15, fq = lane >> 4;
#pragma unroll
        for (int m = 0; m < 8; ++m) {
            u32x4 wv;
#pragma unroll
            for (int j = 0; j < 4; ++j) {
                unsigned t0 = __float_as_uint(fmaf(acc[m][0][j], inv, 12582912.f));
                unsigned t1 = __float_as_uint(fmaf(acc[m][1][j], inv, 12582912.f));
                unsigned t2 = __float_as_uint(fmaf(acc[m][2][j], inv, 12582912.f));
                unsigned t3 = __float_as_uint(fmaf(acc[m][3][j], inv, 12582912.f));
                unsigned w = __builtin_amdgcn_perm(t1, t0, 0x0c0c0400u) | __builtin_amdgcn_perm(t3, t2, 0x04000c0cu);
                wv[j] = bad ? 0x80808080u : w;
            }
            const int row = wr * 128 + m * 16 + fr;
            *(LAS u32x4*)(lds + row * 256 + (((wc * 4 + fq) ^ fr) << 4)) = wv;
        }
    }
    RAW_BAR();
    {
        const int ch = lane & 15;
        const unsigned obase = (unsigned)(256 * ti) * NTOT + p.rank * NL + 256 * tj + ch * 16;
#pragma unroll
        for (int it = 0; it < 8; ++it) {
            const int row = it * 32 + wid * 4 + (lane >> 4);
            u32x4 v = *(const LAS u32x4*)(lds + row * 256 + ((ch ^ (row & 15)) << 4));
            const unsigned off = obase + (unsigned)row * NTOT;
#pragma unroll
            for (int d = 0; d < NR; ++d) st16_wt(p.out[(p.rank + 1 + d + gidx + wid) & 7], off, v);
        }
    }
    WAIT_VM0();
    __builtin_amdgcn_s_barrier();
    STAMP(4);
    if (wid == 0) {
        if (lane < NR) {
            char* dp = p.ws[0];
#pragma unroll
            for (int i = 1; i < NR; ++i) dp = lane == i ? p.ws[i] : dp;
            st_sys(dp + OFF_DONE + gidx * 128 + p.rank * 4, seq);
        }
        const char* dn = myws + OFF_DONE + gidx * 128 + (lane & 7) * 4;
        unsigned long long t0 = __builtin_amdgcn_s_memrealtime();
        for (;;) {
            if (__all(ld_sys(dn) >= seq)) break;
            if ((unsigned)__builtin_amdgcn_readfirstlane(ld_agent(tmo)) == seq ||
                __builtin_amdgcn_s_memrealtime() - t0 > DEADLINE) {
                st_agent(tmo, seq);
                if (lane == 0) *(GAS signed char*)(p.out[p.rank] + (size_t)(256 * ti) * NTOT + p.rank * NL + 256 * tj) = -128;
                break;
            }
            __builtin_amdgcn_s_sleep(2);
        }
    }
    STAMP(5);
}

__global__ __launch_bounds__(NTHR, 2) __attribute__((amdgpu_waves_per_eu(2, 2))) void fused_kernel(Params p_unused) {
    extern __shared__ __attribute__((aligned(1024))) char smem[];
    const CAS Params& p = *(const CAS Params*)__builtin_amdgcn_kernarg_segment_ptr();
    char* myws = p.ws[p.rank];
    STAMP(0);
    char* cnt = myws + OFF_CNT + blockIdx.x * 4;
    const unsigned seq = (unsigned)__builtin_amdgcn_readfirstlane(ld_agent(cnt)) + 1u;
    WAIT_VM0();
    __builtin_amdgcn_s_barrier();
    if (threadIdx.x == 0) st_agent(cnt, seq);
    if (blockIdx.x < NWG - NGEMM) producer(p, blockIdx.x, seq, myws);
    else gemm_role(p, blockIdx.x - (NWG - NGEMM), seq, myws, (LAS char*)smem);
}

}

extern "C" void kernel_launch(int rank, int nranks, ncclComm_t comm,
                              hipStream_t stream,
                              void* const* d_in, const int* in_sizes, int n_in,
                              void* d_out, int out_size,
                              const int* in_dtypes, int out_dtype,
                              void* const* d_out_all,
                              void* const* d_ws_all, size_t ws_bytes) {
    if (nranks != NR || ws_bytes < WS_NEED || (unsigned)rank >= (unsigned)NR) {
        fprintf(stderr, "unexpected configuration\n");
        return;
    }
    HIP_CHECK(hipFuncSetAttribute((const void*)fused_kernel, hipFuncAttributeMaxDynamicSharedMemorySize, LDS_BYTES));
    Params p;
    p.x = (const float*)d_in[0];
    p.w = (const float*)d_in[1];
    for (int i = 0; i < NR; ++i) { p.ws[i] = (char*)d_ws_all[i]; p.out[i] = (char*)d_out_all[i]; }
    p.rank = rank;
    p.pad = 0;
    hipLaunchKernelGGL(fused_kernel, dim3(NWG), dim3(NTHR), LDS_BYTES, stream, p);
    HIP_CHECK(hipGetLastError());
}
```

```cpp
#include <hip/hip_runtime.h>
#include <rccl/rccl.h>
#include <cstdio>
#include <cstdint>

#define HIP_CHECK(expr) do { hipError_t _e = (expr); if (_e != hipSuccess) { \
    fprintf(stderr, "HIP error %s at %s:%d\n", hipGetErrorString(_e), __FILE__, __LINE__); \
    return; } } while (0)

namespace {

typedef unsigned u32x4 __attribute__((ext_vector_type(4)));
typedef unsigned u32x2 __attribute__((ext_vector_type(2)));
typedef float f32x4 __attribute__((ext_vector_type(4)));
typedef float f32x2 __attribute__((ext_vector_type(2)));
typedef __bf16 bf16x8 __attribute__((ext_vector_type(8)));
typedef __bf16 bf16x2 __attribute__((ext_vector_type(2)));
typedef unsigned short u16x2 __attribute__((ext_vector_type(2)));
typedef __amdgpu_buffer_rsrc_t rsrc_t;
#define GAS __attribute__((address_space(1)))
#define LAS __attribute__((address_space(3)))
#define CAS __attribute__((address_space(4)))

constexpr int NR = 8;
constexpr int NTOT = 16384;
constexpr int KL = 512;
constexpr int NL = 2048;
constexpr int PIECE = 32768;
constexpr int PPK = 24;
constexpr int NGEMM = 128;
constexpr int NWG = 256;
constexpr int NTHR = 512;
constexpr int NEXP = 640;
#define PIN_G 12
#define POUT_G 10
constexpr int NPULL = 7 * (PIN_G > POUT_G ? PIN_G : POUT_G);
static_assert(NPULL <= NWG - NGEMM, "puller count");
constexpr int XCAP = 15;
constexpr int OITEM = 8320;

constexpr unsigned OFF_CNT = 0;
constexpr unsigned OFF_TMO = 4096;
constexpr unsigned OFF_AMAX = 8192;
constexpr unsigned OFF_OFLAG = 16384;
constexpr unsigned OFF_TICK = 24576;
constexpr unsigned OFF_XFLAG = 65536;
constexpr unsigned OFF_SFLAG = 262144;
constexpr unsigned OFF_DBG = 512 * 1024;
constexpr unsigned OFF_STAGE = 1u << 20;
constexpr unsigned OFF_XIN = OFF_STAGE + NR * 8 * PPK * PIECE;
constexpr unsigned OFF_XOUT = OFF_XIN + NEXP * 65536;
constexpr unsigned WS_NEED = OFF_XOUT + NGEMM * 8 * OITEM;
constexpr unsigned long long DEADLINE = 50000000ull;
constexpr int LDS_MISC = 131072;
constexpr int LDS_BYTES = LDS_MISC + 1024;
constexpr int AUX_SYS = 17;
#define OUT_AUX 17
#define EXP_PRIO 72
#define EXP_SLEEP 32

template <int V> struct IC { static constexpr int value = V; };
template <int N, int I = 0, class F> __device__ __forceinline__ void static_for(F&& f) {
    if constexpr (I < N) { f(IC<I>{}); static_for<N, I + 1>(f); }
}

struct Params {
    const float* x;
    const float* w;
    char* ws[NR];
    char* out;
    int rank;
    int pad;
};

__device__ __forceinline__ unsigned ld_sys(const char* p) {
    return __hip_atomic_load((const GAS unsigned*)p, __ATOMIC_RELAXED, __HIP_MEMORY_SCOPE_SYSTEM);
}
__device__ __forceinline__ unsigned long long ld_sys64(const char* p) {
    return __hip_atomic_load((const GAS unsigned long long*)p, __ATOMIC_RELAXED, __HIP_MEMORY_SCOPE_SYSTEM);
}
__device__ __forceinline__ void st_sys(char* p, unsigned v) {
    __hip_atomic_store((GAS unsigned*)p, v, __ATOMIC_RELAXED, __HIP_MEMORY_SCOPE_SYSTEM);
}
__device__ __forceinline__ void st_sys64(char* p, unsigned long long v) {
    __hip_atomic_store((GAS unsigned long long*)p, v, __ATOMIC_RELAXED, __HIP_MEMORY_SCOPE_SYSTEM);
}
__device__ __forceinline__ unsigned ld_agent(const char* p) {
    return __hip_atomic_load((const GAS unsigned*)p, __ATOMIC_RELAXED, __HIP_MEMORY_SCOPE_AGENT);
}
__device__ __forceinline__ void st_agent(char* p, unsigned v) {
    __hip_atomic_store((GAS unsigned*)p, v, __ATOMIC_RELAXED, __HIP_MEMORY_SCOPE_AGENT);
}
__device__ __forceinline__ rsrc_t mk_rsrc(const void* p, unsigned bytes) {
    return __builtin_amdgcn_make_buffer_rsrc((void*)p, (short)0, (int)bytes, 0x00020000);
}
__device__ __forceinline__ u32x4 bld16(rsrc_t r, unsigned voff, unsigned soff) {
    return __builtin_amdgcn_raw_buffer_load_b128(r, (int)voff, (int)soff, AUX_SYS);
}
__device__ __forceinline__ void st16(const char* base, unsigned voff, int blk, unsigned soff, u32x4 v) {
    const char* b = base + soff;
    if (blk == 0) asm volatile("global_store_dwordx4 %0, %1, %2 sc0 sc1\n\ts_nop 1" :: "v"(voff), "v"(v), "s"(b) : "memory");
    if (blk == 1) asm volatile("global_store_dwordx4 %0, %1, %2 offset:1024 sc0 sc1\n\ts_nop 1" :: "v"(voff), "v"(v), "s"(b) : "memory");
    if (blk == 2) asm volatile("global_store_dwordx4 %0, %1, %2 offset:2048 sc0 sc1\n\ts_nop 1" :: "v"(voff), "v"(v), "s"(b) : "memory");
    if (blk == 3) asm volatile("global_store_dwordx4 %0, %1, %2 offset:3072 sc0 sc1\n\ts_nop 1" :: "v"(voff), "v"(v), "s"(b) : "memory");
}
#define WAIT_VM0() do { asm volatile("s_waitcnt vmcnt(0)" ::: "memory"); __builtin_amdgcn_s_waitcnt(0x0F70); } while (0)
#define WAIT_LGKM0() asm volatile("s_waitcnt lgkmcnt(0)" ::: "memory")
#define RAW_BAR() do { asm volatile("s_waitcnt lgkmcnt(0)" ::: "memory"); __builtin_amdgcn_s_barrier(); asm volatile("" ::: "memory"); } while (0)
#define SCHED0() __builtin_amdgcn_sched_barrier(0)
#define PERM(a, b, s) __builtin_amdgcn_perm((a), (b), (s))
#define RFL(x) ((unsigned)__builtin_amdgcn_readfirstlane((int)(x)))

template <int OFF> __device__ __forceinline__ void glds_if(unsigned have, unsigned want, const char* src, unsigned loff, unsigned ldst) {
    asm volatile(
        "s_cmp_lt_u32 %0, %5\n\t"
        "s_cbranch_scc1 1f\n\t"
        "s_mov_b32 m0, %3\n\t"
        "s_nop 0\n\t"
        "global_load_lds_dwordx4 %1, %2 offset:%4 sc0 sc1\n"
        "1:"
        :: "s"(have), "v"(loff), "s"(src), "s"(ldst), "n"(OFF), "s"(want) : "memory", "scc");
}
__device__ __forceinline__ void stage8k(const char* src, unsigned loff, unsigned loff2, unsigned ldst) {
    asm volatile(
        "s_mov_b32 m0, %3\n\t"
        "s_nop 0\n\t"
        "global_load_lds_dwordx4 %0, %2 sc0 sc1\n\t"
        "global_load_lds_dwordx4 %0, %2 offset:1024 sc0 sc1\n\t"
        "global_load_lds_dwordx4 %0, %2 offset:2048 sc0 sc1\n\t"
        "global_load_lds_dwordx4 %0, %2 offset:3072 sc0 sc1\n\t"
        "s_add_u32 m0, m0, 0x1000\n\t"
        "s_nop 0\n\t"
        "global_load_lds_dwordx4 %1, %2 sc0 sc1\n\t"
        "global_load_lds_dwordx4 %1, %2 offset:1024 sc0 sc1\n\t"
        "global_load_lds_dwordx4 %1, %2 offset:2048 sc0 sc1\n\t"
        "global_load_lds_dwordx4 %1, %2 offset:3072 sc0 sc1"
        :: "v"(loff), "v"(loff2), "s"(src), "s"(ldst) : "memory", "scc");
}

__device__ __forceinline__ unsigned pk_bf16(float a, float b) {
    f32x2 v = {a, b};
    bf16x2 r = __builtin_convertvector(v, bf16x2);
    return __builtin_bit_cast(unsigned, r);
}

#define STAMP(i) do {} while (0)

__device__ __forceinline__ bool give_up(char* tmo, unsigned seq, unsigned long long t0) {
    if (RFL(ld_agent(tmo)) == seq || __builtin_amdgcn_s_memrealtime() - t0 > DEADLINE) {
        st_agent(tmo, seq);
        return true;
    }
    return false;
}
__device__ __forceinline__ void spin_flag(const char* flag, unsigned seq, char* tmo) {
    unsigned long long t0 = __builtin_amdgcn_s_memrealtime();
    for (unsigned n = 1; RFL(ld_sys(flag)) < seq; ++n) {
        if ((n & 31u) == 0u && give_up(tmo, seq, t0)) break;
        __builtin_amdgcn_s_sleep(1);
    }
    asm volatile("" ::: "memory");
}

__device__ __forceinline__ void exp_decode(int e, int rank, int& kt, int& slot, int& dst) {
    if (e >= 56 && e < 248) {
        const int t = e - 56, r = t / 24, u = t - r * 24;
        kt = r; slot = u; dst = u < 16 ? -1 : rank;
        return;
    }
    const int t = e < 56 ? e : e - 192;
    kt = t / 56;
    const int u = t - kt * 56, j = u / 7;
    slot = 16 + j;
    dst = (rank + 1 + (u - j * 7)) & 7;
}
__device__ __forceinline__ unsigned xin_slot(int kt, int slot, int dst) {
    return slot < 16 ? kt * 16 + slot : 128 + (dst * 8 + kt) * 8 + (slot - 16);
}

__device__ __forceinline__ void load_piece(const CAS Params& p, int dst, int kt, int slot, int wid, int lane, f32x4 (&v)[8]) {
    const int kq = lane >> 4, c = lane & 15;
    if (slot < 16) {
        const GAS float* b = (const GAS float*)p.x + (size_t)(256 * slot + 32 * wid + c) * KL + 64 * kt + 8 * kq;
#pragma unroll
        for (int rb = 0; rb < 2; ++rb)
#pragma unroll
            for (int kb = 0; kb < 2; ++kb) {
                v[(rb * 2 + kb) * 2 + 0] = *(const GAS f32x4*)(b + rb * 16 * KL + kb * 32);
                v[(rb * 2 + kb) * 2 + 1] = *(const GAS f32x4*)(b + rb * 16 * KL + kb * 32 + 4);
            }
    } else {
        const int ch = wid >> 2, kb = (wid >> 1) & 1, g = wid & 1;
        const GAS float* b = (const GAS float*)p.w + (size_t)(64 * kt + 32 * kb + 8 * kq) * NTOT + dst * NL + 256 * (slot - 16) + 128 * ch + 8 * c + 4 * g;
#pragma unroll
        for (int jj = 0; jj < 8; ++jj) v[jj] = *(const GAS f32x4*)(b + (size_t)jj * NTOT);
    }
}

__device__ __forceinline__ unsigned exporter(const CAS Params& p, char* myws, LAS char* lds, int e0, int estep) {
    const int tid = threadIdx.x;
    const int wid = RFL(tid >> 6), lane = tid & 63;
    const int rank = p.rank;
    char* cnt = myws + OFF_CNT + blockIdx.x * 4;
    unsigned seq = ld_agent(cnt);
    LAS unsigned* sh = (LAS unsigned*)(lds + LDS_MISC);
    if (tid == 0) { sh[8] = 0; sh[40] = 0; }
    f32x4 v[8];
    int kt, slot, dst;
    int e = blockIdx.x;
    exp_decode(e, rank, kt, slot, dst);
    if (e >= EXP_PRIO) __builtin_amdgcn_s_sleep(EXP_SLEEP);
    load_piece(p, dst, kt, slot, wid, lane, v);
    seq = RFL(seq) + 1u;
    int par = 0;
    for (;;) {
        u32x4 ch[4];
        if (slot < 16) {
#pragma unroll
            for (int i = 0; i < 4; ++i) {
                ch[i].x = pk_bf16(v[2 * i][0], v[2 * i][1]); ch[i].y = pk_bf16(v[2 * i][2], v[2 * i][3]);
                ch[i].z = pk_bf16(v[2 * i + 1][0], v[2 * i + 1][1]); ch[i].w = pk_bf16(v[2 * i + 1][2], v[2 * i + 1][3]);
            }
        } else {
#pragma unroll
            for (int i = 0; i < 4; ++i) {
                ch[i].x = pk_bf16(v[0][i], v[1][i]); ch[i].y = pk_bf16(v[2][i], v[3][i]);
                ch[i].z = pk_bf16(v[4][i], v[5][i]); ch[i].w = pk_bf16(v[6][i], v[7][i]);
            }
        }
        const int ckt = kt, cslot = slot, cdst = dst;
        const int en = e < NWG ? e0 : e + estep;
        const bool more = en < NEXP;
        if (more) {
            exp_decode(en, rank, kt, slot, dst);
            load_piece(p, dst, kt, slot, wid, lane, v);
        }
        const bool local = cdst < 0 || cdst == rank, remote = cdst != rank;
        const unsigned pidx = (rank * 8 + ckt) * PPK + cslot;
        const unsigned xoff = OFF_XIN + xin_slot(ckt, cslot, cdst) * 65536u;
        LAS unsigned* s = sh + par * 32;
        unsigned base = 0;
        if (local) {
#pragma unroll
            for (int i = 0; i < 4; ++i) st16(myws, lane * 16, i, OFF_STAGE + pidx * PIECE + wid * 4096, ch[i]);
        }
        if (remote) {
            u16x2 mx = {0, 0};
#pragma unroll
            for (int i = 0; i < 4; ++i)
#pragma unroll
                for (int j = 0; j < 4; ++j) mx = __builtin_elementwise_max(mx, __builtin_bit_cast(u16x2, ch[i][j] & 0x7fff7fffu));
            unsigned m = mx.x > mx.y ? mx.x : mx.y;
#pragma unroll
            for (int o = 32; o > 0; o >>= 1) { unsigned t = __shfl_xor(m, o); m = t > m ? t : m; }
            if (lane == 0) s[wid] = m;
            RAW_BAR();
            {
                u32x4 a = *(LAS u32x4*)s, b = *(LAS u32x4*)(s + 4);
                m = max(max(max(a.x, a.y), max(a.z, a.w)), max(max(b.x, b.y), max(b.z, b.w)));
            }
            m = RFL(m) >> 8;
            base = m > 7u ? m - 7u : 0u;
            const unsigned kadd = 0x80808080u - base * 0x01010101u;
            u32x4 lo0, lo1, nib;
            unsigned hh[8], any = 0;
#pragma unroll
            for (int i = 0; i < 4; ++i) {
                const unsigned l0 = PERM(ch[i].y, ch[i].x, 0x06040200u), l1 = PERM(ch[i].w, ch[i].z, 0x06040200u);
                const unsigned h0 = PERM(ch[i].y, ch[i].x, 0x07050301u), h1 = PERM(ch[i].w, ch[i].z, 0x07050301u);
                const unsigned t0 = (h0 & 0x7f7f7f7fu) + kadd, t1 = (h1 & 0x7f7f7f7fu) + kadd;
                nib[i] = (t0 & 0x07070707u) | ((h0 >> 4) & 0x08080808u) | ((t1 & 0x07070707u) << 4) | (h1 & 0x80808080u);
                if (i < 2) { lo0[2 * i] = l0; lo0[2 * i + 1] = l1; } else { lo1[2 * i - 4] = l0; lo1[2 * i - 3] = l1; }
                hh[2 * i] = h0; hh[2 * i + 1] = h1;
                any |= ~(t0 & t1);
            }
            st16(myws, lane * 16, 0, xoff + wid * 3072, lo0);
            st16(myws, lane * 16, 1, xoff + wid * 3072, lo1);
            st16(myws, lane * 16, 2, xoff + wid * 3072, nib);
            if (any & 0x80808080u) {
#pragma unroll
                for (int r = 0; r < 8; ++r)
#pragma unroll
                    for (int y = 0; y < 4; ++y) {
                        const unsigned hb = (hh[r] >> (8 * y)) & 0xffu;
                        if ((hb & 0x7fu) < base) {
                            const unsigned k = __hip_atomic_fetch_add(s + 8, 1u, __ATOMIC_RELAXED, __HIP_MEMORY_SCOPE_WORKGROUP);
                            if (k < XCAP) s[17 + k] = hb | (y << 8) | (((wid & 1) * 8 + r) << 10) | (lane << 14) | ((wid >> 1) << 20);
                        }
                    }
            }
        }
        WAIT_VM0();
        RAW_BAR();
        if (tid == 0 && e < NWG) st_agent(cnt, seq);
        if (remote) {
            const unsigned nex = RFL(s[8]);
            if (nex > XCAP) {
#pragma unroll
                for (int i = 0; i < 4; ++i) st16(myws, lane * 16, i, xoff + 32768 + wid * 4096, ch[i]);
                WAIT_VM0();
                RAW_BAR();
            }
            if (tid < 128) {
                const int d = cdst < 0 ? tid >> 4 : cdst;
                if ((cdst < 0 ? d != rank : tid < 16)) {
                    const int gi = tid & 15;
                    const unsigned pay = gi == 0 ? (base | (nex > XCAP ? 0x100u : nex << 16)) : s[16 + gi];
                    st_sys64(p.ws[d] + OFF_XFLAG + pidx * 128 + gi * 8, ((unsigned long long)seq << 32) | pay);
                }
            }
        }
        if (local && tid >= 128 && tid < 132) st_agent(myws + OFF_SFLAG + pidx * 128 + (tid - 128) * 32, seq);
        if (tid == 192) sh[(par ^ 1) * 32 + 8] = 0;
        if (!more) break;
        e = en; par ^= 1;
    }
    return seq;
}

__device__ __forceinline__ void seq_decode(int q, int delta, int& kt, int& slot) {
    if (q < 8) { kt = 0; slot = 16 + q; return; }
    int qq = q - 8;
    int r = qq / 24, e = qq - r * 24;
    if (r == 7) { kt = 7; slot = (e + 2 * delta) & 15; return; }
    int t3 = e / 3, m3 = e - t3 * 3;
    if (m3 < 2) { kt = r; slot = (2 * t3 + m3 + 2 * delta) & 15; }
    else { kt = r + 1; slot = 16 + t3; }
}

__device__ __forceinline__ void tick_init(char* ctr, unsigned seq, unsigned first) {
    if ((threadIdx.x & 63) == 0)
        __hip_atomic_fetch_max((GAS unsigned*)ctr, (seq << 11) + first, __ATOMIC_RELAXED, __HIP_MEMORY_SCOPE_AGENT);
    WAIT_VM0();
}
__device__ __forceinline__ unsigned tick_draw(char* ctr) {
    return (unsigned)__builtin_amdgcn_raw_ptr_buffer_atomic_add_i32(1, mk_rsrc(ctr, 4), (threadIdx.x & 63) ? 64 : 0, 0, 16);
}

__device__ __forceinline__ void pull_inputs(const CAS Params& p, char* myws, unsigned seq, int link, int g) {
    const int lane = threadIdx.x & 63;
    const int rank = p.rank, delta = link + 1, src = (rank + delta) & 7;
    char* tmo = myws + OFF_TMO;
    const rsrc_t rr = mk_rsrc(p.ws[src], WS_NEED);
    const char* xf = myws + OFF_XFLAG + (size_t)src * (8 * PPK * 128) + (lane & 15) * 8;
    constexpr int NIT = 768;
    u32x4 a[6];
#pragma unroll
    for (int u = 0; u < 6; ++u) a[u] = (u32x4){0u, 0u, 0u, 0u};
    unsigned pay = 0, pidx = 0, xoff = 0;
    unsigned pidx_n, xoff_n;
    unsigned long long mn;
    {
        int kt, slot; seq_decode(g >> 2, delta, kt, slot);
        pidx_n = kt * PPK + slot; xoff_n = OFF_XIN + xin_slot(kt, slot, rank) * 65536u;
        mn = ld_sys64(xf + pidx_n * 128);
    }
    char* ctr = myws + OFF_TICK + link * 128;
    tick_init(ctr, seq, PIN_G);
    unsigned tk = tick_draw(ctr);
    int prev_flag = -1;
    for (int it = -1, nx = g; it < NIT;) {
        WAIT_VM0();
        if (prev_flag >= 0 && lane == 0) st_agent(myws + OFF_SFLAG + prev_flag, seq);
        u32x4 c[6];
#pragma unroll
        for (int u = 0; u < 6; ++u) { c[u] = a[u]; asm volatile("" : "+v"(c[u])); }
        const unsigned cpay = pay, cpidx = pidx, cxoff = xoff;
        int nn = NIT;
        if (nx < NIT) {
            if (!__all((unsigned)(mn >> 32) == seq)) {
                unsigned long long t0 = __builtin_amdgcn_s_memrealtime();
                for (unsigned n = 1;; ++n) {
                    mn = ld_sys64(xf + pidx_n * 128);
                    if (__all((unsigned)(mn >> 32) == seq)) break;
                    if ((n & 31u) == 0u && give_up(tmo, seq, t0)) { mn = 0; break; }
                    __builtin_amdgcn_s_sleep(1);
                }
                asm volatile("" ::: "memory");
            }
            pay = (unsigned)mn; pidx = pidx_n; xoff = xoff_n;
            const int sub = nx & 3;
            if (!(RFL(pay) & 0x100u)) {
#pragma unroll
                for (int u = 0; u < 6; ++u) a[u] = bld16(rr, lane * 16 + (u % 3) * 1024, xoff_n + sub * 6144 + (u / 3) * 3072);
            }
            nn = (int)(RFL(tk) & 0x7ffu);
            if (nn < NIT) {
                int kt, slot; seq_decode(nn >> 2, delta, kt, slot);
                pidx_n = kt * PPK + slot; xoff_n = OFF_XIN + xin_slot(kt, slot, rank) * 65536u;
                mn = ld_sys64(xf + pidx_n * 128);
                tk = tick_draw(ctr);
            }
        }
        SCHED0();
        if (it >= 0) {
            const int sub = it & 3;
            const unsigned meta = RFL(cpay);
            const unsigned soff = OFF_STAGE + (src * (8 * PPK) + cpidx) * PIECE + sub * 8192;
            u32x4 o[8];
            if (meta & 0x100u) {
#pragma unroll
                for (int u = 0; u < 8; ++u) o[u] = bld16(rr, lane * 16 + (u & 3) * 1024, cxoff + 32768 + sub * 8192 + (u >> 2) * 4096);
                WAIT_VM0();
#pragma unroll
                for (int u = 0; u < 8; ++u) asm volatile("" : "+v"(o[u]));
            } else {
                const unsigned base4 = (meta & 0xffu) * 0x01010101u;
                unsigned H[16];
#pragma unroll
                for (int h = 0; h < 2; ++h)
#pragma unroll
                    for (int i = 0; i < 4; ++i) {
                        const unsigned W = c[3 * h + 2][i];
                        H[h * 8 + 2 * i] = ((W & 0x07070707u) + base4) | ((W << 4) & 0x80808080u);
                        H[h * 8 + 2 * i + 1] = (((W >> 4) & 0x07070707u) + base4) | (W & 0x80808080u);
                    }
                const unsigned nex = meta >> 16;
                for (unsigned k = 1; k <= nex; ++k) {
                    const unsigned en = (unsigned)__builtin_amdgcn_readlane((int)cpay, k);
                    if ((int)(en >> 20) != sub) continue;
                    const unsigned sh = ((en >> 8) & 3u) * 8u, r = (en >> 10) & 15u;
                    const unsigned msk = (unsigned)lane == ((en >> 14) & 63u) ? 0xffu << sh : 0u;
                    const unsigned val = (en & 0xffu) << sh;
#pragma unroll
                    for (int q = 0; q < 16; ++q) { const unsigned mq = r == (unsigned)q ? msk : 0u; H[q] = (H[q] & ~mq) | (val & mq); }
                }
#pragma unroll
                for (int h = 0; h < 2; ++h)
#pragma unroll
                    for (int i = 0; i < 4; ++i) {
                        const unsigned l0 = c[3 * h + (i >> 1)][(i & 1) * 2], l1 = c[3 * h + (i >> 1)][(i & 1) * 2 + 1];
                        const unsigned h0 = H[h * 8 + 2 * i], h1 = H[h * 8 + 2 * i + 1];
                        o[h * 4 + i].x = PERM(h0, l0, 0x05010400u); o[h * 4 + i].y = PERM(h0, l0, 0x07030602u);
                        o[h * 4 + i].z = PERM(h1, l1, 0x05010400u); o[h * 4 + i].w = PERM(h1, l1, 0x07030602u);
                    }
            }
#pragma unroll
            for (int u = 0; u < 8; ++u) st16(myws, lane * 16, u & 3, soff + (u >> 2) * 4096, o[u]);
            prev_flag = (src * (8 * PPK) + cpidx) * 128 + sub * 32;
        }
        it = nx; nx = nn;
    }
    WAIT_VM0();
    if (lane == 0) st_agent(myws + OFF_SFLAG + prev_flag, seq);
}

__device__ __forceinline__ void tile_of(int gidx, int& ti, int& tj) {
    const int xcd = gidx & 7, sl = gidx >> 3;
    ti = (xcd >> 1) * 4 + (sl >> 2); tj = (xcd & 1) * 4 + (sl & 3);
}
__device__ __forceinline__ unsigned item_out(int src, int gidx, int w) {
    int ti, tj; tile_of(gidx, ti, tj);
    return (unsigned)(256 * ti + 64 * (w >> 1)) * NTOT + src * NL + 256 * tj + 128 * (w & 1);
}
__device__ __forceinline__ void store_rows(rsrc_t ro, unsigned lane_off, unsigned soff, const unsigned (&q)[32]) {
#pragma unroll
    for (int m = 0; m < 4; ++m)
#pragma unroll
        for (int j = 0; j < 4; ++j) {
            u32x2 v = {q[(m * 4 + j) * 2], q[(m * 4 + j) * 2 + 1]};
            __builtin_amdgcn_raw_buffer_store_b64(v, ro, (int)lane_off, (int)(soff + (m * 16 + j) * NTOT), OUT_AUX);
        }
}

__device__ __forceinline__ void pull_outputs(const CAS Params& p, char* myws, LAS char* lds, unsigned seq, int link, int g) {
    const int lane = threadIdx.x & 63;
    const int rank = p.rank, src = (rank + link + 1) & 7;
    char* tmo = myws + OFF_TMO;
    const rsrc_t rr = mk_rsrc(p.ws[src], WS_NEED), ro = mk_rsrc(p.out, 4096u * NTOT);
    const unsigned lane_off = (unsigned)(4 * (lane >> 4)) * NTOT + 8 * (lane & 15);
    constexpr int NIT = NGEMM * 8;
    char* ctr = myws + OFF_TICK + (8 + link) * 128;
    tick_init(ctr, seq, POUT_G);
    unsigned tk = tick_draw(ctr);
    {
        const char* f = myws + OFF_OFLAG + src * 512 + lane * 8;
        unsigned long long t0 = __builtin_amdgcn_s_memrealtime();
        for (unsigned n = 1;; ++n) {
            unsigned long long v = ld_sys64(f);
            if (__all((unsigned)v >= seq && (unsigned)(v >> 32) >= seq)) break;
            if ((n & 31u) == 0u && give_up(tmo, seq, t0)) break;
            __builtin_amdgcn_s_sleep(2);
        }
        asm volatile("" ::: "memory");
    }
    STAMP(3);
    u32x4 a[7];
    unsigned lst;
#define ISSUE_OUT(IT) do { const unsigned _o = OFF_XOUT + (unsigned)(IT) * OITEM; \
        _Pragma("unroll") for (int u = 0; u < 7; ++u) a[u] = bld16(rr, lane * 16 + (u & 3) * 1024, _o + (u >> 2) * 4096); \
        lst = __builtin_amdgcn_raw_buffer_load_b16(rr, lane * 2, (int)(_o + 7168), AUX_SYS); } while (0)
    ISSUE_OUT(g);
    if (RFL(ld_agent(tmo)) == seq) {
        unsigned q[32];
#pragma unroll
        for (int d = 0; d < 32; ++d) q[d] = 0x80808080u;
        for (int it = g; it < NIT; it += POUT_G) store_rows(ro, lane_off, item_out(src, it >> 3, it & 7), q);
        return;
    }
    LAS u32x4* bm = (LAS u32x4*)lds;
    for (int it = g; it < NIT;) {
        unsigned w[28];
#pragma unroll
        for (int d = 0; d < 28; ++d) w[d] = a[d >> 2][d & 3];
        const unsigned cl = lst;
#pragma unroll
        for (int d = 0; d < 28; ++d) asm volatile("" : "+v"(w[d]));
        const int nx = (int)(RFL(tk) & 0x7ffu);
        if (nx < NIT) { ISSUE_OUT(nx); tk = tick_draw(ctr); }
        SCHED0();
        u32x4 m;
        if (RFL(cl) == 0xfffeu) {
            m = bld16(rr, lane * 16, OFF_XOUT + (unsigned)it * OITEM + 7296);
            WAIT_VM0();
            asm volatile("" : "+v"(m));
        } else {
            bm[lane] = (u32x4){0u, 0u, 0u, 0u};
            WAIT_LGKM0();
            if (cl < 8192u) __hip_atomic_fetch_or((LAS unsigned*)bm + (cl >> 5), 1u << (cl & 31u), __ATOMIC_RELAXED, __HIP_MEMORY_SCOPE_WORKGROUP);
            WAIT_LGKM0();
            m = bm[lane];
        }
        unsigned q[32];
#pragma unroll
        for (int jx = 0; jx < 4; ++jx) {
            unsigned x = 0;
#pragma unroll
            for (int b = 0; b < 7; ++b) x |= (w[7 * jx + b] >> (7 - b)) & (0x01010101u << b);
            q[28 + jx] = x;
        }
#pragma unroll
        for (int d = 0; d < 28; ++d) q[d] = w[d];
#pragma unroll
        for (int d = 0; d < 32; ++d)
            q[d] = (q[d] & 0x7f7f7f7fu) | (((q[d] << 1) ^ (m[d >> 3] << (7 - (d & 7)))) & 0x80808080u);
        store_rows(ro, lane_off, item_out(src, it >> 3, it & 7), q);
        it = nx;
    }
}

__device__ __forceinline__ int step_group(int s, int rank, int d0) {
    if (s < 8) return rank * 8 + s;
    int t = s - 8;
    int kt = t / 7, u = t - kt * 7;
    int v = u < d0 ? u : u + 1;
    int delta = (d0 - v) & 7;
    return ((rank + delta) & 7) * 8 + kt;
}

__device__ __forceinline__ void gemm_role(const CAS Params& p, int gidx, unsigned seq, char* myws, LAS char* lds) {
    const int tid = threadIdx.x;
    const int wid = RFL(tid >> 6), lane = tid & 63;
    const int wr = wid >> 1, wc = wid & 1;
    const int rank = p.rank;
    int ti, tj; tile_of(gidx, ti, tj);
    const int d0 = ti >> 1;
    const int myslot = wid < 4 ? ti : 16 + tj;
    char* tmo = myws + OFF_TMO;
    const char* dsrc = myws + OFF_STAGE + (size_t)myslot * PIECE + (wid & 3) * 8192;
    const char* fsrc = myws + OFF_SFLAG + myslot * 128 + (wid & 3) * 32;
    const unsigned loff = lane * 16, loff2 = loff + 4096;
    const unsigned ldsw = (unsigned)(size_t)lds + wid * 8192;

    f32x4 acc[4][8];
#pragma unroll
    for (int m = 0; m < 4; ++m)
#pragma unroll
        for (int i = 0; i < 8; ++i) acc[m][i] = (f32x4){0.f, 0.f, 0.f, 0.f};

#define GL(n, S) glds_if<((n) & 3) * 1024>(have, want, S, (n) < 4 ? loff : loff2, (n) < 4 ? ld0 : ld1)
#define STAGE_ALL(S, D) stage8k(S, loff, loff2, D)
#define PRIO(n) __builtin_amdgcn_s_setprio(n)
#define RDV 0
#define RD1(t) (((t) & 1) == 0 && (t) < 24 ? (t) / 2 : -1)
#define RD2(t) (((t) & 1) == 0 && (t) >= 2 && (t) < 26 ? (t) / 2 - 1 : -1)
#define GOFF(st) ((unsigned)__builtin_amdgcn_readlane((int)tab, (st)))
#define GSRC(off) (dsrc + (off))
#define GFLAG(off) (fsrc + ((off) >> 8))
    static_assert(PPK * PIECE / 256 == PPK * 128, "flag offset");
    const unsigned tab = (unsigned)step_group(lane, rank, d0) * (PPK * PIECE);
    const LAS char* abase = lds + wr * 8192 + lane * 16;
    const LAS char* bbase = lds + 32768 + wc * 16384 + lane * 16;
    bf16x8 a0[4], b0[8], a1[4], b1[8];

    {
        const unsigned g0 = GOFF(0);
        spin_flag(GFLAG(g0), seq, tmo);
        STAGE_ALL(GSRC(g0), ldsw);
    }
    unsigned gn = GOFF(1);
    unsigned fv = ld_sys(GFLAG(gn));
    WAIT_VM0();
    RAW_BAR();
    STAMP(2);
    bool staged = RFL(fv) >= seq;
    if (staged) { STAGE_ALL(GSRC(gn), ldsw + 65536); gn = GOFF(2); }
    fv = ld_sys(GFLAG(gn));
#pragma unroll
    for (int i = 0; i < 8; ++i) b0[i] = *(const LAS bf16x8*)(bbase + i * 1024);
#pragma unroll
    for (int m = 0; m < 4; ++m) a0[m] = *(const LAS bf16x8*)(abase + m * 2048);

    for (int s = 0; s < 64; ++s) {
        const int cur = (s & 1) * 65536, oth = cur ^ 65536;
        if (!staged) {
            WAIT_VM0();
            if (RFL(fv) >= seq) { STAGE_ALL(GSRC(gn), ldsw + oth); staged = true; gn = GOFF(min(s + 2, 63)); fv = ld_sys(GFLAG(gn)); }
        }
        const LAS char* ab2 = abase + oth;
        const LAS char* bb2 = bbase + oth;
        const unsigned want = s + 2 < 64 ? seq : 0xffffffffu;
        const unsigned g3 = GOFF(min(s + 3, 63));
        const unsigned ld0 = ldsw + cur, ld1 = ld0 + 4096;
        SCHED0();
        {
            const LAS char* ab = abase + cur + 1024;
            const LAS char* bb = bbase + cur + 8192;
            PRIO(1);
            static_for<32>([&](auto tc) __attribute__((always_inline)) {
                constexpr int t = decltype(tc)::value;
                acc[t >> 3][t & 7] = __builtin_amdgcn_mfma_f32_16x16x32_bf16(a0[t >> 3], b0[t & 7], acc[t >> 3][t & 7], 0, 0, 0);
                constexpr int rd = RD1(t);
                if constexpr (rd >= 0 && rd < 8) b1[rd] = *(const LAS bf16x8*)(bb + rd * 1024);
                if constexpr (rd >= 8) a1[rd - 8] = *(const LAS bf16x8*)(ab + (rd - 8) * 2048);
                if constexpr (t == 15) PRIO(0);
                SCHED0();
            });
        }
        if (!staged) {
            spin_flag(GFLAG(gn), seq, tmo);
            STAGE_ALL(GSRC(gn), ldsw + oth);
            gn = GOFF(min(s + 2, 63));
            fv = ld_sys(GFLAG(gn));
        }
        const char* gs = GSRC(gn);
        WAIT_VM0();
        const unsigned have = RFL(fv);
        __builtin_amdgcn_s_waitcnt(0xC07F);
        PRIO(3);
        SCHED0();
        RAW_BAR();
        SCHED0();
        {
            static_for<32>([&](auto tc) __attribute__((always_inline)) {
                constexpr int t = decltype(tc)::value;
                acc[t >> 3][t & 7] = __builtin_amdgcn_mfma_f32_16x16x32_bf16(a1[t >> 3], b1[t & 7], acc[t >> 3][t & 7], 0, 0, 0);
                constexpr int rd = RD2(t);
                if constexpr (rd >= 0 && rd < 8) b0[rd] = *(const LAS bf16x8*)(bb2 + rd * 1024);
                if constexpr (rd >= 8) a0[rd - 8] = *(const LAS bf16x8*)(ab2 + (rd - 8) * 2048);
                if constexpr ((t & 3) == 1) GL(t >> 2, gs);
                if constexpr (t == 15) PRIO(2);
                if constexpr (t == 27) {
                    const bool go = have >= want;
                    staged = go || s + 2 >= 64;
                    gn = go ? g3 : gn;
                }
                if constexpr (t == 30) fv = ld_sys(GFLAG(gn));
                SCHED0();
            });
        }
    }
    STAMP(3);

    float am = 0.f;
#pragma unroll
    for (int m = 0; m < 4; ++m)
#pragma unroll
        for (int i = 0; i < 8; ++i)
#pragma unroll
            for (int j = 0; j < 4; ++j) am = fmaxf(am, fabsf(acc[m][i][j]));
#pragma unroll
    for (int o = 32; o > 0; o >>= 1) am = fmaxf(am, __shfl_xor(am, o));
    LAS float* red = (LAS float*)(lds + LDS_MISC + 512);
    if (lane == 0) red[wid] = am;
    RAW_BAR();
    if (tid < NR) {
        float a = red[0];
#pragma unroll
        for (int i = 1; i < 8; ++i) a = fmaxf(a, red[i]);
        st_sys64(p.ws[tid] + OFF_AMAX + (rank * NGEMM + gidx) * 8, ((unsigned long long)seq << 32) | __float_as_uint(a));
    }
    LAS char* wl = lds + wid * 256;
    if (lane < 9) *(LAS u32x4*)(wl + lane * 16) = lane < 8 ? (u32x4){~0u, ~0u, ~0u, ~0u} : (u32x4){0u, 0u, 0u, 0u};
    {
        const char* gp = myws + OFF_AMAX + tid * 16;
        unsigned long long t0 = __builtin_amdgcn_s_memrealtime();
        unsigned long long a, b;
        for (unsigned n = 1;; ++n) {
            a = ld_sys64(gp);
            b = ld_sys64(gp + 8);
            bool ok = (unsigned)(a >> 32) == seq && (unsigned)(b >> 32) == seq;
            if (__all(ok)) break;
            if ((n & 31u) == 0u && give_up(tmo, seq, t0)) break;
            __builtin_amdgcn_s_sleep(2);
        }
        am = fmaxf(__uint_as_float((unsigned)a), __uint_as_float((unsigned)b));
#pragma unroll
        for (int o = 32; o > 0; o >>= 1) am = fmaxf(am, __shfl_xor(am, o));
        if (lane == 0) red[8 + wid] = am;
        WAIT_VM0();
        RAW_BAR();
#pragma unroll
        for (int i = 0; i < 8; ++i) am = fmaxf(am, red[8 + i]);
    }
    const unsigned tmov = ld_agent(tmo);
    STAMP(4);

    const float inv = am > 0.f ? 127.0f / am : 0.f;
    unsigned q[32];
#pragma unroll
    for (int m = 0; m < 4; ++m)
#pragma unroll
        for (int j = 0; j < 4; ++j)
#pragma unroll
            for (int h = 0; h < 2; ++h) {
                const unsigned t0 = __float_as_uint(fmaf(acc[m][4 * h + 0][j], inv, 12582912.f));
                const unsigned t1 = __float_as_uint(fmaf(acc[m][4 * h + 1][j], inv, 12582912.f));
                const unsigned t2 = __float_as_uint(fmaf(acc[m][4 * h + 2][j], inv, 12582912.f));
                const unsigned t3 = __float_as_uint(fmaf(acc[m][4 * h + 3][j], inv, 12582912.f));
                q[(m * 4 + j) * 2 + h] = PERM(t1, t0, 0x0c0c0400u) | PERM(t3, t2, 0x04000c0cu);
            }
    if (RFL(tmov) == seq) {
#pragma unroll
        for (int d = 0; d < 32; ++d) q[d] = 0x80808080u;
    }
    const unsigned xo = OFF_XOUT + (gidx * 8 + wid) * OITEM;
#pragma unroll
    for (int u = 0; u < 7; ++u) {
        u32x4 v;
#pragma unroll
        for (int k = 0; k < 4; ++k) {
            const int d = 4 * u + k;
            v[k] = (q[d] & 0x7f7f7f7fu) | ((q[28 + d / 7] << (7 - d % 7)) & 0x80808080u);
        }
        st16(myws, lane * 16, u & 3, xo + (u >> 2) * 4096, v);
    }
    {
        u32x4 m;
#pragma unroll
        for (int jm = 0; jm < 4; ++jm) {
            unsigned x = 0;
#pragma unroll
            for (int b = 0; b < 8; ++b) { const unsigned v = q[8 * jm + b]; x |= ((v ^ (v << 1)) >> (7 - b)) & (0x01010101u << b); }
            m[jm] = x;
        }
        const unsigned c = __popc(m.x) + __popc(m.y) + __popc(m.z) + __popc(m.w);
        unsigned o;
        asm volatile("ds_add_rtn_u32 %0, %1, %2\n\ts_waitcnt lgkmcnt(0)" : "=&v"(o) : "v"((unsigned)(size_t)(wl + 128)), "v"(c) : "memory");
        const bool raw = __any(o + c > 64u);
        if (raw) {
            st16(myws, lane * 16, 0, xo + 7296, m);
            if (lane == 0) *(LAS unsigned short*)wl = 0xfffe;
        } else {
#pragma unroll
            for (int jm = 0; jm < 4; ++jm) {
                unsigned x = m[jm];
                while (x) {
                    const unsigned b = __builtin_ctz(x);
                    *(LAS unsigned short*)(wl + 2 * o) = (unsigned short)(lane * 128 + jm * 32 + b);
                    ++o; x &= x - 1;
                }
            }
        }
        WAIT_LGKM0();
        if (lane < 8) st16(myws, lane * 16, 0, xo + 7168, *(LAS u32x4*)(wl + lane * 16));
    }
    WAIT_VM0();
    __builtin_amdgcn_s_barrier();
    if (tid < NR) st_sys(p.ws[tid] + OFF_OFLAG + rank * 512 + gidx * 4, seq);
    STAMP(5);
    store_rows(mk_rsrc(p.out, 4096u * NTOT), (unsigned)(4 * (lane >> 4)) * NTOT + 8 * (lane & 15), item_out(rank, gidx, wid), q);
}

__global__ __launch_bounds__(NTHR, 2) __attribute__((amdgpu_waves_per_eu(2, 2))) void fused_kernel(Params p_unused) {
    extern __shared__ __attribute__((aligned(1024))) char smem[];
    const CAS Params& p = *(const CAS Params*)__builtin_amdgcn_kernarg_segment_ptr();
    char* myws = p.ws[p.rank];
    LAS char* lds = (LAS char*)smem;
    STAMP(0);
    const int b = blockIdx.x;
    const unsigned seq = exporter(p, myws, lds, (b < NPULL || b >= NWG - NGEMM) ? NEXP : NWG + (b - NPULL), NWG - NGEMM - NPULL);
    STAMP(1);
    if (b >= NWG - NGEMM) {
        RAW_BAR();
        gemm_role(p, b - (NWG - NGEMM), seq, myws, lds);
    } else if (b < NPULL && RFL(threadIdx.x) == 0) {
        const int link = b % 7, g = b / 7;
        if (g < PIN_G) pull_inputs(p, myws, seq, link, g);
        STAMP(2);
        if (g < POUT_G) pull_outputs(p, myws, lds, seq, link, g);
        STAMP(4);
    }
}

}

extern "C" void kernel_launch(int rank, int nranks, ncclComm_t comm,
                              hipStream_t stream,
                              void* const* d_in, const int* in_sizes, int n_in,
                              void* d_out, int out_size,
                              const int* in_dtypes, int out_dtype,
                              void* const* d_out_all,
                              void* const* d_ws_all, size_t ws_bytes) {
    if (nranks != NR || ws_bytes < WS_NEED || (unsigned)rank >= (unsigned)NR) {
        fprintf(stderr, "unexpected configuration\n");
        return;
    }
    HIP_CHECK(hipFuncSetAttribute((const void*)fused_kernel, hipFuncAttributeMaxDynamicSharedMemorySize, LDS_BYTES));
    Params p;
    p.x = (const float*)d_in[0];
    p.w = (const float*)d_in[1];
    for (int i = 0; i < NR; ++i) p.ws[i] = (char*)d_ws_all[i];
    p.out = (char*)d_out;
    p.rank = rank;
    p.pad = 0;
    hipLaunchKernelGGL(fused_kernel, dim3(NWG), dim3(NTHR), LDS_BYTES, stream, p);
    HIP_CHECK(hipGetLastError());
}
```

```cpp
#include <hip/hip_runtime.h>
#include <rccl/rccl.h>
#include <cstdio>
#include <cstdint>

#define HIP_CHECK(expr) do { hipError_t _e = (expr); if (_e != hipSuccess) { \
    fprintf(stderr, "HIP error %s at %s:%d\n", hipGetErrorString(_e), __FILE__, __LINE__); \
    return; } } while (0)

namespace {

typedef unsigned u32x4 __attribute__((ext_vector_type(4)));
typedef unsigned u32x2 __attribute__((ext_vector_type(2)));
typedef float f32x4 __attribute__((ext_vector_type(4)));
typedef float f32x2 __attribute__((ext_vector_type(2)));
typedef __bf16 bf16x8 __attribute__((ext_vector_type(8)));
typedef __bf16 bf16x2 __attribute__((ext_vector_type(2)));
typedef unsigned short u16x2 __attribute__((ext_vector_type(2)));
typedef __amdgpu_buffer_rsrc_t rsrc_t;
#define GAS __attribute__((address_space(1)))
#define LAS __attribute__((address_space(3)))
#define CAS __attribute__((address_space(4)))

constexpr int NR = 8;
constexpr int NTOT = 16384;
constexpr int KL = 512;
constexpr int NL = 2048;
constexpr int PIECE = 32768;
constexpr int PPK = 28;
constexpr int NGEMM = 128;
constexpr int NXG = 64;
constexpr int NWG = 256;
constexpr int NTHR = 512;
constexpr int NEXP = 2560;
#define PULL_WG 9
#define PULL_W 8
#define WIN_IN 8
#define WIN_OUT 8
#define EXP_W1 5
#define EXP_SLEEP 3
constexpr int NPWG = 7 * PULL_WG;
static_assert(NPWG <= NWG - NGEMM - NXG, "puller count");
constexpr int NI_MAIN = 768, NI_IN = NI_MAIN + 112, NI_ALL = NI_IN + 960;
constexpr int XI_SZ = 6144;
constexpr int XO_SZ = 7168;

constexpr unsigned OFF_CNT = 0;
constexpr unsigned OFF_TMO = 8192;
constexpr unsigned OFF_TICK = 12288;
constexpr unsigned OFF_AMAX = 16384;
constexpr unsigned OFF_XFLAG = 32768;
constexpr unsigned XFLAG_SZ = NR * 8 * PPK * 32;
constexpr unsigned OFF_OFLAG = 163840;
constexpr unsigned OFF_SFLAG = 262144;
constexpr unsigned OFF_DBG = 512 * 1024;
constexpr unsigned OFF_XIN = 1u << 20;
constexpr unsigned OFF_XOUT = OFF_XIN + 2 * NEXP * XI_SZ;
constexpr unsigned OFF_STAGE = OFF_XOUT + 1024 * XO_SZ;
constexpr unsigned OFF_RIN = OFF_STAGE + NR * 8 * PPK * PIECE;
constexpr unsigned OFF_ROUT = OFF_RIN + 2 * NEXP * 8192;
constexpr unsigned WS_NEED = OFF_ROUT + 1024 * 8192;
static_assert(OFF_XFLAG + 2 * XFLAG_SZ <= OFF_OFLAG && OFF_OFLAG + 65536 <= OFF_SFLAG && OFF_SFLAG + NR * 8 * PPK * 128 <= OFF_DBG, "layout");
constexpr unsigned long long DEADLINE = 50000000ull;
constexpr int LDS_MISC = 131072;
constexpr int LDS_BYTES = LDS_MISC + 1024;
constexpr int AUX_SYS = 17;
constexpr int OUT_AUX = 17;
constexpr unsigned SCR_SZ = 10240;
constexpr unsigned SC_BLOB = 0, SC_A2 = 1152, SC_A3 = 5376, SC_LUT = 9600;
constexpr unsigned M_RAW = 15;

template <int V> struct IC { static constexpr int value = V; };
template <int N, int I = 0, class F> __device__ __forceinline__ void static_for(F&& f) {
    if constexpr (I < N) { f(IC<I>{}); static_for<N, I + 1>(f); }
}

struct Params {
    const float* x;
    const float* w;
    char* ws[NR];
    char* out;
    int rank;
    int pad;
};
#define TUNED(i, dflt) (dflt)

__device__ __forceinline__ unsigned ld_sys(const char* p) {
    return __hip_atomic_load((const GAS unsigned*)p, __ATOMIC_RELAXED, __HIP_MEMORY_SCOPE_SYSTEM);
}
__device__ __forceinline__ unsigned long long ld_sys64(const char* p) {
    return __hip_atomic_load((const GAS unsigned long long*)p, __ATOMIC_RELAXED, __HIP_MEMORY_SCOPE_SYSTEM);
}
__device__ __forceinline__ void st_sys(char* p, unsigned v) {
    __hip_atomic_store((GAS unsigned*)p, v, __ATOMIC_RELAXED, __HIP_MEMORY_SCOPE_SYSTEM);
}
__device__ __forceinline__ void st_sys64(char* p, unsigned long long v) {
    __hip_atomic_store((GAS unsigned long long*)p, v, __ATOMIC_RELAXED, __HIP_MEMORY_SCOPE_SYSTEM);
}
__device__ __forceinline__ unsigned ld_agent(const char* p) {
    return __hip_atomic_load((const GAS unsigned*)p, __ATOMIC_RELAXED, __HIP_MEMORY_SCOPE_AGENT);
}
__device__ __forceinline__ void st_agent(char* p, unsigned v) {
    __hip_atomic_store((GAS unsigned*)p, v, __ATOMIC_RELAXED, __HIP_MEMORY_SCOPE_AGENT);
}
__device__ __forceinline__ rsrc_t mk_rsrc(const void* p, unsigned bytes) {
    return __builtin_amdgcn_make_buffer_rsrc((void*)p, (short)0, (int)bytes, 0x00020000);
}
__device__ __forceinline__ u32x4 bld16(rsrc_t r, unsigned voff, unsigned soff) {
    return __builtin_amdgcn_raw_buffer_load_b128(r, (int)voff, (int)soff, AUX_SYS);
}
__device__ __forceinline__ void st16(const char* base, unsigned voff, int blk, unsigned soff, u32x4 v) {
    const char* b = base + soff;
    if (blk == 0) asm volatile("global_store_dwordx4 %0, %1, %2 sc0 sc1\n\ts_nop 1" :: "v"(voff), "v"(v), "s"(b) : "memory");
    if (blk == 1) asm volatile("global_store_dwordx4 %0, %1, %2 offset:1024 sc0 sc1\n\ts_nop 1" :: "v"(voff), "v"(v), "s"(b) : "memory");
    if (blk == 2) asm volatile("global_store_dwordx4 %0, %1, %2 offset:2048 sc0 sc1\n\ts_nop 1" :: "v"(voff), "v"(v), "s"(b) : "memory");
    if (blk == 3) asm volatile("global_store_dwordx4 %0, %1, %2 offset:3072 sc0 sc1\n\ts_nop 1" :: "v"(voff), "v"(v), "s"(b) : "memory");
}
#define WAIT_VM0() do { asm volatile("s_waitcnt vmcnt(0)" ::: "memory"); __builtin_amdgcn_s_waitcnt(0x0F70); } while (0)
#define WAIT_LGKM0() asm volatile("s_waitcnt lgkmcnt(0)" ::: "memory")
#define RAW_BAR() do { asm volatile("s_waitcnt lgkmcnt(0)" ::: "memory"); __builtin_amdgcn_s_barrier(); asm volatile("" ::: "memory"); } while (0)
#define SCHED0() __builtin_amdgcn_sched_barrier(0)
#define PERM(a, b, s) __builtin_amdgcn_perm((a), (b), (s))
#define RFL(x) ((unsigned)__builtin_amdgcn_readfirstlane((int)(x)))

template <int OFF> __device__ __forceinline__ void glds_if(unsigned have, unsigned want, const char* src, unsigned loff, unsigned ldst) {
    asm volatile(
        "s_cmp_lt_u32 %0, %5\n\t"
        "s_cbranch_scc1 1f\n\t"
        "s_mov_b32 m0, %3\n\t"
        "s_nop 0\n\t"
        "global_load_lds_dwordx4 %1, %2 offset:%4 sc0 sc1\n"
        "1:"
        :: "s"(have), "v"(loff), "s"(src), "s"(ldst), "n"(OFF), "s"(want) : "memory", "scc");
}
__device__ __forceinline__ void stage8k(const char* src, unsigned loff, unsigned loff2, unsigned ldst) {
    asm volatile(
        "s_mov_b32 m0, %3\n\t"
        "s_nop 0\n\t"
        "global_load_lds_dwordx4 %0, %2 sc0 sc1\n\t"
        "global_load_lds_dwordx4 %0, %2 offset:1024 sc0 sc1\n\t"
        "global_load_lds_dwordx4 %0, %2 offset:2048 sc0 sc1\n\t"
        "global_load_lds_dwordx4 %0, %2 offset:3072 sc0 sc1\n\t"
        "s_add_u32 m0, m0, 0x1000\n\t"
        "s_nop 0\n\t"
        "global_load_lds_dwordx4 %1, %2 sc0 sc1\n\t"
        "global_load_lds_dwordx4 %1, %2 offset:1024 sc0 sc1\n\t"
        "global_load_lds_dwordx4 %1, %2 offset:2048 sc0 sc1\n\t"
        "global_load_lds_dwordx4 %1, %2 offset:3072 sc0 sc1"
        :: "v"(loff), "v"(loff2), "s"(src), "s"(ldst) : "memory", "scc");
}

__device__ __forceinline__ unsigned pk_bf16(float a, float b) {
    f32x2 v = {a, b};
    bf16x2 r = __builtin_convertvector(v, bf16x2);
    return __builtin_bit_cast(unsigned, r);
}

#define STAMP(i) do {} while (0)

__device__ __forceinline__ bool give_up(char* tmo, unsigned seq, unsigned long long t0) {
    if (RFL(ld_agent(tmo)) == seq || __builtin_amdgcn_s_memrealtime() - t0 > DEADLINE) {
        st_agent(tmo, seq);
        return true;
    }
    return false;
}
__device__ __forceinline__ void spin_flag(const char* flag, unsigned seq, char* tmo) {
    unsigned long long t0 = __builtin_amdgcn_s_memrealtime();
    for (unsigned n = 1; RFL(ld_sys(flag)) < seq; ++n) {
        if ((n & 31u) == 0u && give_up(tmo, seq, t0)) break;
        __builtin_amdgcn_s_sleep(1);
    }
    asm volatile("" ::: "memory");
}

typedef unsigned u32x2a __attribute__((ext_vector_type(2), aligned(4)));
#define LDS32(a) (*(LAS unsigned*)(size_t)(a))
#define CFENCE() asm volatile("" ::: "memory")

__device__ __forceinline__ unsigned wave_scan(unsigned x) {
    x += (unsigned)__builtin_amdgcn_update_dpp(0, (int)x, 0x111, 0xf, 0xf, false);
    x += (unsigned)__builtin_amdgcn_update_dpp(0, (int)x, 0x112, 0xf, 0xf, false);
    x += (unsigned)__builtin_amdgcn_update_dpp(0, (int)x, 0x114, 0xf, 0xf, false);
    x += (unsigned)__builtin_amdgcn_update_dpp(0, (int)x, 0x118, 0xf, 0xf, false);
    x += (unsigned)__builtin_amdgcn_update_dpp(0, (int)x, 0x142, 0xa, 0xf, false);
    x += (unsigned)__builtin_amdgcn_update_dpp(0, (int)x, 0x143, 0xc, 0xf, false);
    return x;
}
__device__ __forceinline__ unsigned esc_bits(unsigned P) { return __builtin_amdgcn_bitop3_b32(P, P >> 1, 0x55555555u, 0x80); }

__device__ __forceinline__ void lut_init(unsigned scr, int lane) {
    if (lane < 16) {
        unsigned sel = 0x0c0c0c0cu; int k = 0;
#pragma unroll
        for (int y = 0; y < 4; ++y) if ((lane >> y) & 1) { sel = (sel & ~(0xffu << (8 * k))) | ((unsigned)y << (8 * k)); ++k; }
        LDS32(scr + SC_LUT + lane * 4) = sel;
    }
}
#define BFI(m, x, y) __builtin_amdgcn_bitop3_b32((m), (x), (y), 0xCA)
__device__ __forceinline__ void zero_scratch(unsigned scr, int lane) {
#pragma unroll
    for (int i = 0; i < 9; ++i) *(LAS u32x4*)(size_t)(scr + i * 1024 + lane * 16) = (u32x4){0u, 0u, 0u, 0u};
    if (lane < 24) *(LAS u32x4*)(size_t)(scr + 9216 + lane * 16) = (u32x4){0u, 0u, 0u, 0u};
}
__device__ __forceinline__ unsigned digits1(unsigned n02, unsigned n13) {
    const unsigned lo = BFI(0x33333333u, n02, n13 << 2), hi = BFI(0x33333333u, n02 >> 2, n13);
    const unsigned h = (hi | (hi >> 1)) & 0x55555555u;
    return lo | h | (h << 1);
}
__device__ __forceinline__ unsigned sel_addr(unsigned E, int t, unsigned scr) {
    if (t == 0) return __builtin_amdgcn_udot4(E & 0x01010101u, 0x20100804u, scr + SC_LUT, false);
    if (t == 1) return __builtin_amdgcn_udot4(E & 0x04040404u, 0x08040201u, scr + SC_LUT, false);
    if (t == 2) return __builtin_amdgcn_udot4((E >> 4) & 0x01010101u, 0x20100804u, scr + SC_LUT, false);
    return __builtin_amdgcn_udot4((E >> 4) & 0x04040404u, 0x08040201u, scr + SC_LUT, false);
}
template <int NP, bool IN> __device__ __forceinline__ unsigned encode(const unsigned (&n)[2 * NP], unsigned (&P)[NP], unsigned scr, unsigned nb0, int lane) {
    unsigned E[NP], cnt = 0;
#pragma unroll
    for (int j = 0; j < NP; ++j) { P[j] = digits1(n[2 * j], n[2 * j + 1]); E[j] = esc_bits(P[j]); cnt += __popc(E[j]); }
    unsigned sel[2][8];
#pragma unroll
    for (int k = 0; k < 8; ++k) sel[0][k] = LDS32(sel_addr(E[k >> 2], k & 3, scr));
    cnt = max(cnt, 4u);
    const unsigned inc = wave_scan(cnt);
    const unsigned n2d = ((unsigned)__builtin_amdgcn_readlane((int)inc, 63) + 15u) >> 4;
    if (nb0 + n2d > 256u) return 1000u;
    unsigned pos = scr + ((lane & 1) ? SC_A3 : SC_A2) + inc - cnt, carry = 0;
    static_for<NP / 2>([&](auto gc) __attribute__((always_inline)) {
        constexpr int g = decltype(gc)::value;
        SCHED0();
        if constexpr (g + 1 < NP / 2) {
#pragma unroll
            for (int k = 0; k < 8; ++k) sel[(g + 1) & 1][k] = LDS32(sel_addr(E[2 * g + 2 + (k >> 2)], k & 3, scr));
        }
        SCHED0();
#pragma unroll
        for (int k = 0; k < 8; ++k) {
            const int j = 2 * g + (k >> 2), t = k & 3;
            const unsigned z = t < 2 ? n[2 * j + t] : n[2 * j + t - 2] >> 4;
            const unsigned long long v = (unsigned long long)PERM(0u, z, sel[g & 1][k]) << ((pos & 3u) * 8u);
            const unsigned lo = (unsigned)v | carry;
            LDS32(pos & ~3u) = lo;
            const unsigned np = pos + __popc(E[j] & (0x01010101u << (2 * t)));
            carry = ((np ^ pos) & 4u) ? (unsigned)(v >> 32) : lo;
            pos = np;
        }
    });
    LDS32(pos & ~3u) = carry;
    CFENCE();
    unsigned n3 = 0;
    const unsigned s3 = scr + SC_BLOB + (nb0 + n2d) * 4;
    for (unsigned i0 = 0; i0 < n2d; i0 += 64) {
        const unsigned i = i0 + lane;
        const u32x4 v = (*(LAS u32x4*)(size_t)(scr + SC_A2 + i * 16) | *(LAS u32x4*)(size_t)(scr + SC_A3 + i * 16)) & 0x0f0f0f0fu;
        unsigned x = 0;
#pragma unroll
        for (int t = 0; t < 4; ++t) {
            const unsigned z1 = v[t] + 0x01010101u;
            unsigned d = z1, e4;
            if (IN) { e4 = (((v[t] + 0x0b0b0b0bu) ^ z1) >> 4) & 0x01010101u; d |= z1 >> 3; }
            else e4 = ((v[t] + 0x0a0a0a0au) >> 4) & 0x01010101u;
            x |= ((d | e4 | (e4 << 1)) & 0x03030303u) << (2 * t);
        }
        const unsigned E2 = esc_bits(x);
        unsigned sl[4];
#pragma unroll
        for (int t = 0; t < 4; ++t) sl[t] = LDS32(sel_addr(E2, t, scr));
        const unsigned c = __popc(E2), ic = wave_scan(c);
        const unsigned tot = (unsigned)__builtin_amdgcn_readlane((int)ic, 63);
        if (nb0 + n2d + ((n3 + tot + 7u) >> 3) > 256u) return 1000u;
        if (i < n2d) LDS32(scr + SC_BLOB + (nb0 + i) * 4) = x;
        unsigned h[4];
#pragma unroll
        for (int t = 0; t < 4; ++t) { const unsigned y = PERM(0u, v[t], sl[t]); h[t] = PERM(0u, y | (y >> 4), 0x0c0c0200u); }
        const unsigned c0 = __popc(E2 & 0x01010101u), c01 = __popc(E2 & 0x05050505u), c2 = __popc(E2 & 0x10101010u);
        const unsigned long long acc = (unsigned long long)(h[0] | (h[1] << (4u * c0))) | ((unsigned long long)(h[2] | (h[3] << (4u * c2))) << (4u * c01));
        const unsigned p3 = n3 + ic - c, sh = (p3 & 7u) * 4u;
        if (c) {
            LAS unsigned* d = (LAS unsigned*)(size_t)(s3 + (p3 >> 3) * 4);
            const unsigned long long w = acc << sh;
            __hip_atomic_fetch_or(d, (unsigned)w, __ATOMIC_RELAXED, __HIP_MEMORY_SCOPE_WAVEFRONT);
            if (4u * c + sh > 32u) {
                __hip_atomic_fetch_or(d + 1, (unsigned)(w >> 32), __ATOMIC_RELAXED, __HIP_MEMORY_SCOPE_WAVEFRONT);
                __hip_atomic_fetch_or(d + 2, (unsigned)(((acc >> 32) << sh) >> 32), __ATOMIC_RELAXED, __HIP_MEMORY_SCOPE_WAVEFRONT);
            }
        }
        n3 += tot;
    }
    CFENCE();
    return nb0 + n2d + ((n3 + 7u) >> 3);
}
__device__ __forceinline__ unsigned rot8(unsigned e) { return ((e >> 1) & 0x7f7f7f7fu) | ((e << 7) & 0x80808080u); }
__device__ __forceinline__ void selectors(unsigned P, unsigned E, unsigned (&sel)[4]) {
    const unsigned pf = ((((E << 8) + E) << 8) + E) << 8;
    const unsigned S = BFI(E | (E << 1), pf, P);
    const unsigned A = BFI(0x33333333u, S, E << 2), B = BFI(0x33333333u, S >> 2, E);
    sel[0] = A & 0x07070707u; sel[1] = B & 0x07070707u; sel[2] = (A >> 4) & 0x07070707u; sel[3] = (B >> 4) & 0x07070707u;
}
template <int NP, bool IN> __device__ __forceinline__ void decode(const unsigned (&P)[NP], unsigned (&o)[4 * NP], unsigned scr, unsigned nb0, unsigned nbmax, unsigned T, int lane) {
    const unsigned lut1 = IN ? rot8(T | ((T - 1u) << 8) | ((T - 2u) << 16)) : 0x00100800u;
    const unsigned lut2 = IN ? rot8((T - 3u) | ((T - 4u) << 8) | ((T + 1u) << 16)) : 0x00282018u;
    unsigned E[NP], cnt = 0;
#pragma unroll
    for (int j = 0; j < NP; ++j) { E[j] = esc_bits(P[j]); cnt += __popc(E[j]); }
    cnt = max(cnt, 4u);
    const unsigned inc = wave_scan(cnt);
    const unsigned n2d = min(((unsigned)__builtin_amdgcn_readlane((int)inc, 63) + 15u) >> 4, 256u);
    for (unsigned i = lane; nb0 + n2d + i < nbmax; i += 64) {
        const unsigned W = LDS32(scr + SC_BLOB + (nb0 + n2d + i) * 4);
        unsigned a, b;
        if (IN) {
            const unsigned K = (T + 2u) * 0x01010101u;
            a = rot8(K - (((W & 0x0f0f0f0fu) + 0x02020202u) & 0x0f0f0f0fu));
            b = rot8(K - ((((W >> 4) & 0x0f0f0f0fu) + 0x02020202u) & 0x0f0f0f0fu));
        } else {
            a = (W << 3) & 0x78787878u;
            b = (W >> 1) & 0x78787878u;
        }
        *(LAS u32x2*)(size_t)(scr + SC_A3 + i * 8) = (u32x2){PERM(b, a, 0x05010400u), PERM(b, a, 0x07030602u)};
    }
    CFENCE();
    unsigned n3 = 0;
    for (unsigned i0 = 0; i0 < n2d; i0 += 64) {
        const unsigned i = i0 + lane;
        unsigned W = LDS32(scr + SC_BLOB + ((nb0 + i) & 255u) * 4);
        W = i < n2d ? W : 0u;
        const unsigned E2 = esc_bits(W), c = __popc(E2), ic = wave_scan(c);
        unsigned ps[4];
        u32x2a w[4];
        ps[0] = scr + SC_A3 + min(n3 + ic - c, 2048u);
#pragma unroll
        for (int t = 0; t < 4; ++t) {
            w[t] = *(LAS u32x2a*)(size_t)(ps[t] & ~3u);
            if (t < 3) ps[t + 1] = ps[t] + __popc(E2 & (0x01010101u << (2 * t)));
        }
        SCHED0();
        n3 += (unsigned)__builtin_amdgcn_readlane((int)ic, 63);
        unsigned sel[4];
        selectors(W, E2, sel);
        SCHED0();
        u32x4 v;
#pragma unroll
        for (int t = 0; t < 4; ++t) v[t] = PERM(__builtin_amdgcn_alignbyte(w[t].y, w[t].x, ps[t]), lut2, sel[t]);
        *(LAS u32x4*)(size_t)(scr + SC_A2 + i * 16) = v;
    }
    CFENCE();
    unsigned pos = scr + SC_A2 + min(inc - cnt, 4096u);
    unsigned ps[2][8];
    u32x2a w[2][8];
    static_for<NP / 2 + 1>([&](auto gc) __attribute__((always_inline)) {
        constexpr int g = decltype(gc)::value;
        SCHED0();
        if constexpr (g < NP / 2) {
#pragma unroll
            for (int k = 0; k < 8; ++k) {
                ps[g & 1][k] = pos;
                w[g & 1][k] = *(LAS u32x2a*)(size_t)(pos & ~3u);
                pos += __popc(E[2 * g + (k >> 2)] & (0x01010101u << (2 * (k & 3))));
            }
        }
        SCHED0();
        if constexpr (g > 0) {
#pragma unroll
            for (int h = 0; h < 2; ++h) {
                const int j = 2 * (g - 1) + h;
                unsigned sel[4];
                selectors(P[j], E[j], sel);
#pragma unroll
                for (int t = 0; t < 4; ++t) {
                    const u32x2a x = w[(g - 1) & 1][4 * h + t];
                    o[4 * j + t] = PERM(__builtin_amdgcn_alignbyte(x.y, x.x, ps[(g - 1) & 1][4 * h + t]), lut1, sel[t]);
                }
            }
        }
    });
    CFENCE();
}

__device__ __forceinline__ void exp_decode(int e, int rank, int& kt, int& slot, int& sub, int& dst) {
    sub = e & 3;
    const int q = e >> 2;
    int t;
    if (q < 56) t = q;
    else if (q < 152 || (q >= 208 && q < 304)) {
        const int v = q < 152 ? q - 56 : q - 112, r = v / 24, u = v - r * 24;
        kt = r; slot = u; dst = u < 16 ? -1 : rank;
        return;
    } else t = q < 208 ? q - 96 : q - 192;
    kt = t / 56;
    const int u = t - kt * 56, j = u / 7;
    slot = 16 + j;
    dst = (rank + 1 + (u - j * 7)) & 7;
}
__device__ __forceinline__ unsigned xin_item(int kt, int slot, int sub, int dst) {
    return (slot < 16 ? kt * 16 + slot : 128 + (dst * 8 + kt) * 8 + (slot - 16)) * 4 + sub;
}

__device__ __forceinline__ void export_item(const CAS Params& p, char* myws, char* peer, unsigned scr, int e, unsigned& seq, bool first) {
    const int lane = threadIdx.x & 63;
    const int rank = p.rank;
    int kt, slot, sub, dst;
    exp_decode(e, rank, kt, slot, sub, dst);
    const int kq = lane >> 4, c = lane & 15;
    u32x4 ch[8];
    if (slot < 16) {
        const GAS float* b = (const GAS float*)p.x + (size_t)(256 * slot + 64 * sub + c) * KL + 64 * kt + 8 * kq;
        f32x4 v[16];
#pragma unroll
        for (int i = 0; i < 8; ++i) {
            v[2 * i] = *(const GAS f32x4*)(b + (i >> 1) * 16 * KL + (i & 1) * 32);
            v[2 * i + 1] = *(const GAS f32x4*)(b + (i >> 1) * 16 * KL + (i & 1) * 32 + 4);
        }
        zero_scratch(scr, lane);
#pragma unroll
        for (int i = 0; i < 8; ++i) {
            ch[i].x = pk_bf16(v[2 * i][0], v[2 * i][1]); ch[i].y = pk_bf16(v[2 * i][2], v[2 * i][3]);
            ch[i].z = pk_bf16(v[2 * i + 1][0], v[2 * i + 1][1]); ch[i].w = pk_bf16(v[2 * i + 1][2], v[2 * i + 1][3]);
        }
    } else {
        const GAS float* b = (const GAS float*)p.w + (size_t)(64 * kt + 32 * (sub & 1) + 8 * kq) * NTOT + dst * NL + 256 * (slot - 16) + 128 * (sub >> 1) + 8 * c;
        f32x4 v[16];
#pragma unroll
        for (int jj = 0; jj < 8; ++jj) {
            v[2 * jj] = *(const GAS f32x4*)(b + (size_t)jj * NTOT);
            v[2 * jj + 1] = *(const GAS f32x4*)(b + (size_t)jj * NTOT + 4);
        }
        zero_scratch(scr, lane);
#pragma unroll
        for (int i = 0; i < 8; ++i) {
            ch[i].x = pk_bf16(v[0 + (i >> 2)][i & 3], v[2 + (i >> 2)][i & 3]); ch[i].y = pk_bf16(v[4 + (i >> 2)][i & 3], v[6 + (i >> 2)][i & 3]);
            ch[i].z = pk_bf16(v[8 + (i >> 2)][i & 3], v[10 + (i >> 2)][i & 3]); ch[i].w = pk_bf16(v[12 + (i >> 2)][i & 3], v[14 + (i >> 2)][i & 3]);
        }
    }
    if (first) seq = RFL(seq) + 1u;
    const bool dup = slot == 23 && sub >= 2;
    const bool local = dst < 0 || dst == rank || dup, remote = dst != rank || dup;
    unsigned meta = 0;
    if (remote) {
        const unsigned xi = xin_item(kt, slot, sub, dst) + (seq & 1u) * NEXP;
        unsigned ex[16], sum = 0;
#pragma unroll
        for (int u = 0; u < 4; ++u) {
            u32x4 r;
#pragma unroll
            for (int k = 0; k < 4; ++k) {
                const unsigned d0 = ch[2 * u + (k >> 1)][(k & 1) * 2], d1 = ch[2 * u + (k >> 1)][(k & 1) * 2 + 1];
                const unsigned L = PERM(d1, d0, 0x06040200u), H = PERM(d1, d0, 0x07050301u);
                r[k] = (H & 0x80808080u) | (L & 0x7f7f7f7fu);
                ex[4 * u + k] = ((H << 1) & 0xfefefefeu) | ((L >> 7) & 0x01010101u);
                sum = __builtin_amdgcn_sad_u8(ex[4 * u + k], 0u, sum);
            }
            st16(myws, lane * 16, u, OFF_XIN + xi * XI_SZ, r);
        }
        unsigned T = ((unsigned)__builtin_amdgcn_readlane((int)wave_scan(sum), 63) + 7864u) >> 12;
        T = min(max(T, 13u), 253u);
        const unsigned K = (T + 2u) * 0x01010101u;
        unsigned z[16], yy[16], bad = 0;
#pragma unroll
        for (int d = 0; d < 16; ++d) {
            const unsigned y = ((K | 0x80808080u) - (ex[d] & 0x7f7f7f7fu)) ^ (~(K ^ ex[d]) & 0x80808080u);
            bad |= y; yy[d] = y;
            z[d] = ((y & 0x0f0f0f0fu) + 0x0e0e0e0eu) & 0x0f0f0f0fu;
        }
        unsigned nex = 0;
        if (__any((bad & 0xf0f0f0f0u) != 0u)) {
            CFENCE();
            unsigned mk = 0;
#pragma unroll
            for (int d = 0; d < 16; ++d) mk |= (yy[d] & 0xf0f0f0f0u) ? 1u << d : 0u;
            while (mk) {
                const unsigned d = __builtin_ctz(mk);
                mk &= mk - 1u;
                unsigned ev = ex[0];
#pragma unroll
                for (int t = 1; t < 16; ++t) ev = d == (unsigned)t ? ex[t] : ev;
#pragma unroll 1
                for (unsigned y = 0; y < 4; ++y) {
                    const unsigned eb = (ev >> (8 * y)) & 0xffu;
                    if (T + 2u - eb > 15u) {
                        const unsigned k = __hip_atomic_fetch_add((LAS unsigned*)(size_t)(scr + SC_LUT + 64), 1u, __ATOMIC_RELAXED, __HIP_MEMORY_SCOPE_WAVEFRONT);
                        if (k < 15u) LDS32(scr + SC_BLOB + k * 4) = eb | (y << 8) | (d << 10) | (lane << 14);
                    }
                }
            }
            CFENCE();
            nex = RFL(LDS32(scr + SC_LUT + 64));
            CFENCE();
            LDS32(scr + SC_LUT + 64) = 0u;
        }
        unsigned P[4], nb = 1000u;
        if (nex <= 15u) {
            unsigned n[8];
#pragma unroll
            for (int k = 0; k < 8; ++k) n[k] = z[(k >> 1) * 4 + (k & 1)] | (z[(k >> 1) * 4 + (k & 1) + 2] << 4);
            nb = encode<4, true>(n, P, scr, nex, lane);
        }
        if (nb <= 256u) {
            const unsigned lines = (nb + 31u) >> 5;
            st16(myws, lane * 16, 0, OFF_XIN + xi * XI_SZ + 4096, (u32x4){P[0], P[1], P[2], P[3]});
            if ((unsigned)lane < lines * 8u) st16(myws, lane * 16, 1, OFF_XIN + xi * XI_SZ + 4096, *(LAS u32x4*)(size_t)(scr + SC_BLOB + lane * 16));
            meta = T | (lines << 8) | (nex << 12);
        } else {
#pragma unroll
            for (int u = 0; u < 8; ++u) st16(myws, lane * 16, u & 3, OFF_RIN + xi * 8192 + (u >> 2) * 4096, ch[u]);
            meta = M_RAW << 8;
        }
    }
    const int d = lane & 7;
    const int ud = (dst - d - 1) & 7;
    const unsigned at = (rank * 8 + kt) * (PPK * 4) + ((dup && d != dst) ? (24 + (ud >> 1)) * 4 + (ud & 1) * 2 + (sub & 1) : slot * 4 + sub);
    if (local) {
        const unsigned mine = (unsigned)__builtin_amdgcn_readlane((int)at, rank);
#pragma unroll
        for (int u = 0; u < 8; ++u) st16(myws, lane * 16, u & 3, OFF_STAGE + mine * 8192 + (u >> 2) * 4096, ch[u]);
    }
    WAIT_VM0();
    if (lane < NR) {
        if (lane == rank) { if (local) st_agent(myws + OFF_SFLAG + at * 32, seq); }
        else if (dst < 0 || dup || lane == dst) st_sys64(peer + OFF_XFLAG + (seq & 1u) * XFLAG_SZ + at * 8, ((unsigned long long)seq << 32) | meta);
    }
}

__device__ __forceinline__ void tick_init(char* ctr, unsigned seq, unsigned first) {
    if ((threadIdx.x & 63) == 0)
        __hip_atomic_fetch_max((GAS unsigned*)ctr, (seq << 11) + first, __ATOMIC_RELAXED, __HIP_MEMORY_SCOPE_AGENT);
    WAIT_VM0();
}
__device__ __forceinline__ unsigned tick_draw(char* ctr) {
    return (unsigned)__builtin_amdgcn_raw_ptr_buffer_atomic_add_i32(1, mk_rsrc(ctr, 4), (threadIdx.x & 63) ? 64 : 0, 0, 16);
}
static_assert(NI_ALL + 2 * PULL_WG * 8 < 2048, "ticket range");

__device__ __forceinline__ void item_decode(int it, int rank, int delta, unsigned par, unsigned& xi, unsigned& at) {
    if (it >= NI_IN) {
        const int o = it - NI_IN, rb = o / 15;
        xi = at = o + rb;
        return;
    }
    const int src = (rank + delta) & 7;
    if (it >= NI_MAIN) {
        const int o = it - NI_MAIN, kt = o / 14, r = o - kt * 14, u = r >> 1;
        xi = xin_item(kt, 23, 2 + (r & 1), (rank + 1 + u) & 7) + par;
        at = (src * 8 + kt) * (PPK * 4) + 96 + r;
        return;
    }
    const int q = it >> 2;
    int kt, slot;
    if (q < 8) { kt = 0; slot = 16 + q; }
    else {
        const int qq = q - 8, r = qq / 24, e = qq - r * 24;
        if (r == 7) { kt = 7; slot = (e + 2 * delta) & 15; }
        else {
            const int t3 = e / 3, m3 = e - t3 * 3;
            if (m3 < 2) { kt = r; slot = (2 * t3 + m3 + 2 * delta) & 15; }
            else { kt = r + 1; slot = 16 + t3; }
        }
    }
    xi = xin_item(kt, slot, it & 3, rank) + par;
    at = (src * 8 + kt) * (PPK * 4) + slot * 4 + (it & 3);
}

__device__ __forceinline__ void store_rows(rsrc_t ro, unsigned lane_off, unsigned soff, const unsigned (&q)[32]) {
#pragma unroll
    for (int m = 0; m < 4; ++m)
#pragma unroll
        for (int j = 0; j < 4; ++j) {
            u32x2 v = {q[(m * 4 + j) * 2], q[(m * 4 + j) * 2 + 1]};
            __builtin_amdgcn_raw_buffer_store_b64(v, ro, (int)lane_off, (int)(soff + (m * 16 + j) * NTOT), OUT_AUX);
        }
}

__device__ __forceinline__ void puller(const CAS Params& p, char* myws, unsigned scr, unsigned sem, unsigned seq, int link, int gw, int g) {
    const int lane = threadIdx.x & 63;
    const int rank = p.rank, delta = link + 1, src = (rank + delta) & 7;
    char* tmo = myws + OFF_TMO;
    const rsrc_t rr = mk_rsrc(p.ws[src], WS_NEED), ro = mk_rsrc(p.out, 4096u * NTOT);
    const unsigned lane_off = (unsigned)(4 * (lane >> 4)) * NTOT + 8 * (lane & 15);
    char* ctr = myws + OFF_TICK + link * 128;
    const unsigned par = (seq & 1u) * NEXP;
    const int win = TUNED(0, WIN_IN), wout = TUNED(1, WIN_OUT);
    tick_init(ctr, seq, PULL_WG * TUNED(2, PULL_W));
    bool dead = false;
    for (int it = g; it < NI_ALL;) {
        unsigned cxi, cat;
        item_decode(it, rank, delta, par, cxi, cat);
        const char* fl = myws + (it >= NI_IN ? OFF_OFLAG + src * 8192 : OFF_XFLAG + (seq & 1u) * XFLAG_SZ) + cat * 8;
        unsigned long long mn = ld_sys64(fl);
        if (RFL(mn >> 32) != seq) {
            unsigned long long t0 = __builtin_amdgcn_s_memrealtime();
            for (unsigned n = 1;; ++n) {
                mn = ld_sys64(fl);
                if (RFL(mn >> 32) == seq) break;
                if ((n & 31u) == 0u && give_up(tmo, seq, t0)) { mn = 0; dead = true; break; }
                __builtin_amdgcn_s_sleep(1);
            }
        }
        CFENCE();
        const unsigned cmeta = RFL(mn);
        u32x4 c[8];
        unsigned tk;
        {
            const unsigned lines = (cmeta >> 8) & 15u;
            const bool isout = it >= NI_IN, raw = lines == M_RAW;
            const unsigned bo = (unsigned)lane < lines * 8u ? lane * 16u : 0x80000000u;
            const unsigned o = raw ? (isout ? OFF_ROUT : OFF_RIN) + cxi * 8192 : (isout ? OFF_XOUT + cxi * XO_SZ : OFF_XIN + cxi * XI_SZ);
            const unsigned nld = raw ? 8u : (isout ? 7u : 6u);
            unsigned my;
            asm volatile("s_mov_b64 exec, 1\n\tds_add_rtn_u32 %0, %1, %2\n\ts_mov_b64 exec, -1\n\ts_waitcnt lgkmcnt(0)" : "=&v"(my) : "v"(sem), "v"(nld) : "memory");
            my = RFL(my) - max((unsigned)(isout ? wout : win), nld);
            unsigned rel = RFL(__hip_atomic_load((LAS unsigned*)(size_t)(sem + 4), __ATOMIC_RELAXED, __HIP_MEMORY_SCOPE_WORKGROUP));
#define GATE(u) while ((int)(my + (u) - rel) >= 0) { rel = RFL(__hip_atomic_load((LAS unsigned*)(size_t)(sem + 4), __ATOMIC_RELAXED, __HIP_MEMORY_SCOPE_WORKGROUP)); CFENCE(); }
#define RELEASE(n, v) asm volatile("s_waitcnt vmcnt(%0)\n\ts_mov_b64 exec, 1\n\tds_add_u32 %1, %2 offset:4\n\ts_mov_b64 exec, -1" :: "n"(n), "v"(sem), "v"(v) : "memory")
            while ((int)(my - 4u - rel) >= 0) { __builtin_amdgcn_s_sleep(2); rel = RFL(__hip_atomic_load((LAS unsigned*)(size_t)(sem + 4), __ATOMIC_RELAXED, __HIP_MEMORY_SCOPE_WORKGROUP)); }
            __builtin_amdgcn_s_setprio(3);
            GATE(0u); c[4] = bld16(rr, lane * 16, o + 4096);
            GATE(1u); c[5] = bld16(rr, isout ? lane * 16u : bo, o + 5120);
            GATE(2u); c[0] = bld16(rr, lane * 16, o);
            GATE(3u); c[1] = bld16(rr, lane * 16 + 1024, o);
            GATE(4u); c[2] = bld16(rr, lane * 16 + 2048, o);
            GATE(5u); c[3] = bld16(rr, lane * 16 + 3072, o);
            GATE(nld - 1u);
            c[6] = bld16(rr, (isout || raw) ? bo : 0x80000000u, o + 6144);
            c[7] = bld16(rr, raw ? lane * 16u : 0x80000000u, o + 7168);
            tk = tick_draw(ctr);
            const unsigned one = 1u, last = nld - 5u;
            RELEASE(8, one); RELEASE(7, one); RELEASE(6, one); RELEASE(5, one); RELEASE(4, one); RELEASE(1, last);
            __builtin_amdgcn_s_setprio(0);
        }
        {
            const unsigned lines = (cmeta >> 8) & 15u;
            if (it >= NI_IN) {
                unsigned q[32];
                if (dead) {
#pragma unroll
                    for (int d = 0; d < 32; ++d) q[d] = 0x80808080u;
                } else if (lines == M_RAW) {
#pragma unroll
                    for (int d = 0; d < 32; ++d) q[d] = c[d >> 2][d & 3];
                } else {
                    *(LAS u32x4*)(size_t)(scr + SC_BLOB + lane * 16) = c[6];
                    CFENCE();
                    const unsigned P[8] = {c[4].x, c[4].y, c[4].z, c[4].w, c[5].x, c[5].y, c[5].z, c[5].w};
                    decode<8, false>(P, q, scr, 0u, lines * 32u, 0u, lane);
#pragma unroll
                    for (int d = 0; d < 32; ++d) {
                        const unsigned R = c[d >> 3][((d >> 1) & 2) + (d & 1)] >> ((d & 2) * 2);
                        const unsigned s8 = R & 0x08080808u;
                        q[d] = (q[d] ^ ((s8 << 5) - s8)) | (R & 0x07070707u);
                    }
                }
                store_rows(ro, lane_off, (cat >> 4) * (64u * NTOT) + src * NL + (cat & 15u) * 128u, q);
            } else {
                u32x4 o[8];
                if (lines == M_RAW) {
#pragma unroll
                    for (int u = 0; u < 8; ++u) o[u] = c[u];
                } else {
                    *(LAS u32x4*)(size_t)(scr + SC_BLOB + lane * 16) = c[5];
                    CFENCE();
                    const unsigned P[4] = {c[4].x, c[4].y, c[4].z, c[4].w};
                    const unsigned nex = (cmeta >> 12) & 15u;
                    unsigned ex[16];
                    decode<4, true>(P, ex, scr, nex, lines * 32u, cmeta & 0xffu, lane);
                    for (unsigned k = 0; k < nex; ++k) {
                        const unsigned en = RFL(LDS32(scr + SC_BLOB + k * 4));
                        const unsigned sh = ((en >> 8) & 3u) * 8u, r = (en >> 10) & 15u;
                        const unsigned msk = (unsigned)lane == ((en >> 14) & 63u) ? 0xffu << sh : 0u;
                        const unsigned val = rot8(en & 0xffu) << sh;
#pragma unroll
                        for (int d = 0; d < 16; ++d) { const unsigned mq = r == (unsigned)d ? msk : 0u; ex[d] = (ex[d] & ~mq) | (val & mq); }
                    }
#pragma unroll
                    for (int d = 0; d < 16; ++d) {
                        const unsigned r = c[d >> 2][d & 3];
                        const unsigned H = (r & 0x80808080u) | (ex[d] & 0x7f7f7f7fu), L = (ex[d] & 0x80808080u) | (r & 0x7f7f7f7fu);
                        o[d >> 1][(d & 1) * 2] = PERM(H, L, 0x05010400u);
                        o[d >> 1][(d & 1) * 2 + 1] = PERM(H, L, 0x07030602u);
                    }
                }
#pragma unroll
                for (int u = 0; u < 8; ++u) st16(myws, lane * 16, u & 3, OFF_STAGE + cat * 8192 + (u >> 2) * 4096, o[u]);
                WAIT_VM0();
                if (lane == 0) st_agent(myws + OFF_SFLAG + cat * 32, seq);
            }
        }
        const int nx = (int)(RFL(tk) & 0x7ffu);
        it = nx;
    }
}

__device__ __forceinline__ int step_group(int s, int rank, int d0) {
    if (s < 8) return rank * 8 + s;
    int t = s - 8;
    int kt = t / 7, u = t - kt * 7;
    int v = u < d0 ? u : u + 1;
    int delta = (d0 - v) & 7;
    return ((rank + delta) & 7) * 8 + kt;
}

__device__ __forceinline__ void gemm_role(const CAS Params& p, int gidx, unsigned seq, char* myws, LAS char* lds) {
    const int tid = threadIdx.x;
    const int wid = RFL(tid >> 6), lane = tid & 63;
    const int wr = wid >> 1, wc = wid & 1;
    const int rank = p.rank;
    const bool mainw = gidx >= 0;
    int ti, tj;
    if (mainw) { const int xcd = gidx & 7, sl = gidx >> 3; ti = (xcd >> 1) * 4 + (sl >> 2); tj = (xcd & 1) * 4 + (sl & 3); }
    else { const int x = -1 - gidx; ti = (x & 7) * 2 + ((x >> 3) & 1); tj = 8 + (x >> 4); }
    const int d0 = ti >> 1;
    const int myslot = wid < 4 ? ti : 16 + tj;
    char* tmo = myws + OFF_TMO;
    const char* dsrc = myws + OFF_STAGE + (size_t)myslot * PIECE + (wid & 3) * 8192;
    const char* fsrc = myws + OFF_SFLAG + myslot * 128 + (wid & 3) * 32;
    const unsigned loff = lane * 16, loff2 = loff + 4096;
    const unsigned ldsw = (unsigned)(size_t)lds + wid * 8192;

    f32x4 acc[4][8];
#pragma unroll
    for (int m = 0; m < 4; ++m)
#pragma unroll
        for (int i = 0; i < 8; ++i) acc[m][i] = (f32x4){0.f, 0.f, 0.f, 0.f};

#define GL(n, S) glds_if<((n) & 3) * 1024>(have, want, S, (n) < 4 ? loff : loff2, (n) < 4 ? ld0 : ld1)
#define STAGE_ALL(S, D) stage8k(S, loff, loff2, D)
#define PRIO(n) __builtin_amdgcn_s_setprio(n)
#define RDV 0
#define RD1(t) (((t) & 1) == 0 && (t) < 24 ? (t) / 2 : -1)
#define RD2(t) (((t) & 1) == 0 && (t) >= 2 && (t) < 26 ? (t) / 2 - 1 : -1)
#define GOFF(st) ((unsigned)__builtin_amdgcn_readlane((int)tab, (st)))
#define GSRC(off) (dsrc + (off))
#define GFLAG(off) (fsrc + ((off) >> 8))
    static_assert(PPK * PIECE / 256 == PPK * 128, "flag offset");
    const unsigned tab = (unsigned)step_group(lane, rank, d0) * (PPK * PIECE);
    const LAS char* abase = lds + wr * 8192 + lane * 16;
    const LAS char* bbase = lds + 32768 + wc * 16384 + lane * 16;
    bf16x8 a0[4], b0[8], a1[4], b1[8];

    if (!mainw) {
        const char* f = myws + OFF_SFLAG + ((((rank + 1) & 7) * 8 + 7) * PPK + 24) * 128;
        unsigned long long t0 = __builtin_amdgcn_s_memrealtime();
        for (unsigned n = 1; RFL(ld_sys(f)) < seq; ++n) {
            if ((n & 7u) == 0u && give_up(tmo, seq, t0)) break;
            __builtin_amdgcn_s_sleep(64);
        }
    }
    {
        const unsigned g0 = GOFF(0);
        spin_flag(GFLAG(g0), seq, tmo);
        STAGE_ALL(GSRC(g0), ldsw);
    }
    unsigned gn = GOFF(1);
    unsigned fv = ld_sys(GFLAG(gn));
    WAIT_VM0();
    RAW_BAR();
    STAMP(2);
    bool staged = RFL(fv) >= seq;
    if (staged) { STAGE_ALL(GSRC(gn), ldsw + 65536); gn = GOFF(2); }
    fv = ld_sys(GFLAG(gn));
#pragma unroll
    for (int i = 0; i < 8; ++i) b0[i] = *(const LAS bf16x8*)(bbase + i * 1024);
#pragma unroll
    for (int m = 0; m < 4; ++m) a0[m] = *(const LAS bf16x8*)(abase + m * 2048);

    for (int s = 0; s < 64; ++s) {
        const int cur = (s & 1) * 65536, oth = cur ^ 65536;
        if (!staged) {
            WAIT_VM0();
            if (RFL(fv) >= seq) { STAGE_ALL(GSRC(gn), ldsw + oth); staged = true; gn = GOFF(min(s + 2, 63)); fv = ld_sys(GFLAG(gn)); }
        }
        const LAS char* ab2 = abase + oth;
        const LAS char* bb2 = bbase + oth;
        const unsigned want = s + 2 < 64 ? seq : 0xffffffffu;
        const unsigned g3 = GOFF(min(s + 3, 63));
        const unsigned ld0 = ldsw + cur, ld1 = ld0 + 4096;
        SCHED0();
        {
            const LAS char* ab = abase + cur + 1024;
            const LAS char* bb = bbase + cur + 8192;
            PRIO(1);
            static_for<32>([&](auto tc) __attribute__((always_inline)) {
                constexpr int t = decltype(tc)::value;
                acc[t >> 3][t & 7] = __builtin_amdgcn_mfma_f32_16x16x32_bf16(a0[t >> 3], b0[t & 7], acc[t >> 3][t & 7], 0, 0, 0);
                constexpr int rd = RD1(t);
                if constexpr (rd >= 0 && rd < 8) b1[rd] = *(const LAS bf16x8*)(bb + rd * 1024);
                if constexpr (rd >= 8) a1[rd - 8] = *(const LAS bf16x8*)(ab + (rd - 8) * 2048);
                if constexpr (t == 15) PRIO(0);
                SCHED0();
            });
        }
        if (!staged) {
            spin_flag(GFLAG(gn), seq, tmo);
            STAGE_ALL(GSRC(gn), ldsw + oth);
            gn = GOFF(min(s + 2, 63));
            fv = ld_sys(GFLAG(gn));
        }
        const char* gs = GSRC(gn);
        WAIT_VM0();
        const unsigned have = RFL(fv);
        __builtin_amdgcn_s_waitcnt(0xC07F);
        PRIO(3);
        SCHED0();
        RAW_BAR();
        SCHED0();
        {
            static_for<32>([&](auto tc) __attribute__((always_inline)) {
                constexpr int t = decltype(tc)::value;
                acc[t >> 3][t & 7] = __builtin_amdgcn_mfma_f32_16x16x32_bf16(a1[t >> 3], b1[t & 7], acc[t >> 3][t & 7], 0, 0, 0);
                constexpr int rd = RD2(t);
                if constexpr (rd >= 0 && rd < 8) b0[rd] = *(const LAS bf16x8*)(bb2 + rd * 1024);
                if constexpr (rd >= 8) a0[rd - 8] = *(const LAS bf16x8*)(ab2 + (rd - 8) * 2048);
                if constexpr ((t & 3) == 1) GL(t >> 2, gs);
                if constexpr (t == 15) PRIO(2);
                if constexpr (t == 27) {
                    const bool go = have >= want;
                    staged = go || s + 2 >= 64;
                    gn = go ? g3 : gn;
                }
                if constexpr (t == 30) fv = ld_sys(GFLAG(gn));
                SCHED0();
            });
        }
    }
    STAMP(3);

    char* peer = p.ws[lane & 7];
    float am = 0.f;
#pragma unroll
    for (int m = 0; m < 4; ++m)
#pragma unroll
        for (int i = 0; i < 8; ++i)
#pragma unroll
            for (int j = 0; j < 4; ++j) am = fmaxf(am, fabsf(acc[m][i][j]));
#pragma unroll
    for (int o = 32; o > 0; o >>= 1) am = fmaxf(am, __shfl_xor(am, o));
    LAS float* red = (LAS float*)(lds + LDS_MISC + 512);
    if (lane == 0) red[wid] = am;
    RAW_BAR();
    if (tid < NR && mainw) {
        float a = red[0];
#pragma unroll
        for (int i = 1; i < 8; ++i) a = fmaxf(a, red[i]);
        st_sys64(peer + OFF_AMAX + (rank * NGEMM + gidx) * 8, ((unsigned long long)seq << 32) | __float_as_uint(a));
    }
    const unsigned scr = (unsigned)(size_t)lds + wid * SCR_SZ;
    zero_scratch(scr, lane);
    lut_init(scr, lane);
    {
        const char* gp = myws + OFF_AMAX + tid * 16;
        unsigned long long t0 = __builtin_amdgcn_s_memrealtime();
        unsigned long long a, b;
        for (unsigned n = 1;; ++n) {
            a = ld_sys64(gp);
            b = ld_sys64(gp + 8);
            bool ok = (unsigned)(a >> 32) == seq && (unsigned)(b >> 32) == seq;
            if (__all(ok)) break;
            if ((n & 31u) == 0u && give_up(tmo, seq, t0)) break;
            __builtin_amdgcn_s_sleep(2);
        }
        am = fmaxf(__uint_as_float((unsigned)a), __uint_as_float((unsigned)b));
#pragma unroll
        for (int o = 32; o > 0; o >>= 1) am = fmaxf(am, __shfl_xor(am, o));
        if (lane == 0) red[8 + wid] = am;
        WAIT_VM0();
        RAW_BAR();
#pragma unroll
        for (int i = 0; i < 8; ++i) am = fmaxf(am, red[8 + i]);
    }
    const unsigned tmov = ld_agent(tmo);
    STAMP(4);

    const float inv = am > 0.f ? 127.0f / am : 0.f;
    unsigned q[32];
#pragma unroll
    for (int m = 0; m < 4; ++m)
#pragma unroll
        for (int j = 0; j < 4; ++j)
#pragma unroll
            for (int h = 0; h < 2; ++h) {
                const unsigned t0 = __float_as_uint(fmaf(acc[m][4 * h + 0][j], inv, 12582912.f));
                const unsigned t1 = __float_as_uint(fmaf(acc[m][4 * h + 1][j], inv, 12582912.f));
                const unsigned t2 = __float_as_uint(fmaf(acc[m][4 * h + 2][j], inv, 12582912.f));
                const unsigned t3 = __float_as_uint(fmaf(acc[m][4 * h + 3][j], inv, 12582912.f));
                q[(m * 4 + j) * 2 + h] = PERM(t1, t0, 0x0c0c0400u) | PERM(t3, t2, 0x04000c0cu);
            }
    if (RFL(tmov) == seq) {
#pragma unroll
        for (int d = 0; d < 32; ++d) q[d] = 0x80808080u;
    }
    const unsigned lane_off = (unsigned)(4 * (lane >> 4)) * NTOT + 8 * (lane & 15);
    const rsrc_t ro = mk_rsrc(p.out, 4096u * NTOT);
    const unsigned rowoff = (unsigned)(ti * 4 + wr) * (64u * NTOT);
    if (!mainw) {
        const int u = (tj - 8) * 2 + wc;
        if (u < 7) store_rows(ro, lane_off, rowoff + ((rank + 1 + u) & 7) * NL + 15 * 128, q);
        return;
    }
    const int cb = tj * 2 + wc;
    if (cb < 15) {
        const unsigned xi = (ti * 4 + wr) * 16 + cb;
        unsigned n[16];
#pragma unroll
        for (int u = 0; u < 4; ++u) {
            u32x4 r;
#pragma unroll
            for (int k = 0; k < 4; ++k) {
                const unsigned q0 = q[8 * u + (k >> 1) * 4 + (k & 1)], q1 = q[8 * u + (k >> 1) * 4 + (k & 1) + 2];
                r[k] = BFI(0x77777777u, BFI(0x0f0f0f0fu, q0, q1 << 4), BFI(0x0f0f0f0fu, q0 >> 4, q1));
                const unsigned sg = (r[k] >> 3) & 0x11111111u;
                n[4 * u + k] = BFI(0x0f0f0f0fu, q0 >> 3, q1 << 1) ^ ((sg << 4) - sg);
            }
            st16(myws, lane * 16, u, OFF_XOUT + xi * XO_SZ, r);
        }
        unsigned P[8];
        const unsigned nb = encode<8, false>(n, P, scr, 0u, lane);
        unsigned lines = M_RAW;
        if (nb <= 256u) {
            lines = (nb + 31u) >> 5;
            st16(myws, lane * 16, 0, OFF_XOUT + xi * XO_SZ + 4096, (u32x4){P[0], P[1], P[2], P[3]});
            st16(myws, lane * 16, 1, OFF_XOUT + xi * XO_SZ + 4096, (u32x4){P[4], P[5], P[6], P[7]});
            if ((unsigned)lane < lines * 8u) st16(myws, lane * 16, 2, OFF_XOUT + xi * XO_SZ + 4096, *(LAS u32x4*)(size_t)(scr + SC_BLOB + lane * 16));
        } else {
#pragma unroll
            for (int u = 0; u < 8; ++u)
                st16(myws, lane * 16, u & 3, OFF_ROUT + xi * 8192 + (u >> 2) * 4096, (u32x4){q[4 * u], q[4 * u + 1], q[4 * u + 2], q[4 * u + 3]});
        }
        WAIT_VM0();
        if (lane < NR && lane != rank) st_sys64(peer + OFF_OFLAG + rank * 8192 + xi * 8, ((unsigned long long)seq << 32) | (lines << 8));
    }
    STAMP(5);
    store_rows(ro, lane_off, rowoff + rank * NL + cb * 128, q);
}

__global__ __launch_bounds__(NTHR, 2) __attribute__((amdgpu_waves_per_eu(2, 2))) void fused_kernel(Params p_unused) {
    extern __shared__ __attribute__((aligned(1024))) char smem[];
    const CAS Params& p = *(const CAS Params*)__builtin_amdgcn_kernarg_segment_ptr();
    char* myws = p.ws[p.rank];
    LAS char* lds = (LAS char*)smem;
    STAMP(0);
    const int b = blockIdx.x;
    const int wid = RFL(threadIdx.x >> 6), lane = threadIdx.x & 63;
    const unsigned scr = (unsigned)(size_t)lds + wid * SCR_SZ;
    char* cnt = myws + OFF_CNT + (b * 8 + wid) * 4;
    unsigned seq = ld_agent(cnt);
    const unsigned sem = (unsigned)(size_t)lds + LDS_MISC;
    if (threadIdx.x < 2) LDS32(sem + threadIdx.x * 4) = 0u;
    lut_init(scr, lane);
    if (lane == 0) LDS32(scr + SC_LUT + 64) = 0u;
    RAW_BAR();
    {
        constexpr int NS = NWG - NGEMM - NPWG;
        const bool spare = b >= NPWG && b < NWG - NGEMM;
        const int e2 = spare ? EXP_W1 * NWG + (7 - wid) * NS + (b - NPWG) : NEXP;
        int e = wid < EXP_W1 ? wid * NWG + b : e2;
        for (int i = 0; i < min(wid, EXP_W1) * TUNED(3, EXP_SLEEP); ++i) __builtin_amdgcn_s_sleep(4);
        char* peer = p.ws[lane & 7];
        bool first = true;
        while (e < NEXP) {
            export_item(p, myws, peer, scr, e, seq, first);
            first = false;
            e = e < EXP_W1 * NWG ? e2 : e + NS * 8;
        }
        if (first) seq = RFL(seq) + 1u;
    }
    if (lane == 0) st_agent(cnt, seq);
    if (b == NPWG && wid == 0)
        for (int i = lane; i < 128; i += 64) st_agent(myws + OFF_SFLAG + ((i >> 1) * PPK + 27) * 128 + 64 + (i & 1) * 32, seq);
    STAMP(1);
    if (b >= NWG - NGEMM - NXG) {
        RAW_BAR();
        gemm_role(p, b >= NWG - NGEMM ? b - (NWG - NGEMM) : -1 - (b - (NWG - NGEMM - NXG)), seq, myws, lds);
    } else if (b < NPWG && wid < TUNED(2, PULL_W)) {
        puller(p, myws, scr, sem, seq, b % 7, b / 7, (b / 7) + wid * PULL_WG);
        STAMP(4);
    }
}

}

extern "C" void kernel_launch(int rank, int nranks, ncclComm_t comm,
                              hipStream_t stream,
                              void* const* d_in, const int* in_sizes, int n_in,
                              void* d_out, int out_size,
                              const int* in_dtypes, int out_dtype,
                              void* const* d_out_all,
                              void* const* d_ws_all, size_t ws_bytes) {
    if (nranks != NR || ws_bytes < WS_NEED || (unsigned)rank >= (unsigned)NR) {
        fprintf(stderr, "unexpected configuration\n");
        return;
    }
    HIP_CHECK(hipFuncSetAttribute((const void*)fused_kernel, hipFuncAttributeMaxDynamicSharedMemorySize, LDS_BYTES));
    Params p;
    p.x = (const float*)d_in[0];
    p.w = (const float*)d_in[1];
    for (int i = 0; i < NR; ++i) p.ws[i] = (char*)d_ws_all[i];
    p.out = (char*)d_out;
    p.rank = rank;
    p.pad = 0;
    hipLaunchKernelGGL(fused_kernel, dim3(NWG), dim3(NTHR), LDS_BYTES, stream, p);
    HIP_CHECK(hipGetLastError());
}
```

```cpp
#include <hip/hip_runtime.h>
#include <rccl/rccl.h>
#include <cstdio>
#include <cstdint>

#define HIP_CHECK(expr) do { hipError_t _e = (expr); if (_e != hipSuccess) { \
    fprintf(stderr, "HIP error %s at %s:%d\n", hipGetErrorString(_e), __FILE__, __LINE__); \
    return; } } while (0)

namespace {

typedef unsigned u32x4 __attribute__((ext_vector_type(4)));
typedef unsigned u32x2 __attribute__((ext_vector_type(2)));
typedef float f32x4 __attribute__((ext_vector_type(4)));
typedef float f32x2 __attribute__((ext_vector_type(2)));
typedef __bf16 bf16x8 __attribute__((ext_vector_type(8)));
typedef __bf16 bf16x2 __attribute__((ext_vector_type(2)));
typedef unsigned short u16x2 __attribute__((ext_vector_type(2)));
typedef __amdgpu_buffer_rsrc_t rsrc_t;
#define GAS __attribute__((address_space(1)))
#define LAS __attribute__((address_space(3)))
#define CAS __attribute__((address_space(4)))

constexpr int NR = 8;
constexpr int NTOT = 16384;
constexpr int KL = 512;
constexpr int NL = 2048;
constexpr int PIECE = 32768;
#define NDUP 2
#define NPUSH 6
constexpr int NXT = (7 * NDUP + 1) / 2;
constexpr int NCB = 16 - NDUP;
constexpr int PPK = 24 + NXT;
constexpr int NGEMM = 128;
constexpr int NWG = 256;
constexpr int NTHR = 512;
constexpr int NEXP = 2560;
#define PULL_WG 9
#define PULL_W 8
#define WIN_IN 8
#define WIN_OUT 8
#define EXP_SLEEP 5
constexpr int NPWG = 7 * PULL_WG;
static_assert(NPWG < NWG - NGEMM, "puller count");
constexpr int NI_MAIN = 768 - 4 * NPUSH, NI_IN = NI_MAIN + 112 * NDUP, NI_ALL = NI_IN + 64 * NCB;
constexpr int XI_SZ = 6144;
constexpr int XO_SZ = 7168;

constexpr unsigned OFF_CNT = 0;
constexpr unsigned OFF_TMO = 8192;
constexpr unsigned OFF_TICK = 12288;
constexpr unsigned OFF_AMAX = 16384;
constexpr unsigned OFF_XFLAG = 32768;
constexpr unsigned XFLAG_SZ = NR * 8 * PPK * 32;
constexpr unsigned OFF_OFLAG = 163840;
constexpr unsigned OFF_SFLAG = 262144;
constexpr unsigned OFF_DBG = 512 * 1024;
constexpr unsigned OFF_XIN = 1u << 20;
constexpr unsigned OFF_XOUT = OFF_XIN + 2 * NEXP * XI_SZ;
constexpr unsigned OFF_STAGE = OFF_XOUT + 1024 * XO_SZ;
constexpr unsigned OFF_RIN = OFF_STAGE + NR * 8 * PPK * PIECE;
constexpr unsigned OFF_ROUT = OFF_RIN + 2 * NEXP * 8192;
constexpr unsigned WS_NEED = OFF_ROUT + 1024 * 8192;
static_assert(OFF_XFLAG + 2 * XFLAG_SZ <= OFF_OFLAG && OFF_OFLAG + 65536 <= OFF_SFLAG && OFF_SFLAG + NR * 8 * PPK * 128 <= OFF_DBG, "layout");
constexpr unsigned long long DEADLINE = 50000000ull;
constexpr int LDS_MISC = 131072;
constexpr int LDS_BYTES = LDS_MISC + 1024;
constexpr int AUX_SYS = 17;
constexpr int OUT_AUX = 17;
constexpr unsigned SCR_SZ = 10240;
constexpr unsigned SC_BLOB = 0, SC_A2 = 1152, SC_A3 = 5376, SC_LUT = 9600;
constexpr unsigned M_RAW = 15;

template <int V> struct IC { static constexpr int value = V; };
template <int N, int I = 0, class F> __device__ __forceinline__ void static_for(F&& f) {
    if constexpr (I < N) { f(IC<I>{}); static_for<N, I + 1>(f); }
}

struct Params {
    const float* x;
    const float* w;
    char* ws[NR];
    char* out;
    int rank;
    int pad;
};
#define TUNED(i, dflt) (dflt)

__device__ __forceinline__ unsigned ld_sys(const char* p) {
    return __hip_atomic_load((const GAS unsigned*)p, __ATOMIC_RELAXED, __HIP_MEMORY_SCOPE_SYSTEM);
}
__device__ __forceinline__ unsigned long long ld_sys64(const char* p) {
    return __hip_atomic_load((const GAS unsigned long long*)p, __ATOMIC_RELAXED, __HIP_MEMORY_SCOPE_SYSTEM);
}
__device__ __forceinline__ void st_sys(char* p, unsigned v) {
    __hip_atomic_store((GAS unsigned*)p, v, __ATOMIC_RELAXED, __HIP_MEMORY_SCOPE_SYSTEM);
}
__device__ __forceinline__ void st_sys64(char* p, unsigned long long v) {
    __hip_atomic_store((GAS unsigned long long*)p, v, __ATOMIC_RELAXED, __HIP_MEMORY_SCOPE_SYSTEM);
}
__device__ __forceinline__ unsigned ld_agent(const char* p) {
    return __hip_atomic_load((const GAS unsigned*)p, __ATOMIC_RELAXED, __HIP_MEMORY_SCOPE_AGENT);
}
__device__ __forceinline__ void st_agent(char* p, unsigned v) {
    __hip_atomic_store((GAS unsigned*)p, v, __ATOMIC_RELAXED, __HIP_MEMORY_SCOPE_AGENT);
}
__device__ __forceinline__ rsrc_t mk_rsrc(const void* p, unsigned bytes) {
    return __builtin_amdgcn_make_buffer_rsrc((void*)p, (short)0, (int)bytes, 0x00020000);
}
__device__ __forceinline__ u32x4 bld16(rsrc_t r, unsigned voff, unsigned soff) {
    return __builtin_amdgcn_raw_buffer_load_b128(r, (int)voff, (int)soff, AUX_SYS);
}
__device__ __forceinline__ void st16(const char* base, unsigned voff, int blk, unsigned soff, u32x4 v) {
    const char* b = base + soff;
    if (blk == 0) asm volatile("global_store_dwordx4 %0, %1, %2 sc0 sc1\n\ts_nop 1" :: "v"(voff), "v"(v), "s"(b) : "memory");
    if (blk == 1) asm volatile("global_store_dwordx4 %0, %1, %2 offset:1024 sc0 sc1\n\ts_nop 1" :: "v"(voff), "v"(v), "s"(b) : "memory");
    if (blk == 2) asm volatile("global_store_dwordx4 %0, %1, %2 offset:2048 sc0 sc1\n\ts_nop 1" :: "v"(voff), "v"(v), "s"(b) : "memory");
    if (blk == 3) asm volatile("global_store_dwordx4 %0, %1, %2 offset:3072 sc0 sc1\n\ts_nop 1" :: "v"(voff), "v"(v), "s"(b) : "memory");
}
#define WAIT_VM0() do { asm volatile("s_waitcnt vmcnt(0)" ::: "memory"); __builtin_amdgcn_s_waitcnt(0x0F70); } while (0)
#define WAIT_LGKM0() asm volatile("s_waitcnt lgkmcnt(0)" ::: "memory")
#define RAW_BAR() do { asm volatile("s_waitcnt lgkmcnt(0)" ::: "memory"); __builtin_amdgcn_s_barrier(); asm volatile("" ::: "memory"); } while (0)
#define SCHED0() __builtin_amdgcn_sched_barrier(0)
#define PERM(a, b, s) __builtin_amdgcn_perm((a), (b), (s))
#define RFL(x) ((unsigned)__builtin_amdgcn_readfirstlane((int)(x)))

#define GLB "sc0 sc1"
template <int OFF> __device__ __forceinline__ void glds_if(unsigned have, unsigned want, const char* src, unsigned loff, unsigned ldst) {
    asm volatile(
        "s_cmp_lt_u32 %0, %5\n\t"
        "s_cbranch_scc1 1f\n\t"
        "s_mov_b32 m0, %3\n\t"
        "s_nop 0\n\t"
        "global_load_lds_dwordx4 %1, %2 offset:%4 " GLB "\n"
        "1:"
        :: "s"(have), "v"(loff), "s"(src), "s"(ldst), "n"(OFF), "s"(want) : "memory", "scc");
}
__device__ __forceinline__ void stage8k(const char* src, unsigned loff, unsigned loff2, unsigned ldst) {
    asm volatile(
        "s_mov_b32 m0, %3\n\t"
        "s_nop 0\n\t"
        "global_load_lds_dwordx4 %0, %2 " GLB "\n\t"
        "global_load_lds_dwordx4 %0, %2 offset:1024 " GLB "\n\t"
        "global_load_lds_dwordx4 %0, %2 offset:2048 " GLB "\n\t"
        "global_load_lds_dwordx4 %0, %2 offset:3072 " GLB "\n\t"
        "s_add_u32 m0, m0, 0x1000\n\t"
        "s_nop 0\n\t"
        "global_load_lds_dwordx4 %1, %2 " GLB "\n\t"
        "global_load_lds_dwordx4 %1, %2 offset:1024 " GLB "\n\t"
        "global_load_lds_dwordx4 %1, %2 offset:2048 " GLB "\n\t"
        "global_load_lds_dwordx4 %1, %2 offset:3072 " GLB
        :: "v"(loff), "v"(loff2), "s"(src), "s"(ldst) : "memory", "scc");
}

__device__ __forceinline__ unsigned pk_bf16(float a, float b) {
    f32x2 v = {a, b};
    bf16x2 r = __builtin_convertvector(v, bf16x2);
    return __builtin_bit_cast(unsigned, r);
}

#define STAMP(i) do {} while (0)

__device__ __forceinline__ bool give_up(char* tmo, unsigned seq, unsigned long long t0) {
    if (RFL(ld_agent(tmo)) == seq || __builtin_amdgcn_s_memrealtime() - t0 > DEADLINE) {
        st_agent(tmo, seq);
        return true;
    }
    return false;
}
__device__ __forceinline__ void spin_flag(const char* flag, unsigned seq, char* tmo) {
    unsigned long long t0 = __builtin_amdgcn_s_memrealtime();
    for (unsigned n = 1; RFL(ld_sys(flag)) < seq; ++n) {
        if ((n & 31u) == 0u && give_up(tmo, seq, t0)) break;
        __builtin_amdgcn_s_sleep(1);
    }
    asm volatile("" ::: "memory");
}

typedef unsigned u32x2a __attribute__((ext_vector_type(2), aligned(4)));
#define LDS32(a) (*(LAS unsigned*)(size_t)(a))
#define CFENCE() asm volatile("" ::: "memory")

__device__ __forceinline__ unsigned wave_scan(unsigned x) {
    x += (unsigned)__builtin_amdgcn_update_dpp(0, (int)x, 0x111, 0xf, 0xf, false);
    x += (unsigned)__builtin_amdgcn_update_dpp(0, (int)x, 0x112, 0xf, 0xf, false);
    x += (unsigned)__builtin_amdgcn_update_dpp(0, (int)x, 0x114, 0xf, 0xf, false);
    x += (unsigned)__builtin_amdgcn_update_dpp(0, (int)x, 0x118, 0xf, 0xf, false);
    x += (unsigned)__builtin_amdgcn_update_dpp(0, (int)x, 0x142, 0xa, 0xf, false);
    x += (unsigned)__builtin_amdgcn_update_dpp(0, (int)x, 0x143, 0xc, 0xf, false);
    return x;
}
__device__ __forceinline__ unsigned esc_bits(unsigned P) { return __builtin_amdgcn_bitop3_b32(P, P >> 1, 0x55555555u, 0x80); }

__device__ __forceinline__ void lut_init(unsigned scr, int lane) {
    if (lane < 16) {
        unsigned sel = 0x0c0c0c0cu; int k = 0;
#pragma unroll
        for (int y = 0; y < 4; ++y) if ((lane >> y) & 1) { sel = (sel & ~(0xffu << (8 * k))) | ((unsigned)y << (8 * k)); ++k; }
        LDS32(scr + SC_LUT + lane * 4) = sel;
    }
}
#define BFI(m, x, y) __builtin_amdgcn_bitop3_b32((m), (x), (y), 0xCA)
__device__ __forceinline__ void zero_scratch(unsigned scr, int lane) {
#pragma unroll
    for (int i = 0; i < 9; ++i) *(LAS u32x4*)(size_t)(scr + i * 1024 + lane * 16) = (u32x4){0u, 0u, 0u, 0u};
    if (lane < 24) *(LAS u32x4*)(size_t)(scr + 9216 + lane * 16) = (u32x4){0u, 0u, 0u, 0u};
}
__device__ __forceinline__ unsigned digits1(unsigned n02, unsigned n13) {
    const unsigned lo = BFI(0x33333333u, n02, n13 << 2), hi = BFI(0x33333333u, n02 >> 2, n13);
    const unsigned h = (hi | (hi >> 1)) & 0x55555555u;
    return lo | h | (h << 1);
}
__device__ __forceinline__ unsigned sel_addr(unsigned E, int t, unsigned scr) {
    if (t == 0) return __builtin_amdgcn_udot4(E & 0x01010101u, 0x20100804u, scr + SC_LUT, false);
    if (t == 1) return __builtin_amdgcn_udot4(E & 0x04040404u, 0x08040201u, scr + SC_LUT, false);
    if (t == 2) return __builtin_amdgcn_udot4((E >> 4) & 0x01010101u, 0x20100804u, scr + SC_LUT, false);
    return __builtin_amdgcn_udot4((E >> 4) & 0x04040404u, 0x08040201u, scr + SC_LUT, false);
}
template <int NP, bool IN> __device__ __forceinline__ unsigned encode(const unsigned (&n)[2 * NP], unsigned (&P)[NP], unsigned scr, unsigned nb0, int lane) {
    unsigned E[NP], cnt = 0;
#pragma unroll
    for (int j = 0; j < NP; ++j) { P[j] = digits1(n[2 * j], n[2 * j + 1]); E[j] = esc_bits(P[j]); cnt += __popc(E[j]); }
    unsigned sel[2][8];
#pragma unroll
    for (int k = 0; k < 8; ++k) sel[0][k] = LDS32(sel_addr(E[k >> 2], k & 3, scr));
    cnt = max(cnt, 4u);
    const unsigned inc = wave_scan(cnt);
    const unsigned n2d = ((unsigned)__builtin_amdgcn_readlane((int)inc, 63) + 15u) >> 4;
    if (nb0 + n2d > 256u) return 1000u;
    unsigned pos = scr + ((lane & 1) ? SC_A3 : SC_A2) + inc - cnt, carry = 0;
    static_for<NP / 2>([&](auto gc) __attribute__((always_inline)) {
        constexpr int g = decltype(gc)::value;
        SCHED0();
        if constexpr (g + 1 < NP / 2) {
#pragma unroll
            for (int k = 0; k < 8; ++k) sel[(g + 1) & 1][k] = LDS32(sel_addr(E[2 * g + 2 + (k >> 2)], k & 3, scr));
        }
        SCHED0();
#pragma unroll
        for (int k = 0; k < 8; ++k) {
            const int j = 2 * g + (k >> 2), t = k & 3;
            const unsigned z = t < 2 ? n[2 * j + t] : n[2 * j + t - 2] >> 4;
            const unsigned long long v = (unsigned long long)PERM(0u, z, sel[g & 1][k]) << ((pos & 3u) * 8u);
            const unsigned lo = (unsigned)v | carry;
            LDS32(pos & ~3u) = lo;
            const unsigned np = pos + __popc(E[j] & (0x01010101u << (2 * t)));
            carry = ((np ^ pos) & 4u) ? (unsigned)(v >> 32) : lo;
            pos = np;
        }
    });
    LDS32(pos & ~3u) = carry;
    CFENCE();
    unsigned n3 = 0;
    const unsigned s3 = scr + SC_BLOB + (nb0 + n2d) * 4;
    for (unsigned i0 = 0; i0 < n2d; i0 += 64) {
        const unsigned i = i0 + lane;
        const u32x4 v = (*(LAS u32x4*)(size_t)(scr + SC_A2 + i * 16) | *(LAS u32x4*)(size_t)(scr + SC_A3 + i * 16)) & 0x0f0f0f0fu;
        unsigned x = 0;
#pragma unroll
        for (int t = 0; t < 4; ++t) {
            const unsigned z1 = v[t] + 0x01010101u;
            unsigned d = z1, e4;
            if (IN) { e4 = (((v[t] + 0x0b0b0b0bu) ^ z1) >> 4) & 0x01010101u; d |= z1 >> 3; }
            else e4 = ((v[t] + 0x0a0a0a0au) >> 4) & 0x01010101u;
            x |= ((d | e4 | (e4 << 1)) & 0x03030303u) << (2 * t);
        }
        const unsigned E2 = esc_bits(x);
        unsigned sl[4];
#pragma unroll
        for (int t = 0; t < 4; ++t) sl[t] = LDS32(sel_addr(E2, t, scr));
        const unsigned c = __popc(E2), ic = wave_scan(c);
        const unsigned tot = (unsigned)__builtin_amdgcn_readlane((int)ic, 63);
        if (nb0 + n2d + ((n3 + tot + 7u) >> 3) > 256u) return 1000u;
        if (i < n2d) LDS32(scr + SC_BLOB + (nb0 + i) * 4) = x;
        unsigned h[4];
#pragma unroll
        for (int t = 0; t < 4; ++t) { const unsigned y = PERM(0u, v[t], sl[t]); h[t] = PERM(0u, y | (y >> 4), 0x0c0c0200u); }
        const unsigned c0 = __popc(E2 & 0x01010101u), c01 = __popc(E2 & 0x05050505u), c2 = __popc(E2 & 0x10101010u);
        const unsigned long long acc = (unsigned long long)(h[0] | (h[1] << (4u * c0))) | ((unsigned long long)(h[2] | (h[3] << (4u * c2))) << (4u * c01));
        const unsigned p3 = n3 + ic - c, sh = (p3 & 7u) * 4u;
        if (c) {
            LAS unsigned* d = (LAS unsigned*)(size_t)(s3 + (p3 >> 3) * 4);
            const unsigned long long w = acc << sh;
            __hip_atomic_fetch_or(d, (unsigned)w, __ATOMIC_RELAXED, __HIP_MEMORY_SCOPE_WAVEFRONT);
            if (4u * c + sh > 32u) {
                __hip_atomic_fetch_or(d + 1, (unsigned)(w >> 32), __ATOMIC_RELAXED, __HIP_MEMORY_SCOPE_WAVEFRONT);
                __hip_atomic_fetch_or(d + 2, (unsigned)(((acc >> 32) << sh) >> 32), __ATOMIC_RELAXED, __HIP_MEMORY_SCOPE_WAVEFRONT);
            }
        }
        n3 += tot;
    }
    CFENCE();
    return nb0 + n2d + ((n3 + 7u) >> 3);
}
__device__ __forceinline__ unsigned rot8(unsigned e) { return ((e >> 1) & 0x7f7f7f7fu) | ((e << 7) & 0x80808080u); }
__device__ __forceinline__ void selectors(unsigned P, unsigned E, unsigned (&sel)[4]) {
    const unsigned pf = ((((E << 8) + E) << 8) + E) << 8;
    const unsigned S = BFI(E | (E << 1), pf, P);
    const unsigned A = BFI(0x33333333u, S, E << 2), B = BFI(0x33333333u, S >> 2, E);
    sel[0] = A & 0x07070707u; sel[1] = B & 0x07070707u; sel[2] = (A >> 4) & 0x07070707u; sel[3] = (B >> 4) & 0x07070707u;
}
template <int NP, bool IN> __device__ __forceinline__ void decode(const unsigned (&P)[NP], unsigned (&o)[4 * NP], unsigned scr, unsigned nb0, unsigned nbmax, unsigned T, int lane) {
    const unsigned lut1 = IN ? rot8(T | ((T - 1u) << 8) | ((T - 2u) << 16)) : 0x00100800u;
    const unsigned lut2 = IN ? rot8((T - 3u) | ((T - 4u) << 8) | ((T + 1u) << 16)) : 0x00282018u;
    unsigned E[NP], cnt = 0;
#pragma unroll
    for (int j = 0; j < NP; ++j) { E[j] = esc_bits(P[j]); cnt += __popc(E[j]); }
    cnt = max(cnt, 4u);
    const unsigned inc = wave_scan(cnt);
    const unsigned n2d = min(((unsigned)__builtin_amdgcn_readlane((int)inc, 63) + 15u) >> 4, 256u);
    for (unsigned i = lane; nb0 + n2d + i < nbmax; i += 64) {
        const unsigned W = LDS32(scr + SC_BLOB + (nb0 + n2d + i) * 4);
        unsigned a, b;
        if (IN) {
            const unsigned K = (T + 2u) * 0x01010101u;
            a = rot8(K - (((W & 0x0f0f0f0fu) + 0x02020202u) & 0x0f0f0f0fu));
            b = rot8(K - ((((W >> 4) & 0x0f0f0f0fu) + 0x02020202u) & 0x0f0f0f0fu));
        } else {
            a = (W << 3) & 0x78787878u;
            b = (W >> 1) & 0x78787878u;
        }
        *(LAS u32x2*)(size_t)(scr + SC_A3 + i * 8) = (u32x2){PERM(b, a, 0x05010400u), PERM(b, a, 0x07030602u)};
    }
    CFENCE();
    unsigned n3 = 0;
    for (unsigned i0 = 0; i0 < n2d; i0 += 64) {
        const unsigned i = i0 + lane;
        unsigned W = LDS32(scr + SC_BLOB + ((nb0 + i) & 255u) * 4);
        W = i < n2d ? W : 0u;
        const unsigned E2 = esc_bits(W), c = __popc(E2), ic = wave_scan(c);
        unsigned ps[4];
        u32x2a w[4];
        ps[0] = scr + SC_A3 + min(n3 + ic - c, 2048u);
#pragma unroll
        for (int t = 0; t < 4; ++t) {
            w[t] = *(LAS u32x2a*)(size_t)(ps[t] & ~3u);
            if (t < 3) ps[t + 1] = ps[t] + __popc(E2 & (0x01010101u << (2 * t)));
        }
        SCHED0();
        n3 += (unsigned)__builtin_amdgcn_readlane((int)ic, 63);
        unsigned sel[4];
        selectors(W, E2, sel);
        SCHED0();
        u32x4 v;
#pragma unroll
        for (int t = 0; t < 4; ++t) v[t] = PERM(__builtin_amdgcn_alignbyte(w[t].y, w[t].x, ps[t]), lut2, sel[t]);
        *(LAS u32x4*)(size_t)(scr + SC_A2 + i * 16) = v;
    }
    CFENCE();
    unsigned pos = scr + SC_A2 + min(inc - cnt, 4096u);
    unsigned ps[2][8];
    u32x2a w[2][8];
    static_for<NP / 2 + 1>([&](auto gc) __attribute__((always_inline)) {
        constexpr int g = decltype(gc)::value;
        SCHED0();
        if constexpr (g < NP / 2) {
#pragma unroll
            for (int k = 0; k < 8; ++k) {
                ps[g & 1][k] = pos;
                w[g & 1][k] = *(LAS u32x2a*)(size_t)(pos & ~3u);
                pos += __popc(E[2 * g + (k >> 2)] & (0x01010101u << (2 * (k & 3))));
            }
        }
        SCHED0();
        if constexpr (g > 0) {
#pragma unroll
            for (int h = 0; h < 2; ++h) {
                const int j = 2 * (g - 1) + h;
                unsigned sel[4];
                selectors(P[j], E[j], sel);
#pragma unroll
                for (int t = 0; t < 4; ++t) {
                    const u32x2a x = w[(g - 1) & 1][4 * h + t];
                    o[4 * j + t] = PERM(__builtin_amdgcn_alignbyte(x.y, x.x, ps[(g - 1) & 1][4 * h + t]), lut1, sel[t]);
                }
            }
        }
    });
    CFENCE();
}

__device__ __forceinline__ void exp_decode(int e, int rank, int& kt, int& slot, int& sub, int& dst) {
    sub = e & 3;
    const int q = e >> 2;
    if (q < 192) {
        kt = q / 24; slot = q - kt * 24; dst = slot < 16 ? -1 : rank;
        return;
    }
    const int t = q - 192;
    kt = t / 56;
    const int u = t - kt * 56, j = u / 7;
    slot = 16 + j;
    dst = (rank + 1 + (u - j * 7)) & 7;
}
__device__ __forceinline__ unsigned xin_item(int kt, int slot, int sub, int dst) {
    return (slot < 16 ? kt * 16 + slot : 128 + (dst * 8 + kt) * 8 + (slot - 16)) * 4 + sub;
}

__device__ __forceinline__ void export_item(const CAS Params& p, char* myws, char* peer, unsigned scr, int e, unsigned& seq, bool first) {
    const int lane = threadIdx.x & 63;
    const int rank = p.rank;
    int kt, slot, sub, dst;
    exp_decode(e, rank, kt, slot, sub, dst);
    const int kq = lane >> 4, c = lane & 15;
    u32x4 ch[8];
    if (slot < 16) {
        const GAS float* b = (const GAS float*)p.x + (size_t)(256 * slot + 64 * sub + c) * KL + 64 * kt + 8 * kq;
        f32x4 v[16];
#pragma unroll
        for (int i = 0; i < 8; ++i) {
            v[2 * i] = *(const GAS f32x4*)(b + (i >> 1) * 16 * KL + (i & 1) * 32);
            v[2 * i + 1] = *(const GAS f32x4*)(b + (i >> 1) * 16 * KL + (i & 1) * 32 + 4);
        }
        zero_scratch(scr, lane);
#pragma unroll
        for (int i = 0; i < 8; ++i) {
            ch[i].x = pk_bf16(v[2 * i][0], v[2 * i][1]); ch[i].y = pk_bf16(v[2 * i][2], v[2 * i][3]);
            ch[i].z = pk_bf16(v[2 * i + 1][0], v[2 * i + 1][1]); ch[i].w = pk_bf16(v[2 * i + 1][2], v[2 * i + 1][3]);
        }
    } else {
        const GAS float* b = (const GAS float*)p.w + (size_t)(64 * kt + 32 * (sub & 1) + 8 * kq) * NTOT + dst * NL + 256 * (slot - 16) + 128 * (sub >> 1) + 8 * c;
        f32x4 v[16];
#pragma unroll
        for (int jj = 0; jj < 8; ++jj) {
            v[2 * jj] = *(const GAS f32x4*)(b + (size_t)jj * NTOT);
            v[2 * jj + 1] = *(const GAS f32x4*)(b + (size_t)jj * NTOT + 4);
        }
        zero_scratch(scr, lane);
#pragma unroll
        for (int i = 0; i < 8; ++i) {
            ch[i].x = pk_bf16(v[0 + (i >> 2)][i & 3], v[2 + (i >> 2)][i & 3]); ch[i].y = pk_bf16(v[4 + (i >> 2)][i & 3], v[6 + (i >> 2)][i & 3]);
            ch[i].z = pk_bf16(v[8 + (i >> 2)][i & 3], v[10 + (i >> 2)][i & 3]); ch[i].w = pk_bf16(v[12 + (i >> 2)][i & 3], v[14 + (i >> 2)][i & 3]);
        }
    }
    if (first) seq = RFL(seq) + 1u;
    const bool dup = slot == 23 && sub >= 4 - 2 * NDUP;
    const bool local = dst < 0 || dst == rank || dup, remote = dst != rank || dup;
    const int d = lane & 7;
    const unsigned at = (rank * 8 + kt) * (PPK * 4) + ((dup && d != dst) ? 96 + ((dst - d - 1) & 7) * (2 * NDUP) + sub - (4 - 2 * NDUP) : slot * 4 + sub);
    if (kt == 0 && !local && slot < 16 + NPUSH) {
        char* pw = p.ws[dst];
        const unsigned th = rank * 8 * (PPK * 4) + slot * 4 + sub;
#pragma unroll
        for (int u = 0; u < 8; ++u) st16(pw, lane * 16, u & 3, OFF_STAGE + th * 8192 + (u >> 2) * 4096, ch[u]);
        WAIT_VM0();
        if (lane == 0) st_sys(pw + OFF_SFLAG + th * 32, seq);
        if (first) STAMP(12);
        return;
    }
    if (local) {
        const unsigned mine = (unsigned)__builtin_amdgcn_readlane((int)at, rank);
#pragma unroll
        for (int u = 0; u < 8; ++u) st16(myws, lane * 16, u & 3, OFF_STAGE + mine * 8192 + (u >> 2) * 4096, ch[u]);
        if (!remote) {
            WAIT_VM0();
            if (lane == rank) st_agent(myws + OFF_SFLAG + at * 32, seq);
            return;
        }
    }
    unsigned meta = 0;
    {
        const unsigned xi = xin_item(kt, slot, sub, dst) + (seq & 1u) * NEXP;
        unsigned ex[16], sum = 0;
#pragma unroll
        for (int u = 0; u < 4; ++u) {
            u32x4 r;
#pragma unroll
            for (int k = 0; k < 4; ++k) {
                const unsigned d0 = ch[2 * u + (k >> 1)][(k & 1) * 2], d1 = ch[2 * u + (k >> 1)][(k & 1) * 2 + 1];
                const unsigned L = PERM(d1, d0, 0x06040200u), H = PERM(d1, d0, 0x07050301u);
                r[k] = (H & 0x80808080u) | (L & 0x7f7f7f7fu);
                ex[4 * u + k] = ((H << 1) & 0xfefefefeu) | ((L >> 7) & 0x01010101u);
                sum = __builtin_amdgcn_sad_u8(ex[4 * u + k], 0u, sum);
            }
            st16(myws, lane * 16, u, OFF_XIN + xi * XI_SZ, r);
        }
        unsigned T = ((unsigned)__builtin_amdgcn_readlane((int)wave_scan(sum), 63) + 7864u) >> 12;
        T = min(max(T, 13u), 253u);
        const unsigned K = (T + 2u) * 0x01010101u;
        unsigned z[16], yy[16], bad = 0;
#pragma unroll
        for (int d = 0; d < 16; ++d) {
            const unsigned y = ((K | 0x80808080u) - (ex[d] & 0x7f7f7f7fu)) ^ (~(K ^ ex[d]) & 0x80808080u);
            bad |= y; yy[d] = y;
            z[d] = ((y & 0x0f0f0f0fu) + 0x0e0e0e0eu) & 0x0f0f0f0fu;
        }
        unsigned nex = 0;
        if (__any((bad & 0xf0f0f0f0u) != 0u)) {
            CFENCE();
            unsigned mk = 0;
#pragma unroll
            for (int d = 0; d < 16; ++d) mk |= (yy[d] & 0xf0f0f0f0u) ? 1u << d : 0u;
            while (mk) {
                const unsigned d = __builtin_ctz(mk);
                mk &= mk - 1u;
                unsigned ev = ex[0];
#pragma unroll
                for (int t = 1; t < 16; ++t) ev = d == (unsigned)t ? ex[t] : ev;
#pragma unroll 1
                for (unsigned y = 0; y < 4; ++y) {
                    const unsigned eb = (ev >> (8 * y)) & 0xffu;
                    if (T + 2u - eb > 15u) {
                        const unsigned k = __hip_atomic_fetch_add((LAS unsigned*)(size_t)(scr + SC_LUT + 64), 1u, __ATOMIC_RELAXED, __HIP_MEMORY_SCOPE_WAVEFRONT);
                        if (k < 15u) LDS32(scr + SC_BLOB + k * 4) = eb | (y << 8) | (d << 10) | (lane << 14);
                    }
                }
            }
            CFENCE();
            nex = RFL(LDS32(scr + SC_LUT + 64));
            CFENCE();
            LDS32(scr + SC_LUT + 64) = 0u;
        }
        if (local) {
            WAIT_VM0();
            if (lane == rank) st_agent(myws + OFF_SFLAG + at * 32, seq);
        }
        if (first) STAMP(10);
        unsigned P[4], nb = 1000u;
        if (nex <= 15u) {
            unsigned n[8];
#pragma unroll
            for (int k = 0; k < 8; ++k) n[k] = z[(k >> 1) * 4 + (k & 1)] | (z[(k >> 1) * 4 + (k & 1) + 2] << 4);
            nb = encode<4, true>(n, P, scr, nex, lane);
        }
        if (nb <= 256u) {
            const unsigned lines = (nb + 31u) >> 5;
            st16(myws, lane * 16, 0, OFF_XIN + xi * XI_SZ + 4096, (u32x4){P[0], P[1], P[2], P[3]});
            if ((unsigned)lane < lines * 8u) st16(myws, lane * 16, 1, OFF_XIN + xi * XI_SZ + 4096, *(LAS u32x4*)(size_t)(scr + SC_BLOB + lane * 16));
            meta = T | (lines << 8) | (nex << 12);
        } else {
#pragma unroll
            for (int u = 0; u < 8; ++u) st16(myws, lane * 16, u & 3, OFF_RIN + xi * 8192 + (u >> 2) * 4096, ch[u]);
            meta = M_RAW << 8;
        }
    }
    if (first) STAMP(11);
    WAIT_VM0();
    if (first) STAMP(12);
    if (lane < NR && lane != rank && (dst < 0 || dup || lane == dst))
        st_sys64(peer + OFF_XFLAG + (seq & 1u) * XFLAG_SZ + at * 8, ((unsigned long long)seq << 32) | meta);
}

__device__ __forceinline__ void tick_init(char* ctr, unsigned seq, unsigned first) {
    if ((threadIdx.x & 63) == 0)
        __hip_atomic_fetch_max((GAS unsigned*)ctr, (seq << 11) + first, __ATOMIC_RELAXED, __HIP_MEMORY_SCOPE_AGENT);
    WAIT_VM0();
}
__device__ __forceinline__ unsigned tick_draw(char* ctr) {
    return (unsigned)__builtin_amdgcn_raw_ptr_buffer_atomic_add_i32(1, mk_rsrc(ctr, 4), (threadIdx.x & 63) ? 64 : 0, 0, 16);
}
static_assert(NI_ALL + 2 * PULL_WG * 8 < 2048, "ticket range");

__device__ __forceinline__ void item_decode(int it, int rank, int delta, unsigned par, unsigned& xi, unsigned& at) {
    if (it >= NI_IN) {
        const int o = it - NI_IN, r = o / NCB;
        xi = at = (((r & 31) >> 1) * 4 + (r & 1) + 2 * (r >> 5)) * 16 + (o - r * NCB);
        return;
    }
    const int src = (rank + delta) & 7;
    if (it >= NI_MAIN) {
        const int o = it - NI_MAIN, kt = o / (14 * NDUP), r = o - kt * (14 * NDUP), u = r / (2 * NDUP);
        xi = xin_item(kt, 23, 4 - 2 * NDUP + (r & (2 * NDUP - 1)), (rank + 1 + u) & 7) + par;
        at = (src * 8 + kt) * (PPK * 4) + 96 + r;
        return;
    }
    const int q = (it >> 2) + NPUSH;
    int kt, slot;
    if (q < 8) { kt = 0; slot = 16 + q; }
    else {
        const int qq = q - 8, r = qq / 24, e = qq - r * 24;
        if (r == 7) { kt = 7; slot = (e + 2 * delta) & 15; }
        else {
            const int t3 = e / 3, m3 = e - t3 * 3;
            if (m3 < 2) { kt = r; slot = (2 * t3 + m3 + 2 * delta) & 15; }
            else { kt = r + 1; slot = 16 + t3; }
        }
    }
    xi = xin_item(kt, slot, it & 3, rank) + par;
    at = (src * 8 + kt) * (PPK * 4) + slot * 4 + (it & 3);
}

template <int AUX> __device__ __forceinline__ void store_rows(rsrc_t ro, unsigned lane_off, unsigned soff, const unsigned (&q)[32]) {
#pragma unroll
    for (int m = 0; m < 4; ++m)
#pragma unroll
        for (int j = 0; j < 4; ++j) {
            u32x2 v = {q[(m * 4 + j) * 2], q[(m * 4 + j) * 2 + 1]};
            __builtin_amdgcn_raw_buffer_store_b64(v, ro, (int)lane_off, (int)(soff + (m * 16 + j) * NTOT), AUX);
        }
}

__device__ __forceinline__ void puller(const CAS Params& p, char* myws, unsigned scr, unsigned sem, unsigned seq, int link, int gw, int g) {
    const int lane = threadIdx.x & 63;
    const int rank = p.rank, delta = link + 1, src = (rank + delta) & 7;
    char* tmo = myws + OFF_TMO;
    const rsrc_t rr = mk_rsrc(p.ws[src], WS_NEED), ro = mk_rsrc(p.out, 4096u * NTOT);
    const unsigned lane_off = (unsigned)(4 * (lane >> 4)) * NTOT + 8 * (lane & 15);
    char* ctr = myws + OFF_TICK + link * 128;
    const unsigned par = (seq & 1u) * NEXP;
    const int win = TUNED(0, WIN_IN), wout = TUNED(1, WIN_OUT);
    tick_init(ctr, seq, PULL_WG * TUNED(2, PULL_W));
    bool dead = false;
    for (int it = g; it < NI_ALL;) {
        unsigned cxi, cat;
        item_decode(it, rank, delta, par, cxi, cat);
        const char* fl = myws + (it >= NI_IN ? OFF_OFLAG + src * 8192 : OFF_XFLAG + (seq & 1u) * XFLAG_SZ) + cat * 8;
        unsigned long long mn = ld_sys64(fl);
        if (RFL(mn >> 32) != seq) {
            unsigned long long t0 = __builtin_amdgcn_s_memrealtime();
            for (unsigned n = 1;; ++n) {
                mn = ld_sys64(fl);
                if (RFL(mn >> 32) == seq) break;
                if ((n & 31u) == 0u && give_up(tmo, seq, t0)) { mn = 0; dead = true; break; }
                __builtin_amdgcn_s_sleep(1);
            }
        }
        CFENCE();
        const unsigned cmeta = RFL(mn);
        u32x4 c[8];
        unsigned tk;
        {
            const unsigned lines = (cmeta >> 8) & 15u;
            const bool isout = it >= NI_IN, raw = lines == M_RAW;
            const unsigned bo = (unsigned)lane < lines * 8u ? lane * 16u : 0x80000000u;
            const unsigned o = raw ? (isout ? OFF_ROUT : OFF_RIN) + cxi * 8192 : (isout ? OFF_XOUT + cxi * XO_SZ : OFF_XIN + cxi * XI_SZ);
            const unsigned nld = raw ? 8u : (isout ? 7u : 6u);
            unsigned my;
            asm volatile("s_mov_b64 exec, 1\n\tds_add_rtn_u32 %0, %1, %2\n\ts_mov_b64 exec, -1\n\ts_waitcnt lgkmcnt(0)" : "=&v"(my) : "v"(sem), "v"(nld) : "memory");
            my = RFL(my) - max((unsigned)(isout ? wout : win), nld);
            unsigned rel = RFL(__hip_atomic_load((LAS unsigned*)(size_t)(sem + 4), __ATOMIC_RELAXED, __HIP_MEMORY_SCOPE_WORKGROUP));
#define GATE(u) while ((int)(my + (u) - rel) >= 0) { rel = RFL(__hip_atomic_load((LAS unsigned*)(size_t)(sem + 4), __ATOMIC_RELAXED, __HIP_MEMORY_SCOPE_WORKGROUP)); CFENCE(); }
#define RELEASE(n, v) asm volatile("s_waitcnt vmcnt(%0)\n\ts_mov_b64 exec, 1\n\tds_add_u32 %1, %2 offset:4\n\ts_mov_b64 exec, -1" :: "n"(n), "v"(sem), "v"(v) : "memory")
            while ((int)(my - 4u - rel) >= 0) { __builtin_amdgcn_s_sleep(2); rel = RFL(__hip_atomic_load((LAS unsigned*)(size_t)(sem + 4), __ATOMIC_RELAXED, __HIP_MEMORY_SCOPE_WORKGROUP)); }
            __builtin_amdgcn_s_setprio(3);
            GATE(0u); c[4] = bld16(rr, lane * 16, o + 4096);
            GATE(1u); c[5] = bld16(rr, isout ? lane * 16u : bo, o + 5120);
            GATE(2u); c[0] = bld16(rr, lane * 16, o);
            GATE(3u); c[1] = bld16(rr, lane * 16 + 1024, o);
            GATE(4u); c[2] = bld16(rr, lane * 16 + 2048, o);
            GATE(5u); c[3] = bld16(rr, lane * 16 + 3072, o);
            GATE(nld - 1u);
            c[6] = bld16(rr, (isout || raw) ? bo : 0x80000000u, o + 6144);
            c[7] = bld16(rr, raw ? lane * 16u : 0x80000000u, o + 7168);
            tk = tick_draw(ctr);
            const unsigned one = 1u, last = nld - 5u;
            RELEASE(8, one); RELEASE(7, one); RELEASE(6, one); RELEASE(5, one); RELEASE(4, one); RELEASE(1, last);
            __builtin_amdgcn_s_setprio(0);
        }
        {
            const unsigned lines = (cmeta >> 8) & 15u;
            if (it >= NI_IN) {
                unsigned q[32];
                if (dead) {
#pragma unroll
                    for (int d = 0; d < 32; ++d) q[d] = 0x80808080u;
                } else if (lines == M_RAW) {
#pragma unroll
                    for (int d = 0; d < 32; ++d) q[d] = c[d >> 2][d & 3];
                } else {
                    *(LAS u32x4*)(size_t)(scr + SC_BLOB + lane * 16) = c[6];
                    CFENCE();
                    const unsigned P[8] = {c[4].x, c[4].y, c[4].z, c[4].w, c[5].x, c[5].y, c[5].z, c[5].w};
                    decode<8, false>(P, q, scr, 0u, lines * 32u, 0u, lane);
#pragma unroll
                    for (int d = 0; d < 32; ++d) {
                        const unsigned R = c[d >> 3][((d >> 1) & 2) + (d & 1)] >> ((d & 2) * 2);
                        const unsigned s8 = R & 0x08080808u;
                        q[d] = (q[d] ^ ((s8 << 5) - s8)) | (R & 0x07070707u);
                    }
                }
                store_rows<OUT_AUX>(ro, lane_off, (cat >> 4) * (64u * NTOT) + src * NL + (cat & 15u) * 128u, q);
            } else {
                u32x4 o[8];
                if (lines == M_RAW) {
#pragma unroll
                    for (int u = 0; u < 8; ++u) o[u] = c[u];
                } else {
                    *(LAS u32x4*)(size_t)(scr + SC_BLOB + lane * 16) = c[5];
                    CFENCE();
                    const unsigned P[4] = {c[4].x, c[4].y, c[4].z, c[4].w};
                    const unsigned nex = (cmeta >> 12) & 15u;
                    unsigned ex[16];
                    decode<4, true>(P, ex, scr, nex, lines * 32u, cmeta & 0xffu, lane);
                    for (unsigned k = 0; k < nex; ++k) {
                        const unsigned en = RFL(LDS32(scr + SC_BLOB + k * 4));
                        const unsigned sh = ((en >> 8) & 3u) * 8u, r = (en >> 10) & 15u;
                        const unsigned msk = (unsigned)lane == ((en >> 14) & 63u) ? 0xffu << sh : 0u;
                        const unsigned val = rot8(en & 0xffu) << sh;
#pragma unroll
                        for (int d = 0; d < 16; ++d) { const unsigned mq = r == (unsigned)d ? msk : 0u; ex[d] = (ex[d] & ~mq) | (val & mq); }
                    }
#pragma unroll
                    for (int d = 0; d < 16; ++d) {
                        const unsigned r = c[d >> 2][d & 3];
                        const unsigned H = (r & 0x80808080u) | (ex[d] & 0x7f7f7f7fu), L = (ex[d] & 0x80808080u) | (r & 0x7f7f7f7fu);
                        o[d >> 1][(d & 1) * 2] = PERM(H, L, 0x05010400u);
                        o[d >> 1][(d & 1) * 2 + 1] = PERM(H, L, 0x07030602u);
                    }
                }
#pragma unroll
                for (int u = 0; u < 8; ++u) st16(myws, lane * 16, u & 3, OFF_STAGE + cat * 8192 + (u >> 2) * 4096, o[u]);
                WAIT_VM0();
                if (lane == 0) st_agent(myws + OFF_SFLAG + cat * 32, seq);
            }
        }
        const int nx = (int)(RFL(tk) & 0x7ffu);
        it = nx;
    }
}

__device__ __forceinline__ int step_group(int s, int rank, int d0) {
    if (s < 8) return rank * 8 + s;
    int t = s - 8;
    int kt = t / 7, u = t - kt * 7;
    int v = u < d0 ? u : u + 1;
    int delta = (d0 - v) & 7;
    return ((rank + delta) & 7) * 8 + kt;
}

__device__ __forceinline__ void gemm_role(const CAS Params& p, int gidx, unsigned seq, char* myws, LAS char* lds) {
    const int tid = threadIdx.x;
    const int wid = RFL(tid >> 6), lane = tid & 63;
    const int wr = wid >> 1, wc = wid & 1;
    const int rank = p.rank;
    const bool mainw = gidx >= 0;
    int ti, tj;
    if (mainw) { const int xcd = gidx & 7, sl = gidx >> 3; ti = (xcd >> 1) * 4 + (sl >> 2); tj = (xcd & 1) * 4 + (sl & 3); }
    else { const int x = -1 - gidx; ti = (x & 7) * 2 + ((x >> 3) & 1); tj = 8 + (x >> 4); }
    const int d0 = ti >> 1;
    const int myslot = wid < 4 ? ti : 16 + tj;
    char* tmo = myws + OFF_TMO;
    const char* dsrc = myws + OFF_STAGE + (size_t)myslot * PIECE + (wid & 3) * 8192;
    const char* fsrc = myws + OFF_SFLAG + myslot * 128 + (wid & 3) * 32;
    const unsigned loff = lane * 16, loff2 = loff + 4096;
    const unsigned ldsw = (unsigned)(size_t)lds + wid * 8192;

    f32x4 acc[4][8];
#pragma unroll
    for (int m = 0; m < 4; ++m)
#pragma unroll
        for (int i = 0; i < 8; ++i) acc[m][i] = (f32x4){0.f, 0.f, 0.f, 0.f};

#define GL(n, S) glds_if<((n) & 3) * 1024>(have, want, S, (n) < 4 ? loff : loff2, (n) < 4 ? ld0 : ld1)
#define STAGE_ALL(S, D) stage8k(S, loff, loff2, D)
#define PRIO(n) __builtin_amdgcn_s_setprio(n)
#define RDV 0
#define RD1(t) (((t) & 1) == 0 && (t) < 24 ? (t) / 2 : -1)
#define RD2(t) (((t) & 1) == 0 && (t) >= 2 && (t) < 26 ? (t) / 2 - 1 : -1)
#define GOFF(st) ((unsigned)__builtin_amdgcn_readlane((int)tab, (st)))
#define GSRC(off) (dsrc + (off))
#define GFLAG(off) (fsrc + ((off) >> 8))
    static_assert(PPK * PIECE / 256 == PPK * 128, "flag offset");
    const unsigned tab = (unsigned)step_group(lane, rank, d0) * (PPK * PIECE);
    const LAS char* abase = lds + wr * 8192 + lane * 16;
    const LAS char* bbase = lds + 32768 + wc * 16384 + lane * 16;
    bf16x8 a0[4], b0[8], a1[4], b1[8];

    {
        const unsigned g0 = GOFF(0);
        spin_flag(GFLAG(g0), seq, tmo);
        STAGE_ALL(GSRC(g0), ldsw);
    }
    unsigned gn = GOFF(1);
    unsigned fv = ld_sys(GFLAG(gn));
    WAIT_VM0();
    RAW_BAR();
    if (mainw) STAMP(2);
    bool staged = RFL(fv) >= seq;
    if (staged) { STAGE_ALL(GSRC(gn), ldsw + 65536); gn = GOFF(2); }
    fv = ld_sys(GFLAG(gn));
#pragma unroll
    for (int i = 0; i < 8; ++i) b0[i] = *(const LAS bf16x8*)(bbase + i * 1024);
#pragma unroll
    for (int m = 0; m < 4; ++m) a0[m] = *(const LAS bf16x8*)(abase + m * 2048);

    for (int s = 0; s < 64; ++s) {
        const int cur = (s & 1) * 65536, oth = cur ^ 65536;
        if (!staged) {
            WAIT_VM0();
            if (RFL(fv) >= seq) { STAGE_ALL(GSRC(gn), ldsw + oth); staged = true; gn = GOFF(min(s + 2, 63)); fv = ld_sys(GFLAG(gn)); }
        }
        const LAS char* ab2 = abase + oth;
        const LAS char* bb2 = bbase + oth;
        const unsigned want = s + 2 < 64 ? seq : 0xffffffffu;
        const unsigned g3 = GOFF(min(s + 3, 63));
        const unsigned ld0 = ldsw + cur, ld1 = ld0 + 4096;
        SCHED0();
        {
            const LAS char* ab = abase + cur + 1024;
            const LAS char* bb = bbase + cur + 8192;
            PRIO(1);
            static_for<32>([&](auto tc) __attribute__((always_inline)) {
                constexpr int t = decltype(tc)::value;
                acc[t >> 3][t & 7] = __builtin_amdgcn_mfma_f32_16x16x32_bf16(a0[t >> 3], b0[t & 7], acc[t >> 3][t & 7], 0, 0, 0);
                constexpr int rd = RD1(t);
                if constexpr (rd >= 0 && rd < 8) b1[rd] = *(const LAS bf16x8*)(bb + rd * 1024);
                if constexpr (rd >= 8) a1[rd - 8] = *(const LAS bf16x8*)(ab + (rd - 8) * 2048);
                if constexpr (t == 15) PRIO(0);
                SCHED0();
            });
        }
        if (!staged) {
            spin_flag(GFLAG(gn), seq, tmo);
            STAGE_ALL(GSRC(gn), ldsw + oth);
            gn = GOFF(min(s + 2, 63));
            fv = ld_sys(GFLAG(gn));
        }
        const char* gs = GSRC(gn);
        WAIT_VM0();
        const unsigned have = RFL(fv);
        __builtin_amdgcn_s_waitcnt(0xC07F);
        PRIO(3);
        SCHED0();
        RAW_BAR();
        SCHED0();
        {
            static_for<32>([&](auto tc) __attribute__((always_inline)) {
                constexpr int t = decltype(tc)::value;
                acc[t >> 3][t & 7] = __builtin_amdgcn_mfma_f32_16x16x32_bf16(a1[t >> 3], b1[t & 7], acc[t >> 3][t & 7], 0, 0, 0);
                constexpr int rd = RD2(t);
                if constexpr (rd >= 0 && rd < 8) b0[rd] = *(const LAS bf16x8*)(bb2 + rd * 1024);
                if constexpr (rd >= 8) a0[rd - 8] = *(const LAS bf16x8*)(ab2 + (rd - 8) * 2048);
                if constexpr ((t & 3) == 1) GL(t >> 2, gs);
                if constexpr (t == 15) PRIO(2);
                if constexpr (t == 27) {
                    const bool go = have >= want;
                    staged = go || s + 2 >= 64;
                    gn = go ? g3 : gn;
                }
                if constexpr (t == 30) fv = ld_sys(GFLAG(gn));
                SCHED0();
            });
        }
    }
    if (mainw) STAMP(3);

    char* peer = p.ws[lane & 7];
    float am = 0.f;
#pragma unroll
    for (int m = 0; m < 4; ++m)
#pragma unroll
        for (int i = 0; i < 8; ++i)
#pragma unroll
            for (int j = 0; j < 4; ++j) am = fmaxf(am, fabsf(acc[m][i][j]));
#pragma unroll
    for (int o = 32; o > 0; o >>= 1) am = fmaxf(am, __shfl_xor(am, o));
    LAS float* red = (LAS float*)(lds + LDS_MISC + 512);
    if (lane == 0) red[wid] = am;
    RAW_BAR();
    if (tid < NR && mainw) {
        float a = red[0];
#pragma unroll
        for (int i = 1; i < 8; ++i) a = fmaxf(a, red[i]);
        st_sys64(peer + OFF_AMAX + (rank * NGEMM + gidx) * 8, ((unsigned long long)seq << 32) | __float_as_uint(a));
    }
    const unsigned scr = (unsigned)(size_t)lds + wid * SCR_SZ;
    zero_scratch(scr, lane);
    lut_init(scr, lane);
    {
        const char* gp = myws + OFF_AMAX + tid * 16;
        unsigned long long t0 = __builtin_amdgcn_s_memrealtime();
        unsigned long long a, b;
        for (unsigned n = 1;; ++n) {
            a = ld_sys64(gp);
            b = ld_sys64(gp + 8);
            bool ok = (unsigned)(a >> 32) == seq && (unsigned)(b >> 32) == seq;
            if (__all(ok)) break;
            if ((n & 31u) == 0u && give_up(tmo, seq, t0)) break;
            __builtin_amdgcn_s_sleep(8);
        }
        am = fmaxf(__uint_as_float((unsigned)a), __uint_as_float((unsigned)b));
#pragma unroll
        for (int o = 32; o > 0; o >>= 1) am = fmaxf(am, __shfl_xor(am, o));
        if (lane == 0) red[8 + wid] = am;
        WAIT_VM0();
        RAW_BAR();
#pragma unroll
        for (int i = 0; i < 8; ++i) am = fmaxf(am, red[8 + i]);
    }
    const unsigned tmov = ld_agent(tmo);
    if (mainw) STAMP(4);

    const float inv = am > 0.f ? 127.0f / am : 0.f;
    unsigned q[32];
#pragma unroll
    for (int m = 0; m < 4; ++m)
#pragma unroll
        for (int j = 0; j < 4; ++j)
#pragma unroll
            for (int h = 0; h < 2; ++h) {
                const unsigned t0 = __float_as_uint(fmaf(acc[m][4 * h + 0][j], inv, 12582912.f));
                const unsigned t1 = __float_as_uint(fmaf(acc[m][4 * h + 1][j], inv, 12582912.f));
                const unsigned t2 = __float_as_uint(fmaf(acc[m][4 * h + 2][j], inv, 12582912.f));
                const unsigned t3 = __float_as_uint(fmaf(acc[m][4 * h + 3][j], inv, 12582912.f));
                q[(m * 4 + j) * 2 + h] = PERM(t1, t0, 0x0c0c0400u) | PERM(t3, t2, 0x04000c0cu);
            }
    if (RFL(tmov) == seq) {
#pragma unroll
        for (int d = 0; d < 32; ++d) q[d] = 0x80808080u;
    }
    const unsigned lane_off = (unsigned)(4 * (lane >> 4)) * NTOT + 8 * (lane & 15);
    const rsrc_t ro = mk_rsrc(p.out, 4096u * NTOT);
    const unsigned rowoff = (unsigned)(ti * 4 + wr) * (64u * NTOT);
    if (!mainw) {
        const int h = (tj - 8) * 2 + wc;
        if (h < 7 * NDUP) store_rows<OUT_AUX>(ro, lane_off, rowoff + ((rank + 1 + h / NDUP) & 7) * NL + (NCB + h % NDUP) * 128, q);
        STAMP(6);
        return;
    }
    const int cb = tj * 2 + wc;
    if (cb < NCB) {
        const unsigned xi = (ti * 4 + wr) * 16 + cb;
        unsigned n[16];
#pragma unroll
        for (int u = 0; u < 4; ++u) {
            u32x4 r;
#pragma unroll
            for (int k = 0; k < 4; ++k) {
                const unsigned q0 = q[8 * u + (k >> 1) * 4 + (k & 1)], q1 = q[8 * u + (k >> 1) * 4 + (k & 1) + 2];
                r[k] = BFI(0x77777777u, BFI(0x0f0f0f0fu, q0, q1 << 4), BFI(0x0f0f0f0fu, q0 >> 4, q1));
                const unsigned sg = (r[k] >> 3) & 0x11111111u;
                n[4 * u + k] = BFI(0x0f0f0f0fu, q0 >> 3, q1 << 1) ^ ((sg << 4) - sg);
            }
            st16(myws, lane * 16, u, OFF_XOUT + xi * XO_SZ, r);
        }
        unsigned P[8];
        const unsigned nb = encode<8, false>(n, P, scr, 0u, lane);
        unsigned lines = M_RAW;
        if (nb <= 256u) {
            lines = (nb + 31u) >> 5;
            st16(myws, lane * 16, 0, OFF_XOUT + xi * XO_SZ + 4096, (u32x4){P[0], P[1], P[2], P[3]});
            st16(myws, lane * 16, 1, OFF_XOUT + xi * XO_SZ + 4096, (u32x4){P[4], P[5], P[6], P[7]});
            if ((unsigned)lane < lines * 8u) st16(myws, lane * 16, 2, OFF_XOUT + xi * XO_SZ + 4096, *(LAS u32x4*)(size_t)(scr + SC_BLOB + lane * 16));
        } else {
#pragma unroll
            for (int u = 0; u < 8; ++u)
                st16(myws, lane * 16, u & 3, OFF_ROUT + xi * 8192 + (u >> 2) * 4096, (u32x4){q[4 * u], q[4 * u + 1], q[4 * u + 2], q[4 * u + 3]});
        }
        WAIT_VM0();
        if (lane < NR && lane != rank) st_sys64(peer + OFF_OFLAG + rank * 8192 + xi * 8, ((unsigned long long)seq << 32) | (lines << 8));
    }
    STAMP(5);
    store_rows<0>(ro, lane_off, rowoff + rank * NL + cb * 128, q);
}

__global__ __launch_bounds__(NTHR, 2) __attribute__((amdgpu_waves_per_eu(2, 2))) void fused_kernel(Params p_unused) {
    extern __shared__ __attribute__((aligned(1024))) char smem[];
    const CAS Params& p = *(const CAS Params*)__builtin_amdgcn_kernarg_segment_ptr();
    char* myws = p.ws[p.rank];
    LAS char* lds = (LAS char*)smem;
    STAMP(0);
    const int b = blockIdx.x;
    const int wid = RFL(threadIdx.x >> 6), lane = threadIdx.x & 63;
    const unsigned scr = (unsigned)(size_t)lds + wid * SCR_SZ;
    char* cnt = myws + OFF_CNT + (b * 8 + wid) * 4;
    unsigned seq = ld_agent(cnt);
    const unsigned sem = (unsigned)(size_t)lds + LDS_MISC;
    if (threadIdx.x < 2) LDS32(sem + threadIdx.x * 4) = 0u;
    lut_init(scr, lane);
    if (lane == 0) LDS32(scr + SC_LUT + 64) = 0u;
    RAW_BAR();
    {
        constexpr int NS = NWG - NGEMM - NPWG, NP0 = 28 * NPUSH, G3 = 384 - 192 - (224 - NP0);
        const bool gw = b >= NWG - NGEMM;
        int q = gw ? (wid < 3 ? wid * NGEMM + (b - (NWG - NGEMM)) : NEXP) : (b >= NPWG ? wid * NS + (b - NPWG) : NEXP);
        const int lag = gw ? wid : (q < NP0 ? 0 : wid - 1);
        for (int i = 0; i < lag * TUNED(3, EXP_SLEEP); ++i) __builtin_amdgcn_s_sleep(4);
        char* peer = p.ws[lane & 7];
        STAMP(8);
        bool first = true;
        while (q < NEXP - 384) {
            int e;
            if (gw) e = q < 96 ? q : (q < 320 - NP0 ? 768 + NP0 + (q - 96) : (q < 416 - NP0 ? 96 + (q - (320 - NP0)) : 992 + (q - (416 - NP0))));
            else e = q < NP0 ? 768 + q : (q < NP0 + 224 - G3 ? 992 + G3 + (q - NP0) : (q < 832 ? 192 + (q - 256) : 1216 + (q - 832)));
            static_assert(NP0 + 224 - G3 == 256 && G3 >= 0, "export lists");
            export_item(p, myws, peer, scr, e, seq, first);
            first = false;
            if (gw) break;
            q += NS * 8;
        }
        if (first) seq = RFL(seq) + 1u;
    }
    if (lane == 0) st_agent(cnt, seq);
    if (NDUP == 1 && b == NPWG && wid == 0)
        for (int i = lane; i < 128; i += 64) st_agent(myws + OFF_SFLAG + ((i >> 1) * PPK + 27) * 128 + 64 + (i & 1) * 32, seq);
    STAMP(1);
    if (b >= NWG - NGEMM) {
        const int gidx = b - (NWG - NGEMM);
#pragma unroll 1
        for (int g = gidx;; g = -1 - gidx) {
            RAW_BAR();
            gemm_role(p, g, seq, myws, lds);
            if (g < 0 || gidx >= 16 * NXT) break;
        }
    } else if (b < NPWG && wid < TUNED(2, PULL_W)) {
        puller(p, myws, scr, sem, seq, b % 7, b / 7, (b / 7) + wid * PULL_WG);
        STAMP(4);
    }
}

}

extern "C" void kernel_launch(int rank, int nranks, ncclComm_t comm,
                              hipStream_t stream,
                              void* const* d_in, const int* in_sizes, int n_in,
                              void* d_out, int out_size,
                              const int* in_dtypes, int out_dtype,
                              void* const* d_out_all,
                              void* const* d_ws_all, size_t ws_bytes) {
    if (nranks != NR || ws_bytes < WS_NEED || (unsigned)rank >= (unsigned)NR) {
        fprintf(stderr, "unexpected configuration\n");
        return;
    }
    HIP_CHECK(hipFuncSetAttribute((const void*)fused_kernel, hipFuncAttributeMaxDynamicSharedMemorySize, LDS_BYTES));
    Params p;
    p.x = (const float*)d_in[0];
    p.w = (const float*)d_in[1];
    for (int i = 0; i < NR; ++i) p.ws[i] = (char*)d_ws_all[i];
    p.out = (char*)d_out;
    p.rank = rank;
    p.pad = 0;
    hipLaunchKernelGGL(fused_kernel, dim3(NWG), dim3(NTHR), LDS_BYTES, stream, p);
    HIP_CHECK(hipGetLastError());
}
```

```cpp
#include <hip/hip_runtime.h>
#include <rccl/rccl.h>
#include <cstdio>
#include <cstdint>

#define HIP_CHECK(expr) do { hipError_t _e = (expr); if (_e != hipSuccess) { \
    fprintf(stderr, "HIP error %s at %s:%d\n", hipGetErrorString(_e), __FILE__, __LINE__); \
    return; } } while (0)

namespace {

typedef unsigned u32x4 __attribute__((ext_vector_type(4)));
typedef unsigned u32x2 __attribute__((ext_vector_type(2)));
typedef float f32x4 __attribute__((ext_vector_type(4)));
typedef float f32x2 __attribute__((ext_vector_type(2)));
typedef __bf16 bf16x8 __attribute__((ext_vector_type(8)));
typedef __bf16 bf16x2 __attribute__((ext_vector_type(2)));
typedef unsigned short u16x2 __attribute__((ext_vector_type(2)));
typedef __amdgpu_buffer_rsrc_t rsrc_t;
#define GAS __attribute__((address_space(1)))
#define LAS __attribute__((address_space(3)))
#define CAS __attribute__((address_space(4)))

constexpr int NR = 8;
constexpr int NTOT = 16384;
constexpr int KL = 512;
constexpr int NL = 2048;
constexpr int PIECE = 32768;
constexpr int NDUP = 2;
#define NPUSH 6
constexpr int NXT = (7 * NDUP + 1) / 2;
constexpr int NCB = 16 - NDUP;
constexpr int PPK = 24 + NXT;
constexpr int NGEMM = 128;
constexpr int NWG = 256;
constexpr int NTHR = 512;
constexpr int NEXP = 2560;
#define PULL_WG 9
#define PULL_W 8
#define WIN_IN 8
#define WIN_OUT 8
#define PUSH_WIN 2
constexpr unsigned CODE_BYTES = 36u * 1024u;
#define EXP_GL 3
#define EXP_LW 2
#define EXP_LG 6
#define EXP_LP 20
#define EXP_LS 6
constexpr int NPWG = 7 * PULL_WG;
static_assert(NPWG < NWG - NGEMM, "puller count");
constexpr int NI_MAIN = 768 - 4 * NPUSH, NI_IN = NI_MAIN + 112 * NDUP, NI_ALL = NI_IN + 64 * NCB;
constexpr int XI_SZ = 6144;
constexpr int XO_SZ = 7168;

constexpr unsigned OFF_CNT = 0;
constexpr unsigned OFF_TMO = 8192;
constexpr unsigned OFF_TICK = 12288;
constexpr unsigned OFF_AMAX = 16384;
constexpr unsigned OFF_XFLAG = 32768;
constexpr unsigned XFLAG_SZ = NR * 8 * PPK * 16;
constexpr unsigned OFF_OFLAG = 163840;
constexpr unsigned OFF_SFLAG = 262144;
constexpr unsigned OFF_DBG = 512 * 1024;
constexpr unsigned OFF_XIN = 1u << 20;
constexpr unsigned OFF_XOUT = OFF_XIN + 2 * NEXP * XI_SZ;
constexpr unsigned OFF_STAGE = OFF_XOUT + 1024 * XO_SZ;
constexpr unsigned OFF_RIN = OFF_STAGE + NR * 8 * PPK * PIECE;
constexpr unsigned OFF_ROUT = OFF_RIN + 2 * NEXP * 8192;
constexpr unsigned WS_NEED = OFF_ROUT + 1024 * 8192;
static_assert(OFF_XFLAG + 2 * XFLAG_SZ <= OFF_OFLAG && OFF_OFLAG + 65536 <= OFF_SFLAG && OFF_SFLAG + NR * 8 * PPK * 128 <= OFF_DBG, "layout");
constexpr unsigned long long DEADLINE = 50000000ull;
constexpr int LDS_MISC = 131072;
constexpr int LDS_BYTES = LDS_MISC + 1024;
constexpr int AUX_SYS = 17;
constexpr int OUT_AUX = 17;
constexpr unsigned SCR_SZ = 10240;
constexpr unsigned SC_BLOB = 0, SC_A2 = 1152, SC_A3 = 5376, SC_LUT = 9600;
constexpr unsigned M_RAW = 15;

template <int V> struct IC { static constexpr int value = V; };
template <int N, int I = 0, class F> __device__ __forceinline__ void static_for(F&& f) {
    if constexpr (I < N) { f(IC<I>{}); static_for<N, I + 1>(f); }
}

struct Params {
    const float* x;
    const float* w;
    char* ws[NR];
    char* out;
    char* self;
    int rank;
    int pad;
};
#define TUNED(i, dflt) (dflt)

__device__ __forceinline__ unsigned ld_sys(const char* p) {
    return __hip_atomic_load((const GAS unsigned*)p, __ATOMIC_RELAXED, __HIP_MEMORY_SCOPE_SYSTEM);
}
__device__ __forceinline__ unsigned long long ld_sys64(const char* p) {
    return __hip_atomic_load((const GAS unsigned long long*)p, __ATOMIC_RELAXED, __HIP_MEMORY_SCOPE_SYSTEM);
}
__device__ __forceinline__ void st_sys(char* p, unsigned v) {
    __hip_atomic_store((GAS unsigned*)p, v, __ATOMIC_RELAXED, __HIP_MEMORY_SCOPE_SYSTEM);
}
__device__ __forceinline__ void st_sys64(char* p, unsigned long long v) {
    __hip_atomic_store((GAS unsigned long long*)p, v, __ATOMIC_RELAXED, __HIP_MEMORY_SCOPE_SYSTEM);
}
__device__ __forceinline__ unsigned ld_agent(const char* p) {
    return __hip_atomic_load((const GAS unsigned*)p, __ATOMIC_RELAXED, __HIP_MEMORY_SCOPE_AGENT);
}
__device__ __forceinline__ void st_agent(char* p, unsigned v) {
    __hip_atomic_store((GAS unsigned*)p, v, __ATOMIC_RELAXED, __HIP_MEMORY_SCOPE_AGENT);
}
__device__ __forceinline__ rsrc_t mk_rsrc(const void* p, unsigned bytes) {
    return __builtin_amdgcn_make_buffer_rsrc((void*)p, (short)0, (int)bytes, 0x00020000);
}
__device__ __forceinline__ u32x4 bld16(rsrc_t r, unsigned voff, unsigned soff) {
    return __builtin_amdgcn_raw_buffer_load_b128(r, (int)voff, (int)soff, AUX_SYS);
}
__device__ __forceinline__ void st16(const char* base, unsigned voff, int blk, unsigned soff, u32x4 v) {
    const char* b = base + soff;
    if (blk == 0) asm volatile("global_store_dwordx4 %0, %1, %2 sc0 sc1\n\ts_nop 1" :: "v"(voff), "v"(v), "s"(b) : "memory");
    if (blk == 1) asm volatile("global_store_dwordx4 %0, %1, %2 offset:1024 sc0 sc1\n\ts_nop 1" :: "v"(voff), "v"(v), "s"(b) : "memory");
    if (blk == 2) asm volatile("global_store_dwordx4 %0, %1, %2 offset:2048 sc0 sc1\n\ts_nop 1" :: "v"(voff), "v"(v), "s"(b) : "memory");
    if (blk == 3) asm volatile("global_store_dwordx4 %0, %1, %2 offset:3072 sc0 sc1\n\ts_nop 1" :: "v"(voff), "v"(v), "s"(b) : "memory");
}
#define WAIT_VM0() do { asm volatile("s_waitcnt vmcnt(0)" ::: "memory"); __builtin_amdgcn_s_waitcnt(0x0F70); } while (0)
#define WAIT_LGKM0() asm volatile("s_waitcnt lgkmcnt(0)" ::: "memory")
#define RAW_BAR() do { asm volatile("s_waitcnt lgkmcnt(0)" ::: "memory"); __builtin_amdgcn_s_barrier(); asm volatile("" ::: "memory"); } while (0)
#define SCHED0() __builtin_amdgcn_sched_barrier(0)
#define PERM(a, b, s) __builtin_amdgcn_perm((a), (b), (s))
#define RFL(x) ((unsigned)__builtin_amdgcn_readfirstlane((int)(x)))

#define GLB "sc0 sc1"
template <int OFF> __device__ __forceinline__ void glds_if(unsigned have, unsigned want, const char* src, unsigned loff, unsigned ldst) {
    asm volatile(
        "s_cmp_lt_u32 %0, %5\n\t"
        "s_cbranch_scc1 1f\n\t"
        "s_mov_b32 m0, %3\n\t"
        "s_nop 0\n\t"
        "global_load_lds_dwordx4 %1, %2 offset:%4 " GLB "\n"
        "1:"
        :: "s"(have), "v"(loff), "s"(src), "s"(ldst), "n"(OFF), "s"(want) : "memory", "scc");
}
__device__ __forceinline__ void stage8k(const char* src, unsigned loff, unsigned loff2, unsigned ldst) {
    asm volatile(
        "s_mov_b32 m0, %3\n\t"
        "s_nop 0\n\t"
        "global_load_lds_dwordx4 %0, %2 " GLB "\n\t"
        "global_load_lds_dwordx4 %0, %2 offset:1024 " GLB "\n\t"
        "global_load_lds_dwordx4 %0, %2 offset:2048 " GLB "\n\t"
        "global_load_lds_dwordx4 %0, %2 offset:3072 " GLB "\n\t"
        "s_add_u32 m0, m0, 0x1000\n\t"
        "s_nop 0\n\t"
        "global_load_lds_dwordx4 %1, %2 " GLB "\n\t"
        "global_load_lds_dwordx4 %1, %2 offset:1024 " GLB "\n\t"
        "global_load_lds_dwordx4 %1, %2 offset:2048 " GLB "\n\t"
        "global_load_lds_dwordx4 %1, %2 offset:3072 " GLB
        :: "v"(loff), "v"(loff2), "s"(src), "s"(ldst) : "memory", "scc");
}

__device__ __forceinline__ unsigned pk_bf16(float a, float b) {
    f32x2 v = {a, b};
    bf16x2 r = __builtin_convertvector(v, bf16x2);
    return __builtin_bit_cast(unsigned, r);
}

#define STAMP(i) do {} while (0)

__device__ __forceinline__ bool give_up(char* tmo, unsigned seq, unsigned long long t0) {
    if (RFL(ld_agent(tmo)) == seq || __builtin_amdgcn_s_memrealtime() - t0 > DEADLINE) {
        st_agent(tmo, seq);
        return true;
    }
    return false;
}
__device__ __forceinline__ void spin_flag(const char* flag, unsigned seq, char* tmo) {
    unsigned long long t0 = __builtin_amdgcn_s_memrealtime();
    for (unsigned n = 1; RFL(ld_sys(flag)) < seq; ++n) {
        if ((n & 31u) == 0u && give_up(tmo, seq, t0)) break;
        __builtin_amdgcn_s_sleep(1);
    }
    asm volatile("" ::: "memory");
}

typedef unsigned u32x2a __attribute__((ext_vector_type(2), aligned(4)));
#define LDS32(a) (*(LAS unsigned*)(size_t)(a))
#define CFENCE() asm volatile("" ::: "memory")

__device__ __forceinline__ unsigned wave_scan(unsigned x) {
    x += (unsigned)__builtin_amdgcn_update_dpp(0, (int)x, 0x111, 0xf, 0xf, false);
    x += (unsigned)__builtin_amdgcn_update_dpp(0, (int)x, 0x112, 0xf, 0xf, false);
    x += (unsigned)__builtin_amdgcn_update_dpp(0, (int)x, 0x114, 0xf, 0xf, false);
    x += (unsigned)__builtin_amdgcn_update_dpp(0, (int)x, 0x118, 0xf, 0xf, false);
    x += (unsigned)__builtin_amdgcn_update_dpp(0, (int)x, 0x142, 0xa, 0xf, false);
    x += (unsigned)__builtin_amdgcn_update_dpp(0, (int)x, 0x143, 0xc, 0xf, false);
    return x;
}
__device__ __forceinline__ unsigned esc_bits(unsigned P) { return __builtin_amdgcn_bitop3_b32(P, P >> 1, 0x55555555u, 0x80); }

__device__ __forceinline__ void lut_init(unsigned scr, int lane) {
    if (lane < 16) {
        unsigned sel = 0x0c0c0c0cu; int k = 0;
#pragma unroll
        for (int y = 0; y < 4; ++y) if ((lane >> y) & 1) { sel = (sel & ~(0xffu << (8 * k))) | ((unsigned)y << (8 * k)); ++k; }
        LDS32(scr + SC_LUT + lane * 4) = sel;
    }
}
#define BFI(m, x, y) __builtin_amdgcn_bitop3_b32((m), (x), (y), 0xCA)
__device__ __forceinline__ void zero_scratch(unsigned scr, int lane) {
#pragma unroll
    for (int i = 0; i < 9; ++i) *(LAS u32x4*)(size_t)(scr + i * 1024 + lane * 16) = (u32x4){0u, 0u, 0u, 0u};
    if (lane < 24) *(LAS u32x4*)(size_t)(scr + 9216 + lane * 16) = (u32x4){0u, 0u, 0u, 0u};
}
__device__ __forceinline__ unsigned digits1(unsigned n02, unsigned n13) {
    const unsigned lo = BFI(0x33333333u, n02, n13 << 2), hi = BFI(0x33333333u, n02 >> 2, n13);
    const unsigned h = (hi | (hi >> 1)) & 0x55555555u;
    return lo | h | (h << 1);
}
__device__ __forceinline__ unsigned sel_addr(unsigned E, int t, unsigned scr) {
    if (t == 0) return __builtin_amdgcn_udot4(E & 0x01010101u, 0x20100804u, scr + SC_LUT, false);
    if (t == 1) return __builtin_amdgcn_udot4(E & 0x04040404u, 0x08040201u, scr + SC_LUT, false);
    if (t == 2) return __builtin_amdgcn_udot4((E >> 4) & 0x01010101u, 0x20100804u, scr + SC_LUT, false);
    return __builtin_amdgcn_udot4((E >> 4) & 0x04040404u, 0x08040201u, scr + SC_LUT, false);
}
template <int NP, bool IN> __device__ __forceinline__ unsigned encode(const unsigned (&n)[2 * NP], unsigned (&P)[NP], unsigned scr, unsigned nb0, int lane) {
    unsigned E[NP], cnt = 0;
#pragma unroll
    for (int j = 0; j < NP; ++j) { P[j] = digits1(n[2 * j], n[2 * j + 1]); E[j] = esc_bits(P[j]); cnt += __popc(E[j]); }
    unsigned sel[2][8];
#pragma unroll
    for (int k = 0; k < 8; ++k) sel[0][k] = LDS32(sel_addr(E[k >> 2], k & 3, scr));
    cnt = max(cnt, 4u);
    const unsigned inc = wave_scan(cnt);
    const unsigned n2d = ((unsigned)__builtin_amdgcn_readlane((int)inc, 63) + 15u) >> 4;
    if (nb0 + n2d > 256u) return 1000u;
    unsigned pos = scr + ((lane & 1) ? SC_A3 : SC_A2) + inc - cnt, carry = 0;
    static_for<NP / 2>([&](auto gc) __attribute__((always_inline)) {
        constexpr int g = decltype(gc)::value;
        SCHED0();
        if constexpr (g + 1 < NP / 2) {
#pragma unroll
            for (int k = 0; k < 8; ++k) sel[(g + 1) & 1][k] = LDS32(sel_addr(E[2 * g + 2 + (k >> 2)], k & 3, scr));
        }
        SCHED0();
#pragma unroll
        for (int k = 0; k < 8; ++k) {
            const int j = 2 * g + (k >> 2), t = k & 3;
            const unsigned z = t < 2 ? n[2 * j + t] : n[2 * j + t - 2] >> 4;
            const unsigned long long v = (unsigned long long)PERM(0u, z, sel[g & 1][k]) << ((pos & 3u) * 8u);
            const unsigned lo = (unsigned)v | carry;
            LDS32(pos & ~3u) = lo;
            const unsigned np = pos + __popc(E[j] & (0x01010101u << (2 * t)));
            carry = ((np ^ pos) & 4u) ? (unsigned)(v >> 32) : lo;
            pos = np;
        }
    });
    LDS32(pos & ~3u) = carry;
    CFENCE();
    unsigned n3 = 0;
    const unsigned s3 = scr + SC_BLOB + (nb0 + n2d) * 4;
    for (unsigned i0 = 0; i0 < n2d; i0 += 64) {
        const unsigned i = i0 + lane;
        const u32x4 v = (*(LAS u32x4*)(size_t)(scr + SC_A2 + i * 16) | *(LAS u32x4*)(size_t)(scr + SC_A3 + i * 16)) & 0x0f0f0f0fu;
        unsigned x = 0;
#pragma unroll
        for (int t = 0; t < 4; ++t) {
            const unsigned z1 = v[t] + 0x01010101u;
            unsigned d = z1, e4;
            if (IN) { e4 = (((v[t] + 0x0b0b0b0bu) ^ z1) >> 4) & 0x01010101u; d |= z1 >> 3; }
            else e4 = ((v[t] + 0x0a0a0a0au) >> 4) & 0x01010101u;
            x |= ((d | e4 | (e4 << 1)) & 0x03030303u) << (2 * t);
        }
        const unsigned E2 = esc_bits(x);
        unsigned sl[4];
#pragma unroll
        for (int t = 0; t < 4; ++t) sl[t] = LDS32(sel_addr(E2, t, scr));
        const unsigned c = __popc(E2), ic = wave_scan(c);
        const unsigned tot = (unsigned)__builtin_amdgcn_readlane((int)ic, 63);
        if (nb0 + n2d + ((n3 + tot + 7u) >> 3) > 256u) return 1000u;
        if (i < n2d) LDS32(scr + SC_BLOB + (nb0 + i) * 4) = x;
        unsigned h[4];
#pragma unroll
        for (int t = 0; t < 4; ++t) { const unsigned y = PERM(0u, v[t], sl[t]); h[t] = PERM(0u, y | (y >> 4), 0x0c0c0200u); }
        const unsigned c0 = __popc(E2 & 0x01010101u), c01 = __popc(E2 & 0x05050505u), c2 = __popc(E2 & 0x10101010u);
        const unsigned long long acc = (unsigned long long)(h[0] | (h[1] << (4u * c0))) | ((unsigned long long)(h[2] | (h[3] << (4u * c2))) << (4u * c01));
        const unsigned p3 = n3 + ic - c, sh = (p3 & 7u) * 4u;
        if (c) {
            LAS unsigned* d = (LAS unsigned*)(size_t)(s3 + (p3 >> 3) * 4);
            const unsigned long long w = acc << sh;
            __hip_atomic_fetch_or(d, (unsigned)w, __ATOMIC_RELAXED, __HIP_MEMORY_SCOPE_WAVEFRONT);
            if (4u * c + sh > 32u) {
                __hip_atomic_fetch_or(d + 1, (unsigned)(w >> 32), __ATOMIC_RELAXED, __HIP_MEMORY_SCOPE_WAVEFRONT);
                __hip_atomic_fetch_or(d + 2, (unsigned)(((acc >> 32) << sh) >> 32), __ATOMIC_RELAXED, __HIP_MEMORY_SCOPE_WAVEFRONT);
            }
        }
        n3 += tot;
    }
    CFENCE();
    return nb0 + n2d + ((n3 + 7u) >> 3);
}
__device__ __forceinline__ unsigned rot8(unsigned e) { return ((e >> 1) & 0x7f7f7f7fu) | ((e << 7) & 0x80808080u); }
__device__ __forceinline__ void selectors(unsigned P, unsigned E, unsigned (&sel)[4]) {
    const unsigned pf = ((((E << 8) + E) << 8) + E) << 8;
    const unsigned S = BFI(E | (E << 1), pf, P);
    const unsigned A = BFI(0x33333333u, S, E << 2), B = BFI(0x33333333u, S >> 2, E);
    sel[0] = A & 0x07070707u; sel[1] = B & 0x07070707u; sel[2] = (A >> 4) & 0x07070707u; sel[3] = (B >> 4) & 0x07070707u;
}
template <int NP, bool IN> __device__ __forceinline__ void decode(const unsigned (&P)[NP], unsigned (&o)[4 * NP], unsigned scr, unsigned nb0, unsigned nbmax, unsigned T, int lane) {
    const unsigned lut1 = IN ? rot8(T | ((T - 1u) << 8) | ((T - 2u) << 16)) : 0x00100800u;
    const unsigned lut2 = IN ? rot8((T - 3u) | ((T - 4u) << 8) | ((T + 1u) << 16)) : 0x00282018u;
    unsigned E[NP], cnt = 0;
#pragma unroll
    for (int j = 0; j < NP; ++j) { E[j] = esc_bits(P[j]); cnt += __popc(E[j]); }
    cnt = max(cnt, 4u);
    const unsigned inc = wave_scan(cnt);
    const unsigned n2d = min(((unsigned)__builtin_amdgcn_readlane((int)inc, 63) + 15u) >> 4, 256u);
    for (unsigned i = lane; nb0 + n2d + i < nbmax; i += 64) {
        const unsigned W = LDS32(scr + SC_BLOB + (nb0 + n2d + i) * 4);
        unsigned a, b;
        if (IN) {
            const unsigned K = (T + 2u) * 0x01010101u;
            a = rot8(K - (((W & 0x0f0f0f0fu) + 0x02020202u) & 0x0f0f0f0fu));
            b = rot8(K - ((((W >> 4) & 0x0f0f0f0fu) + 0x02020202u) & 0x0f0f0f0fu));
        } else {
            a = (W << 3) & 0x78787878u;
            b = (W >> 1) & 0x78787878u;
        }
        *(LAS u32x2*)(size_t)(scr + SC_A3 + i * 8) = (u32x2){PERM(b, a, 0x05010400u), PERM(b, a, 0x07030602u)};
    }
    CFENCE();
    unsigned n3 = 0;
    for (unsigned i0 = 0; i0 < n2d; i0 += 64) {
        const unsigned i = i0 + lane;
        unsigned W = LDS32(scr + SC_BLOB + ((nb0 + i) & 255u) * 4);
        W = i < n2d ? W : 0u;
        const unsigned E2 = esc_bits(W), c = __popc(E2), ic = wave_scan(c);
        unsigned ps[4];
        u32x2a w[4];
        ps[0] = scr + SC_A3 + min(n3 + ic - c, 2048u);
#pragma unroll
        for (int t = 0; t < 4; ++t) {
            w[t] = *(LAS u32x2a*)(size_t)(ps[t] & ~3u);
            if (t < 3) ps[t + 1] = ps[t] + __popc(E2 & (0x01010101u << (2 * t)));
        }
        SCHED0();
        n3 += (unsigned)__builtin_amdgcn_readlane((int)ic, 63);
        unsigned sel[4];
        selectors(W, E2, sel);
        SCHED0();
        u32x4 v;
#pragma unroll
        for (int t = 0; t < 4; ++t) v[t] = PERM(__builtin_amdgcn_alignbyte(w[t].y, w[t].x, ps[t]), lut2, sel[t]);
        *(LAS u32x4*)(size_t)(scr + SC_A2 + i * 16) = v;
    }
    CFENCE();
    unsigned pos = scr + SC_A2 + min(inc - cnt, 4096u);
    unsigned ps[2][8];
    u32x2a w[2][8];
    static_for<NP / 2 + 1>([&](auto gc) __attribute__((always_inline)) {
        constexpr int g = decltype(gc)::value;
        SCHED0();
        if constexpr (g < NP / 2) {
#pragma unroll
            for (int k = 0; k < 8; ++k) {
                ps[g & 1][k] = pos;
                w[g & 1][k] = *(LAS u32x2a*)(size_t)(pos & ~3u);
                pos += __popc(E[2 * g + (k >> 2)] & (0x01010101u << (2 * (k & 3))));
            }
        }
        SCHED0();
        if constexpr (g > 0) {
#pragma unroll
            for (int h = 0; h < 2; ++h) {
                const int j = 2 * (g - 1) + h;
                unsigned sel[4];
                selectors(P[j], E[j], sel);
#pragma unroll
                for (int t = 0; t < 4; ++t) {
                    const u32x2a x = w[(g - 1) & 1][4 * h + t];
                    o[4 * j + t] = PERM(__builtin_amdgcn_alignbyte(x.y, x.x, ps[(g - 1) & 1][4 * h + t]), lut1, sel[t]);
                }
            }
        }
    });
    CFENCE();
}

__device__ __forceinline__ bool gather4(unsigned cell, unsigned long long v, unsigned long long& tot) {
    unsigned n;
    const unsigned one = 1u;
    asm volatile("s_mov_b64 exec, 1\n\tds_or_b64 %2, %3\n\tds_add_rtn_u32 %0, %2, %4 offset:8\n\ts_mov_b64 exec, -1\n\t"
                 "ds_read_b64 %1, %2\n\ts_waitcnt lgkmcnt(0)" : "=&v"(n), "=&v"(tot) : "v"(cell), "v"(v), "v"(one) : "memory");
    return RFL(n) == 3u;
}

__device__ __forceinline__ void exp_decode(int e, int rank, int& kt, int& slot, int& sub, int& dst) {
    sub = e & 3;
    const int q = e >> 2;
    if (q < 192) {
        kt = q / 24; slot = q - kt * 24; dst = slot < 16 ? -1 : rank;
        return;
    }
    const int t = q - 192;
    kt = t / 56;
    const int u = t - kt * 56, j = u / 7;
    slot = 16 + j;
    dst = (rank + 1 + (u - j * 7)) & 7;
}
__device__ __forceinline__ unsigned xin_item(int kt, int slot, int sub, int dst) {
    return (slot < 16 ? kt * 16 + slot : 128 + (dst * 8 + kt) * 8 + (slot - 16)) * 4 + sub;
}

__device__ __forceinline__ void export_item(const CAS Params& p, char* myws, const float* px, const float* pw0, int rank, unsigned scr, unsigned grp, int e, bool stage_only, unsigned& seq, unsigned seqv, bool first, bool bar) {
    const int lane = threadIdx.x & 63;
    int kt, slot, sub, dst;
    exp_decode(e, rank, kt, slot, sub, dst);
    const int kq = lane >> 4, c = lane & 15;
    u32x4 ch[8];
    if (slot < 16) {
        const GAS float* b = (const GAS float*)px + (size_t)(256 * slot + 64 * sub + (lane >> 2)) * KL + 64 * kt + 8 * ((lane & 3) ^ ((lane >> 4) & 2));
        f32x4 v[16];
#pragma unroll
        for (int i = 0; i < 8; ++i) {
            v[2 * i] = *(const GAS f32x4*)(b + (i >> 1) * 16 * KL + (i & 1) * 32);
            v[2 * i + 1] = *(const GAS f32x4*)(b + (i >> 1) * 16 * KL + (i & 1) * 32 + 4);
        }
        if (!stage_only) zero_scratch(scr, lane);
#pragma unroll
        for (int i = 0; i < 8; ++i) {
            ch[i].x = pk_bf16(v[2 * i][0], v[2 * i][1]); ch[i].y = pk_bf16(v[2 * i][2], v[2 * i][3]);
            ch[i].z = pk_bf16(v[2 * i + 1][0], v[2 * i + 1][1]); ch[i].w = pk_bf16(v[2 * i + 1][2], v[2 * i + 1][3]);
        }
    } else {
        const GAS float* b = (const GAS float*)pw0 + (size_t)(64 * kt + 32 * (sub & 1) + 8 * kq) * NTOT + dst * NL + 256 * (slot - 16) + 128 * (sub >> 1) + 8 * c;
        f32x4 v[16];
#pragma unroll
        for (int jj = 0; jj < 8; ++jj) {
            v[2 * jj] = *(const GAS f32x4*)(b + (size_t)jj * NTOT);
            v[2 * jj + 1] = *(const GAS f32x4*)(b + (size_t)jj * NTOT + 4);
        }
        if (!stage_only) zero_scratch(scr, lane);
#pragma unroll
        for (int i = 0; i < 8; ++i) {
            ch[i].x = pk_bf16(v[0 + (i >> 2)][i & 3], v[2 + (i >> 2)][i & 3]); ch[i].y = pk_bf16(v[4 + (i >> 2)][i & 3], v[6 + (i >> 2)][i & 3]);
            ch[i].z = pk_bf16(v[8 + (i >> 2)][i & 3], v[10 + (i >> 2)][i & 3]); ch[i].w = pk_bf16(v[12 + (i >> 2)][i & 3], v[14 + (i >> 2)][i & 3]);
        }
    }
    if (first) { asm volatile("s_waitcnt vmcnt(0)\n\tv_readfirstlane_b32 %0, %1\n\ts_nop 4" : "=s"(seq) : "v"(seqv) : "memory"); seq += 1u; }
    if (bar) RAW_BAR();
    const bool dup = slot == 23 && sub >= 4 - 2 * NDUP;
    const bool local = stage_only || (dup && dst != rank);
    const int d = lane & 7;
    const unsigned at = (rank * 8 + kt) * (PPK * 4) + ((dup && d != dst) ? 96 + ((dst - d - 1) & 7) * (2 * NDUP) + sub - (4 - 2 * NDUP) : slot * 4 + sub);
    if (kt == 0 && !stage_only && slot >= 16 && slot < 16 + NPUSH) {
        char* pw = p.ws[dst];
        const unsigned th = rank * 8 * (PPK * 4) + slot * 4 + sub;
#pragma unroll
        for (int u = 0; u < 8; ++u) {
            st16(pw, lane * 16, u & 3, OFF_STAGE + th * 8192 + (u >> 2) * 4096, ch[u]);
            asm volatile("s_waitcnt vmcnt(%0)" :: "n"(PUSH_WIN) : "memory");
        }
        WAIT_VM0();
        if (lane == 0) st_sys(pw + OFF_SFLAG + th * 32, seq);
        if (first) STAMP(12);
        return;
    }
    if (local) {
        const unsigned mine = (unsigned)__builtin_amdgcn_readlane((int)at, rank);
#pragma unroll
        for (int u = 0; u < 8; ++u) st16(myws, lane * 16, u & 3, OFF_STAGE + mine * 8192 + (u >> 2) * 4096, ch[u]);
        if (stage_only) {
            WAIT_VM0();
            if (lane == rank) st_agent(myws + OFF_SFLAG + at * 32, seq);
            return;
        }
    }
    int l7 = lane & 7;
    asm volatile("" : "+v"(l7));
    char* peer = p.ws[l7];
    unsigned meta = 0;
    {
        const unsigned xi = xin_item(kt, slot, sub, dst) + (seq & 1u) * NEXP;
        unsigned ex[16], sum = 0;
#pragma unroll
        for (int u = 0; u < 4; ++u) {
            u32x4 r;
#pragma unroll
            for (int k = 0; k < 4; ++k) {
                const unsigned d0 = ch[2 * u + (k >> 1)][(k & 1) * 2], d1 = ch[2 * u + (k >> 1)][(k & 1) * 2 + 1];
                const unsigned L = PERM(d1, d0, 0x06040200u), H = PERM(d1, d0, 0x07050301u);
                r[k] = (H & 0x80808080u) | (L & 0x7f7f7f7fu);
                ex[4 * u + k] = ((H << 1) & 0xfefefefeu) | ((L >> 7) & 0x01010101u);
                sum = __builtin_amdgcn_sad_u8(ex[4 * u + k], 0u, sum);
            }
            st16(myws, lane * 16, u, OFF_XIN + xi * XI_SZ, r);
        }
        unsigned T = ((unsigned)__builtin_amdgcn_readlane((int)wave_scan(sum), 63) + 7864u) >> 12;
        T = min(max(T, 13u), 253u);
        const unsigned K = (T + 2u) * 0x01010101u;
        unsigned z[16], yy[16], bad = 0;
#pragma unroll
        for (int d = 0; d < 16; ++d) {
            const unsigned y = ((K | 0x80808080u) - (ex[d] & 0x7f7f7f7fu)) ^ (~(K ^ ex[d]) & 0x80808080u);
            bad |= y; yy[d] = y;
            z[d] = ((y & 0x0f0f0f0fu) + 0x0e0e0e0eu) & 0x0f0f0f0fu;
        }
        unsigned nex = 0;
        if (__any((bad & 0xf0f0f0f0u) != 0u)) {
            CFENCE();
            unsigned mk = 0;
#pragma unroll
            for (int d = 0; d < 16; ++d) mk |= (yy[d] & 0xf0f0f0f0u) ? 1u << d : 0u;
            while (mk) {
                const unsigned d = __builtin_ctz(mk);
                mk &= mk - 1u;
                unsigned ev = ex[0];
#pragma unroll
                for (int t = 1; t < 16; ++t) ev = d == (unsigned)t ? ex[t] : ev;
#pragma unroll 1
                for (unsigned y = 0; y < 4; ++y) {
                    const unsigned eb = (ev >> (8 * y)) & 0xffu;
                    if (T + 2u - eb > 15u) {
                        const unsigned k = __hip_atomic_fetch_add((LAS unsigned*)(size_t)(scr + SC_LUT + 64), 1u, __ATOMIC_RELAXED, __HIP_MEMORY_SCOPE_WAVEFRONT);
                        if (k < 15u) LDS32(scr + SC_BLOB + k * 4) = eb | (y << 8) | (d << 10) | (lane << 14);
                    }
                }
            }
            CFENCE();
            nex = RFL(LDS32(scr + SC_LUT + 64));
            CFENCE();
            LDS32(scr + SC_LUT + 64) = 0u;
        }
        if (first) STAMP(10);
        unsigned P[4], nb = 1000u;
        if (nex <= 15u) {
            unsigned n[8];
#pragma unroll
            for (int k = 0; k < 8; ++k) n[k] = z[(k >> 1) * 4 + (k & 1)] | (z[(k >> 1) * 4 + (k & 1) + 2] << 4);
            nb = encode<4, true>(n, P, scr, nex, lane);
        }
        if (nb <= 256u) {
            const unsigned lines = (nb + 31u) >> 5;
            st16(myws, lane * 16, 0, OFF_XIN + xi * XI_SZ + 4096, (u32x4){P[0], P[1], P[2], P[3]});
            if ((unsigned)lane < lines * 8u) st16(myws, lane * 16, 1, OFF_XIN + xi * XI_SZ + 4096, *(LAS u32x4*)(size_t)(scr + SC_BLOB + lane * 16));
            meta = T | (lines << 8) | (nex << 12);
        } else {
#pragma unroll
            for (int u = 0; u < 8; ++u) st16(myws, lane * 16, u & 3, OFF_RIN + xi * 8192 + (u >> 2) * 4096, ch[u]);
            meta = M_RAW << 8;
        }
    }
    if (first) STAMP(11);
    WAIT_VM0();
    if (first) STAMP(12);
    if (local && lane == rank) st_agent(myws + OFF_SFLAG + at * 32, seq);
    unsigned long long tot;
    if (gather4(grp, (unsigned long long)meta << (16 * sub), tot) && lane < NR && lane != rank && (dst < 0 || dup || lane == dst)) {
        const u32x4 g = {(unsigned)tot, seq, (unsigned)(tot >> 32), seq};
        asm volatile("global_store_dwordx4 %0, %1, off sc0 sc1\n\ts_nop 1" :: "v"(peer + OFF_XFLAG + (seq & 1u) * XFLAG_SZ + (at >> 2) * 16), "v"(g) : "memory");
    }
}

__device__ __forceinline__ void tick_init(char* ctr, unsigned seq, unsigned first) {
    if ((threadIdx.x & 63) == 0)
        __hip_atomic_fetch_max((GAS unsigned*)ctr, (seq << 11) + first, __ATOMIC_RELAXED, __HIP_MEMORY_SCOPE_AGENT);
    WAIT_VM0();
}
__device__ __forceinline__ unsigned tick_draw(char* ctr) {
    return (unsigned)__builtin_amdgcn_raw_ptr_buffer_atomic_add_i32(1, mk_rsrc(ctr, 4), (threadIdx.x & 63) ? 64 : 0, 0, 16);
}
static_assert(NI_ALL + 2 * PULL_WG * 8 < 2048, "ticket range");

__device__ __forceinline__ void item_decode(int it, int rank, int delta, unsigned par, unsigned& xi, unsigned& at) {
    if (it >= NI_IN) {
        const int o = it - NI_IN, r = o / NCB;
        xi = at = (((r & 31) >> 1) * 4 + (r & 1) + 2 * (r >> 5)) * 16 + (o - r * NCB);
        return;
    }
    const int src = (rank + delta) & 7;
    if (it >= NI_MAIN) {
        const int o = it - NI_MAIN, kt = o / (14 * NDUP), r = o - kt * (14 * NDUP), u = r / (2 * NDUP);
        xi = xin_item(kt, 23, 4 - 2 * NDUP + (r & (2 * NDUP - 1)), (rank + 1 + u) & 7) + par;
        at = (src * 8 + kt) * (PPK * 4) + 96 + r;
        return;
    }
    const int q = (it >> 2) + NPUSH;
    int kt, slot;
    if (q < 8) { kt = 0; slot = 16 + q; }
    else {
        const int qq = q - 8, r = qq / 24, e = qq - r * 24;
        if (r == 7 || (r == 0 && e < 16)) { kt = r; slot = (e + 2 * delta) & 15; }
        else if (r == 0) { kt = 1; slot = e; }
        else {
            const int t3 = e / 3, m3 = e - t3 * 3;
            if (m3 < 2) { kt = r; slot = (2 * t3 + m3 + 2 * delta) & 15; }
            else { kt = r + 1; slot = 16 + t3; }
        }
    }
    xi = xin_item(kt, slot, it & 3, rank) + par;
    at = (src * 8 + kt) * (PPK * 4) + slot * 4 + (it & 3);
}

template <int AUX> __device__ __forceinline__ void store_rows(rsrc_t ro, unsigned lane_off, unsigned soff, const unsigned (&q)[32]) {
#pragma unroll
    for (int m = 0; m < 4; ++m)
#pragma unroll
        for (int j = 0; j < 4; ++j) {
            u32x2 v = {q[(m * 4 + j) * 2], q[(m * 4 + j) * 2 + 1]};
            __builtin_amdgcn_raw_buffer_store_b64(v, ro, (int)lane_off, (int)(soff + (m * 16 + j) * NTOT), AUX);
        }
}

__device__ __forceinline__ void puller(const CAS Params& p, char* myws, unsigned scr, unsigned sem, unsigned seq, int link, int gw, int g) {
    const int lane = threadIdx.x & 63;
    const int rank = p.rank, delta = link + 1, src = (rank + delta) & 7;
    char* tmo = myws + OFF_TMO;
    const rsrc_t rr = mk_rsrc(p.ws[src], WS_NEED), ro = mk_rsrc(p.out, 4096u * NTOT);
    const unsigned lane_off = (unsigned)(4 * (lane >> 4)) * NTOT + 8 * (lane & 15);
    char* ctr = myws + OFF_TICK + link * 128;
    const unsigned par = (seq & 1u) * NEXP;
    const int win = TUNED(0, WIN_IN), wout = TUNED(1, WIN_OUT);
    tick_init(ctr, seq, PULL_WG * TUNED(2, PULL_W));
    bool dead = false;
    for (int it = g; it < NI_ALL;) {
        unsigned cxi, cat;
        item_decode(it, rank, delta, par, cxi, cat);
        const char* fl = myws + (it >= NI_IN ? OFF_OFLAG + src * 2048 + (((cat >> 6) * 8 + ((cat >> 1) & 7)) * 2 + ((cat >> 5) & 1)) * 8
                                             : OFF_XFLAG + (seq & 1u) * XFLAG_SZ + (cat >> 1) * 8);
        unsigned long long mn = ld_sys64(fl);
        if (RFL(mn >> 32) != seq) {
            unsigned long long t0 = __builtin_amdgcn_s_memrealtime();
            for (unsigned n = 1;; ++n) {
                mn = ld_sys64(fl);
                if (RFL(mn >> 32) == seq) break;
                if ((n & 31u) == 0u && give_up(tmo, seq, t0)) { mn = 0; dead = true; break; }
                __builtin_amdgcn_s_sleep(1);
            }
        }
        CFENCE();
        const unsigned cmeta = it >= NI_IN ? ((RFL(mn) >> (((cat >> 4) & 1) * 8 + (cat & 1) * 4)) & 15u) << 8 : (RFL(mn) >> ((cat & 1) * 16)) & 0xffffu;
        u32x4 c[8];
        unsigned tk;
        {
            const unsigned lines = (cmeta >> 8) & 15u;
            const bool isout = it >= NI_IN, raw = lines == M_RAW;
            const unsigned bo = (unsigned)lane < lines * 8u ? lane * 16u : 0x80000000u;
            const unsigned o = raw ? (isout ? OFF_ROUT : OFF_RIN) + cxi * 8192 : (isout ? OFF_XOUT + cxi * XO_SZ : OFF_XIN + cxi * XI_SZ);
            const unsigned nld = raw ? 8u : (isout ? 7u : 6u);
            unsigned my;
            asm volatile("s_mov_b64 exec, 1\n\tds_add_rtn_u32 %0, %1, %2\n\ts_mov_b64 exec, -1\n\ts_waitcnt lgkmcnt(0)" : "=&v"(my) : "v"(sem), "v"(nld) : "memory");
            my = RFL(my) - max((unsigned)(isout ? wout : win), nld);
            unsigned rel = RFL(__hip_atomic_load((LAS unsigned*)(size_t)(sem + 4), __ATOMIC_RELAXED, __HIP_MEMORY_SCOPE_WORKGROUP));
#define GATE(u) while ((int)(my + (u) - rel) >= 0) { rel = RFL(__hip_atomic_load((LAS unsigned*)(size_t)(sem + 4), __ATOMIC_RELAXED, __HIP_MEMORY_SCOPE_WORKGROUP)); CFENCE(); }
#define RELEASE(n, v) asm volatile("s_waitcnt vmcnt(%0)\n\ts_mov_b64 exec, 1\n\tds_add_u32 %1, %2 offset:4\n\ts_mov_b64 exec, -1" :: "n"(n), "v"(sem), "v"(v) : "memory")
            while ((int)(my - 4u - rel) >= 0) { __builtin_amdgcn_s_sleep(2); rel = RFL(__hip_atomic_load((LAS unsigned*)(size_t)(sem + 4), __ATOMIC_RELAXED, __HIP_MEMORY_SCOPE_WORKGROUP)); }
            __builtin_amdgcn_s_setprio(3);
            GATE(0u); c[4] = bld16(rr, lane * 16, o + 4096);
            GATE(1u); c[5] = bld16(rr, isout ? lane * 16u : bo, o + 5120);
            GATE(2u); c[0] = bld16(rr, lane * 16, o);
            GATE(3u); c[1] = bld16(rr, lane * 16 + 1024, o);
            GATE(4u); c[2] = bld16(rr, lane * 16 + 2048, o);
            GATE(5u); c[3] = bld16(rr, lane * 16 + 3072, o);
            GATE(nld - 1u);
            c[6] = bld16(rr, (isout || raw) ? bo : 0x80000000u, o + 6144);
            c[7] = bld16(rr, raw ? lane * 16u : 0x80000000u, o + 7168);
            tk = tick_draw(ctr);
            const unsigned one = 1u, last = nld - 5u;
            RELEASE(8, one); RELEASE(7, one); RELEASE(6, one); RELEASE(5, one); RELEASE(4, one); RELEASE(1, last);
            __builtin_amdgcn_s_setprio(0);
        }
        {
            const unsigned lines = (cmeta >> 8) & 15u;
            if (it >= NI_IN) {
                unsigned q[32];
                if (dead) {
#pragma unroll
                    for (int d = 0; d < 32; ++d) q[d] = 0x80808080u;
                } else if (lines == M_RAW) {
#pragma unroll
                    for (int d = 0; d < 32; ++d) q[d] = c[d >> 2][d & 3];
                } else {
                    *(LAS u32x4*)(size_t)(scr + SC_BLOB + lane * 16) = c[6];
                    CFENCE();
                    const unsigned P[8] = {c[4].x, c[4].y, c[4].z, c[4].w, c[5].x, c[5].y, c[5].z, c[5].w};
                    decode<8, false>(P, q, scr, 0u, lines * 32u, 0u, lane);
#pragma unroll
                    for (int d = 0; d < 32; ++d) {
                        const unsigned R = c[d >> 3][((d >> 1) & 2) + (d & 1)] >> ((d & 2) * 2);
                        const unsigned s8 = R & 0x08080808u;
                        q[d] = (q[d] ^ ((s8 << 5) - s8)) | (R & 0x07070707u);
                    }
                }
                store_rows<OUT_AUX>(ro, lane_off, (cat >> 4) * (64u * NTOT) + src * NL + (cat & 15u) * 128u, q);
            } else {
                u32x4 o[8];
                if (lines == M_RAW) {
#pragma unroll
                    for (int u = 0; u < 8; ++u) o[u] = c[u];
                } else {
                    *(LAS u32x4*)(size_t)(scr + SC_BLOB + lane * 16) = c[5];
                    CFENCE();
                    const unsigned P[4] = {c[4].x, c[4].y, c[4].z, c[4].w};
                    const unsigned nex = (cmeta >> 12) & 15u;
                    unsigned ex[16];
                    decode<4, true>(P, ex, scr, nex, lines * 32u, cmeta & 0xffu, lane);
                    for (unsigned k = 0; k < nex; ++k) {
                        const unsigned en = RFL(LDS32(scr + SC_BLOB + k * 4));
                        const unsigned sh = ((en >> 8) & 3u) * 8u, r = (en >> 10) & 15u;
                        const unsigned msk = (unsigned)lane == ((en >> 14) & 63u) ? 0xffu << sh : 0u;
                        const unsigned val = rot8(en & 0xffu) << sh;
#pragma unroll
                        for (int d = 0; d < 16; ++d) { const unsigned mq = r == (unsigned)d ? msk : 0u; ex[d] = (ex[d] & ~mq) | (val & mq); }
                    }
#pragma unroll
                    for (int d = 0; d < 16; ++d) {
                        const unsigned r = c[d >> 2][d & 3];
                        const unsigned H = (r & 0x80808080u) | (ex[d] & 0x7f7f7f7fu), L = (ex[d] & 0x80808080u) | (r & 0x7f7f7f7fu);
                        o[d >> 1][(d & 1) * 2] = PERM(H, L, 0x05010400u);
                        o[d >> 1][(d & 1) * 2 + 1] = PERM(H, L, 0x07030602u);
                    }
                }
#pragma unroll
                for (int u = 0; u < 8; ++u) st16(myws, lane * 16, u & 3, OFF_STAGE + cat * 8192 + (u >> 2) * 4096, o[u]);
                WAIT_VM0();
                if (lane == 0) st_agent(myws + OFF_SFLAG + cat * 32, seq);
            }
        }
        const int nx = (int)(RFL(tk) & 0x7ffu);
        it = nx;
    }
}

__device__ __forceinline__ int step_group(int s, int rank, int d0) {
    if (s < 8) return rank * 8 + s;
    int t = s - 8;
    int kt = t / 7, u = t - kt * 7;
    int v = u < d0 ? u : u + 1;
    int delta = (d0 - v) & 7;
    return ((rank + delta) & 7) * 8 + kt;
}

__device__ __forceinline__ void gemm_role(const CAS Params& p, int gidx, unsigned seq, char* myws, LAS char* lds) {
    int tid = threadIdx.x;
    asm volatile("" : "+v"(tid));
    const int wid = RFL(tid >> 6);
    int lane = tid & 63;
    const int wr = wid >> 1, wc = wid & 1;
    const int rank = p.rank;
    const bool mainw = gidx >= 0;
    int ti, tj;
    if (mainw) { const int xcd = gidx & 7, sl = gidx >> 3; ti = (xcd >> 1) * 4 + (sl >> 2); tj = (xcd & 1) * 4 + (sl & 3); }
    else { const int x = -1 - gidx; ti = (x & 7) * 2 + ((x >> 3) & 1); tj = 8 + (x >> 4); }
    const int d0 = ti >> 1;
    const int myslot = wid < 4 ? ti : 16 + tj;
    char* tmo = myws + OFF_TMO;
    const char* dsrc = myws + OFF_STAGE + (size_t)myslot * PIECE + (wid & 3) * 8192;
    const char* fsrc = myws + OFF_SFLAG + myslot * 128 + (wid & 3) * 32;
    const unsigned loff = lane * 16, loff2 = loff + 4096;
    const unsigned ldsw = (unsigned)(size_t)lds + wid * 8192;

    f32x4 acc[4][8];
#pragma unroll
    for (int m = 0; m < 4; ++m)
#pragma unroll
        for (int i = 0; i < 8; ++i) acc[m][i] = (f32x4){0.f, 0.f, 0.f, 0.f};

#define GL(n, S) glds_if<((n) & 3) * 1024>(have, want, S, (n) < 4 ? loff : loff2, (n) < 4 ? ld0 : ld1)
#define STAGE_ALL(S, D) stage8k(S, loff, loff2, D)
#define PRIO(n) __builtin_amdgcn_s_setprio(n)
#define RDV 0
#define RD1(t) (((t) & 1) == 0 && (t) < 24 ? (t) / 2 : -1)
#define RD2(t) (((t) & 1) == 0 && (t) >= 2 && (t) < 26 ? (t) / 2 - 1 : -1)
#define GOFF(st) ((unsigned)__builtin_amdgcn_readlane((int)tab, (st)))
#define GSRC(off) (dsrc + (off))
#define GFLAG(off) (fsrc + ((off) >> 8))
    static_assert(PPK * PIECE / 256 == PPK * 128, "flag offset");
    const unsigned tab = (unsigned)step_group(lane, rank, d0) * (PPK * PIECE);
    const LAS char* abase = lds + wr * 8192 + (lane & 15) * 64 + (((lane >> 4) ^ ((lane >> 2) & 2)) << 4);
    const LAS char* bbase = lds + 32768 + wc * 16384 + lane * 16;
    bf16x8 a0[4], b0[8], a1[4], b1[8];

    {
        const unsigned g0 = GOFF(0);
        spin_flag(GFLAG(g0), seq, tmo);
        STAGE_ALL(GSRC(g0), ldsw);
    }
    unsigned gn = GOFF(1);
    unsigned fv = ld_sys(GFLAG(gn));
    WAIT_VM0();
    RAW_BAR();
    if (mainw) STAMP(2);
    bool staged = RFL(fv) >= seq;
    if (staged) { STAGE_ALL(GSRC(gn), ldsw + 65536); gn = GOFF(2); }
    fv = ld_sys(GFLAG(gn));
#pragma unroll
    for (int i = 0; i < 8; ++i) b0[i] = *(const LAS bf16x8*)(bbase + i * 1024);
#pragma unroll
    for (int m = 0; m < 4; ++m) a0[m] = *(const LAS bf16x8*)(abase + m * 2048);

    for (int s = 0; s < 64; ++s) {
        const int cur = (s & 1) * 65536, oth = cur ^ 65536;
        if (!staged) {
            WAIT_VM0();
            if (RFL(fv) >= seq) { STAGE_ALL(GSRC(gn), ldsw + oth); staged = true; gn = GOFF(min(s + 2, 63)); fv = ld_sys(GFLAG(gn)); }
        }
        const LAS char* ab2 = abase + oth;
        const LAS char* bb2 = bbase + oth;
        const unsigned want = s + 2 < 64 ? seq : 0xffffffffu;
        const unsigned g3 = GOFF(min(s + 3, 63));
        const unsigned ld0 = ldsw + cur, ld1 = ld0 + 4096;
        SCHED0();
        {
            const LAS char* ab = abase + cur + 1024;
            const LAS char* bb = bbase + cur + 8192;
            PRIO(1);
            static_for<32>([&](auto tc) __attribute__((always_inline)) {
                constexpr int t = decltype(tc)::value;
                acc[t >> 3][t & 7] = __builtin_amdgcn_mfma_f32_16x16x32_bf16(a0[t >> 3], b0[t & 7], acc[t >> 3][t & 7], 0, 0, 0);
                constexpr int rd = RD1(t);
                if constexpr (rd >= 0 && rd < 8) b1[rd] = *(const LAS bf16x8*)(bb + rd * 1024);
                if constexpr (rd >= 8) a1[rd - 8] = *(const LAS bf16x8*)(ab + (rd - 8) * 2048);
                if constexpr (t == 15) PRIO(0);
                SCHED0();
            });
        }
        if (!staged) {
            spin_flag(GFLAG(gn), seq, tmo);
            STAGE_ALL(GSRC(gn), ldsw + oth);
            gn = GOFF(min(s + 2, 63));
            fv = ld_sys(GFLAG(gn));
        }
        const char* gs = GSRC(gn);
        WAIT_VM0();
        const unsigned have = RFL(fv);
        __builtin_amdgcn_s_waitcnt(0xC07F);
        PRIO(3);
        SCHED0();
        RAW_BAR();
        SCHED0();
        {
            static_for<32>([&](auto tc) __attribute__((always_inline)) {
                constexpr int t = decltype(tc)::value;
                acc[t >> 3][t & 7] = __builtin_amdgcn_mfma_f32_16x16x32_bf16(a1[t >> 3], b1[t & 7], acc[t >> 3][t & 7], 0, 0, 0);
                constexpr int rd = RD2(t);
                if constexpr (rd >= 0 && rd < 8) b0[rd] = *(const LAS bf16x8*)(bb2 + rd * 1024);
                if constexpr (rd >= 8) a0[rd - 8] = *(const LAS bf16x8*)(ab2 + (rd - 8) * 2048);
                if constexpr ((t & 3) == 1) GL(t >> 2, gs);
                if constexpr (t == 15) PRIO(2);
                if constexpr (t == 27) {
                    const bool go = have >= want;
                    staged = go || s + 2 >= 64;
                    gn = go ? g3 : gn;
                }
                if constexpr (t == 30) fv = ld_sys(GFLAG(gn));
                SCHED0();
            });
        }
    }
    if (mainw) STAMP(3);

    asm volatile("" : "+v"(tid));
    lane = tid & 63;
    char* peer = p.ws[lane & 7];
    float am = 0.f;
#pragma unroll
    for (int m = 0; m < 4; ++m)
#pragma unroll
        for (int i = 0; i < 8; ++i)
#pragma unroll
            for (int j = 0; j < 4; ++j) am = fmaxf(am, fabsf(acc[m][i][j]));
#pragma unroll
    for (int o = 32; o > 0; o >>= 1) am = fmaxf(am, __shfl_xor(am, o));
    LAS float* red = (LAS float*)(lds + LDS_MISC + 512);
    if (lane == 0) red[wid] = am;
    RAW_BAR();
    if (tid < NR && mainw) {
        float a = red[0];
#pragma unroll
        for (int i = 1; i < 8; ++i) a = fmaxf(a, red[i]);
        st_sys64(peer + OFF_AMAX + (rank * NGEMM + gidx) * 8, ((unsigned long long)seq << 32) | __float_as_uint(a));
    }
    const unsigned scr = (unsigned)(size_t)lds + wid * SCR_SZ;
    zero_scratch(scr, lane);
    lut_init(scr, lane);
    {
        const char* gp = myws + OFF_AMAX + tid * 16;
        unsigned long long t0 = __builtin_amdgcn_s_memrealtime();
        unsigned long long a, b;
        for (unsigned n = 1;; ++n) {
            a = ld_sys64(gp);
            b = ld_sys64(gp + 8);
            bool ok = (unsigned)(a >> 32) == seq && (unsigned)(b >> 32) == seq;
            if (__all(ok)) break;
            if ((n & 31u) == 0u && give_up(tmo, seq, t0)) break;
            __builtin_amdgcn_s_sleep(8);
        }
        am = fmaxf(__uint_as_float((unsigned)a), __uint_as_float((unsigned)b));
#pragma unroll
        for (int o = 32; o > 0; o >>= 1) am = fmaxf(am, __shfl_xor(am, o));
        if (lane == 0) red[8 + wid] = am;
        WAIT_VM0();
        RAW_BAR();
#pragma unroll
        for (int i = 0; i < 8; ++i) am = fmaxf(am, red[8 + i]);
    }
    const unsigned tmov = ld_agent(tmo);
    if (mainw) STAMP(4);

    const float inv = am > 0.f ? 127.0f / am : 0.f;
    unsigned q[32];
#pragma unroll
    for (int m = 0; m < 4; ++m)
#pragma unroll
        for (int j = 0; j < 4; ++j)
#pragma unroll
            for (int h = 0; h < 2; ++h) {
                const unsigned t0 = __float_as_uint(fmaf(acc[m][4 * h + 0][j], inv, 12582912.f));
                const unsigned t1 = __float_as_uint(fmaf(acc[m][4 * h + 1][j], inv, 12582912.f));
                const unsigned t2 = __float_as_uint(fmaf(acc[m][4 * h + 2][j], inv, 12582912.f));
                const unsigned t3 = __float_as_uint(fmaf(acc[m][4 * h + 3][j], inv, 12582912.f));
                q[(m * 4 + j) * 2 + h] = PERM(t1, t0, 0x0c0c0400u) | PERM(t3, t2, 0x04000c0cu);
            }
    if (RFL(tmov) == seq) {
#pragma unroll
        for (int d = 0; d < 32; ++d) q[d] = 0x80808080u;
    }
    const unsigned lane_off = (unsigned)(4 * (lane >> 4)) * NTOT + 8 * (lane & 15);
    const rsrc_t ro = mk_rsrc(p.out, 4096u * NTOT);
    const unsigned rowoff = (unsigned)(ti * 4 + wr) * (64u * NTOT);
    if (!mainw) {
        const int h = (tj - 8) * 2 + wc;
        if (h < 7 * NDUP) store_rows<OUT_AUX>(ro, lane_off, rowoff + ((rank + 1 + h / NDUP) & 7) * NL + (NCB + h % NDUP) * 128, q);
        STAMP(6);
        return;
    }
    const int cb = tj * 2 + wc;
    if (cb < NCB) {
        const unsigned xi = (ti * 4 + wr) * 16 + cb;
        unsigned n[16];
#pragma unroll
        for (int u = 0; u < 4; ++u) {
            u32x4 r;
#pragma unroll
            for (int k = 0; k < 4; ++k) {
                const unsigned q0 = q[8 * u + (k >> 1) * 4 + (k & 1)], q1 = q[8 * u + (k >> 1) * 4 + (k & 1) + 2];
                r[k] = BFI(0x77777777u, BFI(0x0f0f0f0fu, q0, q1 << 4), BFI(0x0f0f0f0fu, q0 >> 4, q1));
                const unsigned sg = (r[k] >> 3) & 0x11111111u;
                n[4 * u + k] = BFI(0x0f0f0f0fu, q0 >> 3, q1 << 1) ^ ((sg << 4) - sg);
            }
            st16(myws, lane * 16, u, OFF_XOUT + xi * XO_SZ, r);
        }
        unsigned P[8];
        const unsigned nb = encode<8, false>(n, P, scr, 0u, lane);
        unsigned lines = M_RAW;
        if (nb <= 256u) {
            lines = (nb + 31u) >> 5;
            st16(myws, lane * 16, 0, OFF_XOUT + xi * XO_SZ + 4096, (u32x4){P[0], P[1], P[2], P[3]});
            st16(myws, lane * 16, 1, OFF_XOUT + xi * XO_SZ + 4096, (u32x4){P[4], P[5], P[6], P[7]});
            if ((unsigned)lane < lines * 8u) st16(myws, lane * 16, 2, OFF_XOUT + xi * XO_SZ + 4096, *(LAS u32x4*)(size_t)(scr + SC_BLOB + lane * 16));
        } else {
#pragma unroll
            for (int u = 0; u < 8; ++u)
                st16(myws, lane * 16, u & 3, OFF_ROUT + xi * 8192 + (u >> 2) * 4096, (u32x4){q[4 * u], q[4 * u + 1], q[4 * u + 2], q[4 * u + 3]});
        }
        WAIT_VM0();
        unsigned long long tot;
        if (gather4((unsigned)(size_t)lds + LDS_MISC + 64 + (wid >> 2) * 16, (unsigned long long)(lines << ((wid & 3) * 4)), tot) && lane < NR && lane != rank)
            st_sys64(peer + OFF_OFLAG + rank * 2048 + ((ti * 8 + tj) * 2 + (wid >> 2)) * 8, ((unsigned long long)seq << 32) | (unsigned)tot);
    }
    STAMP(5);
    store_rows<0>(ro, lane_off, rowoff + rank * NL + cb * 128, q);
}

__global__ __launch_bounds__(NTHR, 2) __attribute__((amdgpu_waves_per_eu(2, 2))) void fused_kernel(Params p_unused) {
    extern __shared__ __attribute__((aligned(1024))) char smem[];
    const CAS Params& p = *(const CAS Params*)__builtin_amdgcn_kernarg_segment_ptr();
    char* myws = p.self;
    const float* px = p.x;
    const float* pw = p.w;
    int rank = p.rank;
    asm volatile("" : "+s"(myws), "+s"(px), "+s"(pw), "+s"(rank));
    LAS char* lds = (LAS char*)smem;
    STAMP(0);
    const int b = blockIdx.x;
    const int wid = RFL(threadIdx.x >> 6), lane = threadIdx.x & 63;
    const unsigned scr = (unsigned)(size_t)lds + wid * SCR_SZ;
    char* cnt = myws + OFF_CNT + (b * 8 + wid) * 4;
    unsigned seq = 0, seqv;
    asm volatile("global_load_dword %0, %1, %2 sc1" : "=v"(seqv) : "v"(0u), "s"(cnt) : "memory");
    if (b < 16) {
        const char* pc = (const char*)__builtin_amdgcn_s_getpc();
        const unsigned o = threadIdx.x * 128u;
        if (o < CODE_BYTES) { unsigned t = *(const volatile GAS unsigned*)(pc + o); asm volatile("" :: "v"(t)); }
    }
    const unsigned sem = (unsigned)(size_t)lds + LDS_MISC;
    if (threadIdx.x < 128) LDS32(sem + threadIdx.x * 4) = 0u;
    lut_init(scr, lane);
    if (lane == 0) LDS32(scr + SC_LUT + 64) = 0u;
    {
        constexpr int NS = NWG - NGEMM - NPWG, NP0 = 7 * NPUSH, NXL = 128 + 448 + 8;
        const bool gw = b >= NWG - NGEMM, spare = !gw && b >= NPWG;
        const int og = TUNED(7, EXP_GL) * NGEMM;
        int q = gw ? wid * NGEMM + (b - (NWG - NGEMM)) : (spare ? (wid >> 2) * NS + (b - NPWG) : og + (7 - wid) * NPWG + b);
        const int lag = spare ? (q < 72 ? 0 : TUNED(6, EXP_LS)) : (gw ? TUNED(4, EXP_LG) + wid * TUNED(3, EXP_LW) : TUNED(5, EXP_LP));
        const bool late_bar = spare && q < 72;
        if (!late_bar) RAW_BAR();
        if (q < (spare ? NXL : (gw ? og : 768))) for (int i = 0; i < lag; ++i) __builtin_amdgcn_s_sleep(4);
        STAMP(8);
        bool first = true;
        unsigned grp = sem + 64 + (wid >> 2) * 16;
        while (q < (spare ? NXL : (gw ? og : 768))) {
            int e = q;
            if (spare) {
                if (q < 16) e = q;
                else if (q < 72 - NP0) e = 192 + NP0 + (q - 16);
                else if (q < 72) e = 192 + (q - (72 - NP0));
                else if (q < 576) { const int r = (q - 72) / 72, u = q - 72 - r * 72; e = u < 56 ? 248 + r * 56 + u : (r + 1) * 24 + (u - 56); }
                else e = (q - 576) * 24 + 23;
                e = e * 4 + (wid & 3);
            }
            export_item(p, myws, px, pw, rank, scr, grp, e, !spare, seq, seqv, first, first && late_bar);
            first = false;
            if (!spare) break;
            q += NS * 2; grp += 32;
        }
        static_assert((NXL + 2 * NS - 1) / (2 * NS) * 32 + 64 <= 512, "gather cells");
        if (first) { asm volatile("s_waitcnt vmcnt(0)\n\tv_readfirstlane_b32 %0, %1\n\ts_nop 4" : "=s"(seq) : "v"(seqv) : "memory"); seq += 1u; }
    }
    if (lane == 0) st_agent(cnt, seq);
    STAMP(1);
    if (b >= NWG - NGEMM) {
        const int gidx = b - (NWG - NGEMM);
#pragma unroll 1
        for (int g = gidx;; g = -1 - gidx) {
            RAW_BAR();
            gemm_role(p, g, seq, myws, lds);
            if (g < 0 || gidx >= 16 * NXT) break;
        }
    } else if (b < NPWG && wid < TUNED(2, PULL_W)) {
        puller(p, myws, scr, sem, seq, b % 7, b / 7, (b / 7) + wid * PULL_WG);
        STAMP(4);
    }
}

}

extern "C" void kernel_launch(int rank, int nranks, ncclComm_t comm,
                              hipStream_t stream,
                              void* const* d_in, const int* in_sizes, int n_in,
                              void* d_out, int out_size,
                              const int* in_dtypes, int out_dtype,
                              void* const* d_out_all,
                              void* const* d_ws_all, size_t ws_bytes) {
    if (nranks != NR || ws_bytes < WS_NEED || (unsigned)rank >= (unsigned)NR) {
        fprintf(stderr, "unexpected configuration\n");
        return;
    }
    HIP_CHECK(hipFuncSetAttribute((const void*)fused_kernel, hipFuncAttributeMaxDynamicSharedMemorySize, LDS_BYTES));
    Params p;
    p.x = (const float*)d_in[0];
    p.w = (const float*)d_in[1];
    for (int i = 0; i < NR; ++i) p.ws[i] = (char*)d_ws_all[i];
    p.out = (char*)d_out;
    p.self = (char*)d_ws_all[rank];
    p.rank = rank;
    p.pad = 0;
    hipLaunchKernelGGL(fused_kernel, dim3(NWG), dim3(NTHR), LDS_BYTES, stream, p);
    HIP_CHECK(hipGetLastError());
}
```

```cpp
#include <hip/hip_runtime.h>
#include <rccl/rccl.h>
#include <cstdio>
#include <cstdint>

#define HIP_CHECK(expr) do { hipError_t _e = (expr); if (_e != hipSuccess) { \
    fprintf(stderr, "HIP error %s at %s:%d\n", hipGetErrorString(_e), __FILE__, __LINE__); \
    return; } } while (0)

namespace {

typedef unsigned u32x4 __attribute__((ext_vector_type(4)));
typedef unsigned u32x2 __attribute__((ext_vector_type(2)));
typedef float f32x4 __attribute__((ext_vector_type(4)));
typedef float f32x2 __attribute__((ext_vector_type(2)));
typedef __bf16 bf16x8 __attribute__((ext_vector_type(8)));
typedef __bf16 bf16x2 __attribute__((ext_vector_type(2)));
typedef unsigned short u16x2 __attribute__((ext_vector_type(2)));
typedef __amdgpu_buffer_rsrc_t rsrc_t;
#define GAS __attribute__((address_space(1)))
#define LAS __attribute__((address_space(3)))
#define CAS __attribute__((address_space(4)))

constexpr int NR = 8;
constexpr int NTOT = 16384;
constexpr int KL = 512;
constexpr int NL = 2048;
constexpr int PIECE = 32768;
constexpr int NDUP = 2;
constexpr int NPUSH = 16;
constexpr int NXT = (7 * NDUP + 1) / 2;
constexpr int NCB = 16 - NDUP;
constexpr int PPK = 24 + NXT;
constexpr int NGEMM = 128;
constexpr int NWG = 256;
constexpr int NTHR = 512;
constexpr int NEXP = 2560;
#define PULL_WG 9
#define PULL_W 8
#define WIN_IN 8
#define WIN_OUT 8
#define PUSH_WIN 4
constexpr unsigned CODE_BYTES = 36u * 1024u;
#define EXP_GL 4
#define EXP_LW 2
#define EXP_LG 4
#define EXP_LP 10
#define EXP_LS 2
#define RAW_TAIL 2
#define LATE_AT 24
#define IBX_PULL 0
#define EXP_L2 16
#define EXP_L3 6
#define EXP_LX 0
#define EXP_LPUSH 2
constexpr int NPWG = 7 * PULL_WG;
static_assert(NPWG < NWG - NGEMM, "puller count");
constexpr int NI_MAIN = 768 - 4 * NPUSH, NI_IN = NI_MAIN + 112 * NDUP, NI_ALL = NI_IN + 64 * NCB;
constexpr int XI_SZ = 6144;
constexpr int XO_SZ = 7168;

constexpr unsigned OFF_CNT = 0;
constexpr unsigned OFF_TMO = 8192;
constexpr unsigned OFF_TICK = 12288;
constexpr unsigned OFF_AMAX = 16384;
constexpr unsigned OFF_IFLAG = 24576;
constexpr unsigned OFF_XFLAG = 32768;
constexpr unsigned XFLAG_SZ = NR * 8 * PPK * 16;
constexpr unsigned OFF_OFLAG = 163840;
constexpr unsigned OFF_SFLAG = 262144;
constexpr unsigned OFF_DBG = 512 * 1024;
constexpr unsigned OFF_XIN = 1u << 20;
constexpr unsigned OFF_XOUT = OFF_XIN + 2 * NEXP * XI_SZ;
constexpr unsigned OFF_IBX = OFF_XOUT + 1024 * XO_SZ;
constexpr unsigned OFF_STAGE = OFF_IBX + 512 * XI_SZ;
constexpr unsigned OFF_RIN = OFF_STAGE + NR * 8 * PPK * PIECE;
constexpr unsigned OFF_ROUT = OFF_RIN + 2 * NEXP * 8192;
constexpr unsigned OFF_RIBX = OFF_ROUT + 1024 * 8192;
constexpr unsigned WS_NEED = OFF_RIBX + 512 * 8192;
static_assert(OFF_XFLAG + 2 * XFLAG_SZ <= OFF_OFLAG && OFF_OFLAG + 65536 <= OFF_SFLAG && OFF_SFLAG + NR * 8 * PPK * 128 <= OFF_DBG, "layout");
constexpr unsigned long long DEADLINE = 50000000ull;
constexpr int LDS_MISC = 131072;
constexpr int LDS_BYTES = LDS_MISC + 1024;
constexpr int AUX_SYS = 17;
constexpr int OUT_AUX = 17;
constexpr unsigned SCR_SZ = 10240;
constexpr unsigned SC_BLOB = 0, SC_A2 = 1152, SC_A3 = 5376, SC_LUT = 9600;
constexpr unsigned M_RAW = 15;

template <int V> struct IC { static constexpr int value = V; };
template <int N, int I = 0, class F> __device__ __forceinline__ void static_for(F&& f) {
    if constexpr (I < N) { f(IC<I>{}); static_for<N, I + 1>(f); }
}

struct Params {
    const float* x;
    const float* w;
    char* ws[NR];
    char* out;
    char* self;
    int rank;
    int pad;
};
#define TUNED(i, dflt) (dflt)

__device__ __forceinline__ unsigned ld_sys(const char* p) {
    return __hip_atomic_load((const GAS unsigned*)p, __ATOMIC_RELAXED, __HIP_MEMORY_SCOPE_SYSTEM);
}
__device__ __forceinline__ unsigned long long ld_sys64(const char* p) {
    return __hip_atomic_load((const GAS unsigned long long*)p, __ATOMIC_RELAXED, __HIP_MEMORY_SCOPE_SYSTEM);
}
__device__ __forceinline__ void st_sys(char* p, unsigned v) {
    __hip_atomic_store((GAS unsigned*)p, v, __ATOMIC_RELAXED, __HIP_MEMORY_SCOPE_SYSTEM);
}
__device__ __forceinline__ void st_sys64(char* p, unsigned long long v) {
    __hip_atomic_store((GAS unsigned long long*)p, v, __ATOMIC_RELAXED, __HIP_MEMORY_SCOPE_SYSTEM);
}
__device__ __forceinline__ unsigned ld_agent(const char* p) {
    return __hip_atomic_load((const GAS unsigned*)p, __ATOMIC_RELAXED, __HIP_MEMORY_SCOPE_AGENT);
}
__device__ __forceinline__ void st_agent(char* p, unsigned v) {
    __hip_atomic_store((GAS unsigned*)p, v, __ATOMIC_RELAXED, __HIP_MEMORY_SCOPE_AGENT);
}
__device__ __forceinline__ rsrc_t mk_rsrc(const void* p, unsigned bytes) {
    return __builtin_amdgcn_make_buffer_rsrc((void*)p, (short)0, (int)bytes, 0x00020000);
}
__device__ __forceinline__ u32x4 bld16(rsrc_t r, unsigned voff, unsigned soff) {
    return __builtin_amdgcn_raw_buffer_load_b128(r, (int)voff, (int)soff, AUX_SYS);
}
#define STB "sc0 sc1"
__device__ __forceinline__ void st16(const char* base, unsigned voff, int blk, unsigned soff, u32x4 v) {
    const char* b = base + soff;
    if (blk == 0) asm volatile("s_nop 4\n\tglobal_store_dwordx4 %0, %1, %2 " STB "\n\ts_nop 1" :: "v"(voff), "v"(v), "s"(b) : "memory");
    if (blk == 1) asm volatile("s_nop 4\n\tglobal_store_dwordx4 %0, %1, %2 offset:1024 " STB "\n\ts_nop 1" :: "v"(voff), "v"(v), "s"(b) : "memory");
    if (blk == 2) asm volatile("s_nop 4\n\tglobal_store_dwordx4 %0, %1, %2 offset:2048 " STB "\n\ts_nop 1" :: "v"(voff), "v"(v), "s"(b) : "memory");
    if (blk == 3) asm volatile("s_nop 4\n\tglobal_store_dwordx4 %0, %1, %2 offset:3072 " STB "\n\ts_nop 1" :: "v"(voff), "v"(v), "s"(b) : "memory");
}
#define WAIT_VM0() do { asm volatile("s_waitcnt vmcnt(0)" ::: "memory"); __builtin_amdgcn_s_waitcnt(0x0F70); } while (0)
#define WAIT_LGKM0() asm volatile("s_waitcnt lgkmcnt(0)" ::: "memory")
#define RAW_BAR() do { asm volatile("s_waitcnt lgkmcnt(0)" ::: "memory"); __builtin_amdgcn_s_barrier(); asm volatile("" ::: "memory"); } while (0)
#define SCHED0() __builtin_amdgcn_sched_barrier(0)
#define PERM(a, b, s) __builtin_amdgcn_perm((a), (b), (s))
#define RFL(x) ((unsigned)__builtin_amdgcn_readfirstlane((int)(x)))

#define GLB "sc0 sc1"
template <int OFF> __device__ __forceinline__ void glds_if(unsigned have, unsigned want, const char* src, unsigned loff, unsigned ldst) {
    asm volatile(
        "s_cmp_lt_u32 %0, %5\n\t"
        "s_cbranch_scc1 1f\n\t"
        "s_mov_b32 m0, %3\n\t"
        "s_nop 0\n\t"
        "global_load_lds_dwordx4 %1, %2 offset:%4 " GLB "\n"
        "1:"
        :: "s"(have), "v"(loff), "s"(src), "s"(ldst), "n"(OFF), "s"(want) : "memory", "scc");
}
__device__ __forceinline__ void stage8k(const char* src, unsigned loff, unsigned loff2, unsigned ldst) {
    asm volatile(
        "s_mov_b32 m0, %3\n\t"
        "s_nop 3\n\t"
        "global_load_lds_dwordx4 %0, %2 " GLB "\n\t"
        "global_load_lds_dwordx4 %0, %2 offset:1024 " GLB "\n\t"
        "global_load_lds_dwordx4 %0, %2 offset:2048 " GLB "\n\t"
        "global_load_lds_dwordx4 %0, %2 offset:3072 " GLB "\n\t"
        "s_add_u32 m0, m0, 0x1000\n\t"
        "s_nop 0\n\t"
        "global_load_lds_dwordx4 %1, %2 " GLB "\n\t"
        "global_load_lds_dwordx4 %1, %2 offset:1024 " GLB "\n\t"
        "global_load_lds_dwordx4 %1, %2 offset:2048 " GLB "\n\t"
        "global_load_lds_dwordx4 %1, %2 offset:3072 " GLB
        :: "v"(loff), "v"(loff2), "s"(src), "s"(ldst) : "memory", "scc");
}

__device__ __forceinline__ unsigned pk_bf16(float a, float b) {
    f32x2 v = {a, b};
    bf16x2 r = __builtin_convertvector(v, bf16x2);
    return __builtin_bit_cast(unsigned, r);
}

#define STAMP(i) do {} while (0)

__device__ __forceinline__ bool give_up(char* tmo, unsigned seq, unsigned long long t0) {
    if (RFL(ld_agent(tmo)) == seq || __builtin_amdgcn_s_memrealtime() - t0 > DEADLINE) {
        st_agent(tmo, seq);
        return true;
    }
    return false;
}
__device__ __forceinline__ void spin_flag(const char* flag, unsigned seq, char* tmo) {
    unsigned long long t0 = __builtin_amdgcn_s_memrealtime();
    for (unsigned n = 1; RFL(ld_sys(flag)) < seq; ++n) {
        if ((n & 31u) == 0u && give_up(tmo, seq, t0)) break;
        __builtin_amdgcn_s_sleep(1);
    }
    asm volatile("" ::: "memory");
}

typedef unsigned u32x2a __attribute__((ext_vector_type(2), aligned(4)));
#define LDS32(a) (*(LAS unsigned*)(size_t)(a))
#define CFENCE() asm volatile("" ::: "memory")

__device__ __forceinline__ unsigned wave_scan(unsigned x) {
    x += (unsigned)__builtin_amdgcn_update_dpp(0, (int)x, 0x111, 0xf, 0xf, false);
    x += (unsigned)__builtin_amdgcn_update_dpp(0, (int)x, 0x112, 0xf, 0xf, false);
    x += (unsigned)__builtin_amdgcn_update_dpp(0, (int)x, 0x114, 0xf, 0xf, false);
    x += (unsigned)__builtin_amdgcn_update_dpp(0, (int)x, 0x118, 0xf, 0xf, false);
    x += (unsigned)__builtin_amdgcn_update_dpp(0, (int)x, 0x142, 0xa, 0xf, false);
    x += (unsigned)__builtin_amdgcn_update_dpp(0, (int)x, 0x143, 0xc, 0xf, false);
    return x;
}
__device__ __forceinline__ unsigned esc_bits(unsigned P) { return __builtin_amdgcn_bitop3_b32(P, P >> 1, 0x55555555u, 0x80); }

__device__ __forceinline__ void lut_init(unsigned scr, int lane) {
    if (lane < 16) {
        unsigned sel = 0x0c0c0c0cu; int k = 0;
#pragma unroll
        for (int y = 0; y < 4; ++y) if ((lane >> y) & 1) { sel = (sel & ~(0xffu << (8 * k))) | ((unsigned)y << (8 * k)); ++k; }
        LDS32(scr + SC_LUT + lane * 4) = sel;
    }
}
#define BFI(m, x, y) __builtin_amdgcn_bitop3_b32((m), (x), (y), 0xCA)
__device__ __forceinline__ void zero_scratch(unsigned scr, int lane) {
#pragma unroll
    for (int i = 0; i < 9; ++i) *(LAS u32x4*)(size_t)(scr + i * 1024 + lane * 16) = (u32x4){0u, 0u, 0u, 0u};
    if (lane < 24) *(LAS u32x4*)(size_t)(scr + 9216 + lane * 16) = (u32x4){0u, 0u, 0u, 0u};
}
__device__ __forceinline__ unsigned digits1(unsigned n02, unsigned n13) {
    const unsigned lo = BFI(0x33333333u, n02, n13 << 2), hi = BFI(0x33333333u, n02 >> 2, n13);
    const unsigned h = (hi | (hi >> 1)) & 0x55555555u;
    return lo | h | (h << 1);
}
__device__ __forceinline__ unsigned sel_addr(unsigned E, int t, unsigned scr) {
    if (t == 0) return __builtin_amdgcn_udot4(E & 0x01010101u, 0x20100804u, scr + SC_LUT, false);
    if (t == 1) return __builtin_amdgcn_udot4(E & 0x04040404u, 0x08040201u, scr + SC_LUT, false);
    if (t == 2) return __builtin_amdgcn_udot4((E >> 4) & 0x01010101u, 0x20100804u, scr + SC_LUT, false);
    return __builtin_amdgcn_udot4((E >> 4) & 0x04040404u, 0x08040201u, scr + SC_LUT, false);
}
template <int NP, bool IN> __device__ __forceinline__ unsigned encode(const unsigned (&n)[2 * NP], unsigned (&P)[NP], unsigned scr, unsigned nb0, int lane) {
    unsigned E[NP], cnt = 0;
#pragma unroll
    for (int j = 0; j < NP; ++j) { P[j] = digits1(n[2 * j], n[2 * j + 1]); E[j] = esc_bits(P[j]); cnt += __popc(E[j]); }
    unsigned sel[2][8];
#pragma unroll
    for (int k = 0; k < 8; ++k) sel[0][k] = LDS32(sel_addr(E[k >> 2], k & 3, scr));
    cnt = max(cnt, 4u);
    const unsigned inc = wave_scan(cnt);
    const unsigned n2d = ((unsigned)__builtin_amdgcn_readlane((int)inc, 63) + 15u) >> 4;
    if (nb0 + n2d > 256u) return 1000u;
    unsigned pos = scr + ((lane & 1) ? SC_A3 : SC_A2) + inc - cnt, carry = 0;
    static_for<NP / 2>([&](auto gc) __attribute__((always_inline)) {
        constexpr int g = decltype(gc)::value;
        SCHED0();
        if constexpr (g + 1 < NP / 2) {
#pragma unroll
            for (int k = 0; k < 8; ++k) sel[(g + 1) & 1][k] = LDS32(sel_addr(E[2 * g + 2 + (k >> 2)], k & 3, scr));
        }
        SCHED0();
#pragma unroll
        for (int k = 0; k < 8; ++k) {
            const int j = 2 * g + (k >> 2), t = k & 3;
            const unsigned z = t < 2 ? n[2 * j + t] : n[2 * j + t - 2] >> 4;
            const unsigned long long v = (unsigned long long)PERM(0u, z, sel[g & 1][k]) << ((pos & 3u) * 8u);
            const unsigned lo = (unsigned)v | carry;
            LDS32(pos & ~3u) = lo;
            const unsigned np = pos + __popc(E[j] & (0x01010101u << (2 * t)));
            carry = ((np ^ pos) & 4u) ? (unsigned)(v >> 32) : lo;
            pos = np;
        }
    });
    LDS32(pos & ~3u) = carry;
    CFENCE();
    unsigned n3 = 0;
    const unsigned s3 = scr + SC_BLOB + (nb0 + n2d) * 4;
    for (unsigned i0 = 0; i0 < n2d; i0 += 64) {
        const unsigned i = i0 + lane;
        const u32x4 v = (*(LAS u32x4*)(size_t)(scr + SC_A2 + i * 16) | *(LAS u32x4*)(size_t)(scr + SC_A3 + i * 16)) & 0x0f0f0f0fu;
        unsigned x = 0;
#pragma unroll
        for (int t = 0; t < 4; ++t) {
            const unsigned z1 = v[t] + 0x01010101u;
            unsigned d = z1, e4;
            if (IN) { e4 = (((v[t] + 0x0b0b0b0bu) ^ z1) >> 4) & 0x01010101u; d |= z1 >> 3; }
            else e4 = ((v[t] + 0x0a0a0a0au) >> 4) & 0x01010101u;
            x |= ((d | e4 | (e4 << 1)) & 0x03030303u) << (2 * t);
        }
        const unsigned E2 = esc_bits(x);
        unsigned sl[4];
#pragma unroll
        for (int t = 0; t < 4; ++t) sl[t] = LDS32(sel_addr(E2, t, scr));
        const unsigned c = __popc(E2), ic = wave_scan(c);
        const unsigned tot = (unsigned)__builtin_amdgcn_readlane((int)ic, 63);
        if (nb0 + n2d + ((n3 + tot + 7u) >> 3) > 256u) return 1000u;
        if (i < n2d) LDS32(scr + SC_BLOB + (nb0 + i) * 4) = x;
        unsigned h[4];
#pragma unroll
        for (int t = 0; t < 4; ++t) { const unsigned y = PERM(0u, v[t], sl[t]); h[t] = PERM(0u, y | (y >> 4), 0x0c0c0200u); }
        const unsigned c0 = __popc(E2 & 0x01010101u), c01 = __popc(E2 & 0x05050505u), c2 = __popc(E2 & 0x10101010u);
        const unsigned long long acc = (unsigned long long)(h[0] | (h[1] << (4u * c0))) | ((unsigned long long)(h[2] | (h[3] << (4u * c2))) << (4u * c01));
        const unsigned p3 = n3 + ic - c, sh = (p3 & 7u) * 4u;
        if (c) {
            LAS unsigned* d = (LAS unsigned*)(size_t)(s3 + (p3 >> 3) * 4);
            const unsigned long long w = acc << sh;
            __hip_atomic_fetch_or(d, (unsigned)w, __ATOMIC_RELAXED, __HIP_MEMORY_SCOPE_WAVEFRONT);
            if (4u * c + sh > 32u) {
                __hip_atomic_fetch_or(d + 1, (unsigned)(w >> 32), __ATOMIC_RELAXED, __HIP_MEMORY_SCOPE_WAVEFRONT);
                __hip_atomic_fetch_or(d + 2, (unsigned)(((acc >> 32) << sh) >> 32), __ATOMIC_RELAXED, __HIP_MEMORY_SCOPE_WAVEFRONT);
            }
        }
        n3 += tot;
    }
    CFENCE();
    return nb0 + n2d + ((n3 + 7u) >> 3);
}
__device__ __forceinline__ unsigned rot8(unsigned e) { return ((e >> 1) & 0x7f7f7f7fu) | ((e << 7) & 0x80808080u); }
__device__ __forceinline__ void selectors(unsigned P, unsigned E, unsigned (&sel)[4]) {
    const unsigned pf = ((((E << 8) + E) << 8) + E) << 8;
    const unsigned S = BFI(E | (E << 1), pf, P);
    const unsigned A = BFI(0x33333333u, S, E << 2), B = BFI(0x33333333u, S >> 2, E);
    sel[0] = A & 0x07070707u; sel[1] = B & 0x07070707u; sel[2] = (A >> 4) & 0x07070707u; sel[3] = (B >> 4) & 0x07070707u;
}
template <int NP, bool IN> __device__ __forceinline__ void decode(const unsigned (&P)[NP], unsigned (&o)[4 * NP], unsigned scr, unsigned nb0, unsigned nbmax, unsigned T, int lane) {
    const unsigned lut1 = IN ? rot8(T | ((T - 1u) << 8) | ((T - 2u) << 16)) : 0x00100800u;
    const unsigned lut2 = IN ? rot8((T - 3u) | ((T - 4u) << 8) | ((T + 1u) << 16)) : 0x00282018u;
    unsigned E[NP], cnt = 0;
#pragma unroll
    for (int j = 0; j < NP; ++j) { E[j] = esc_bits(P[j]); cnt += __popc(E[j]); }
    cnt = max(cnt, 4u);
    const unsigned inc = wave_scan(cnt);
    const unsigned n2d = min(((unsigned)__builtin_amdgcn_readlane((int)inc, 63) + 15u) >> 4, 256u);
    for (unsigned i = lane; nb0 + n2d + i < nbmax; i += 64) {
        const unsigned W = LDS32(scr + SC_BLOB + (nb0 + n2d + i) * 4);
        unsigned a, b;
        if (IN) {
            const unsigned K = (T + 2u) * 0x01010101u;
            a = rot8(K - (((W & 0x0f0f0f0fu) + 0x02020202u) & 0x0f0f0f0fu));
            b = rot8(K - ((((W >> 4) & 0x0f0f0f0fu) + 0x02020202u) & 0x0f0f0f0fu));
        } else {
            a = (W << 3) & 0x78787878u;
            b = (W >> 1) & 0x78787878u;
        }
        *(LAS u32x2*)(size_t)(scr + SC_A3 + i * 8) = (u32x2){PERM(b, a, 0x05010400u), PERM(b, a, 0x07030602u)};
    }
    CFENCE();
    unsigned n3 = 0;
    for (unsigned i0 = 0; i0 < n2d; i0 += 64) {
        const unsigned i = i0 + lane;
        unsigned W = LDS32(scr + SC_BLOB + ((nb0 + i) & 255u) * 4);
        W = i < n2d ? W : 0u;
        const unsigned E2 = esc_bits(W), c = __popc(E2), ic = wave_scan(c);
        unsigned ps[4];
        u32x2a w[4];
        ps[0] = scr + SC_A3 + min(n3 + ic - c, 2048u);
#pragma unroll
        for (int t = 0; t < 4; ++t) {
            w[t] = *(LAS u32x2a*)(size_t)(ps[t] & ~3u);
            if (t < 3) ps[t + 1] = ps[t] + __popc(E2 & (0x01010101u << (2 * t)));
        }
        SCHED0();
        n3 += (unsigned)__builtin_amdgcn_readlane((int)ic, 63);
        unsigned sel[4];
        selectors(W, E2, sel);
        SCHED0();
        u32x4 v;
#pragma unroll
        for (int t = 0; t < 4; ++t) v[t] = PERM(__builtin_amdgcn_alignbyte(w[t].y, w[t].x, ps[t]), lut2, sel[t]);
        *(LAS u32x4*)(size_t)(scr + SC_A2 + i * 16) = v;
    }
    CFENCE();
    unsigned pos = scr + SC_A2 + min(inc - cnt, 4096u);
    unsigned ps[2][8];
    u32x2a w[2][8];
    static_for<NP / 2 + 1>([&](auto gc) __attribute__((always_inline)) {
        constexpr int g = decltype(gc)::value;
        SCHED0();
        if constexpr (g < NP / 2) {
#pragma unroll
            for (int k = 0; k < 8; ++k) {
                ps[g & 1][k] = pos;
                w[g & 1][k] = *(LAS u32x2a*)(size_t)(pos & ~3u);
                pos += __popc(E[2 * g + (k >> 2)] & (0x01010101u << (2 * (k & 3))));
            }
        }
        SCHED0();
        if constexpr (g > 0) {
#pragma unroll
            for (int h = 0; h < 2; ++h) {
                const int j = 2 * (g - 1) + h;
                unsigned sel[4];
                selectors(P[j], E[j], sel);
#pragma unroll
                for (int t = 0; t < 4; ++t) {
                    const u32x2a x = w[(g - 1) & 1][4 * h + t];
                    o[4 * j + t] = PERM(__builtin_amdgcn_alignbyte(x.y, x.x, ps[(g - 1) & 1][4 * h + t]), lut1, sel[t]);
                }
            }
        }
    });
    CFENCE();
}

__device__ __forceinline__ unsigned gather(unsigned cell, unsigned long long v, unsigned long long& tot) {
    unsigned n;
    const unsigned one = 1u;
    asm volatile("s_mov_b64 exec, 1\n\tds_or_b64 %2, %3\n\tds_add_rtn_u32 %0, %2, %4 offset:8\n\ts_mov_b64 exec, -1\n\t"
                 "ds_read_b64 %1, %2\n\ts_waitcnt lgkmcnt(0)" : "=&v"(n), "=&v"(tot) : "v"(cell), "v"(v), "v"(one) : "memory");
    return RFL(n);
}

__device__ __forceinline__ void exp_decode(int e, int rank, int& kt, int& slot, int& sub, int& dst) {
    sub = e & 3;
    const int q = e >> 2;
    if (q < 192) {
        kt = q / 24; slot = q - kt * 24; dst = slot < 16 ? -1 : rank;
        return;
    }
    const int t = q - 192;
    kt = t / 56;
    const int u = t - kt * 56, j = u / 7;
    slot = 16 + j;
    dst = (rank + 1 + (u - j * 7)) & 7;
}
__device__ __forceinline__ unsigned xin_item(int kt, int slot, int sub, int dst) {
    return (slot < 16 ? kt * 16 + slot : 128 + (dst * 8 + kt) * 8 + (slot - 16)) * 4 + sub;
}

#define LDW(p) __builtin_nontemporal_load(p)
#define LDX(p) __builtin_nontemporal_load(p)
__device__ __forceinline__ void export_item(const CAS Params& p, char* myws, const float* px, const float* pw0, int rank, unsigned scr, unsigned grp, int e, bool stage_only, bool push, unsigned& seq, unsigned seqv, bool first, bool pair, bool bar) {
    const int lane = threadIdx.x & 63;
    int kt, slot, sub, dst;
    exp_decode(e, rank, kt, slot, sub, dst);
    const int kq = lane >> 4, c = lane & 15;
    u32x4 ch[8];
    if (slot < 16) {
        const GAS float* b = (const GAS float*)px + (size_t)(256 * slot + 64 * sub + (lane >> 3)) * KL + 64 * kt;
        const int g0 = 4 * ((lane & 7) ^ (lane >> 4));
        f32x4 v[16];
#pragma unroll
        for (int i = 0; i < 8; ++i) {
            v[2 * i] = LDX((const GAS f32x4*)(b + i * 8 * KL + (g0 ^ ((i & 1) * 16))));
            v[2 * i + 1] = LDX((const GAS f32x4*)(b + i * 8 * KL + (g0 ^ ((i & 1) * 16)) + 32));
        }
        if (!stage_only) zero_scratch(scr, lane);
#pragma unroll
        for (int i = 0; i < 8; ++i) {
            ch[i].x = pk_bf16(v[2 * i][0], v[2 * i][1]); ch[i].y = pk_bf16(v[2 * i][2], v[2 * i][3]);
            ch[i].z = pk_bf16(v[2 * i + 1][0], v[2 * i + 1][1]); ch[i].w = pk_bf16(v[2 * i + 1][2], v[2 * i + 1][3]);
        }
    } else {
        const GAS float* b = (const GAS float*)pw0 + (size_t)(64 * kt + 16 * (sub & 1) + 4 * kq) * NTOT + dst * NL + 256 * (slot - 16) + 128 * (sub >> 1) + 4 * (c ^ (kq & 1));
        f32x4 v[16];
#pragma unroll
        for (int jj = 0; jj < 8; ++jj) {
            v[2 * jj] = LDW((const GAS f32x4*)(b + (size_t)((jj & 3) + (jj >> 2) * 32) * NTOT));
            v[2 * jj + 1] = LDW((const GAS f32x4*)(b + (size_t)((jj & 3) + (jj >> 2) * 32) * NTOT + 64));
        }
        if (!stage_only) zero_scratch(scr, lane);
#pragma unroll
        for (int i = 0; i < 8; ++i) {
            ch[i].x = pk_bf16(v[0 + (i >> 2)][i & 3], v[2 + (i >> 2)][i & 3]); ch[i].y = pk_bf16(v[4 + (i >> 2)][i & 3], v[6 + (i >> 2)][i & 3]);
            ch[i].z = pk_bf16(v[8 + (i >> 2)][i & 3], v[10 + (i >> 2)][i & 3]); ch[i].w = pk_bf16(v[12 + (i >> 2)][i & 3], v[14 + (i >> 2)][i & 3]);
        }
    }
    if (first) { asm volatile("s_waitcnt vmcnt(0)\n\tv_readfirstlane_b32 %0, %1\n\ts_nop 4" : "=s"(seq) : "v"(seqv) : "memory"); seq += 1u; }
    if (bar) RAW_BAR();
    const bool dup = slot == 23 && sub >= 4 - 2 * NDUP;
    const bool local = stage_only || (dup && dst != rank && !push);
    const int d = lane & 7;
    const unsigned at = (rank * 8 + kt) * (PPK * 4) + ((dup && d != dst) ? 96 + ((dst - d - 1) & 7) * (2 * NDUP) + sub - (4 - 2 * NDUP) : slot * 4 + sub);
    if (local) {
        const unsigned mine = (unsigned)__builtin_amdgcn_readlane((int)at, rank);
#pragma unroll
        for (int u = 0; u < 8; ++u) st16(myws, lane * 16, u & 3, OFF_STAGE + mine * 8192 + (u >> 2) * 4096, ch[u]);
        if (stage_only) {
            WAIT_VM0();
            if (lane == rank) st_agent(myws + OFF_SFLAG + at * 32, seq);
            return;
        }
    }
    int l7 = lane & 7;
    asm volatile("" : "+v"(l7));
    char* peer = p.ws[l7];
    unsigned meta = 0;
    const unsigned xi = xin_item(kt, slot, sub, dst) + (seq & 1u) * NEXP, ibx = rank * 64 + kt * 32 + (slot - 16) * 4 + sub;
    {
        const char* xb = push ? p.ws[dst] : myws;
        const unsigned xo = push ? OFF_IBX + ibx * XI_SZ : OFF_XIN + xi * XI_SZ;
        unsigned ex[16], sum = 0;
#pragma unroll
        for (int u = 0; u < 4; ++u) {
            u32x4 r;
#pragma unroll
            for (int k = 0; k < 4; ++k) {
                const unsigned d0 = ch[2 * u + (k >> 1)][(k & 1) * 2], d1 = ch[2 * u + (k >> 1)][(k & 1) * 2 + 1];
                const unsigned L = PERM(d1, d0, 0x06040200u), H = PERM(d1, d0, 0x07050301u);
                r[k] = (H & 0x80808080u) | (L & 0x7f7f7f7fu);
                ex[4 * u + k] = ((H << 1) & 0xfefefefeu) | ((L >> 7) & 0x01010101u);
                sum = __builtin_amdgcn_sad_u8(ex[4 * u + k], 0u, sum);
            }
            st16(xb, lane * 16, u, xo, r);
        }
        unsigned T = ((unsigned)__builtin_amdgcn_readlane((int)wave_scan(sum), 63) + 7864u) >> 12;
        T = min(max(T, 13u), 253u);
        const unsigned K = (T + 2u) * 0x01010101u;
        unsigned z[16], yy[16], bad = 0;
#pragma unroll
        for (int d = 0; d < 16; ++d) {
            const unsigned y = ((K | 0x80808080u) - (ex[d] & 0x7f7f7f7fu)) ^ (~(K ^ ex[d]) & 0x80808080u);
            bad |= y; yy[d] = y;
            z[d] = ((y & 0x0f0f0f0fu) + 0x0e0e0e0eu) & 0x0f0f0f0fu;
        }
        unsigned nex = 0;
        if (__any((bad & 0xf0f0f0f0u) != 0u)) {
            CFENCE();
            unsigned mk = 0;
#pragma unroll
            for (int d = 0; d < 16; ++d) mk |= (yy[d] & 0xf0f0f0f0u) ? 1u << d : 0u;
            while (mk) {
                const unsigned d = __builtin_ctz(mk);
                mk &= mk - 1u;
                unsigned ev = ex[0];
#pragma unroll
                for (int t = 1; t < 16; ++t) ev = d == (unsigned)t ? ex[t] : ev;
#pragma unroll 1
                for (unsigned y = 0; y < 4; ++y) {
                    const unsigned eb = (ev >> (8 * y)) & 0xffu;
                    if (T + 2u - eb > 15u) {
                        const unsigned k = __hip_atomic_fetch_add((LAS unsigned*)(size_t)(scr + SC_LUT + 64), 1u, __ATOMIC_RELAXED, __HIP_MEMORY_SCOPE_WAVEFRONT);
                        if (k < 15u) LDS32(scr + SC_BLOB + k * 4) = eb | (y << 8) | (d << 10) | (lane << 14);
                    }
                }
            }
            CFENCE();
            nex = RFL(LDS32(scr + SC_LUT + 64));
            CFENCE();
            LDS32(scr + SC_LUT + 64) = 0u;
        }
        if (first) STAMP(10);
        unsigned P[4], nb = 1000u;
        if (nex <= 15u) {
            unsigned n[8];
#pragma unroll
            for (int k = 0; k < 8; ++k) n[k] = z[(k >> 1) * 4 + (k & 1)] | (z[(k >> 1) * 4 + (k & 1) + 2] << 4);
            nb = encode<4, true>(n, P, scr, nex, lane);
        }
        if (nb <= 256u) {
            const unsigned lines = (nb + 31u) >> 5;
            st16(xb, lane * 16, 0, xo + 4096, (u32x4){P[0], P[1], P[2], P[3]});
            if ((unsigned)lane < lines * 8u) st16(xb, lane * 16, 1, xo + 4096, *(LAS u32x4*)(size_t)(scr + SC_BLOB + lane * 16));
            meta = T | (lines << 8) | (nex << 12);
        } else {
#pragma unroll
            for (int u = 0; u < 8; ++u) st16(xb, lane * 16, u & 3, (push ? OFF_RIBX + ibx * 8192 : OFF_RIN + xi * 8192) + (u >> 2) * 4096, ch[u]);
            meta = M_RAW << 8;
        }
    }
    if (first) STAMP(11);
    WAIT_VM0();
    if (first) STAMP(12);
    if (local && lane == rank) st_agent(myws + OFF_SFLAG + at * 32, seq);
    if (push) {
        if (lane == dst) st_sys64(peer + OFF_IFLAG + ibx * 8, ((unsigned long long)seq << 32) | meta);
        return;
    }
    unsigned long long tot;
    if (gather(grp, (unsigned long long)meta << (16 * sub), tot) == (pair ? 1u : 3u) && lane < NR && lane != rank && (dst < 0 || dup || lane == dst)) {
        char* fp = peer + OFF_XFLAG + (seq & 1u) * XFLAG_SZ + (at >> 2) * 16;
        if (pair) {
            st_sys64(fp + (sub >> 1) * 8, ((unsigned long long)seq << 32) | (unsigned)tot | (unsigned)(tot >> 32));
        } else {
            const u32x4 g = {(unsigned)tot, seq, (unsigned)(tot >> 32), seq};
            asm volatile("global_store_dwordx4 %0, %1, off sc0 sc1\n\ts_nop 1" :: "v"(fp), "v"(g) : "memory");
        }
    }
}

__device__ __forceinline__ unsigned tick_draw(char* ctr, bool off) {
    return (unsigned)__builtin_amdgcn_raw_ptr_buffer_atomic_add_i32(1, mk_rsrc(ctr, 4), ((threadIdx.x & 63) || off) ? 64 : 0, 0, 16);
}

__device__ __forceinline__ void item_decode(int it, int rank, int delta, unsigned par, unsigned& xi, unsigned& at) {
    if (it >= NI_IN) {
        const int o = it - NI_IN, r = o / NCB;
        xi = at = (((r & 31) >> 1) * 4 + (r & 1) + 2 * (r >> 5)) * 16 + (o - r * NCB);
        return;
    }
    const int src = (rank + delta) & 7;
    if (it >= NI_MAIN) {
        const int o = it - NI_MAIN, kt = o / (14 * NDUP), r = o - kt * (14 * NDUP), u = r / (2 * NDUP);
        xi = xin_item(kt, 23, 4 - 2 * NDUP + (r & (2 * NDUP - 1)), (rank + 1 + u) & 7) + par;
        at = (src * 8 + kt) * (PPK * 4) + 96 + r;
        return;
    }
    if (it < 0) {
        const int k = -1 - it;
        xi = src * 64 + k;
        at = (src * 8 + (k >> 5)) * (PPK * 4) + 64 + (k & 31);
        return;
    }
    const int q = (it >> 2) + 8;
    int kt, slot;
    if (q < 56 || q >= 168) { kt = q < 56 ? (q - 8) >> 4 : 7; slot = (q - 8 + 2 * delta) & 15; }
    else if (q < 72) { kt = q < 64 ? 2 : 3; slot = 16 + (q & 7); }
    else {
        const int r = q / 24, e = q - r * 24, t3 = e / 3, m3 = e - t3 * 3;
        if (m3 < 2) { kt = r; slot = (2 * t3 + m3 + 2 * delta) & 15; }
        else { kt = r + 1; slot = 16 + t3; }
    }
    xi = xin_item(kt, slot, it & 3, rank) + par;
    at = (src * 8 + kt) * (PPK * 4) + slot * 4 + (it & 3);
}

template <int AUX> __device__ __forceinline__ void store_rows(rsrc_t ro, unsigned lane_off, unsigned soff, const unsigned (&q)[32]) {
#pragma unroll
    for (int m = 0; m < 4; ++m)
#pragma unroll
        for (int j = 0; j < 4; ++j) {
            u32x2 v = {q[(m * 4 + j) * 2], q[(m * 4 + j) * 2 + 1]};
            __builtin_amdgcn_raw_buffer_store_b64(v, ro, (int)lane_off, (int)(soff + (m * 16 + j) * NTOT), AUX);
        }
}

__device__ __forceinline__ void puller(const CAS Params& p, char* myws, unsigned scr, unsigned sem, unsigned seq, int link, int g) {
    const int lane = threadIdx.x & 63;
    const int rank = p.rank, delta = link + 1, src = (rank + delta) & 7;
    char* tmo = myws + OFF_TMO;
    const bool once = g < 0;
    const rsrc_t rr = mk_rsrc(once ? myws : p.ws[src], WS_NEED), ro = mk_rsrc(p.out, 4096u * NTOT);
    const unsigned lane_off = (unsigned)(4 * (lane >> 4)) * NTOT + 8 * (lane & 15);
    char* ctr = myws + OFF_TICK + link * 128;
    const unsigned par = (seq & 1u) * NEXP;
    const int win = once ? 1024 : TUNED(0, WIN_IN), wout = TUNED(1, WIN_OUT);
    const unsigned tbase = (seq - 1u) * (unsigned)NI_ALL - (unsigned)(PULL_WG * 8);
    bool dead = false;
    for (int it = g; it < NI_ALL;) {
        unsigned cxi, cat;
        item_decode(it, rank, delta, par, cxi, cat);
        const char* fl = myws + (it >= NI_IN ? OFF_OFLAG + src * 2048 + (((cat >> 6) * 8 + ((cat >> 1) & 7)) * 2 + ((cat >> 5) & 1)) * 8
                                             : (once ? OFF_IFLAG + cxi * 8 : OFF_XFLAG + (seq & 1u) * XFLAG_SZ + (cat >> 1) * 8));
        unsigned long long mn = ld_sys64(fl);
        if (RFL(mn >> 32) != seq) {
            unsigned long long t0 = __builtin_amdgcn_s_memrealtime();
            for (unsigned n = 1;; ++n) {
                mn = ld_sys64(fl);
                if (RFL(mn >> 32) == seq) break;
                if ((n & 31u) == 0u && give_up(tmo, seq, t0)) { mn = 0; dead = true; break; }
                __builtin_amdgcn_s_sleep(1);
            }
        }
        CFENCE();
        const unsigned cmeta = it >= NI_IN ? ((RFL(mn) >> (((cat >> 4) & 1) * 8 + (cat & 1) * 4)) & 15u) << 8 : (RFL(mn) >> (once ? 0 : (cat & 1) * 16)) & 0xffffu;
        u32x4 c[8];
        unsigned tk;
        {
            const unsigned lines = (cmeta >> 8) & 15u;
            const bool isout = it >= NI_IN, raw = lines == M_RAW;
            const unsigned bo = (unsigned)lane < lines * 8u ? lane * 16u : 0x80000000u;
            const unsigned o = raw ? (isout ? OFF_ROUT : (once ? OFF_RIBX : OFF_RIN)) + cxi * 8192
                                   : (isout ? OFF_XOUT + cxi * XO_SZ : (once ? OFF_IBX : OFF_XIN) + cxi * XI_SZ);
            const unsigned nld = once ? 0u : (raw ? 8u : (isout ? 7u : 6u));
            unsigned my;
            asm volatile("s_mov_b64 exec, 1\n\tds_add_rtn_u32 %0, %1, %2\n\ts_mov_b64 exec, -1\n\ts_waitcnt lgkmcnt(0)" : "=&v"(my) : "v"(sem), "v"(nld) : "memory");
            my = RFL(my) - max((unsigned)(isout ? wout : win), nld);
            unsigned rel = RFL(__hip_atomic_load((LAS unsigned*)(size_t)(sem + 4), __ATOMIC_RELAXED, __HIP_MEMORY_SCOPE_WORKGROUP));
#define GATE(u) while ((int)(my + (u) - rel) >= 0) { rel = RFL(__hip_atomic_load((LAS unsigned*)(size_t)(sem + 4), __ATOMIC_RELAXED, __HIP_MEMORY_SCOPE_WORKGROUP)); CFENCE(); }
#define RELEASE(n, v) asm volatile("s_waitcnt vmcnt(%0)\n\ts_mov_b64 exec, 1\n\tds_add_u32 %1, %2 offset:4\n\ts_mov_b64 exec, -1" :: "n"(n), "v"(sem), "v"(v) : "memory")
            while ((int)(my - 4u - rel) >= 0) { __builtin_amdgcn_s_sleep(2); rel = RFL(__hip_atomic_load((LAS unsigned*)(size_t)(sem + 4), __ATOMIC_RELAXED, __HIP_MEMORY_SCOPE_WORKGROUP)); }
            __builtin_amdgcn_s_setprio(3);
            GATE(0u); c[4] = bld16(rr, lane * 16, o + 4096);
            GATE(1u); c[5] = bld16(rr, isout ? lane * 16u : bo, o + 5120);
            GATE(2u); c[0] = bld16(rr, lane * 16, o);
            GATE(3u); c[1] = bld16(rr, lane * 16 + 1024, o);
            GATE(4u); c[2] = bld16(rr, lane * 16 + 2048, o);
            GATE(5u); c[3] = bld16(rr, lane * 16 + 3072, o);
            GATE(nld - 1u);
            c[6] = bld16(rr, (isout || raw) ? bo : 0x80000000u, o + 6144);
            c[7] = bld16(rr, raw ? lane * 16u : 0x80000000u, o + 7168);
            tk = tick_draw(ctr, once);
            const unsigned one = once ? 0u : 1u, last = once ? 0u : nld - 5u;
            RELEASE(8, one); RELEASE(7, one); RELEASE(6, one); RELEASE(5, one); RELEASE(4, one); RELEASE(1, last);
            __builtin_amdgcn_s_setprio(0);
        }
        {
            const unsigned lines = (cmeta >> 8) & 15u;
            if (it >= NI_IN) {
                unsigned q[32];
                if (dead) {
#pragma unroll
                    for (int d = 0; d < 32; ++d) q[d] = 0x80808080u;
                } else if (lines == M_RAW) {
#pragma unroll
                    for (int d = 0; d < 32; ++d) q[d] = c[d >> 2][d & 3];
                } else {
                    *(LAS u32x4*)(size_t)(scr + SC_BLOB + lane * 16) = c[6];
                    CFENCE();
                    const unsigned P[8] = {c[4].x, c[4].y, c[4].z, c[4].w, c[5].x, c[5].y, c[5].z, c[5].w};
                    decode<8, false>(P, q, scr, 0u, lines * 32u, 0u, lane);
#pragma unroll
                    for (int d = 0; d < 32; ++d) {
                        const unsigned R = c[d >> 3][((d >> 1) & 2) + (d & 1)] >> ((d & 2) * 2);
                        const unsigned s8 = R & 0x08080808u;
                        q[d] = (q[d] ^ ((s8 << 5) - s8)) | (R & 0x07070707u);
                    }
                }
                store_rows<OUT_AUX>(ro, lane_off, (cat >> 4) * (64u * NTOT) + src * NL + (cat & 15u) * 128u, q);
            } else {
                u32x4 o[8];
                if (lines == M_RAW) {
#pragma unroll
                    for (int u = 0; u < 8; ++u) o[u] = c[u];
                } else {
                    *(LAS u32x4*)(size_t)(scr + SC_BLOB + lane * 16) = c[5];
                    CFENCE();
                    const unsigned P[4] = {c[4].x, c[4].y, c[4].z, c[4].w};
                    const unsigned nex = (cmeta >> 12) & 15u;
                    unsigned ex[16];
                    decode<4, true>(P, ex, scr, nex, lines * 32u, cmeta & 0xffu, lane);
                    for (unsigned k = 0; k < nex; ++k) {
                        const unsigned en = RFL(LDS32(scr + SC_BLOB + k * 4));
                        const unsigned sh = ((en >> 8) & 3u) * 8u, r = (en >> 10) & 15u;
                        const unsigned msk = (unsigned)lane == ((en >> 14) & 63u) ? 0xffu << sh : 0u;
                        const unsigned val = rot8(en & 0xffu) << sh;
#pragma unroll
                        for (int d = 0; d < 16; ++d) { const unsigned mq = r == (unsigned)d ? msk : 0u; ex[d] = (ex[d] & ~mq) | (val & mq); }
                    }
#pragma unroll
                    for (int d = 0; d < 16; ++d) {
                        const unsigned r = c[d >> 2][d & 3];
                        const unsigned H = (r & 0x80808080u) | (ex[d] & 0x7f7f7f7fu), L = (ex[d] & 0x80808080u) | (r & 0x7f7f7f7fu);
                        o[d >> 1][(d & 1) * 2] = PERM(H, L, 0x05010400u);
                        o[d >> 1][(d & 1) * 2 + 1] = PERM(H, L, 0x07030602u);
                    }
                }
#pragma unroll
                for (int u = 0; u < 8; ++u) st16(myws, lane * 16, u & 3, OFF_STAGE + cat * 8192 + (u >> 2) * 4096, o[u]);
                WAIT_VM0();
                if (lane == 0) st_agent(myws + OFF_SFLAG + cat * 32, seq);
            }
        }
        if (once) break;
        const int nx = (int)min(RFL(tk) - tbase, (unsigned)NI_ALL);
        it = nx;
    }
}

__device__ __forceinline__ int step_group(int s, int rank, int d0) {
    if (s < 8) return rank * 8 + s;
    int t = s - 8;
    int kt = t / 7, u = t - kt * 7;
    int v = u < d0 ? u : u + 1;
    int delta = (d0 - v) & 7;
    return ((rank + delta) & 7) * 8 + kt;
}

__device__ __forceinline__ void gemm_role(const CAS Params& p, int gidx, unsigned seq, char* myws, LAS char* lds) {
    int tid = threadIdx.x;
    asm volatile("" : "+v"(tid));
    const int wid = RFL(tid >> 6);
    int lane = tid & 63;
    const int wr = wid >> 1, wc = wid & 1;
    const int rank = p.rank;
    const bool mainw = gidx >= 0;
    int ti, tj;
    if (mainw) { const int xcd = gidx & 7, sl = gidx >> 3; ti = (xcd >> 1) * 4 + (sl >> 2); tj = (xcd & 1) * 4 + (sl & 3); }
    else { const int x = -1 - gidx; ti = (x & 7) * 2 + ((x >> 3) & 1); tj = 8 + (x >> 4); }
    const int d0 = ti >> 1;
    const int myslot = wid < 4 ? ti : 16 + tj;
    char* tmo = myws + OFF_TMO;
    const char* dsrc = myws + OFF_STAGE + (size_t)myslot * PIECE + (wid & 3) * 8192;
    const char* fsrc = myws + OFF_SFLAG + myslot * 128 + (wid & 3) * 32;
    const unsigned loff = lane * 16, loff2 = loff + 4096;
    const unsigned ldsw = (unsigned)(size_t)lds + wid * 8192;

    f32x4 acc[4][8];
#pragma unroll
    for (int m = 0; m < 4; ++m)
#pragma unroll
        for (int i = 0; i < 8; ++i) acc[m][i] = (f32x4){0.f, 0.f, 0.f, 0.f};

#define GL(n, S) glds_if<((n) & 3) * 1024>(have, want, S, (n) < 4 ? loff : loff2, (n) < 4 ? ld0 : ld1)
#define STAGE_ALL(S, D) stage8k(S, loff, loff2, D)
#define PRIO(n) __builtin_amdgcn_s_setprio(n)
#define SPLIT_AT 0
#define RD1(t) (((t) & 1) == 0 && (t) < 24 ? (t) / 2 : -1)
#define P2AT 0
#define P1AT 0
#define RD2(t) (((t) & 1) == 0 && (t) >= 2 && (t) < 26 ? (t) / 2 - 1 : -1)
#define GL2(t) (((t) & 3) == 1 ? (t) >> 2 : -1)
    const unsigned barw = (unsigned)(size_t)lds + LDS_MISC + 32;
    unsigned bart = mainw ? 0u : 512u;
#define GOFF(st) ((unsigned)__builtin_amdgcn_readlane((int)tab, (st)))
#define GSRC(off) (dsrc + (off))
#define GFLAG(off) (fsrc + ((off) >> 8))
    static_assert(PPK * PIECE / 256 == PPK * 128, "flag offset");
    const unsigned tab = (unsigned)step_group(lane, rank, d0) * (PPK * PIECE);
    const unsigned abase = (unsigned)(size_t)lds + wr * 8192 + (lane & 15) * 128 + (((lane >> 4) ^ ((lane >> 1) & 7)) << 4);
    const unsigned bbase = (unsigned)(size_t)lds + 32768 + wc * 16384 + ((lane >> 3) & 1) * 4096 + (lane >> 4) * 256 + (lane & 7) * 32 + ((lane >> 4) & 1) * 16;
#define FRAG(a) (*(const LAS bf16x8*)(size_t)(a))
    bf16x8 a0[4], b0[8], a1[4], b1[8];

    {
        const unsigned g0 = GOFF(0);
        spin_flag(GFLAG(g0), seq, tmo);
        STAGE_ALL(GSRC(g0), ldsw);
    }
    unsigned gn = GOFF(1);
    unsigned fv = ld_sys(GFLAG(gn));
    WAIT_VM0();
    RAW_BAR();
    if (mainw) STAMP(2);
    bool staged = RFL(fv) >= seq;
    if (staged) { STAGE_ALL(GSRC(gn), ldsw + 65536); gn = GOFF(2); }
    fv = ld_sys(GFLAG(gn));
#pragma unroll
    for (int i = 0; i < 8; ++i) b0[i] = FRAG((bbase ^ ((i >> 2) * 16)) + (i & 3) * 1024);
#pragma unroll
    for (int m = 0; m < 4; ++m) a0[m] = FRAG(abase + m * 2048);

    for (int s = 0; s < 64; ++s) {
        const int cur = (s & 1) * 65536, oth = cur ^ 65536;
        if (!staged) {
            WAIT_VM0();
            if (RFL(fv) >= seq) { STAGE_ALL(GSRC(gn), ldsw + oth); staged = true; gn = GOFF(min(s + 2, 63)); fv = ld_sys(GFLAG(gn)); }
        }
        const unsigned ab2 = abase + oth, bb2 = bbase + oth;
        const unsigned want = s + 2 < 64 ? seq : 0xffffffffu;
        const unsigned g3 = GOFF(min(s + 3, 63));
        const unsigned ld0 = ldsw + cur, ld1 = ld0 + 4096;
        SCHED0();
        {
            const unsigned ab = (abase + cur) ^ 64u, bb = bbase + cur;
            if (SPLIT_AT == 0) PRIO(1);
            static_for<32>([&](auto tc) __attribute__((always_inline)) {
                constexpr int t = decltype(tc)::value;
                acc[t >> 3][t & 7] = __builtin_amdgcn_mfma_f32_16x16x32_bf16(a0[t >> 3], b0[t & 7], acc[t >> 3][t & 7], 0, 0, 0);
                constexpr int rd = RD1(t);
                if constexpr (rd >= 0 && rd < 8) b1[rd] = FRAG((bb ^ ((rd >> 2) * 16)) + (rd & 3) * 1024 + 8192);
                if constexpr (rd >= 8) a1[rd - 8] = FRAG(ab + (rd - 8) * 2048);
                if constexpr (t == (SPLIT_AT > 0 ? P1AT : 15)) PRIO(SPLIT_AT > 0 ? 1 : 0);
                SCHED0();
            });
        }
        if (!staged) {
            spin_flag(GFLAG(gn), seq, tmo);
            STAGE_ALL(GSRC(gn), ldsw + oth);
            gn = GOFF(min(s + 2, 63));
            fv = ld_sys(GFLAG(gn));
        }
        const char* gs = GSRC(gn);
        WAIT_VM0();
        const unsigned have = RFL(fv);
        __builtin_amdgcn_s_waitcnt(0xC07F);
        PRIO(SPLIT_AT > 0 ? 0 : 3);
        SCHED0();
        RAW_BAR();
        SCHED0();
        {
            static_for<32>([&](auto tc) __attribute__((always_inline)) {
                constexpr int t = decltype(tc)::value;
                acc[t >> 3][t & 7] = __builtin_amdgcn_mfma_f32_16x16x32_bf16(a1[t >> 3], b1[t & 7], acc[t >> 3][t & 7], 0, 0, 0);
                constexpr int rd = RD2(t);
                if constexpr (rd >= 0 && rd < 8) b0[rd] = FRAG((bb2 ^ ((rd >> 2) * 16)) + (rd & 3) * 1024);
                if constexpr (rd >= 8) a0[rd - 8] = FRAG(ab2 + (rd - 8) * 2048);
                if constexpr (GL2(t) >= 0) GL(GL2(t), gs);
                if constexpr (t == (SPLIT_AT > 0 ? P2AT : 15)) PRIO(2);
                if constexpr (t == 30) {
                    const bool go = have >= want;
                    staged = go || s + 2 >= 64;
                    gn = go ? g3 : gn;
                }
                if constexpr (t == 31) fv = ld_sys(GFLAG(gn));
                SCHED0();
            });
        }
    }
    if (mainw) STAMP(3);

    asm volatile("" : "+v"(tid));
    lane = tid & 63;
    char* peer = p.ws[lane & 7];
    float am = 0.f;
#pragma unroll
    for (int m = 0; m < 4; ++m)
#pragma unroll
        for (int i = 0; i < 8; ++i)
#pragma unroll
            for (int j = 0; j < 4; ++j) am = fmaxf(am, fabsf(acc[m][i][j]));
#pragma unroll
    for (int o = 32; o > 0; o >>= 1) am = fmaxf(am, __shfl_xor(am, o));
    LAS float* red = (LAS float*)(lds + LDS_MISC + 512);
    if (lane == 0) red[wid] = am;
    RAW_BAR();
    if (tid < NR && mainw) {
        float a = red[0];
#pragma unroll
        for (int i = 1; i < 8; ++i) a = fmaxf(a, red[i]);
        st_sys64(peer + OFF_AMAX + (rank * NGEMM + gidx) * 8, ((unsigned long long)seq << 32) | __float_as_uint(a));
    }
    const unsigned scr = (unsigned)(size_t)lds + wid * SCR_SZ;
    zero_scratch(scr, lane);
    lut_init(scr, lane);
    {
        const char* gp = myws + OFF_AMAX + tid * 16;
        unsigned long long t0 = __builtin_amdgcn_s_memrealtime();
        unsigned long long a, b;
        for (unsigned n = 1;; ++n) {
            a = ld_sys64(gp);
            b = ld_sys64(gp + 8);
            bool ok = (unsigned)(a >> 32) == seq && (unsigned)(b >> 32) == seq;
            if (__all(ok)) break;
            if ((n & 31u) == 0u && give_up(tmo, seq, t0)) break;
            __builtin_amdgcn_s_sleep(8);
        }
        am = fmaxf(__uint_as_float((unsigned)a), __uint_as_float((unsigned)b));
#pragma unroll
        for (int o = 32; o > 0; o >>= 1) am = fmaxf(am, __shfl_xor(am, o));
        if (lane == 0) red[8 + wid] = am;
        WAIT_VM0();
        RAW_BAR();
#pragma unroll
        for (int i = 0; i < 8; ++i) am = fmaxf(am, red[8 + i]);
    }
    const unsigned tmov = ld_agent(tmo);
    const unsigned prog = ld_agent(myws + OFF_TICK);
    if (mainw) STAMP(4);

    const float inv = am > 0.f ? 127.0f / am : 0.f;
    unsigned q[32];
#pragma unroll
    for (int m = 0; m < 4; ++m)
#pragma unroll
        for (int j = 0; j < 4; ++j)
#pragma unroll
            for (int h = 0; h < 2; ++h) {
                const unsigned t0 = __float_as_uint(fmaf(acc[m][4 * h + 0][j], inv, 12582912.f));
                const unsigned t1 = __float_as_uint(fmaf(acc[m][4 * h + 1][j], inv, 12582912.f));
                const unsigned t2 = __float_as_uint(fmaf(acc[m][4 * h + 2][j], inv, 12582912.f));
                const unsigned t3 = __float_as_uint(fmaf(acc[m][4 * h + 3][j], inv, 12582912.f));
                q[(m * 4 + j) * 2 + h] = PERM(t1, t0, 0x0c0c0400u) | PERM(t3, t2, 0x04000c0cu);
            }
    if (RFL(tmov) == seq) {
#pragma unroll
        for (int d = 0; d < 32; ++d) q[d] = 0x80808080u;
    }
    const unsigned lane_off = (unsigned)(4 * (lane >> 4)) * NTOT + 8 * (lane & 15);
    const rsrc_t ro = mk_rsrc(p.out, 4096u * NTOT);
    const unsigned rowoff = (unsigned)(ti * 4 + wr) * (64u * NTOT);
    if (!mainw) {
        const int h = (tj - 8) * 2 + wc;
        if (h < 7 * NDUP) store_rows<OUT_AUX>(ro, lane_off, rowoff + ((rank + 1 + h / NDUP) & 7) * NL + (NCB + h % NDUP) * 128, q);
        STAMP(6);
        return;
    }
    const int cb = tj * 2 + wc;
    if (cb < NCB) {
        const unsigned xi = (ti * 4 + wr) * 16 + cb;
        const bool late = RFL(prog) - (seq - 1u) * (unsigned)NI_ALL + PULL_WG * 8 > (unsigned)(NI_IN + LATE_AT);
        unsigned P[8], nb = 1000u;
        if (!(ti * 4 + wr >= 64 - RAW_TAIL || (ti == 0 && wr < 2 && late))) {
        unsigned n[16];
#pragma unroll
        for (int u = 0; u < 4; ++u) {
            u32x4 r;
#pragma unroll
            for (int k = 0; k < 4; ++k) {
                const unsigned q0 = q[8 * u + (k >> 1) * 4 + (k & 1)], q1 = q[8 * u + (k >> 1) * 4 + (k & 1) + 2];
                r[k] = BFI(0x77777777u, BFI(0x0f0f0f0fu, q0, q1 << 4), BFI(0x0f0f0f0fu, q0 >> 4, q1));
                const unsigned sg = (r[k] >> 3) & 0x11111111u;
                n[4 * u + k] = BFI(0x0f0f0f0fu, q0 >> 3, q1 << 1) ^ ((sg << 4) - sg);
            }
            st16(myws, lane * 16, u, OFF_XOUT + xi * XO_SZ, r);
        }
        nb = encode<8, false>(n, P, scr, 0u, lane);
        }
        unsigned lines = M_RAW;
        if (nb <= 256u) {
            lines = (nb + 31u) >> 5;
            st16(myws, lane * 16, 0, OFF_XOUT + xi * XO_SZ + 4096, (u32x4){P[0], P[1], P[2], P[3]});
            st16(myws, lane * 16, 1, OFF_XOUT + xi * XO_SZ + 4096, (u32x4){P[4], P[5], P[6], P[7]});
            if ((unsigned)lane < lines * 8u) st16(myws, lane * 16, 2, OFF_XOUT + xi * XO_SZ + 4096, *(LAS u32x4*)(size_t)(scr + SC_BLOB + lane * 16));
        } else {
#pragma unroll
            for (int u = 0; u < 8; ++u)
                st16(myws, lane * 16, u & 3, OFF_ROUT + xi * 8192 + (u >> 2) * 4096, (u32x4){q[4 * u], q[4 * u + 1], q[4 * u + 2], q[4 * u + 3]});
        }
        WAIT_VM0();
        unsigned long long tot;
        if (gather((unsigned)(size_t)lds + LDS_MISC + 64 + (wid >> 2) * 16, (unsigned long long)(lines << ((wid & 3) * 4)), tot) == 3u && lane < NR && lane != rank)
            st_sys64(peer + OFF_OFLAG + rank * 2048 + ((ti * 8 + tj) * 2 + (wid >> 2)) * 8, ((unsigned long long)seq << 32) | (unsigned)tot);
    }
    STAMP(5);
    store_rows<0>(ro, lane_off, rowoff + rank * NL + cb * 128, q);
}

__global__ __launch_bounds__(NTHR, 2) __attribute__((amdgpu_waves_per_eu(2, 2))) void fused_kernel(Params p_unused) {
    extern __shared__ __attribute__((aligned(1024))) char smem[];
    const CAS Params& p = *(const CAS Params*)__builtin_amdgcn_kernarg_segment_ptr();
    char* myws = p.self;
    const float* px = p.x;
    const float* pw = p.w;
    int rank = p.rank;
    asm volatile("" : "+s"(myws), "+s"(px), "+s"(pw), "+s"(rank));
    LAS char* lds = (LAS char*)smem;
    STAMP(0);
    const int b = blockIdx.x;
    const int wid = RFL(threadIdx.x >> 6), lane = threadIdx.x & 63;
    const unsigned scr = (unsigned)(size_t)lds + wid * SCR_SZ;
    char* cnt = myws + OFF_CNT + (b * 8 + wid) * 4;
    unsigned seq = 0, seqv;
    asm volatile("global_load_dword %0, %1, %2 sc1" : "=v"(seqv) : "v"(0u), "s"(cnt) : "memory");
    if (b < 16) {
        const char* pc = (const char*)__builtin_amdgcn_s_getpc();
        const unsigned o = threadIdx.x * 128u;
        if (o < CODE_BYTES) { unsigned t = *(const volatile GAS unsigned*)(pc + o); asm volatile("" :: "v"(t)); }
    }
    const unsigned sem = (unsigned)(size_t)lds + LDS_MISC;
    if (threadIdx.x < 128) LDS32(sem + threadIdx.x * 4) = 0u;
    lut_init(scr, lane);
    if (lane == 0) LDS32(scr + SC_LUT + 64) = 0u;
    {
        constexpr int NS = NWG - NGEMM - NPWG, NXL = 438;
        static_assert(2 * NPWG >= 96 && EXP_GL * NGEMM + 4 * NPWG + 8 * NS - 448 >= 768 && 8 * NS >= 448 && 4 * NS >= 224 && 4 * NPWG >= 224, "first tasks");
        const bool gw = b >= NWG - NGEMM, spare = !gw && b >= NPWG;
        const int sp = b - NPWG;
        int e = -1, lag = 0, own = 768;
        if (gw) { if (wid < EXP_GL) own = wid * NGEMM + (b - (NWG - NGEMM)); lag = wid ? TUNED(4, EXP_LG) + wid * TUNED(3, EXP_LW) : 0; }
        else if (spare) {
            const int k = wid * NS + sp;
            if (k < 448) { e = 768 + k; lag = k < 224 ? wid * TUNED(5, EXP_LPUSH) : TUNED(7, EXP_L2) + (wid - 3) * TUNED(2, EXP_L3); }
            else { own = EXP_GL * NGEMM + 4 * NPWG + k - 448; lag = 8; }
        } else if (wid >= 4) { const int pr = wid >= 6 ? b : NPWG + b; if (pr < 96) e = ((pr >> 1) + (pr >> 5) * 8) * 4 + (pr & 1) * 2 + (wid & 1); lag = wid >= 6 ? 0 : EXP_LX; }
        else { own = EXP_GL * NGEMM + (3 - wid) * NPWG + b; lag = TUNED(6, EXP_LS); }
        const bool pair = !spare && e >= 0, late_bar = pair && wid >= 6;
        bool push = spare && e >= 0, stage_only = own < 768;
        if (stage_only) e = own;
        if (!late_bar) RAW_BAR();
        if (e >= 0) for (int i = 0; i < lag; ++i) __builtin_amdgcn_s_sleep(4);
        STAMP(8);
        unsigned grp = sem + 16 + (wid >> 1) * 16;
        int q = spare ? (wid < 4 ? 0 : NS) + sp - 2 * NS : NXL;
        int ib = spare ? (wid & 3) * NS + sp : 224;
#pragma unroll 1
        for (bool first = true;; first = false) {
            if (e >= 0) export_item(p, myws, px, pw, rank, scr, grp, e, stage_only, push, seq, seqv, first, pair, late_bar);
            else if (first) { asm volatile("s_waitcnt vmcnt(0)\n\tv_readfirstlane_b32 %0, %1\n\ts_nop 4" : "=s"(seq) : "v"(seqv) : "memory"); seq += 1u; }
            if (gw) break;
            if (ib < 224 || !spare) {
                if (!spare && first) STAMP(1);
                const bool box = ib < 224;
                puller(p, myws, scr, sem, seq, box ? (ib >> 2) % 7 : b % 7, box ? -1 - ((ib / 28) * 4 + (ib & 3) + (wid >= 4 ? 32 : 0)) : (b / 7) + (7 - wid) * PULL_WG);
                ib = 224;
                if (box && !spare) { e = -1; continue; }
            }
            q += 2 * NS;
            if (q >= NXL) break;
            const int qq = q < 56 ? q + 72 : q + 88, r = qq / 72, u = qq - r * 72;
            e = (r == 7 ? (u < 8 ? u * 24 + 23 : (u < 15 ? 233 : 282) + u) : (u < 56 ? 248 + r * 56 + u : (r + 1) * 24 + (u - 56))) * 4 + (wid & 3);
            push = false; stage_only = false;
            grp = sem + 64 + (q / NS) * 16;
        }
        static_assert((NXL / NS + 1) * 16 + 64 <= 512, "gather cells");
    }
    if (lane == 0) st_agent(cnt, seq);
    if (b >= NPWG) STAMP(1);
    if (b >= NWG - NGEMM) {
        const int gidx = b - (NWG - NGEMM);
#pragma unroll 1
        for (int g = gidx;; g = -1 - gidx) {
            RAW_BAR();
            gemm_role(p, g, seq, myws, lds);
            if (g < 0 || gidx >= 16 * NXT) break;
        }
    } else if (b < NPWG) {
        STAMP(4);
    }
}

}

extern "C" void kernel_launch(int rank, int nranks, ncclComm_t comm,
                              hipStream_t stream,
                              void* const* d_in, const int* in_sizes, int n_in,
                              void* d_out, int out_size,
                              const int* in_dtypes, int out_dtype,
                              void* const* d_out_all,
                              void* const* d_ws_all, size_t ws_bytes) {
    if (nranks != NR || ws_bytes < WS_NEED || (unsigned)rank >= (unsigned)NR) {
        fprintf(stderr, "unexpected configuration\n");
        return;
    }
    HIP_CHECK(hipFuncSetAttribute((const void*)fused_kernel, hipFuncAttributeMaxDynamicSharedMemorySize, LDS_BYTES));
    Params p;
    p.x = (const float*)d_in[0];
    p.w = (const float*)d_in[1];
    for (int i = 0; i < NR; ++i) p.ws[i] = (char*)d_ws_all[i];
    p.out = (char*)d_out;
    p.self = (char*)d_ws_all[rank];
    p.rank = rank;
    p.pad = 0;
    hipLaunchKernelGGL(fused_kernel, dim3(NWG), dim3(NTHR), LDS_BYTES, stream, p);
    HIP_CHECK(hipGetLastError());
}
```

```cpp
#include <hip/hip_runtime.h>
#include <rccl/rccl.h>
#include <cstdio>
#include <cstdint>

#define HIP_CHECK(expr) do { hipError_t _e = (expr); if (_e != hipSuccess) { \
    fprintf(stderr, "HIP error %s at %s:%d\n", hipGetErrorString(_e), __FILE__, __LINE__); \
    return; } } while (0)

namespace {

typedef unsigned u32x4 __attribute__((ext_vector_type(4)));
typedef unsigned u32x2 __attribute__((ext_vector_type(2)));
typedef float f32x4 __attribute__((ext_vector_type(4)));
typedef float f32x2 __attribute__((ext_vector_type(2)));
typedef __bf16 bf16x8 __attribute__((ext_vector_type(8)));
typedef __bf16 bf16x2 __attribute__((ext_vector_type(2)));
typedef unsigned short u16x2 __attribute__((ext_vector_type(2)));
typedef __amdgpu_buffer_rsrc_t rsrc_t;
#define GAS __attribute__((address_space(1)))
#define LAS __attribute__((address_space(3)))
#define CAS __attribute__((address_space(4)))

constexpr int NR = 8;
constexpr int NTOT = 16384;
constexpr int KL = 512;
constexpr int NL = 2048;
constexpr int PIECE = 32768;
constexpr int NDUP = 2;
constexpr int NPUSH = 16;
constexpr int NXT = (7 * NDUP + 1) / 2;
constexpr int NCB = 16 - NDUP;
constexpr int PPK = 24 + NXT;
constexpr int NGEMM = 128;
constexpr int NWG = 256;
constexpr int NTHR = 512;
constexpr int NEXP = 2560;
#define PULL_WG 9
#define PULL_W 8
#define WIN_IN 7
#define WIN_OUT 7
#define PUSH_WIN 4
constexpr unsigned CODE_BYTES = 36u * 1024u;
#define EXP_GL 4
#define EXP_LW 2
#define EXP_LG 4
#define EXP_LP 10
#define EXP_LS 2
#define GPUSH 0
#define GP_L0 0
#define RAW_TAIL 2
#define LATE_AT 24
#define IBX_PULL 0
#define EXP_L2 16
#define EXP_L3 6
#define EXP_LX 0
#define EXP_LPUSH 2
constexpr int NPWG = 7 * PULL_WG;
static_assert(NPWG < NWG - NGEMM, "puller count");
constexpr int NI_MAIN = 768 - 4 * NPUSH, NI_IN = NI_MAIN + 112 * NDUP, NI_ALL = NI_IN + 64 * NCB;
constexpr int XI_SZ = 6144;
constexpr int XO_SZ = 7168;

constexpr unsigned OFF_CNT = 0;
constexpr unsigned OFF_TMO = 8192;
constexpr unsigned OFF_TICK = 12288;
constexpr unsigned OFF_AMAX = 16384;
constexpr unsigned OFF_IFLAG = 24576;
constexpr unsigned OFF_XFLAG = 32768;
constexpr unsigned XFLAG_SZ = NR * 8 * PPK * 16;
constexpr unsigned OFF_OFLAG = 163840;
constexpr unsigned OFF_SFLAG = 262144;
constexpr unsigned OFF_DBG = 512 * 1024;
constexpr unsigned OFF_XIN = 1u << 20;
constexpr unsigned OFF_XOUT = OFF_XIN + 2 * NEXP * XI_SZ;
constexpr unsigned OFF_IBX = OFF_XOUT + 1024 * XO_SZ;
constexpr unsigned OFF_STAGE = OFF_IBX + 512 * XI_SZ;
constexpr unsigned OFF_RIN = OFF_STAGE + NR * 8 * PPK * PIECE;
constexpr unsigned OFF_ROUT = OFF_RIN + 2 * NEXP * 8192;
constexpr unsigned OFF_RIBX = OFF_ROUT + 1024 * 8192;
constexpr unsigned WS_NEED = OFF_RIBX + 512 * 8192;
static_assert(OFF_XFLAG + 2 * XFLAG_SZ <= OFF_OFLAG && OFF_OFLAG + 65536 <= OFF_SFLAG && OFF_SFLAG + NR * 8 * PPK * 128 <= OFF_DBG, "layout");
constexpr unsigned long long DEADLINE = 50000000ull;
constexpr int LDS_MISC = 131072;
constexpr int LDS_BYTES = LDS_MISC + 1024 + 2048;
constexpr int AUX_SYS = 17;
#define OUT_AUX 17
#define OWN_AUX 0
#define AM_SLEEP 8
constexpr unsigned SCR_SZ = 10240;
constexpr unsigned SC_BLOB = 16, SC_A2 = 1152, SC_A3 = 5376, SC_LUT = 9600;
constexpr unsigned M_RAW = 15;

template <int V> struct IC { static constexpr int value = V; };
template <int N, int I = 0, class F> __device__ __forceinline__ void static_for(F&& f) {
    if constexpr (I < N) { f(IC<I>{}); static_for<N, I + 1>(f); }
}

struct Params {
    const float* x;
    const float* w;
    char* ws[NR];
    char* out;
    char* self;
    int rank;
    int pad;
};
#define TUNED(i, dflt) (dflt)

__device__ __forceinline__ unsigned ld_sys(const char* p) {
    return __hip_atomic_load((const GAS unsigned*)p, __ATOMIC_RELAXED, __HIP_MEMORY_SCOPE_SYSTEM);
}
__device__ __forceinline__ unsigned long long ld_sys64(const char* p) {
    return __hip_atomic_load((const GAS unsigned long long*)p, __ATOMIC_RELAXED, __HIP_MEMORY_SCOPE_SYSTEM);
}
__device__ __forceinline__ void st_sys(char* p, unsigned v) {
    __hip_atomic_store((GAS unsigned*)p, v, __ATOMIC_RELAXED, __HIP_MEMORY_SCOPE_SYSTEM);
}
__device__ __forceinline__ void st_sys64(char* p, unsigned long long v) {
    __hip_atomic_store((GAS unsigned long long*)p, v, __ATOMIC_RELAXED, __HIP_MEMORY_SCOPE_SYSTEM);
}
__device__ __forceinline__ unsigned ld_agent(const char* p) {
    return __hip_atomic_load((const GAS unsigned*)p, __ATOMIC_RELAXED, __HIP_MEMORY_SCOPE_AGENT);
}
__device__ __forceinline__ void st_agent(char* p, unsigned v) {
    __hip_atomic_store((GAS unsigned*)p, v, __ATOMIC_RELAXED, __HIP_MEMORY_SCOPE_AGENT);
}
__device__ __forceinline__ rsrc_t mk_rsrc(const void* p, unsigned bytes) {
    return __builtin_amdgcn_make_buffer_rsrc((void*)p, (short)0, (int)bytes, 0x00020000);
}
__device__ __forceinline__ u32x4 bld16(rsrc_t r, unsigned voff, unsigned soff) {
    return __builtin_amdgcn_raw_buffer_load_b128(r, (int)voff, (int)soff, AUX_SYS);
}
#define STB "sc0 sc1"
__device__ __forceinline__ void st16(const char* base, unsigned voff, int blk, unsigned soff, u32x4 v) {
    const char* b = base + soff;
    if (blk == 0) asm volatile("s_nop 4\n\tglobal_store_dwordx4 %0, %1, %2 " STB "\n\ts_nop 1" :: "v"(voff), "v"(v), "s"(b) : "memory");
    if (blk == 1) asm volatile("s_nop 4\n\tglobal_store_dwordx4 %0, %1, %2 offset:1024 " STB "\n\ts_nop 1" :: "v"(voff), "v"(v), "s"(b) : "memory");
    if (blk == 2) asm volatile("s_nop 4\n\tglobal_store_dwordx4 %0, %1, %2 offset:2048 " STB "\n\ts_nop 1" :: "v"(voff), "v"(v), "s"(b) : "memory");
    if (blk == 3) asm volatile("s_nop 4\n\tglobal_store_dwordx4 %0, %1, %2 offset:3072 " STB "\n\ts_nop 1" :: "v"(voff), "v"(v), "s"(b) : "memory");
}
#define WAIT_VM0() do { asm volatile("s_waitcnt vmcnt(0)" ::: "memory"); __builtin_amdgcn_s_waitcnt(0x0F70); } while (0)
#define WAIT_LGKM0() asm volatile("s_waitcnt lgkmcnt(0)" ::: "memory")
#define RAW_BAR() do { asm volatile("s_waitcnt lgkmcnt(0)" ::: "memory"); __builtin_amdgcn_s_barrier(); asm volatile("" ::: "memory"); } while (0)
#define SCHED0() __builtin_amdgcn_sched_barrier(0)
#define PERM(a, b, s) __builtin_amdgcn_perm((a), (b), (s))
#define RFL(x) ((unsigned)__builtin_amdgcn_readfirstlane((int)(x)))

#define GLB "sc0 sc1"
template <int OFF> __device__ __forceinline__ void glds_if(unsigned have, unsigned want, const char* src, unsigned loff, unsigned ldst) {
    asm volatile(
        "s_cmp_lt_u32 %0, %5\n\t"
        "s_cbranch_scc1 1f\n\t"
        "s_mov_b32 m0, %3\n\t"
        "s_nop 0\n\t"
        "global_load_lds_dwordx4 %1, %2 offset:%4 " GLB "\n"
        "1:"
        :: "s"(have), "v"(loff), "s"(src), "s"(ldst), "n"(OFF), "s"(want) : "memory", "scc");
}
__device__ __forceinline__ void stage8k(const char* src, unsigned loff, unsigned loff2, unsigned ldst) {
    asm volatile(
        "s_mov_b32 m0, %3\n\t"
        "s_nop 3\n\t"
        "global_load_lds_dwordx4 %0, %2 " GLB "\n\t"
        "global_load_lds_dwordx4 %0, %2 offset:1024 " GLB "\n\t"
        "global_load_lds_dwordx4 %0, %2 offset:2048 " GLB "\n\t"
        "global_load_lds_dwordx4 %0, %2 offset:3072 " GLB "\n\t"
        "s_add_u32 m0, m0, 0x1000\n\t"
        "s_nop 0\n\t"
        "global_load_lds_dwordx4 %1, %2 " GLB "\n\t"
        "global_load_lds_dwordx4 %1, %2 offset:1024 " GLB "\n\t"
        "global_load_lds_dwordx4 %1, %2 offset:2048 " GLB "\n\t"
        "global_load_lds_dwordx4 %1, %2 offset:3072 " GLB
        :: "v"(loff), "v"(loff2), "s"(src), "s"(ldst) : "memory", "scc");
}

__device__ __forceinline__ unsigned pk_bf16(float a, float b) {
    f32x2 v = {a, b};
    bf16x2 r = __builtin_convertvector(v, bf16x2);
    return __builtin_bit_cast(unsigned, r);
}

#define STAMP(i) do {} while (0)

__device__ __forceinline__ bool give_up(char* tmo, unsigned seq, unsigned long long t0) {
    if (RFL(ld_agent(tmo)) == seq || __builtin_amdgcn_s_memrealtime() - t0 > DEADLINE) {
        st_agent(tmo, seq);
        return true;
    }
    return false;
}
__device__ __forceinline__ void spin_flag(const char* flag, unsigned seq, char* tmo) {
    unsigned long long t0 = __builtin_amdgcn_s_memrealtime();
    for (unsigned n = 1; RFL(ld_sys(flag)) < seq; ++n) {
        if ((n & 31u) == 0u && give_up(tmo, seq, t0)) break;
        __builtin_amdgcn_s_sleep(1);
    }
    asm volatile("" ::: "memory");
}

typedef unsigned u32x2a __attribute__((ext_vector_type(2), aligned(4)));
#define LDS32(a) (*(LAS unsigned*)(size_t)(a))
#define CFENCE() asm volatile("" ::: "memory")

__device__ __forceinline__ unsigned wave_scan(unsigned x) {
    x += (unsigned)__builtin_amdgcn_update_dpp(0, (int)x, 0x111, 0xf, 0xf, false);
    x += (unsigned)__builtin_amdgcn_update_dpp(0, (int)x, 0x112, 0xf, 0xf, false);
    x += (unsigned)__builtin_amdgcn_update_dpp(0, (int)x, 0x114, 0xf, 0xf, false);
    x += (unsigned)__builtin_amdgcn_update_dpp(0, (int)x, 0x118, 0xf, 0xf, false);
    x += (unsigned)__builtin_amdgcn_update_dpp(0, (int)x, 0x142, 0xa, 0xf, false);
    x += (unsigned)__builtin_amdgcn_update_dpp(0, (int)x, 0x143, 0xc, 0xf, false);
    return x;
}
__device__ __forceinline__ unsigned esc_bits(unsigned P) { return __builtin_amdgcn_bitop3_b32(P, P >> 1, 0x55555555u, 0x80); }

__device__ __forceinline__ void lut_init(unsigned scr, int lane) {
    if (lane < 16) {
        unsigned sel = 0x0c0c0c0cu; int k = 0;
#pragma unroll
        for (int y = 0; y < 4; ++y) if ((lane >> y) & 1) { sel = (sel & ~(0xffu << (8 * k))) | ((unsigned)y << (8 * k)); ++k; }
        LDS32(scr + SC_LUT + lane * 4) = sel;
    }
}
#define BFI(m, x, y) __builtin_amdgcn_bitop3_b32((m), (x), (y), 0xCA)
__device__ __forceinline__ unsigned opaque(unsigned x) { asm("" : "+v"(x)); return x; }
__device__ __forceinline__ void unary_init(unsigned tab, int t) {
    if (t < 256) {
        unsigned long long v = 0; unsigned k = 0, gap = 3;
#pragma unroll
        for (int y = 0; y < 8; ++y) { if ((t >> y) & 1) { v |= (unsigned long long)(gap << 3) << k; k += 8; gap = 3; } else ++gap; }
        *(LAS unsigned long long*)(size_t)(tab + t * 8) = v;
    }
}
__device__ __forceinline__ void zero_scratch(unsigned scr, int lane) {
#pragma unroll
    for (int i = 0; i < 9; ++i) *(LAS u32x4*)(size_t)(scr + i * 1024 + lane * 16) = (u32x4){0u, 0u, 0u, 0u};
    if (lane < 24) *(LAS u32x4*)(size_t)(scr + 9216 + lane * 16) = (u32x4){0u, 0u, 0u, 0u};
}
__device__ __forceinline__ unsigned digits1(unsigned n02, unsigned n13) {
    const unsigned lo = BFI(0x33333333u, n02, n13 << 2), hi = BFI(0x33333333u, n02 >> 2, n13);
    const unsigned h = (hi | (hi >> 1)) & 0x55555555u;
    return lo | h | (h << 1);
}
__device__ __forceinline__ unsigned sel_addr(unsigned E, int t, unsigned scr) {
    if (t == 0) return __builtin_amdgcn_udot4(E & 0x01010101u, 0x20100804u, scr + SC_LUT, false);
    if (t == 1) return __builtin_amdgcn_udot4(E & 0x04040404u, 0x08040201u, scr + SC_LUT, false);
    if (t == 2) return __builtin_amdgcn_udot4((E >> 4) & 0x01010101u, 0x20100804u, scr + SC_LUT, false);
    return __builtin_amdgcn_udot4((E >> 4) & 0x04040404u, 0x08040201u, scr + SC_LUT, false);
}
template <int NP, bool IN> __device__ __forceinline__ unsigned encode(const unsigned (&n)[2 * NP], unsigned (&P)[NP], unsigned scr, unsigned nb0, int lane) {
    unsigned E[NP], cnt = 0;
#pragma unroll
    for (int j = 0; j < NP; ++j) { P[j] = digits1(n[2 * j], n[2 * j + 1]); E[j] = esc_bits(P[j]); cnt += __popc(E[j]); }
    unsigned sel[2][8];
#pragma unroll
    for (int k = 0; k < 8; ++k) sel[0][k] = LDS32(sel_addr(E[k >> 2], k & 3, scr));
    cnt = max(cnt, 4u);
    const unsigned inc = wave_scan(cnt);
    const unsigned n2d = ((unsigned)__builtin_amdgcn_readlane((int)inc, 63) + 15u) >> 4;
    if (nb0 + n2d > 256u) return 1000u;
    unsigned pos = scr + ((lane & 1) ? SC_A3 : SC_A2) + inc - cnt, carry = 0;
    static_for<NP / 2>([&](auto gc) __attribute__((always_inline)) {
        constexpr int g = decltype(gc)::value;
        SCHED0();
        if constexpr (g + 1 < NP / 2) {
#pragma unroll
            for (int k = 0; k < 8; ++k) sel[(g + 1) & 1][k] = LDS32(sel_addr(E[2 * g + 2 + (k >> 2)], k & 3, scr));
        }
        SCHED0();
#pragma unroll
        for (int k = 0; k < 8; ++k) {
            const int j = 2 * g + (k >> 2), t = k & 3;
            const unsigned z = t < 2 ? n[2 * j + t] : n[2 * j + t - 2] >> 4;
            const unsigned long long v = (unsigned long long)PERM(0u, z, sel[g & 1][k]) << ((pos & 3u) * 8u);
            const unsigned lo = (unsigned)v | carry;
            LDS32(pos & ~3u) = lo;
            const unsigned np = pos + __popc(E[j] & (0x01010101u << (2 * t)));
            carry = ((np ^ pos) & 4u) ? (unsigned)(v >> 32) : lo;
            pos = np;
        }
    });
    LDS32(pos & ~3u) = carry;
    CFENCE();
    unsigned n3 = 0;
    if constexpr (!IN) {
        for (unsigned i0 = 0; i0 < n2d; i0 += 64) {
            const unsigned i = i0 + lane;
            const u32x4 v = *(LAS u32x4*)(size_t)(scr + SC_A2 + i * 16) | *(LAS u32x4*)(size_t)(scr + SC_A3 + i * 16);
            unsigned long long acc = 0;
            unsigned c = 0, mx = 0;
#pragma unroll
            for (int t = 0; t < 4; ++t) {
                const unsigned y = (v[t] & 0x0f0f0f0fu) + 0x0d0d0d0du, m = y & 0x10101010u;
                unsigned l = y & opaque(m - (m >> 4));
                l = opaque((l << 8) + l);
                l = opaque((l << 16) + l) + 0x03020100u;
                const unsigned bits = (1u << (l & 31u)) | (1u << ((l >> 8) & 31u)) | (1u << ((l >> 16) & 31u)) | (1u << ((l >> 24) & 31u));
                acc |= (unsigned long long)bits << (c & 63u);
                c += (l >> 24) + 1u;
                mx |= l;
            }
            c = i < n2d ? c : 0u;
            const unsigned ic = wave_scan(c);
            const unsigned tot = (unsigned)__builtin_amdgcn_readlane((int)ic, 63);
            if (n3 + tot > 8192u || __any(c > 64u || mx >= 0x20000000u)) return 1000u;
            if (i < n2d) {
                const unsigned p3 = n3 + ic - c, sh = p3 & 31u;
                LAS unsigned* d = (LAS unsigned*)(size_t)(scr + SC_BLOB + (p3 >> 5) * 4);
                const unsigned long long w = acc << sh;
                __hip_atomic_fetch_or(d, (unsigned)w, __ATOMIC_RELAXED, __HIP_MEMORY_SCOPE_WAVEFRONT);
                __hip_atomic_fetch_or(d + 1, (unsigned)(w >> 32), __ATOMIC_RELAXED, __HIP_MEMORY_SCOPE_WAVEFRONT);
                if (c + sh > 64u) __hip_atomic_fetch_or(d + 2, (unsigned)(((acc >> 32) << sh) >> 32), __ATOMIC_RELAXED, __HIP_MEMORY_SCOPE_WAVEFRONT);
            }
            n3 += tot;
        }
        CFENCE();
        return (n3 + 31u) >> 5;
    }
    const unsigned s3 = scr + SC_BLOB + (nb0 + n2d) * 4;
    for (unsigned i0 = 0; i0 < n2d; i0 += 64) {
        const unsigned i = i0 + lane;
        const u32x4 v = (*(LAS u32x4*)(size_t)(scr + SC_A2 + i * 16) | *(LAS u32x4*)(size_t)(scr + SC_A3 + i * 16)) & 0x0f0f0f0fu;
        unsigned x = 0;
#pragma unroll
        for (int t = 0; t < 4; ++t) {
            const unsigned z1 = v[t] + 0x01010101u;
            unsigned d = z1, e4;
            if (IN) { e4 = (((v[t] + 0x0b0b0b0bu) ^ z1) >> 4) & 0x01010101u; d |= z1 >> 3; }
            else e4 = ((v[t] + 0x0a0a0a0au) >> 4) & 0x01010101u;
            x |= ((d | e4 | (e4 << 1)) & 0x03030303u) << (2 * t);
        }
        const unsigned E2 = esc_bits(x);
        unsigned sl[4];
#pragma unroll
        for (int t = 0; t < 4; ++t) sl[t] = LDS32(sel_addr(E2, t, scr));
        const unsigned c = __popc(E2), ic = wave_scan(c);
        const unsigned tot = (unsigned)__builtin_amdgcn_readlane((int)ic, 63);
        if (nb0 + n2d + ((n3 + tot + 7u) >> 3) > 256u) return 1000u;
        if (i < n2d) LDS32(scr + SC_BLOB + (nb0 + i) * 4) = x;
        unsigned h[4];
#pragma unroll
        for (int t = 0; t < 4; ++t) { const unsigned y = PERM(0u, v[t], sl[t]); h[t] = PERM(0u, y | (y >> 4), 0x0c0c0200u); }
        const unsigned c0 = __popc(E2 & 0x01010101u), c01 = __popc(E2 & 0x05050505u), c2 = __popc(E2 & 0x10101010u);
        const unsigned long long acc = (unsigned long long)(h[0] | (h[1] << (4u * c0))) | ((unsigned long long)(h[2] | (h[3] << (4u * c2))) << (4u * c01));
        const unsigned p3 = n3 + ic - c, sh = (p3 & 7u) * 4u;
        if (c) {
            LAS unsigned* d = (LAS unsigned*)(size_t)(s3 + (p3 >> 3) * 4);
            const unsigned long long w = acc << sh;
            __hip_atomic_fetch_or(d, (unsigned)w, __ATOMIC_RELAXED, __HIP_MEMORY_SCOPE_WAVEFRONT);
            if (4u * c + sh > 32u) {
                __hip_atomic_fetch_or(d + 1, (unsigned)(w >> 32), __ATOMIC_RELAXED, __HIP_MEMORY_SCOPE_WAVEFRONT);
                __hip_atomic_fetch_or(d + 2, (unsigned)(((acc >> 32) << sh) >> 32), __ATOMIC_RELAXED, __HIP_MEMORY_SCOPE_WAVEFRONT);
            }
        }
        n3 += tot;
    }
    CFENCE();
    return nb0 + n2d + ((n3 + 7u) >> 3);
}
__device__ __forceinline__ unsigned rot8(unsigned e) { return ((e >> 1) & 0x7f7f7f7fu) | ((e << 7) & 0x80808080u); }
__device__ __forceinline__ void selectors(unsigned P, unsigned E, unsigned (&sel)[4]) {
    const unsigned pf = (opaque((opaque((E << 8) + E) << 8) + E)) << 8;
    const unsigned S = BFI(E | (E << 1), pf, P);
    const unsigned A = BFI(0x33333333u, S, E << 2), B = BFI(0x33333333u, S >> 2, E);
    sel[0] = A & 0x07070707u; sel[1] = B & 0x07070707u; sel[2] = (A >> 4) & 0x07070707u; sel[3] = (B >> 4) & 0x07070707u;
}
template <int NP, bool IN> __device__ __forceinline__ void decode(const unsigned (&P)[NP], unsigned (&o)[4 * NP], unsigned scr, unsigned nb0, unsigned nbmax, unsigned T, int lane) {
    const unsigned lut1 = IN ? rot8(T | ((T - 1u) << 8) | ((T - 2u) << 16)) : 0x00100800u;
    const unsigned lut2 = IN ? rot8((T - 3u) | ((T - 4u) << 8) | ((T + 1u) << 16)) : 0x00282018u;
    unsigned E[NP], cnt = 0;
#pragma unroll
    for (int j = 0; j < NP; ++j) { E[j] = esc_bits(P[j]); cnt += __popc(E[j]); }
    cnt = max(cnt, 4u);
    const unsigned inc = wave_scan(cnt);
    const unsigned n2d = min(((unsigned)__builtin_amdgcn_readlane((int)inc, 63) + 15u) >> 4, 256u);
    if constexpr (!IN) {
        for (unsigned i = lane; i <= n2d; i += 64) *(LAS u32x4*)(size_t)(scr + SC_A2 + i * 16) = (u32x4){0u, 0u, 0u, 0u};
        unsigned run = 0;
        for (unsigned i0 = 0; i0 < nbmax; i0 += 64) {
            const unsigned i = i0 + lane;
            const u32x2a ww = *(LAS u32x2a*)(size_t)(scr + SC_BLOB + i * 4 - 4);
            const unsigned W = ww.y, Wp = i ? ww.x : 0x80000000u;
            u32x2 tb[4];
#pragma unroll
            for (int y = 0; y < 4; ++y) tb[y] = *(LAS u32x2*)(size_t)(T + ((W >> (8 * y)) & 0xffu) * 8);
            const unsigned c = __popc(W), ic = wave_scan(c);
            unsigned pos = scr + SC_A2 + min(run + ic - c, 4096u);
            run += (unsigned)__builtin_amdgcn_readlane((int)ic, 63);
#pragma unroll
            for (int y = 0; y < 4; ++y) {
                const unsigned zr = (unsigned)__clz((int)(y ? __builtin_amdgcn_alignbit(W, Wp, 8 * y) : Wp)) & 15u;
                const unsigned lo = tb[y].x ? tb[y].x + (zr << 3) : 0u, sh = (pos & 3u) * 8u;
                const unsigned long long a = (((unsigned long long)tb[y].y << 32) | lo) << sh;
                LAS unsigned* d = (LAS unsigned*)(size_t)(pos & ~3u);
                if (W) {
                    __hip_atomic_fetch_or(d, (unsigned)a, __ATOMIC_RELAXED, __HIP_MEMORY_SCOPE_WAVEFRONT);
                    __hip_atomic_fetch_or(d + 1, (unsigned)(a >> 32), __ATOMIC_RELAXED, __HIP_MEMORY_SCOPE_WAVEFRONT);
                    __hip_atomic_fetch_or(d + 2, (unsigned)(((unsigned long long)tb[y].y << sh) >> 32), __ATOMIC_RELAXED, __HIP_MEMORY_SCOPE_WAVEFRONT);
                }
                pos += __popc(W & (0xffu << (8 * y)));
            }
        }
    } else {
    for (unsigned i = lane; nb0 + n2d + i < nbmax; i += 64) {
        const unsigned W = LDS32(scr + SC_BLOB + (nb0 + n2d + i) * 4);
        unsigned a, b;
        if (IN) {
            const unsigned K = (T + 2u) * 0x01010101u;
            a = rot8(K - (((W & 0x0f0f0f0fu) + 0x02020202u) & 0x0f0f0f0fu));
            b = rot8(K - ((((W >> 4) & 0x0f0f0f0fu) + 0x02020202u) & 0x0f0f0f0fu));
        } else {
            a = (W << 3) & 0x78787878u;
            b = (W >> 1) & 0x78787878u;
        }
        *(LAS u32x2*)(size_t)(scr + SC_A3 + i * 8) = (u32x2){PERM(b, a, 0x05010400u), PERM(b, a, 0x07030602u)};
    }
    CFENCE();
    unsigned n3 = 0;
    for (unsigned i0 = 0; i0 < n2d; i0 += 64) {
        const unsigned i = i0 + lane;
        unsigned W = LDS32(scr + SC_BLOB + ((nb0 + i) & 255u) * 4);
        W = i < n2d ? W : 0u;
        const unsigned E2 = esc_bits(W), c = __popc(E2), ic = wave_scan(c);
        unsigned ps[4];
        u32x2a w[4];
        ps[0] = scr + SC_A3 + min(n3 + ic - c, 2048u);
#pragma unroll
        for (int t = 0; t < 4; ++t) {
            w[t] = *(LAS u32x2a*)(size_t)(ps[t] & ~3u);
            if (t < 3) ps[t + 1] = ps[t] + __popc(E2 & (0x01010101u << (2 * t)));
        }
        SCHED0();
        n3 += (unsigned)__builtin_amdgcn_readlane((int)ic, 63);
        unsigned sel[4];
        selectors(W, E2, sel);
        SCHED0();
        u32x4 v;
#pragma unroll
        for (int t = 0; t < 4; ++t) v[t] = PERM(__builtin_amdgcn_alignbyte(w[t].y, w[t].x, ps[t]), lut2, sel[t]);
        *(LAS u32x4*)(size_t)(scr + SC_A2 + i * 16) = v;
    }
    }
    CFENCE();
    unsigned pos = scr + SC_A2 + min(inc - cnt, 4096u);
    unsigned ps[2][8];
    u32x2a w[2][8];
    static_for<NP / 2 + 1>([&](auto gc) __attribute__((always_inline)) {
        constexpr int g = decltype(gc)::value;
        SCHED0();
        if constexpr (g < NP / 2) {
#pragma unroll
            for (int k = 0; k < 8; ++k) {
                ps[g & 1][k] = pos;
                w[g & 1][k] = *(LAS u32x2a*)(size_t)(pos & ~3u);
                pos += __popc(E[2 * g + (k >> 2)] & (0x01010101u << (2 * (k & 3))));
            }
        }
        SCHED0();
        if constexpr (g > 0) {
#pragma unroll
            for (int h = 0; h < 2; ++h) {
                const int j = 2 * (g - 1) + h;
                unsigned sel[4];
                selectors(P[j], E[j], sel);
#pragma unroll
                for (int t = 0; t < 4; ++t) {
                    const u32x2a x = w[(g - 1) & 1][4 * h + t];
                    o[4 * j + t] = PERM(__builtin_amdgcn_alignbyte(x.y, x.x, ps[(g - 1) & 1][4 * h + t]), lut1, sel[t]);
                }
            }
        }
    });
    CFENCE();
}

__device__ __forceinline__ unsigned gather(unsigned cell, unsigned long long v, unsigned long long& tot) {
    unsigned n;
    const unsigned one = 1u;
    asm volatile("s_mov_b64 exec, 1\n\tds_or_b64 %2, %3\n\tds_add_rtn_u32 %0, %2, %4 offset:8\n\ts_mov_b64 exec, -1\n\t"
                 "ds_read_b64 %1, %2\n\ts_waitcnt lgkmcnt(0)" : "=&v"(n), "=&v"(tot) : "v"(cell), "v"(v), "v"(one) : "memory");
    return RFL(n);
}

__device__ __forceinline__ void exp_decode(int e, int rank, int& kt, int& slot, int& sub, int& dst) {
    sub = e & 3;
    const int q = e >> 2;
    if (q < 192) {
        kt = q / 24; slot = q - kt * 24; dst = slot < 16 ? -1 : rank;
        return;
    }
    const int t = q - 192;
    kt = t / 56;
    const int u = t - kt * 56, j = u / 7;
    slot = 16 + j;
    dst = (rank + 1 + (u - j * 7)) & 7;
}
__device__ __forceinline__ unsigned xin_item(int kt, int slot, int sub, int dst) {
    return (slot < 16 ? kt * 16 + slot : 128 + (dst * 8 + kt) * 8 + (slot - 16)) * 4 + sub;
}

#define LDW(p) __builtin_nontemporal_load(p)
#define LDX(p) __builtin_nontemporal_load(p)
__device__ __forceinline__ void export_item(const CAS Params& p, char* myws, const float* px, const float* pw0, int rank, unsigned scr, unsigned grp, int e, bool stage_only, bool push, unsigned& seq, unsigned seqv, bool first, bool pair, bool bar) {
    const int lane = threadIdx.x & 63;
    int kt, slot, sub, dst;
    exp_decode(e, rank, kt, slot, sub, dst);
    const int kq = lane >> 4, c = lane & 15;
    u32x4 ch[8];
    if (slot < 16) {
        const GAS float* b = (const GAS float*)px + (size_t)(256 * slot + 64 * sub + (lane >> 3)) * KL + 64 * kt;
        const int g0 = 4 * ((lane & 7) ^ (lane >> 4));
        f32x4 v[16];
#pragma unroll
        for (int i = 0; i < 8; ++i) {
            v[2 * i] = LDX((const GAS f32x4*)(b + i * 8 * KL + (g0 ^ ((i & 1) * 16))));
            v[2 * i + 1] = LDX((const GAS f32x4*)(b + i * 8 * KL + (g0 ^ ((i & 1) * 16)) + 32));
        }
        if (!stage_only) zero_scratch(scr, lane);
#pragma unroll
        for (int i = 0; i < 8; ++i) {
            ch[i].x = pk_bf16(v[2 * i][0], v[2 * i][1]); ch[i].y = pk_bf16(v[2 * i][2], v[2 * i][3]);
            ch[i].z = pk_bf16(v[2 * i + 1][0], v[2 * i + 1][1]); ch[i].w = pk_bf16(v[2 * i + 1][2], v[2 * i + 1][3]);
        }
    } else {
        const GAS float* b = (const GAS float*)pw0 + (size_t)(64 * kt + 16 * (sub & 1) + 4 * kq) * NTOT + dst * NL + 256 * (slot - 16) + 128 * (sub >> 1) + 4 * (c ^ (kq & 1));
        f32x4 v[16];
#pragma unroll
        for (int jj = 0; jj < 8; ++jj) {
            v[2 * jj] = LDW((const GAS f32x4*)(b + (size_t)((jj & 3) + (jj >> 2) * 32) * NTOT));
            v[2 * jj + 1] = LDW((const GAS f32x4*)(b + (size_t)((jj & 3) + (jj >> 2) * 32) * NTOT + 64));
        }
        if (!stage_only) zero_scratch(scr, lane);
#pragma unroll
        for (int i = 0; i < 8; ++i) {
            ch[i].x = pk_bf16(v[0 + (i >> 2)][i & 3], v[2 + (i >> 2)][i & 3]); ch[i].y = pk_bf16(v[4 + (i >> 2)][i & 3], v[6 + (i >> 2)][i & 3]);
            ch[i].z = pk_bf16(v[8 + (i >> 2)][i & 3], v[10 + (i >> 2)][i & 3]); ch[i].w = pk_bf16(v[12 + (i >> 2)][i & 3], v[14 + (i >> 2)][i & 3]);
        }
    }
    if (first) { asm volatile("s_waitcnt vmcnt(0)\n\tv_readfirstlane_b32 %0, %1\n\ts_nop 4" : "=s"(seq) : "v"(seqv) : "memory"); seq += 1u; }
    if (bar) RAW_BAR();
    const bool dup = slot == 23 && sub >= 4 - 2 * NDUP;
    const bool local = stage_only || (dup && dst != rank && !push);
    const int d = lane & 7;
    const unsigned at = (rank * 8 + kt) * (PPK * 4) + ((dup && d != dst) ? 96 + ((dst - d - 1) & 7) * (2 * NDUP) + sub - (4 - 2 * NDUP) : slot * 4 + sub);
    if (local) {
        const unsigned mine = (unsigned)__builtin_amdgcn_readlane((int)at, rank);
#pragma unroll
        for (int u = 0; u < 8; ++u) st16(myws, lane * 16, u & 3, OFF_STAGE + mine * 8192 + (u >> 2) * 4096, ch[u]);
        if (stage_only) {
            WAIT_VM0();
            if (lane == rank) st_agent(myws + OFF_SFLAG + at * 32, seq);
            return;
        }
    }
    int l7 = lane & 7;
    asm volatile("" : "+v"(l7));
    char* peer = p.ws[l7];
    unsigned meta = 0;
    const unsigned xi = xin_item(kt, slot, sub, dst) + (seq & 1u) * NEXP, ibx = rank * 64 + kt * 32 + (slot - 16) * 4 + sub;
    {
        const char* xb = push ? p.ws[dst] : myws;
        const unsigned xo = push ? OFF_IBX + ibx * XI_SZ : OFF_XIN + xi * XI_SZ;
        unsigned ex[16], sum = 0;
#pragma unroll
        for (int u = 0; u < 4; ++u) {
            u32x4 r;
#pragma unroll
            for (int k = 0; k < 4; ++k) {
                const unsigned d0 = ch[2 * u + (k >> 1)][(k & 1) * 2], d1 = ch[2 * u + (k >> 1)][(k & 1) * 2 + 1];
                const unsigned L = PERM(d1, d0, 0x06040200u), H = PERM(d1, d0, 0x07050301u);
                r[k] = (H & 0x80808080u) | (L & 0x7f7f7f7fu);
                ex[4 * u + k] = ((H << 1) & 0xfefefefeu) | ((L >> 7) & 0x01010101u);
                sum = __builtin_amdgcn_sad_u8(ex[4 * u + k], 0u, sum);
            }
            st16(xb, lane * 16, u, xo, r);
        }
        unsigned T = ((unsigned)__builtin_amdgcn_readlane((int)wave_scan(sum), 63) + 7864u) >> 12;
        T = min(max(T, 13u), 253u);
        const unsigned K = (T + 2u) * 0x01010101u;
        unsigned z[16], yy[16], bad = 0;
#pragma unroll
        for (int d = 0; d < 16; ++d) {
            const unsigned y = ((K | 0x80808080u) - (ex[d] & 0x7f7f7f7fu)) ^ (~(K ^ ex[d]) & 0x80808080u);
            bad |= y; yy[d] = y;
            z[d] = ((y & 0x0f0f0f0fu) + 0x0e0e0e0eu) & 0x0f0f0f0fu;
        }
        unsigned nex = 0;
        if (__any((bad & 0xf0f0f0f0u) != 0u)) {
            CFENCE();
            unsigned mk = 0;
#pragma unroll
            for (int d = 0; d < 16; ++d) mk |= (yy[d] & 0xf0f0f0f0u) ? 1u << d : 0u;
            while (mk) {
                const unsigned d = __builtin_ctz(mk);
                mk &= mk - 1u;
                unsigned ev = ex[0];
#pragma unroll
                for (int t = 1; t < 16; ++t) ev = d == (unsigned)t ? ex[t] : ev;
#pragma unroll 1
                for (unsigned y = 0; y < 4; ++y) {
                    const unsigned eb = (ev >> (8 * y)) & 0xffu;
                    if (T + 2u - eb > 15u) {
                        const unsigned k = __hip_atomic_fetch_add((LAS unsigned*)(size_t)(scr + SC_LUT + 64), 1u, __ATOMIC_RELAXED, __HIP_MEMORY_SCOPE_WAVEFRONT);
                        if (k < 15u) LDS32(scr + SC_BLOB + k * 4) = eb | (y << 8) | (d << 10) | (lane << 14);
                    }
                }
            }
            CFENCE();
            nex = RFL(LDS32(scr + SC_LUT + 64));
            CFENCE();
            LDS32(scr + SC_LUT + 64) = 0u;
        }
        if (first) STAMP(10);
        unsigned P[4], nb = 1000u;
        if (nex <= 15u) {
            unsigned n[8];
#pragma unroll
            for (int k = 0; k < 8; ++k) n[k] = z[(k >> 1) * 4 + (k & 1)] | (z[(k >> 1) * 4 + (k & 1) + 2] << 4);
            nb = encode<4, true>(n, P, scr, nex, lane);
        }
        if (nb <= 256u) {
            const unsigned lines = (nb + 31u) >> 5;
            st16(xb, lane * 16, 0, xo + 4096, (u32x4){P[0], P[1], P[2], P[3]});
            if ((unsigned)lane < lines * 8u) st16(xb, lane * 16, 1, xo + 4096, *(LAS u32x4*)(size_t)(scr + SC_BLOB + lane * 16));
            meta = T | (lines << 8) | (nex << 12);
        } else {
#pragma unroll
            for (int u = 0; u < 8; ++u) st16(xb, lane * 16, u & 3, (push ? OFF_RIBX + ibx * 8192 : OFF_RIN + xi * 8192) + (u >> 2) * 4096, ch[u]);
            meta = M_RAW << 8;
        }
    }
    if (first) STAMP(11);
    WAIT_VM0();
    if (first) STAMP(12);
    if (local && lane == rank) st_agent(myws + OFF_SFLAG + at * 32, seq);
    if (push) {
        if (lane == dst) st_sys64(peer + OFF_IFLAG + ibx * 8, ((unsigned long long)seq << 32) | meta);
        return;
    }
    unsigned long long tot;
    if (gather(grp, (unsigned long long)meta << (16 * sub), tot) == (pair ? 1u : 3u) && lane < NR && lane != rank && (dst < 0 || dup || lane == dst)) {
        char* fp = peer + OFF_XFLAG + (seq & 1u) * XFLAG_SZ + (at >> 2) * 16;
        if (pair) {
            st_sys64(fp + (sub >> 1) * 8, ((unsigned long long)seq << 32) | (unsigned)tot | (unsigned)(tot >> 32));
        } else {
            const u32x4 g = {(unsigned)tot, seq, (unsigned)(tot >> 32), seq};
            asm volatile("global_store_dwordx4 %0, %1, off sc0 sc1\n\ts_nop 1" :: "v"(fp), "v"(g) : "memory");
        }
    }
}

__device__ __forceinline__ unsigned tick_draw(char* ctr, bool off) {
    return (unsigned)__builtin_amdgcn_raw_ptr_buffer_atomic_add_i32(1, mk_rsrc(ctr, 4), ((threadIdx.x & 63) || off) ? 64 : 0, 0, 16);
}

__device__ __forceinline__ void item_decode(int it, int rank, int delta, unsigned par, unsigned& xi, unsigned& at) {
    if (it >= NI_IN) {
        const int o = it - NI_IN, r = o / NCB;
        xi = at = (((r & 31) >> 1) * 4 + (r & 1) + 2 * (r >> 5)) * 16 + (o - r * NCB);
        return;
    }
    const int src = (rank + delta) & 7;
    if (it >= NI_MAIN) {
        const int o = it - NI_MAIN, kt = o / (14 * NDUP), r = o - kt * (14 * NDUP), u = r / (2 * NDUP);
        xi = xin_item(kt, 23, 4 - 2 * NDUP + (r & (2 * NDUP - 1)), (rank + 1 + u) & 7) + par;
        at = (src * 8 + kt) * (PPK * 4) + 96 + r;
        return;
    }
    if (it < 0) {
        const int k = -1 - it;
        xi = src * 64 + k;
        at = (src * 8 + (k >> 5)) * (PPK * 4) + 64 + (k & 31);
        return;
    }
    const int q = (it >> 2) + 8;
    int kt, slot;
    if (q < 56 || q >= 168) { kt = q < 56 ? (q - 8) >> 4 : 7; slot = (q - 8 + 2 * delta) & 15; }
    else if (q < 72) { kt = q < 64 ? 2 : 3; slot = 16 + (q & 7); }
    else {
        const int r = q / 24, e = q - r * 24, t3 = e / 3, m3 = e - t3 * 3;
        if (m3 < 2) { kt = r; slot = (2 * t3 + m3 + 2 * delta) & 15; }
        else { kt = r + 1; slot = 16 + t3; }
    }
    xi = xin_item(kt, slot, it & 3, rank) + par;
    at = (src * 8 + kt) * (PPK * 4) + slot * 4 + (it & 3);
}

template <int AUX> __device__ __forceinline__ void store_rows(rsrc_t ro, unsigned lane_off, unsigned soff, const unsigned (&q)[32]) {
#pragma unroll
    for (int m = 0; m < 4; ++m)
#pragma unroll
        for (int j = 0; j < 4; ++j) {
            u32x2 v = {q[(m * 4 + j) * 2], q[(m * 4 + j) * 2 + 1]};
            __builtin_amdgcn_raw_buffer_store_b64(v, ro, (int)lane_off, (int)(soff + (m * 16 + j) * NTOT), AUX);
        }
}

__device__ __forceinline__ void puller(const CAS Params& p, char* myws, unsigned scr, unsigned sem, unsigned seq, int link, int g) {
    const int lane = threadIdx.x & 63;
    const int rank = p.rank, delta = link + 1, src = (rank + delta) & 7;
    char* tmo = myws + OFF_TMO;
    const bool once = g < 0;
    const rsrc_t rr = mk_rsrc(once ? myws : p.ws[src], WS_NEED), ro = mk_rsrc(p.out, 4096u * NTOT);
    const unsigned lane_off = (unsigned)(4 * (lane >> 4)) * NTOT + 8 * (lane & 15);
    char* ctr = myws + OFF_TICK + link * 128;
    const unsigned par = (seq & 1u) * NEXP;
    const int win = once ? 1024 : TUNED(0, WIN_IN), wout = TUNED(1, WIN_OUT);
    const unsigned tbase = (seq - 1u) * (unsigned)NI_ALL - (unsigned)(PULL_WG * 8);
    bool dead = false;
    for (int it = g; it < NI_ALL;) {
        unsigned cxi, cat;
        item_decode(it, rank, delta, par, cxi, cat);
        const char* fl = myws + (it >= NI_IN ? OFF_OFLAG + src * 2048 + (((cat >> 6) * 8 + ((cat >> 1) & 7)) * 2 + ((cat >> 5) & 1)) * 8
                                             : (once ? OFF_IFLAG + cxi * 8 : OFF_XFLAG + (seq & 1u) * XFLAG_SZ + (cat >> 1) * 8));
        unsigned long long mn = ld_sys64(fl);
        if (RFL(mn >> 32) != seq) {
            unsigned long long t0 = __builtin_amdgcn_s_memrealtime();
            for (unsigned n = 1;; ++n) {
                mn = ld_sys64(fl);
                if (RFL(mn >> 32) == seq) break;
                if ((n & 31u) == 0u && give_up(tmo, seq, t0)) { mn = 0; dead = true; break; }
                __builtin_amdgcn_s_sleep(1);
            }
        }
        CFENCE();
        const unsigned cmeta = it >= NI_IN ? ((RFL(mn) >> (((cat >> 4) & 1) * 8 + (cat & 1) * 4)) & 15u) << 8 : (RFL(mn) >> (once ? 0 : (cat & 1) * 16)) & 0xffffu;
        u32x4 c[8];
        unsigned tk;
        {
            const unsigned lines = (cmeta >> 8) & 15u;
            const bool isout = it >= NI_IN, raw = lines == M_RAW;
            const unsigned bo = (unsigned)lane < lines * 8u ? lane * 16u : 0x80000000u;
            const unsigned o = raw ? (isout ? OFF_ROUT : (once ? OFF_RIBX : OFF_RIN)) + cxi * 8192
                                   : (isout ? OFF_XOUT + cxi * XO_SZ : (once ? OFF_IBX : OFF_XIN) + cxi * XI_SZ);
            const unsigned nld = once ? 0u : (raw ? 8u : (isout ? 7u : 6u));
            unsigned my;
            asm volatile("s_mov_b64 exec, 1\n\tds_add_rtn_u32 %0, %1, %2\n\ts_mov_b64 exec, -1\n\ts_waitcnt lgkmcnt(0)" : "=&v"(my) : "v"(sem), "v"(nld) : "memory");
            my = RFL(my) - max((unsigned)(isout ? wout : win), nld);
            unsigned rel = RFL(__hip_atomic_load((LAS unsigned*)(size_t)(sem + 4), __ATOMIC_RELAXED, __HIP_MEMORY_SCOPE_WORKGROUP));
#define GATE(u) while ((int)(my + (u) - rel) >= 0) { rel = RFL(__hip_atomic_load((LAS unsigned*)(size_t)(sem + 4), __ATOMIC_RELAXED, __HIP_MEMORY_SCOPE_WORKGROUP)); CFENCE(); }
#define RELEASE(n, v) asm volatile("s_waitcnt vmcnt(%0)\n\ts_mov_b64 exec, 1\n\tds_add_u32 %1, %2 offset:4\n\ts_mov_b64 exec, -1" :: "n"(n), "v"(sem), "v"(v) : "memory")
            while ((int)(my - 4u - rel) >= 0) { __builtin_amdgcn_s_sleep(2); rel = RFL(__hip_atomic_load((LAS unsigned*)(size_t)(sem + 4), __ATOMIC_RELAXED, __HIP_MEMORY_SCOPE_WORKGROUP)); }
            __builtin_amdgcn_s_setprio(3);
            GATE(0u); c[4] = bld16(rr, lane * 16, o + 4096);
            GATE(1u); c[5] = bld16(rr, isout ? lane * 16u : bo, o + 5120);
            GATE(2u); c[0] = bld16(rr, lane * 16, o);
            GATE(3u); c[1] = bld16(rr, lane * 16 + 1024, o);
            GATE(4u); c[2] = bld16(rr, lane * 16 + 2048, o);
            GATE(5u); c[3] = bld16(rr, lane * 16 + 3072, o);
            GATE(nld - 1u);
            c[6] = bld16(rr, (isout || raw) ? bo : 0x80000000u, o + 6144);
            c[7] = bld16(rr, raw ? lane * 16u : 0x80000000u, o + 7168);
            tk = tick_draw(ctr, once);
            const unsigned one = once ? 0u : 1u, last = once ? 0u : nld - 5u;
            RELEASE(8, one); RELEASE(7, one); RELEASE(6, one); RELEASE(5, one); RELEASE(4, one); RELEASE(1, last);
            __builtin_amdgcn_s_setprio(0);
        }
        {
            const unsigned lines = (cmeta >> 8) & 15u;
            if (it >= NI_IN) {
                unsigned q[32];
                if (dead) {
#pragma unroll
                    for (int d = 0; d < 32; ++d) q[d] = 0x80808080u;
                } else if (lines == M_RAW) {
#pragma unroll
                    for (int d = 0; d < 32; ++d) q[d] = c[d >> 2][d & 3];
                } else {
                    *(LAS u32x4*)(size_t)(scr + SC_BLOB + lane * 16) = c[6];
                    CFENCE();
                    const unsigned P[8] = {c[4].x, c[4].y, c[4].z, c[4].w, c[5].x, c[5].y, c[5].z, c[5].w};
                    decode<8, false>(P, q, scr, 0u, lines * 32u, sem + 1024, lane);
#pragma unroll
                    for (int d = 0; d < 32; ++d) {
                        const unsigned R = c[d >> 3][((d >> 1) & 2) + (d & 1)] >> ((d & 2) * 2);
                        const unsigned s8 = R & 0x08080808u;
                        q[d] = (q[d] ^ (opaque(s8 << 5) - s8)) | (R & 0x07070707u);
                    }
                }
                store_rows<OUT_AUX>(ro, lane_off, (cat >> 4) * (64u * NTOT) + src * NL + (cat & 15u) * 128u, q);
            } else {
                u32x4 o[8];
                if (lines == M_RAW) {
#pragma unroll
                    for (int u = 0; u < 8; ++u) o[u] = c[u];
                } else {
                    *(LAS u32x4*)(size_t)(scr + SC_BLOB + lane * 16) = c[5];
                    CFENCE();
                    const unsigned P[4] = {c[4].x, c[4].y, c[4].z, c[4].w};
                    const unsigned nex = (cmeta >> 12) & 15u;
                    unsigned ex[16];
                    decode<4, true>(P, ex, scr, nex, lines * 32u, cmeta & 0xffu, lane);
                    for (unsigned k = 0; k < nex; ++k) {
                        const unsigned en = RFL(LDS32(scr + SC_BLOB + k * 4));
                        const unsigned sh = ((en >> 8) & 3u) * 8u, r = (en >> 10) & 15u;
                        const unsigned msk = (unsigned)lane == ((en >> 14) & 63u) ? 0xffu << sh : 0u;
                        const unsigned val = rot8(en & 0xffu) << sh;
#pragma unroll
                        for (int d = 0; d < 16; ++d) { const unsigned mq = r == (unsigned)d ? msk : 0u; ex[d] = (ex[d] & ~mq) | (val & mq); }
                    }
#pragma unroll
                    for (int d = 0; d < 16; ++d) {
                        const unsigned r = c[d >> 2][d & 3];
                        const unsigned H = (r & 0x80808080u) | (ex[d] & 0x7f7f7f7fu), L = (ex[d] & 0x80808080u) | (r & 0x7f7f7f7fu);
                        o[d >> 1][(d & 1) * 2] = PERM(H, L, 0x05010400u);
                        o[d >> 1][(d & 1) * 2 + 1] = PERM(H, L, 0x07030602u);
                    }
                }
#pragma unroll
                for (int u = 0; u < 8; ++u) st16(myws, lane * 16, u & 3, OFF_STAGE + cat * 8192 + (u >> 2) * 4096, o[u]);
                WAIT_VM0();
                if (lane == 0) st_agent(myws + OFF_SFLAG + cat * 32, seq);
            }
        }
        if (once) break;
        const int nx = (int)min(RFL(tk) - tbase, (unsigned)NI_ALL);
        it = nx;
    }
}

__device__ __forceinline__ int step_group(int s, int rank, int d0) {
    if (s < 8) return rank * 8 + s;
    int t = s - 8;
    int kt = t / 7, u = t - kt * 7;
    int v = u < d0 ? u : u + 1;
    int delta = (d0 - v) & 7;
    return ((rank + delta) & 7) * 8 + kt;
}

__device__ __forceinline__ void gemm_role(const CAS Params& p, int gidx, unsigned seq, char* myws, LAS char* lds) {
    int tid = threadIdx.x;
    asm volatile("" : "+v"(tid));
    const int wid = RFL(tid >> 6);
    int lane = tid & 63;
    const int wr = wid >> 1, wc = wid & 1;
    const int rank = p.rank;
    const bool mainw = gidx >= 0;
    int ti, tj;
    if (mainw) { const int xcd = gidx & 7, sl = gidx >> 3; ti = (xcd >> 1) * 4 + (sl >> 2); tj = (xcd & 1) * 4 + (sl & 3); }
    else { const int x = -1 - gidx; ti = (x & 7) * 2 + ((x >> 3) & 1); tj = 8 + (x >> 4); }
    const int d0 = ti >> 1;
    const int myslot = wid < 4 ? ti : 16 + tj;
    char* tmo = myws + OFF_TMO;
    const char* dsrc = myws + OFF_STAGE + (size_t)myslot * PIECE + (wid & 3) * 8192;
    const char* fsrc = myws + OFF_SFLAG + myslot * 128 + (wid & 3) * 32;
    const unsigned loff = lane * 16, loff2 = loff + 4096;
    const unsigned ldsw = (unsigned)(size_t)lds + wid * 8192;

    f32x4 acc[4][8];
#pragma unroll
    for (int m = 0; m < 4; ++m)
#pragma unroll
        for (int i = 0; i < 8; ++i) acc[m][i] = (f32x4){0.f, 0.f, 0.f, 0.f};

#define GL(n, S) glds_if<((n) & 3) * 1024>(have, want, S, (n) < 4 ? loff : loff2, (n) < 4 ? ld0 : ld1)
#define STAGE_ALL(S, D) stage8k(S, loff, loff2, D)
#define PRIO(n) __builtin_amdgcn_s_setprio(n)
#define DUP2
#define T1 24
#define LV 0
#define PA 15
#define PB 15
#define RD1(t) (((t) & 1) == 0 && (t) < 24 ? (t) / 2 : -1)
#define RD2(t) (((t) & 1) == 0 && (t) >= 2 && (t) < 26 ? (t) / 2 - 1 : -1)
#define GL2(t) (((t) & 3) == 1 ? (t) >> 2 : -1)
#define GOFF(st) ((unsigned)__builtin_amdgcn_readlane((int)tab, (st)))
#define GSRC(off) (dsrc + (off))
#define GFLAG(off) (fsrc + ((off) >> 8))
    static_assert(PPK * PIECE / 256 == PPK * 128, "flag offset");
    const unsigned tab = (unsigned)step_group(lane, rank, d0) * (PPK * PIECE);
    const unsigned abase = (unsigned)(size_t)lds + wr * 8192 + (lane & 15) * 128 + (((lane >> 4) ^ ((lane >> 1) & 7)) << 4);
    const unsigned bbase = (unsigned)(size_t)lds + 32768 + wc * 16384 + ((lane >> 3) & 1) * 4096 + (lane >> 4) * 256 + (lane & 7) * 32 + ((lane >> 4) & 1) * 16;
#define FRAG(a) (*(const LAS bf16x8*)(size_t)(a))
    bf16x8 a0[4], b0[8], a1[4], b1[8];

    {
        const unsigned g0 = GOFF(0);
        spin_flag(GFLAG(g0), seq, tmo);
        STAGE_ALL(GSRC(g0), ldsw);
    }
    unsigned gn = GOFF(1);
    unsigned fv = ld_sys(GFLAG(gn));
    WAIT_VM0();
    RAW_BAR();
    if (mainw) STAMP(2);
    bool staged = RFL(fv) >= seq;
    if (staged) { STAGE_ALL(GSRC(gn), ldsw + 65536); gn = GOFF(2); }
    fv = ld_sys(GFLAG(gn));
#pragma unroll
    for (int i = 0; i < 8; ++i) b0[i] = FRAG((bbase ^ ((i >> 2) * 16)) + (i & 3) * 1024);
#pragma unroll
    for (int m = 0; m < 4; ++m) a0[m] = FRAG(abase + m * 2048);

    for (int s = 0; s < 64; ++s) {
        const int cur = (s & 1) * 65536, oth = cur ^ 65536;
        if (!staged) {
            WAIT_VM0();
            if (RFL(fv) >= seq) { STAGE_ALL(GSRC(gn), ldsw + oth); staged = true; gn = GOFF(min(s + 2, 63)); fv = ld_sys(GFLAG(gn)); }
        }
        const unsigned ab2 = abase + oth, bb2 = bbase + oth;
        const unsigned want = s + 2 < 64 ? seq : 0xffffffffu;
        const unsigned g3 = GOFF(min(s + 3, 63));
        const unsigned ld0 = ldsw + cur, ld1 = ld0 + 4096;
        SCHED0();
        {
            const unsigned ab = (abase + cur) ^ 64u, bb = bbase + cur;
            PRIO(1);
            static_for<T1>([&](auto tc) __attribute__((always_inline)) {
                constexpr int t = decltype(tc)::value;
                acc[t >> 3][t & 7] = __builtin_amdgcn_mfma_f32_16x16x32_bf16(a0[t >> 3], b0[t & 7], acc[t >> 3][t & 7], 0, 0, 0);
                constexpr int rd = RD1(t);
                if constexpr (rd >= 0 && rd < 8) b1[rd] = FRAG((bb ^ ((rd >> 2) * 16)) + (rd & 3) * 1024 + 8192);
                if constexpr (rd >= 8) a1[rd - 8] = FRAG(ab + (rd - 8) * 2048);
                if constexpr (t == PA) PRIO(0);
                SCHED0();
            });
        }
        if (!staged) {
            spin_flag(GFLAG(gn), seq, tmo);
            STAGE_ALL(GSRC(gn), ldsw + oth);
            gn = GOFF(min(s + 2, 63));
            fv = ld_sys(GFLAG(gn));
        }
        const unsigned gcur = GOFF(s);
        WAIT_VM0();
        const unsigned have = RFL(fv);
        const char* gs = GSRC(have >= want ? gn : gcur);
        asm volatile("s_mov_b32 m0, %0" :: "s"(ld0), "s"(gs) : "memory");
        SCHED0();
        static_for<32, T1>([&](auto tc) __attribute__((always_inline)) {
            constexpr int t = decltype(tc)::value;
            acc[t >> 3][t & 7] = __builtin_amdgcn_mfma_f32_16x16x32_bf16(a0[t >> 3], b0[t & 7], acc[t >> 3][t & 7], 0, 0, 0);
            if constexpr (t == PA) PRIO(0);
            SCHED0();
        });
        __builtin_amdgcn_s_waitcnt(0xC07F);
        PRIO(3);
        SCHED0();
        RAW_BAR();
        SCHED0();
        auto part2 = [&](auto mode) __attribute__((always_inline)) {
            static_for<32>([&](auto tc) __attribute__((always_inline)) {
                constexpr int t = decltype(tc)::value, md = decltype(mode)::value;
                acc[t >> 3][t & 7] = __builtin_amdgcn_mfma_f32_16x16x32_bf16(a1[t >> 3], b1[t & 7], acc[t >> 3][t & 7], 0, 0, 0);
                constexpr int rd = RD2(t);
                if constexpr (rd >= 0 && rd < 8) b0[rd] = FRAG((bb2 ^ ((rd >> 2) * 16)) + (rd & 3) * 1024);
                if constexpr (rd >= 8) a0[rd - 8] = FRAG(ab2 + (rd - 8) * 2048);
                if constexpr (md == 0 && GL2(t) >= 0) GL(GL2(t), gs);
                if constexpr (md == 1 && GL2(t + 1) == 4) asm volatile("s_add_u32 m0, m0, 0x1000" ::: "memory", "scc");
                if constexpr (md == 1 && GL2(t) >= 0)
                    asm volatile("global_load_lds_dwordx4 %0, %1 offset:%2 " GLB :: "v"(GL2(t) < 4 ? loff : loff2), "s"(gs), "n"((GL2(t) & 3) * 1024) : "memory");
                if constexpr (t == PB) PRIO(2);
                if constexpr (t == 30) {
                    const bool go = have >= want;
                    staged = go || s + 2 >= 64;
                    gn = go ? g3 : gn;
                }
                if constexpr (t == 31) fv = ld_sys(GFLAG(gn));
                SCHED0();
            });
        };
        part2(IC<1>{});
    }
    if (mainw) STAMP(3);

    asm volatile("" : "+v"(tid));
    lane = tid & 63;
    char* peer = p.ws[lane & 7];
    float am = 0.f;
#pragma unroll
    for (int m = 0; m < 4; ++m)
#pragma unroll
        for (int i = 0; i < 8; ++i)
#pragma unroll
            for (int j = 0; j < 4; ++j) am = fmaxf(am, fabsf(acc[m][i][j]));
#pragma unroll
    for (int o = 32; o > 0; o >>= 1) am = fmaxf(am, __shfl_xor(am, o));
    LAS float* red = (LAS float*)(lds + LDS_MISC + 512);
    if (lane == 0) red[wid] = am;
    WAIT_VM0();
    RAW_BAR();
    if (tid < NR && mainw) {
        float a = red[0];
#pragma unroll
        for (int i = 1; i < 8; ++i) a = fmaxf(a, red[i]);
        st_sys64(peer + OFF_AMAX + (rank * NGEMM + gidx) * 8, ((unsigned long long)seq << 32) | __float_as_uint(a));
    }
    const unsigned scr = (unsigned)(size_t)lds + wid * SCR_SZ;
    zero_scratch(scr, lane);
    lut_init(scr, lane);
    {
        const char* gp = myws + OFF_AMAX + tid * 16;
        unsigned long long t0 = __builtin_amdgcn_s_memrealtime();
        unsigned long long a, b;
        for (unsigned n = 1;; ++n) {
            a = ld_sys64(gp);
            b = ld_sys64(gp + 8);
            bool ok = (unsigned)(a >> 32) == seq && (unsigned)(b >> 32) == seq;
            if (__all(ok)) break;
            if ((n & 31u) == 0u && give_up(tmo, seq, t0)) break;
            __builtin_amdgcn_s_sleep(AM_SLEEP);
        }
        am = fmaxf(__uint_as_float((unsigned)a), __uint_as_float((unsigned)b));
#pragma unroll
        for (int o = 32; o > 0; o >>= 1) am = fmaxf(am, __shfl_xor(am, o));
        if (lane == 0) red[8 + wid] = am;
        WAIT_VM0();
        RAW_BAR();
#pragma unroll
        for (int i = 0; i < 8; ++i) am = fmaxf(am, red[8 + i]);
    }
    const unsigned tmov = ld_agent(tmo);
    const unsigned prog = ld_agent(myws + OFF_TICK);
    if (mainw) STAMP(4);

    const float inv = am > 0.f ? 127.0f / am : 0.f;
    unsigned q[32];
#pragma unroll
    for (int m = 0; m < 4; ++m)
#pragma unroll
        for (int j = 0; j < 4; ++j)
#pragma unroll
            for (int h = 0; h < 2; ++h) {
                const unsigned t0 = __float_as_uint(fmaf(acc[m][4 * h + 0][j], inv, 12582912.f));
                const unsigned t1 = __float_as_uint(fmaf(acc[m][4 * h + 1][j], inv, 12582912.f));
                const unsigned t2 = __float_as_uint(fmaf(acc[m][4 * h + 2][j], inv, 12582912.f));
                const unsigned t3 = __float_as_uint(fmaf(acc[m][4 * h + 3][j], inv, 12582912.f));
                q[(m * 4 + j) * 2 + h] = PERM(t1, t0, 0x0c0c0400u) | PERM(t3, t2, 0x04000c0cu);
            }
    if (RFL(tmov) == seq) {
#pragma unroll
        for (int d = 0; d < 32; ++d) q[d] = 0x80808080u;
    }
    const unsigned lane_off = (unsigned)(4 * (lane >> 4)) * NTOT + 8 * (lane & 15);
    const rsrc_t ro = mk_rsrc(p.out, 4096u * NTOT);
    const unsigned rowoff = (unsigned)(ti * 4 + wr) * (64u * NTOT);
    if (!mainw) {
        const int h = (tj - 8) * 2 + wc;
        if (h < 7 * NDUP) store_rows<OUT_AUX>(ro, lane_off, rowoff + ((rank + 1 + h / NDUP) & 7) * NL + (NCB + h % NDUP) * 128, q);
        STAMP(6);
        return;
    }
    const int cb = tj * 2 + wc;
    if (cb < NCB) {
        const unsigned xi = (ti * 4 + wr) * 16 + cb;
        const bool late = RFL(prog) - (seq - 1u) * (unsigned)NI_ALL + PULL_WG * 8 > (unsigned)(NI_IN + LATE_AT);
        unsigned P[8], nb = 1000u;
        if (!((wr >= 2 && ti * 2 + wr - 2 >= 32 - RAW_TAIL) || (ti == 0 && wr < 2 && late))) {
        unsigned n[16];
#pragma unroll
        for (int u = 0; u < 4; ++u) {
            u32x4 r;
#pragma unroll
            for (int k = 0; k < 4; ++k) {
                const unsigned q0 = q[8 * u + (k >> 1) * 4 + (k & 1)], q1 = q[8 * u + (k >> 1) * 4 + (k & 1) + 2];
                r[k] = BFI(0x77777777u, BFI(0x0f0f0f0fu, q0, q1 << 4), BFI(0x0f0f0f0fu, q0 >> 4, q1));
                const unsigned sg = (r[k] >> 3) & 0x11111111u;
                n[4 * u + k] = BFI(0x0f0f0f0fu, q0 >> 3, q1 << 1) ^ (opaque(sg << 4) - sg);
            }
            st16(myws, lane * 16, u, OFF_XOUT + xi * XO_SZ, r);
        }
        nb = encode<8, false>(n, P, scr, 0u, lane);
        }
        unsigned lines = M_RAW;
        if (nb <= 256u) {
            lines = (nb + 31u) >> 5;
            st16(myws, lane * 16, 0, OFF_XOUT + xi * XO_SZ + 4096, (u32x4){P[0], P[1], P[2], P[3]});
            st16(myws, lane * 16, 1, OFF_XOUT + xi * XO_SZ + 4096, (u32x4){P[4], P[5], P[6], P[7]});
            if ((unsigned)lane < lines * 8u) st16(myws, lane * 16, 2, OFF_XOUT + xi * XO_SZ + 4096, *(LAS u32x4*)(size_t)(scr + SC_BLOB + lane * 16));
        } else {
#pragma unroll
            for (int u = 0; u < 8; ++u)
                st16(myws, lane * 16, u & 3, OFF_ROUT + xi * 8192 + (u >> 2) * 4096, (u32x4){q[4 * u], q[4 * u + 1], q[4 * u + 2], q[4 * u + 3]});
        }
        WAIT_VM0();
        unsigned long long tot;
        if (gather((unsigned)(size_t)lds + LDS_MISC + 64 + (wid >> 2) * 16, (unsigned long long)(lines << ((wid & 3) * 4)), tot) == 3u && lane < NR && lane != rank)
            st_sys64(peer + OFF_OFLAG + rank * 2048 + ((ti * 8 + tj) * 2 + (wid >> 2)) * 8, ((unsigned long long)seq << 32) | (unsigned)tot);
    }
    STAMP(5);
    store_rows<OWN_AUX>(ro, lane_off, rowoff + rank * NL + cb * 128, q);
}

__global__ __launch_bounds__(NTHR, 2) __attribute__((amdgpu_waves_per_eu(2, 2))) void fused_kernel(Params p_unused) {
    extern __shared__ __attribute__((aligned(1024))) char smem[];
    const CAS Params& p = *(const CAS Params*)__builtin_amdgcn_kernarg_segment_ptr();
    char* myws = p.self;
    const float* px = p.x;
    const float* pw = p.w;
    int rank = p.rank;
    asm volatile("" : "+s"(myws), "+s"(px), "+s"(pw), "+s"(rank));
    LAS char* lds = (LAS char*)smem;
    STAMP(0);
    const int b = blockIdx.x;
    const int wid = RFL(threadIdx.x >> 6), lane = threadIdx.x & 63;
    const unsigned scr = (unsigned)(size_t)lds + wid * SCR_SZ;
    char* cnt = myws + OFF_CNT + (b * 8 + wid) * 4;
    unsigned seq = 0, seqv;
    asm volatile("global_load_dword %0, %1, %2 sc1" : "=v"(seqv) : "v"(0u), "s"(cnt) : "memory");
    if (b < 16) {
        const char* pc = (const char*)__builtin_amdgcn_s_getpc();
        const unsigned o = threadIdx.x * 128u;
        if (o < CODE_BYTES) { unsigned t = *(const volatile GAS unsigned*)(pc + o); asm volatile("" :: "v"(t)); }
    }
    const unsigned sem = (unsigned)(size_t)lds + LDS_MISC;
    if (threadIdx.x < 128) LDS32(sem + threadIdx.x * 4) = 0u;
    if (b < NPWG) unary_init(sem + 1024, threadIdx.x);
    lut_init(scr, lane);
    if (lane == 0) LDS32(scr + SC_LUT + 64) = 0u;
    {
        constexpr int NS = NWG - NGEMM - NPWG, NXL = 438;
        static_assert(2 * NPWG >= 96 && EXP_GL * NGEMM + 4 * NPWG + 8 * NS - 448 >= 768 && 8 * NS >= 448 && 4 * NS >= 224 && 4 * NPWG >= 224, "first tasks");
        const bool gw = b >= NWG - NGEMM, spare = !gw && b >= NPWG;
        const int sp = b - NPWG;
        int e = -1, lag = 0, own = 768;
        if (gw) {
            if (wid < EXP_GL) { own = wid * NGEMM + (b - (NWG - NGEMM)); lag = wid ? TUNED(4, EXP_LG) + wid * TUNED(3, EXP_LW) : GP_L0; }
            else if (wid == 7 && b - (NWG - NGEMM) < GPUSH) e = 768 + b - (NWG - NGEMM);
        } else if (spare) {
            const int k = GPUSH + wid * NS + sp;
            if (k < 448) { e = 768 + k; lag = k < 224 ? wid * TUNED(5, EXP_LPUSH) : TUNED(7, EXP_L2) + (wid - (224 - GPUSH) / NS) * TUNED(2, EXP_L3); }
            else { own = EXP_GL * NGEMM + 4 * NPWG + k - 448; lag = 8; }
        } else if (wid >= 4) { const int pr = wid >= 6 ? b : NPWG + b; if (pr < 96) e = ((pr >> 1) + (pr >> 5) * 8) * 4 + (pr & 1) * 2 + (wid & 1); lag = wid >= 6 ? 0 : EXP_LX; }
        else { own = EXP_GL * NGEMM + (3 - wid) * NPWG + b; lag = TUNED(6, EXP_LS); }
        const bool pair = b < NPWG && e >= 0, late_bar = pair && wid >= 6;
        bool push = b >= NPWG && e >= 0, stage_only = own < 768;
        if (stage_only) e = own;
        if (!late_bar) RAW_BAR();
        if (e >= 0) for (int i = 0; i < lag; ++i) __builtin_amdgcn_s_sleep(4);
        STAMP(8);
        unsigned grp = sem + 16 + (wid >> 1) * 16;
        int q = spare ? (wid < 4 ? 0 : NS) + sp - 2 * NS : NXL;
        int ib = spare ? (wid & 3) * NS + sp : 224;
#pragma unroll 1
        for (bool first = true;; first = false) {
            if (e >= 0) export_item(p, myws, px, pw, rank, scr, grp, e, stage_only, push, seq, seqv, first, pair, late_bar);
            else if (first) { asm volatile("s_waitcnt vmcnt(0)\n\tv_readfirstlane_b32 %0, %1\n\ts_nop 4" : "=s"(seq) : "v"(seqv) : "memory"); seq += 1u; }
            if (gw) break;
            if (ib < 224 || !spare) {
                if (!spare && first) STAMP(1);
                const bool box = ib < 224;
                puller(p, myws, scr, sem, seq, box ? (ib >> 2) % 7 : b % 7, box ? -1 - ((ib / 28) * 4 + (ib & 3) + (wid >= 4 ? 32 : 0)) : (b / 7) + (7 - wid) * PULL_WG);
                ib = 224;
                if (box && !spare) { e = -1; continue; }
            }
            q += 2 * NS;
            if (q >= NXL) break;
            const int qq = q < 56 ? q + 72 : q + 88, r = qq / 72, u = qq - r * 72;
            e = (r == 7 ? (u < 8 ? u * 24 + 23 : (u < 15 ? 233 : 282) + u) : (u < 56 ? 248 + r * 56 + u : (r + 1) * 24 + (u - 56))) * 4 + (wid & 3);
            push = false; stage_only = false;
            grp = sem + 64 + (q / NS) * 16;
        }
        static_assert((NXL / NS + 1) * 16 + 64 <= 512, "gather cells");
    }
    if (lane == 0) st_agent(cnt, seq);
    if (b >= NPWG) STAMP(1);
    if (b >= NWG - NGEMM) {
        const int gidx = b - (NWG - NGEMM);
#pragma unroll 1
        for (int g = gidx;; g = -1 - gidx) {
            RAW_BAR();
            gemm_role(p, g, seq, myws, lds);
            if (g < 0 || gidx >= 16 * NXT) break;
        }
    } else if (b < NPWG) {
        STAMP(4);
    }
}

}

extern "C" void kernel_launch(int rank, int nranks, ncclComm_t comm,
                              hipStream_t stream,
                              void* const* d_in, const int* in_sizes, int n_in,
                              void* d_out, int out_size,
                              const int* in_dtypes, int out_dtype,
                              void* const* d_out_all,
                              void* const* d_ws_all, size_t ws_bytes) {
    if (nranks != NR || ws_bytes < WS_NEED || (unsigned)rank >= (unsigned)NR) {
        fprintf(stderr, "unexpected configuration\n");
        return;
    }
    HIP_CHECK(hipFuncSetAttribute((const void*)fused_kernel, hipFuncAttributeMaxDynamicSharedMemorySize, LDS_BYTES));
    Params p;
    p.x = (const float*)d_in[0];
    p.w = (const float*)d_in[1];
    for (int i = 0; i < NR; ++i) p.ws[i] = (char*)d_ws_all[i];
    p.out = (char*)d_out;
    p.self = (char*)d_ws_all[rank];
    p.rank = rank;
    p.pad = 0;
    hipLaunchKernelGGL(fused_kernel, dim3(NWG), dim3(NTHR), LDS_BYTES, stream, p);
    HIP_CHECK(hipGetLastError());
}
```

```cpp
#include <hip/hip_runtime.h>
#include <rccl/rccl.h>
#include <cstdio>
#include <cstdint>

#define HIP_CHECK(expr) do { hipError_t _e = (expr); if (_e != hipSuccess) { \
    fprintf(stderr, "HIP error %s at %s:%d\n", hipGetErrorString(_e), __FILE__, __LINE__); \
    return; } } while (0)

namespace {

typedef unsigned u32x4 __attribute__((ext_vector_type(4)));
typedef unsigned u32x2 __attribute__((ext_vector_type(2)));
typedef float f32x4 __attribute__((ext_vector_type(4)));
typedef float f32x2 __attribute__((ext_vector_type(2)));
typedef __bf16 bf16x8 __attribute__((ext_vector_type(8)));
typedef __bf16 bf16x2 __attribute__((ext_vector_type(2)));
typedef unsigned short u16x2 __attribute__((ext_vector_type(2)));
typedef __amdgpu_buffer_rsrc_t rsrc_t;
#define GAS __attribute__((address_space(1)))
#define LAS __attribute__((address_space(3)))
#define CAS __attribute__((address_space(4)))

constexpr int NR = 8;
constexpr int NTOT = 16384;
constexpr int KL = 512;
constexpr int NL = 2048;
constexpr int PIECE = 32768;
#define NDUP 1
constexpr int NPUSH = 16;
constexpr int NXT = (7 * NDUP + 1) / 2;
constexpr int NCB = 16 - NDUP;
constexpr int PPK = 24 + NXT;
constexpr int NGEMM = 128;
constexpr int NWG = 256;
constexpr int NTHR = 512;
constexpr int NEXP = 2560;
#define PULL_WG 9
#define WIN_IN 7
#define WIN_OUT 7
constexpr unsigned CODE_BYTES = 44u * 1024u;
#define EXP_GL 4
#define EXP_LW 2
#define EXP_LG 4
#define EXP_LS 2
#define FIX0 3
#define RAW_TAIL 2
#define EXP_L2 16
#define EXP_L3 6
#define EXP_LPUSH 2
constexpr int NPWG = 7 * PULL_WG;
static_assert(NPWG < NWG - NGEMM, "puller count");
#define NFAST 4
constexpr int FSTEP = 16 / NFAST;
constexpr int NSGT = (NCB + 1) / 2;
constexpr int NI_MAIN = 768 - 4 * NPUSH, NI_DUP = NI_MAIN + 112 * NDUP, NI_IN = NI_DUP + 16 * NSGT, NI_ALL = NI_IN + 64 * NCB;
constexpr int NSTG = NR * 8 * PPK * 4;
constexpr int XI_SZ = 6144;
constexpr int XO_SZ = 6144;

constexpr unsigned OFF_CNT = 0;
constexpr unsigned OFF_TMO = 8192;
constexpr unsigned OFF_TICK = 12288;
constexpr unsigned OFF_IFLAG = 24576;
constexpr unsigned OFF_XFLAG = 32768;
constexpr unsigned XFLAG_SZ = NR * 8 * PPK * 16;
constexpr unsigned OFF_OFLAG = 163840;
constexpr unsigned OFF_GFLAG = 180224;
constexpr unsigned OFF_SFLAG = 196608;
constexpr unsigned OFF_DBG = 512 * 1024;
constexpr unsigned OFF_XIN = 1u << 20;
constexpr unsigned OFF_XOUT = OFF_XIN + 2 * NEXP * XI_SZ;
constexpr unsigned OFF_IBX = OFF_XOUT + 1024 * XO_SZ;
constexpr unsigned OFF_STAGE = OFF_IBX + 512 * XI_SZ;
constexpr unsigned OFF_RIN = OFF_STAGE + (NSTG + 1024) * 8192;
constexpr unsigned OFF_ROUT = OFF_RIN + (2 * NEXP + 128) * 8192;
constexpr unsigned OFF_RIBX = OFF_ROUT + 1024 * 8192;
constexpr unsigned WS_NEED = OFF_RIBX + 512 * 8192;
static_assert(OFF_XFLAG + 2 * XFLAG_SZ <= OFF_OFLAG && OFF_OFLAG + 16384 <= OFF_GFLAG && OFF_SFLAG + (NSTG + 1024) * 32 <= OFF_DBG, "layout");
constexpr unsigned long long DEADLINE = 50000000ull;
constexpr int LDS_MISC = 131072;
constexpr int LDS_BYTES = LDS_MISC + 1024 + 2048;
constexpr int AUX_SYS = 17;
#define OUT_AUX 17
#define AM_SLEEP 8
constexpr unsigned SCR_SZ = 10240;
constexpr unsigned SC_BLOB = 16, SC_A2 = 1152, SC_A3 = 5376, SC_LUT = 9600;
constexpr unsigned M_RAW = 15;

template <int V> struct IC { static constexpr int value = V; };
template <int N, int I = 0, class F> __device__ __forceinline__ void static_for(F&& f) {
    if constexpr (I < N) { f(IC<I>{}); static_for<N, I + 1>(f); }
}

struct Params {
    const float* x;
    const float* w;
    char* ws[NR];
    char* out;
    char* self;
    int rank;
    int pad;
};

__device__ __forceinline__ unsigned ld_sys(const char* p) {
    return __hip_atomic_load((const GAS unsigned*)p, __ATOMIC_RELAXED, __HIP_MEMORY_SCOPE_SYSTEM);
}
__device__ __forceinline__ unsigned long long ld_sys64(const char* p) {
    return __hip_atomic_load((const GAS unsigned long long*)p, __ATOMIC_RELAXED, __HIP_MEMORY_SCOPE_SYSTEM);
}
__device__ __forceinline__ void st_sys(char* p, unsigned v) {
    __hip_atomic_store((GAS unsigned*)p, v, __ATOMIC_RELAXED, __HIP_MEMORY_SCOPE_SYSTEM);
}
__device__ __forceinline__ void st_sys64(char* p, unsigned long long v) {
    __hip_atomic_store((GAS unsigned long long*)p, v, __ATOMIC_RELAXED, __HIP_MEMORY_SCOPE_SYSTEM);
}
__device__ __forceinline__ unsigned ld_agent(const char* p) {
    return __hip_atomic_load((const GAS unsigned*)p, __ATOMIC_RELAXED, __HIP_MEMORY_SCOPE_AGENT);
}
__device__ __forceinline__ void st_agent(char* p, unsigned v) {
    __hip_atomic_store((GAS unsigned*)p, v, __ATOMIC_RELAXED, __HIP_MEMORY_SCOPE_AGENT);
}
__device__ __forceinline__ rsrc_t mk_rsrc(const void* p, unsigned bytes) {
    return __builtin_amdgcn_make_buffer_rsrc((void*)p, (short)0, (int)bytes, 0x00020000);
}
__device__ __forceinline__ u32x4 bld16(rsrc_t r, unsigned voff, unsigned soff) {
    return __builtin_amdgcn_raw_buffer_load_b128(r, (int)voff, (int)soff, AUX_SYS);
}
#define STB "sc0 sc1"
__device__ __forceinline__ void st16(const char* base, unsigned voff, int blk, unsigned soff, u32x4 v) {
    const char* b = base + soff;
    if (blk == 0) asm volatile("s_nop 4\n\tglobal_store_dwordx4 %0, %1, %2 " STB "\n\ts_nop 1" :: "v"(voff), "v"(v), "s"(b) : "memory");
    if (blk == 1) asm volatile("s_nop 4\n\tglobal_store_dwordx4 %0, %1, %2 offset:1024 " STB "\n\ts_nop 1" :: "v"(voff), "v"(v), "s"(b) : "memory");
    if (blk == 2) asm volatile("s_nop 4\n\tglobal_store_dwordx4 %0, %1, %2 offset:2048 " STB "\n\ts_nop 1" :: "v"(voff), "v"(v), "s"(b) : "memory");
    if (blk == 3) asm volatile("s_nop 4\n\tglobal_store_dwordx4 %0, %1, %2 offset:3072 " STB "\n\ts_nop 1" :: "v"(voff), "v"(v), "s"(b) : "memory");
}
#define WAIT_VM0() do { asm volatile("s_waitcnt vmcnt(0)" ::: "memory"); __builtin_amdgcn_s_waitcnt(0x0F70); } while (0)
#define WAIT_LGKM0() asm volatile("s_waitcnt lgkmcnt(0)" ::: "memory")
#define RAW_BAR() do { asm volatile("s_waitcnt lgkmcnt(0)" ::: "memory"); __builtin_amdgcn_s_barrier(); asm volatile("" ::: "memory"); } while (0)
#define SCHED0() __builtin_amdgcn_sched_barrier(0)
#define PERM(a, b, s) __builtin_amdgcn_perm((a), (b), (s))
#define RFL(x) ((unsigned)__builtin_amdgcn_readfirstlane((int)(x)))

#define GLB "sc0 sc1"
__device__ __forceinline__ void stage8k(const char* src, unsigned loff, unsigned loff2, unsigned ldst) {
    asm volatile(
        "s_mov_b32 m0, %3\n\t"
        "s_nop 3\n\t"
        "global_load_lds_dwordx4 %0, %2 " GLB "\n\t"
        "global_load_lds_dwordx4 %0, %2 offset:1024 " GLB "\n\t"
        "global_load_lds_dwordx4 %0, %2 offset:2048 " GLB "\n\t"
        "global_load_lds_dwordx4 %0, %2 offset:3072 " GLB "\n\t"
        "s_add_u32 m0, m0, 0x1000\n\t"
        "s_nop 0\n\t"
        "global_load_lds_dwordx4 %1, %2 " GLB "\n\t"
        "global_load_lds_dwordx4 %1, %2 offset:1024 " GLB "\n\t"
        "global_load_lds_dwordx4 %1, %2 offset:2048 " GLB "\n\t"
        "global_load_lds_dwordx4 %1, %2 offset:3072 " GLB
        :: "v"(loff), "v"(loff2), "s"(src), "s"(ldst) : "memory", "scc");
}

__device__ __forceinline__ unsigned pk_bf16(float a, float b) {
    f32x2 v = {a, b};
    bf16x2 r = __builtin_convertvector(v, bf16x2);
    return __builtin_bit_cast(unsigned, r);
}


__device__ __forceinline__ bool give_up(char* tmo, unsigned seq, unsigned long long t0) {
    if (RFL(ld_agent(tmo)) == seq || __builtin_amdgcn_s_memrealtime() - t0 > DEADLINE) {
        st_agent(tmo, seq);
        return true;
    }
    return false;
}
__device__ __forceinline__ void spin_flag(const char* flag, unsigned seq, char* tmo) {
    unsigned long long t0 = __builtin_amdgcn_s_memrealtime();
    for (unsigned n = 1; RFL(ld_sys(flag)) < seq; ++n) {
        if ((n & 31u) == 0u && give_up(tmo, seq, t0)) break;
        __builtin_amdgcn_s_sleep(1);
    }
    asm volatile("" ::: "memory");
}

typedef unsigned u32x2a __attribute__((ext_vector_type(2), aligned(4)));
#define LDS32(a) (*(LAS unsigned*)(size_t)(a))
#define CFENCE() asm volatile("" ::: "memory")

__device__ __forceinline__ unsigned wave_scan(unsigned x) {
    x += (unsigned)__builtin_amdgcn_update_dpp(0, (int)x, 0x111, 0xf, 0xf, false);
    x += (unsigned)__builtin_amdgcn_update_dpp(0, (int)x, 0x112, 0xf, 0xf, false);
    x += (unsigned)__builtin_amdgcn_update_dpp(0, (int)x, 0x114, 0xf, 0xf, false);
    x += (unsigned)__builtin_amdgcn_update_dpp(0, (int)x, 0x118, 0xf, 0xf, false);
    x += (unsigned)__builtin_amdgcn_update_dpp(0, (int)x, 0x142, 0xa, 0xf, false);
    x += (unsigned)__builtin_amdgcn_update_dpp(0, (int)x, 0x143, 0xc, 0xf, false);
    return x;
}
__device__ __forceinline__ unsigned esc_bits(unsigned P) { return __builtin_amdgcn_bitop3_b32(P, P >> 1, 0x55555555u, 0x80); }

__device__ __forceinline__ void lut_init(unsigned scr, int lane) {
    if (lane < 16) {
        unsigned sel = 0x0c0c0c0cu; int k = 0;
#pragma unroll
        for (int y = 0; y < 4; ++y) if ((lane >> y) & 1) { sel = (sel & ~(0xffu << (8 * k))) | ((unsigned)y << (8 * k)); ++k; }
        LDS32(scr + SC_LUT + lane * 4) = sel;
        LDS32(scr + SC_LUT + 64) = 0u;
    }
}
#define BFI(m, x, y) __builtin_amdgcn_bitop3_b32((m), (x), (y), 0xCA)
__device__ __forceinline__ unsigned opaque(unsigned x) { asm("" : "+v"(x)); return x; }
__device__ __forceinline__ void unary_init(unsigned tab, int t) {
    if (t < 256) {
        unsigned long long v = 0; unsigned k = 0, gap = 3;
#pragma unroll
        for (int y = 0; y < 8; ++y) { if ((t >> y) & 1) { v |= (unsigned long long)(gap << 3) << k; k += 8; gap = 3; } else ++gap; }
        *(LAS unsigned long long*)(size_t)(tab + t * 8) = v;
    }
}
__device__ __forceinline__ void zero_scratch(unsigned scr, int lane) {
#pragma unroll
    for (int i = 0; i < 9; ++i) *(LAS u32x4*)(size_t)(scr + i * 1024 + lane * 16) = (u32x4){0u, 0u, 0u, 0u};
    if (lane < 24) *(LAS u32x4*)(size_t)(scr + 9216 + lane * 16) = (u32x4){0u, 0u, 0u, 0u};
}
__device__ __forceinline__ unsigned digits1(unsigned n02, unsigned n13) {
    const unsigned lo = BFI(0x33333333u, n02, n13 << 2), hi = BFI(0x33333333u, n02 >> 2, n13);
    const unsigned h = (hi | (hi >> 1)) & 0x55555555u;
    return lo | h | (h << 1);
}
__device__ __forceinline__ unsigned sel_addr(unsigned E, int t, unsigned scr) {
    if (t == 0) return __builtin_amdgcn_udot4(E & 0x01010101u, 0x20100804u, scr + SC_LUT, false);
    if (t == 1) return __builtin_amdgcn_udot4(E & 0x04040404u, 0x08040201u, scr + SC_LUT, false);
    if (t == 2) return __builtin_amdgcn_udot4((E >> 4) & 0x01010101u, 0x20100804u, scr + SC_LUT, false);
    return __builtin_amdgcn_udot4((E >> 4) & 0x04040404u, 0x08040201u, scr + SC_LUT, false);
}
template <int NP, bool IN> __device__ __forceinline__ unsigned encode(const unsigned (&n)[2 * NP], unsigned (&P)[NP], unsigned scr, unsigned nb0, int lane) {
    unsigned E[NP], cnt = 0;
#pragma unroll
    for (int j = 0; j < NP; ++j) { P[j] = digits1(n[2 * j], n[2 * j + 1]); E[j] = esc_bits(P[j]); cnt += __popc(E[j]); }
    unsigned sel[2][8];
#pragma unroll
    for (int k = 0; k < 8; ++k) sel[0][k] = LDS32(sel_addr(E[k >> 2], k & 3, scr));
    cnt = max(cnt, 4u);
    const unsigned inc = wave_scan(cnt);
    const unsigned n2d = ((unsigned)__builtin_amdgcn_readlane((int)inc, 63) + 15u) >> 4;
    if (nb0 + n2d > 256u) return 1000u;
    unsigned pos = scr + ((lane & 1) ? SC_A3 : SC_A2) + inc - cnt, carry = 0;
    static_for<NP / 2>([&](auto gc) __attribute__((always_inline)) {
        constexpr int g = decltype(gc)::value;
        SCHED0();
        if constexpr (g + 1 < NP / 2) {
#pragma unroll
            for (int k = 0; k < 8; ++k) sel[(g + 1) & 1][k] = LDS32(sel_addr(E[2 * g + 2 + (k >> 2)], k & 3, scr));
        }
        SCHED0();
#pragma unroll
        for (int k = 0; k < 8; ++k) {
            const int j = 2 * g + (k >> 2), t = k & 3;
            const unsigned z = t < 2 ? n[2 * j + t] : n[2 * j + t - 2] >> 4;
            const unsigned long long v = (unsigned long long)PERM(0u, z, sel[g & 1][k]) << ((pos & 3u) * 8u);
            const unsigned lo = (unsigned)v | carry;
            LDS32(pos & ~3u) = lo;
            const unsigned np = pos + __popc(E[j] & (0x01010101u << (2 * t)));
            carry = ((np ^ pos) & 4u) ? (unsigned)(v >> 32) : lo;
            pos = np;
        }
    });
    LDS32(pos & ~3u) = carry;
    CFENCE();
    unsigned n3 = 0;
    if constexpr (!IN) {
        for (unsigned i0 = 0; i0 < n2d; i0 += 64) {
            const unsigned i = i0 + lane;
            const u32x4 v = *(LAS u32x4*)(size_t)(scr + SC_A2 + i * 16) | *(LAS u32x4*)(size_t)(scr + SC_A3 + i * 16);
            unsigned long long acc = 0;
            unsigned c = 0, mx = 0;
#pragma unroll
            for (int t = 0; t < 4; ++t) {
                const unsigned y = (v[t] & 0x0f0f0f0fu) + 0x0d0d0d0du, m = y & 0x10101010u;
                unsigned l = y & opaque(m - (m >> 4));
                l = opaque((l << 8) + l);
                l = opaque((l << 16) + l) + 0x03020100u;
                const unsigned bits = (1u << (l & 31u)) | (1u << ((l >> 8) & 31u)) | (1u << ((l >> 16) & 31u)) | (1u << ((l >> 24) & 31u));
                acc |= (unsigned long long)bits << (c & 63u);
                c += (l >> 24) + 1u;
                mx |= l;
            }
            c = i < n2d ? c : 0u;
            const unsigned ic = wave_scan(c);
            const unsigned tot = (unsigned)__builtin_amdgcn_readlane((int)ic, 63);
            if (n3 + tot > 8192u || __any(c > 64u || mx >= 0x20000000u)) return 1000u;
            if (i < n2d) {
                const unsigned p3 = n3 + ic - c, sh = p3 & 31u;
                LAS unsigned* d = (LAS unsigned*)(size_t)(scr + SC_BLOB + (p3 >> 5) * 4);
                const unsigned long long w = acc << sh;
                __hip_atomic_fetch_or(d, (unsigned)w, __ATOMIC_RELAXED, __HIP_MEMORY_SCOPE_WAVEFRONT);
                __hip_atomic_fetch_or(d + 1, (unsigned)(w >> 32), __ATOMIC_RELAXED, __HIP_MEMORY_SCOPE_WAVEFRONT);
                if (c + sh > 64u) __hip_atomic_fetch_or(d + 2, (unsigned)(((acc >> 32) << sh) >> 32), __ATOMIC_RELAXED, __HIP_MEMORY_SCOPE_WAVEFRONT);
            }
            n3 += tot;
        }
        CFENCE();
        return (n3 + 31u) >> 5;
    }
    const unsigned s3 = scr + SC_BLOB + (nb0 + n2d) * 4;
    for (unsigned i0 = 0; i0 < n2d; i0 += 64) {
        const unsigned i = i0 + lane;
        const u32x4 v = (*(LAS u32x4*)(size_t)(scr + SC_A2 + i * 16) | *(LAS u32x4*)(size_t)(scr + SC_A3 + i * 16)) & 0x0f0f0f0fu;
        unsigned x = 0;
#pragma unroll
        for (int t = 0; t < 4; ++t) {
            const unsigned z1 = v[t] + 0x01010101u;
            unsigned d = z1, e4;
            if (IN) { e4 = (((v[t] + 0x0b0b0b0bu) ^ z1) >> 4) & 0x01010101u; d |= z1 >> 3; }
            else e4 = ((v[t] + 0x0a0a0a0au) >> 4) & 0x01010101u;
            x |= ((d | e4 | (e4 << 1)) & 0x03030303u) << (2 * t);
        }
        const unsigned E2 = esc_bits(x);
        unsigned sl[4];
#pragma unroll
        for (int t = 0; t < 4; ++t) sl[t] = LDS32(sel_addr(E2, t, scr));
        const unsigned c = __popc(E2), ic = wave_scan(c);
        const unsigned tot = (unsigned)__builtin_amdgcn_readlane((int)ic, 63);
        if (nb0 + n2d + ((n3 + tot + 7u) >> 3) > 256u) return 1000u;
        if (i < n2d) LDS32(scr + SC_BLOB + (nb0 + i) * 4) = x;
        unsigned h[4];
#pragma unroll
        for (int t = 0; t < 4; ++t) { const unsigned y = PERM(0u, v[t], sl[t]); h[t] = PERM(0u, y | (y >> 4), 0x0c0c0200u); }
        const unsigned c0 = __popc(E2 & 0x01010101u), c01 = __popc(E2 & 0x05050505u), c2 = __popc(E2 & 0x10101010u);
        const unsigned long long acc = (unsigned long long)(h[0] | (h[1] << (4u * c0))) | ((unsigned long long)(h[2] | (h[3] << (4u * c2))) << (4u * c01));
        const unsigned p3 = n3 + ic - c, sh = (p3 & 7u) * 4u;
        if (c) {
            LAS unsigned* d = (LAS unsigned*)(size_t)(s3 + (p3 >> 3) * 4);
            const unsigned long long w = acc << sh;
            __hip_atomic_fetch_or(d, (unsigned)w, __ATOMIC_RELAXED, __HIP_MEMORY_SCOPE_WAVEFRONT);
            if (4u * c + sh > 32u) {
                __hip_atomic_fetch_or(d + 1, (unsigned)(w >> 32), __ATOMIC_RELAXED, __HIP_MEMORY_SCOPE_WAVEFRONT);
                __hip_atomic_fetch_or(d + 2, (unsigned)(((acc >> 32) << sh) >> 32), __ATOMIC_RELAXED, __HIP_MEMORY_SCOPE_WAVEFRONT);
            }
        }
        n3 += tot;
    }
    CFENCE();
    return nb0 + n2d + ((n3 + 7u) >> 3);
}
__device__ __forceinline__ unsigned rot8(unsigned e) { return ((e >> 1) & 0x7f7f7f7fu) | ((e << 7) & 0x80808080u); }
__device__ __forceinline__ void selectors(unsigned P, unsigned E, unsigned (&sel)[4]) {
    const unsigned pf = (opaque((opaque((E << 8) + E) << 8) + E)) << 8;
    const unsigned S = BFI(E | (E << 1), pf, P);
    const unsigned A = BFI(0x33333333u, S, E << 2), B = BFI(0x33333333u, S >> 2, E);
    sel[0] = A & 0x07070707u; sel[1] = B & 0x07070707u; sel[2] = (A >> 4) & 0x07070707u; sel[3] = (B >> 4) & 0x07070707u;
}
template <int NP, bool IN> __device__ __forceinline__ void decode(const unsigned (&P)[NP], unsigned (&o)[4 * NP], unsigned scr, unsigned nb0, unsigned nbmax, unsigned T, int lane) {
    const unsigned lut1 = IN ? rot8(T | ((T - 1u) << 8) | ((T - 2u) << 16)) : 0x00100800u;
    const unsigned lut2 = IN ? rot8((T - 3u) | ((T - 4u) << 8) | ((T + 1u) << 16)) : 0x00282018u;
    unsigned uw[4], uc = 0, uic = 0;
    u32x2 ut[12];
    auto ufetch = [&](unsigned i0) __attribute__((always_inline)) {
        const unsigned i = i0 + 3 * lane, ad = scr + SC_BLOB + min(i, 255u) * 4 - 4;
        const u32x2a w0 = *(LAS u32x2a*)(size_t)ad, w1 = *(LAS u32x2a*)(size_t)(ad + 8);
        uw[0] = i ? w0.x : 0x80000000u; uw[1] = i < 256u ? w0.y : 0u; uw[2] = i < 255u ? w1.x : 0u; uw[3] = i < 254u ? w1.y : 0u;
#pragma unroll
        for (int y = 0; y < 12; ++y) ut[y] = *(LAS u32x2*)(size_t)(T + ((uw[1 + (y >> 2)] >> (8 * (y & 3))) & 0xffu) * 8);
        uc = __popc(uw[1]) + __popc(uw[2]) + __popc(uw[3]);
        uic = wave_scan(uc);
#pragma unroll
        for (int y = 0; y < 12; ++y) asm volatile("" : "+v"(ut[y]));
    };
    if constexpr (!IN) {
#pragma unroll
        for (int i = 0; i < 3; ++i) *(LAS u32x4*)(size_t)(scr + SC_A2 + (i * 64 + lane) * 16) = (u32x4){0u, 0u, 0u, 0u};
    }
    unsigned E[NP], cnt = 0;
#pragma unroll
    for (int j = 0; j < NP; ++j) { E[j] = esc_bits(P[j]); cnt += __popc(E[j]); }
    cnt = max(cnt, 4u);
    const unsigned inc = wave_scan(cnt);
    if constexpr (!IN) ufetch(0u);
    const unsigned n2d = min(((unsigned)__builtin_amdgcn_readlane((int)inc, 63) + 15u) >> 4, 256u);
    if constexpr (!IN) {
        for (unsigned i = 192 + lane; i <= n2d; i += 64) *(LAS u32x4*)(size_t)(scr + SC_A2 + i * 16) = (u32x4){0u, 0u, 0u, 0u};
        unsigned run = 0;
        for (unsigned i0 = 0;;) {
            unsigned pos = scr + SC_A2 + min(run + uic - uc, 4096u);
            if (uc) {
#pragma unroll
                for (int y = 0; y < 12; ++y) {
                    const unsigned W = uw[1 + (y >> 2)], Wp = uw[y >> 2];
                    const unsigned zr = (unsigned)__builtin_clz((y & 3) ? __builtin_amdgcn_alignbit(W, Wp, 8 * (y & 3)) : Wp) & 15u;
                    const unsigned lo = ut[y].x ? ut[y].x + (zr << 3) : 0u, sh = (pos & 3u) * 8u;
                    const unsigned long long a = (((unsigned long long)ut[y].y << 32) | lo) << sh;
                    LAS unsigned* d = (LAS unsigned*)(size_t)(pos & ~3u);
                    __hip_atomic_fetch_or(d, (unsigned)a, __ATOMIC_RELAXED, __HIP_MEMORY_SCOPE_WAVEFRONT);
                    __hip_atomic_fetch_or(d + 1, (unsigned)(a >> 32), __ATOMIC_RELAXED, __HIP_MEMORY_SCOPE_WAVEFRONT);
                    __hip_atomic_fetch_or(d + 2, (unsigned)(((unsigned long long)ut[y].y << sh) >> 32), __ATOMIC_RELAXED, __HIP_MEMORY_SCOPE_WAVEFRONT);
                    pos += __popc(W & (0xffu << (8 * (y & 3))));
                }
            }
            i0 += 192;
            if (i0 >= nbmax) break;
            run += (unsigned)__builtin_amdgcn_readlane((int)uic, 63);
            ufetch(i0);
        }
    } else {
    for (unsigned i = lane; nb0 + n2d + i < nbmax; i += 64) {
        const unsigned W = LDS32(scr + SC_BLOB + (nb0 + n2d + i) * 4);
        unsigned a, b;
        if (IN) {
            const unsigned K = (T + 2u) * 0x01010101u;
            a = rot8(K - (((W & 0x0f0f0f0fu) + 0x02020202u) & 0x0f0f0f0fu));
            b = rot8(K - ((((W >> 4) & 0x0f0f0f0fu) + 0x02020202u) & 0x0f0f0f0fu));
        } else {
            a = (W << 3) & 0x78787878u;
            b = (W >> 1) & 0x78787878u;
        }
        *(LAS u32x2*)(size_t)(scr + SC_A3 + i * 8) = (u32x2){PERM(b, a, 0x05010400u), PERM(b, a, 0x07030602u)};
    }
    CFENCE();
    unsigned n3 = 0;
    for (unsigned i0 = 0; i0 < n2d; i0 += 64) {
        const unsigned i = i0 + lane;
        unsigned W = LDS32(scr + SC_BLOB + ((nb0 + i) & 255u) * 4);
        W = i < n2d ? W : 0u;
        const unsigned E2 = esc_bits(W), c = __popc(E2), ic = wave_scan(c);
        unsigned ps[4];
        u32x2a w[4];
        ps[0] = scr + SC_A3 + min(n3 + ic - c, 2048u);
#pragma unroll
        for (int t = 0; t < 4; ++t) {
            w[t] = *(LAS u32x2a*)(size_t)(ps[t] & ~3u);
            if (t < 3) ps[t + 1] = ps[t] + __popc(E2 & (0x01010101u << (2 * t)));
        }
        SCHED0();
        n3 += (unsigned)__builtin_amdgcn_readlane((int)ic, 63);
        unsigned sel[4];
        selectors(W, E2, sel);
        SCHED0();
        u32x4 v;
#pragma unroll
        for (int t = 0; t < 4; ++t) v[t] = PERM(__builtin_amdgcn_alignbyte(w[t].y, w[t].x, ps[t]), lut2, sel[t]);
        *(LAS u32x4*)(size_t)(scr + SC_A2 + i * 16) = v;
    }
    }
    CFENCE();
    unsigned pos = scr + SC_A2 + min(inc - cnt, 4096u);
    unsigned ps[2][8];
    u32x2a w[2][8];
    static_for<NP / 2 + 1>([&](auto gc) __attribute__((always_inline)) {
        constexpr int g = decltype(gc)::value;
        SCHED0();
        if constexpr (g < NP / 2) {
#pragma unroll
            for (int k = 0; k < 8; ++k) {
                ps[g & 1][k] = pos;
                w[g & 1][k] = *(LAS u32x2a*)(size_t)(pos & ~3u);
                pos += __popc(E[2 * g + (k >> 2)] & (0x01010101u << (2 * (k & 3))));
            }
        }
        SCHED0();
        if constexpr (g > 0) {
#pragma unroll
            for (int h = 0; h < 2; ++h) {
                const int j = 2 * (g - 1) + h;
                unsigned sel[4];
                selectors(P[j], E[j], sel);
#pragma unroll
                for (int t = 0; t < 4; ++t) {
                    const u32x2a x = w[(g - 1) & 1][4 * h + t];
                    o[4 * j + t] = PERM(__builtin_amdgcn_alignbyte(x.y, x.x, ps[(g - 1) & 1][4 * h + t]), lut1, sel[t]);
                }
            }
        }
    });
    CFENCE();
}

__device__ __forceinline__ unsigned gather(unsigned cell, unsigned long long v, unsigned long long& tot) {
    unsigned n;
    const unsigned one = 1u;
    asm volatile("s_mov_b64 exec, 1\n\tds_or_b64 %2, %3\n\tds_add_rtn_u32 %0, %2, %4 offset:8\n\ts_mov_b64 exec, -1\n\t"
                 "ds_read_b64 %1, %2\n\ts_waitcnt lgkmcnt(0)" : "=&v"(n), "=&v"(tot) : "v"(cell), "v"(v), "v"(one) : "memory");
    return RFL(n);
}

__device__ __forceinline__ void exp_decode(int e, int rank, int& kt, int& slot, int& sub, int& dst) {
    sub = e & 3;
    const int q = e >> 2;
    if (q < 192) {
        kt = q / 24; slot = q - kt * 24; dst = slot < 16 ? -1 : rank;
        return;
    }
    const int t = q - 192;
    kt = t / 56;
    const int u = t - kt * 56, j = u / 7;
    slot = 16 + j;
    dst = (rank + 1 + (u - j * 7)) & 7;
}
__device__ __forceinline__ unsigned xin_item(int kt, int slot, int sub, int dst) {
    return (slot < 16 ? kt * 16 + slot : 128 + (dst * 8 + kt) * 8 + (slot - 16)) * 4 + sub;
}

#define LDW(p) __builtin_nontemporal_load(p)
#define LDX(p) __builtin_nontemporal_load(p)
__device__ __forceinline__ void export_item(const CAS Params& p, char* myws, const float* px, const float* pw0, int rank, unsigned scr, unsigned grp, int e, bool stage_only, bool push, unsigned& seq, unsigned seqv, bool first, bool pair, bool bar) {
    int lane = threadIdx.x & 63;
    asm volatile("" : "+v"(lane), "+s"(scr));
    int kt, slot, sub, dst;
    exp_decode(e, rank, kt, slot, sub, dst);
    const int kq = lane >> 4, c = lane & 15;
    u32x4 ch[8];
    if (slot < 16) {
        const GAS float* b = (const GAS float*)px + (size_t)(256 * slot + 64 * sub + (lane >> 3)) * KL + 64 * kt;
        const int g0 = 4 * ((lane & 7) ^ (lane >> 4));
        f32x4 v[16];
#pragma unroll
        for (int i = 0; i < 8; ++i) {
            v[2 * i] = LDX((const GAS f32x4*)(b + i * 8 * KL + (g0 ^ ((i & 1) * 16))));
            v[2 * i + 1] = LDX((const GAS f32x4*)(b + i * 8 * KL + (g0 ^ ((i & 1) * 16)) + 32));
        }
        if (!stage_only) { zero_scratch(scr, lane); lut_init(scr, lane); }
#pragma unroll
        for (int i = 0; i < 8; ++i) {
            ch[i].x = pk_bf16(v[2 * i][0], v[2 * i][1]); ch[i].y = pk_bf16(v[2 * i][2], v[2 * i][3]);
            ch[i].z = pk_bf16(v[2 * i + 1][0], v[2 * i + 1][1]); ch[i].w = pk_bf16(v[2 * i + 1][2], v[2 * i + 1][3]);
        }
    } else {
        const GAS float* b = (const GAS float*)pw0 + (size_t)(64 * kt + 16 * (sub & 1) + 4 * kq) * NTOT + dst * NL + 256 * (slot - 16) + 128 * (sub >> 1) + 4 * (c ^ (kq & 1));
        f32x4 v[16];
#pragma unroll
        for (int jj = 0; jj < 8; ++jj) {
            v[2 * jj] = LDW((const GAS f32x4*)(b + (size_t)((jj & 3) + (jj >> 2) * 32) * NTOT));
            v[2 * jj + 1] = LDW((const GAS f32x4*)(b + (size_t)((jj & 3) + (jj >> 2) * 32) * NTOT + 64));
        }
        if (!stage_only) { zero_scratch(scr, lane); lut_init(scr, lane); }
#pragma unroll
        for (int i = 0; i < 8; ++i) {
            ch[i].x = pk_bf16(v[0 + (i >> 2)][i & 3], v[2 + (i >> 2)][i & 3]); ch[i].y = pk_bf16(v[4 + (i >> 2)][i & 3], v[6 + (i >> 2)][i & 3]);
            ch[i].z = pk_bf16(v[8 + (i >> 2)][i & 3], v[10 + (i >> 2)][i & 3]); ch[i].w = pk_bf16(v[12 + (i >> 2)][i & 3], v[14 + (i >> 2)][i & 3]);
        }
    }
    if (first) { asm volatile("s_waitcnt vmcnt(0)\n\tv_readfirstlane_b32 %0, %1\n\ts_nop 4" : "=s"(seq) : "v"(seqv) : "memory"); seq += 1u; }
    if (bar) RAW_BAR();
    const bool dup = slot == 23 && sub >= 4 - 2 * NDUP;
    const bool local = stage_only || (dup && dst != rank && !push);
    const int d = lane & 7;
    const unsigned at = (rank * 8 + kt) * (PPK * 4) + ((dup && d != dst) ? 96 + ((dst - d - 1) & 7) * (2 * NDUP) + sub - (4 - 2 * NDUP) : slot * 4 + sub);
    if (local) {
        const unsigned mine = (unsigned)__builtin_amdgcn_readlane((int)at, rank);
#pragma unroll
        for (int u = 0; u < 8; ++u) st16(myws, lane * 16, u & 3, OFF_STAGE + mine * 8192 + (u >> 2) * 4096, ch[u]);
        if (stage_only) {
            WAIT_VM0();
            if (lane == rank) st_agent(myws + OFF_SFLAG + at * 32, seq);
            return;
        }
    }
    int l7 = lane & 7;
    asm volatile("" : "+v"(l7));
    char* peer = p.ws[l7];
    unsigned meta = 0;
    const unsigned xi = xin_item(kt, slot, sub, dst) + (seq & 1u) * NEXP, ibx = rank * 64 + kt * 32 + (slot - 16) * 4 + sub;
    {
        const char* xb = push ? p.ws[dst] : myws;
        const unsigned xo = push ? OFF_IBX + ibx * XI_SZ : OFF_XIN + xi * XI_SZ;
        unsigned ex[16], sum = 0;
#pragma unroll
        for (int u = 0; u < 4; ++u) {
            u32x4 r;
#pragma unroll
            for (int k = 0; k < 4; ++k) {
                const unsigned d0 = ch[2 * u + (k >> 1)][(k & 1) * 2], d1 = ch[2 * u + (k >> 1)][(k & 1) * 2 + 1];
                const unsigned L = PERM(d1, d0, 0x06040200u), H = PERM(d1, d0, 0x07050301u);
                r[k] = (H & 0x80808080u) | (L & 0x7f7f7f7fu);
                ex[4 * u + k] = ((H << 1) & 0xfefefefeu) | ((L >> 7) & 0x01010101u);
                sum = __builtin_amdgcn_sad_u8(ex[4 * u + k], 0u, sum);
            }
            st16(xb, lane * 16, u, xo, r);
        }
        unsigned T = ((unsigned)__builtin_amdgcn_readlane((int)wave_scan(sum), 63) + 7864u) >> 12;
        T = min(max(T, 13u), 253u);
        const unsigned K = (T + 2u) * 0x01010101u;
        unsigned z[16], yy[16], bad = 0;
#pragma unroll
        for (int d = 0; d < 16; ++d) {
            const unsigned y = ((K | 0x80808080u) - (ex[d] & 0x7f7f7f7fu)) ^ (~(K ^ ex[d]) & 0x80808080u);
            bad |= y; yy[d] = y;
            z[d] = ((y & 0x0f0f0f0fu) + 0x0e0e0e0eu) & 0x0f0f0f0fu;
        }
        unsigned nex = 0;
        if (__any((bad & 0xf0f0f0f0u) != 0u)) {
            CFENCE();
            unsigned mk = 0;
#pragma unroll
            for (int d = 0; d < 16; ++d) mk |= (yy[d] & 0xf0f0f0f0u) ? 1u << d : 0u;
            while (mk) {
                const unsigned d = __builtin_ctz(mk);
                mk &= mk - 1u;
                unsigned ev = ex[0];
#pragma unroll
                for (int t = 1; t < 16; ++t) ev = d == (unsigned)t ? ex[t] : ev;
#pragma unroll 1
                for (unsigned y = 0; y < 4; ++y) {
                    const unsigned eb = (ev >> (8 * y)) & 0xffu;
                    if (T + 2u - eb > 15u) {
                        const unsigned k = __hip_atomic_fetch_add((LAS unsigned*)(size_t)(scr + SC_LUT + 64), 1u, __ATOMIC_RELAXED, __HIP_MEMORY_SCOPE_WAVEFRONT);
                        if (k < 15u) LDS32(scr + SC_BLOB + k * 4) = eb | (y << 8) | (d << 10) | (lane << 14);
                    }
                }
            }
            CFENCE();
            nex = RFL(LDS32(scr + SC_LUT + 64));
            CFENCE();
            LDS32(scr + SC_LUT + 64) = 0u;
        }
        unsigned P[4], nb = 1000u;
        if (nex <= 15u) {
            unsigned n[8];
#pragma unroll
            for (int k = 0; k < 8; ++k) n[k] = z[(k >> 1) * 4 + (k & 1)] | (z[(k >> 1) * 4 + (k & 1) + 2] << 4);
            nb = encode<4, true>(n, P, scr, nex, lane);
        }
        if (nb <= 256u) {
            const unsigned lines = (nb + 31u) >> 5;
            st16(xb, lane * 16, 0, xo + 4096, (u32x4){P[0], P[1], P[2], P[3]});
            if ((unsigned)lane < lines * 8u) st16(xb, lane * 16, 1, xo + 4096, *(LAS u32x4*)(size_t)(scr + SC_BLOB + lane * 16));
            meta = T | (lines << 8) | (nex << 12);
        } else {
#pragma unroll
            for (int u = 0; u < 8; ++u) st16(xb, lane * 16, u & 3, (push ? OFF_RIBX + ibx * 8192 : OFF_RIN + xi * 8192) + (u >> 2) * 4096, ch[u]);
            meta = M_RAW << 8;
        }
    }
    WAIT_VM0();
    if (local && lane == rank) st_agent(myws + OFF_SFLAG + at * 32, seq);
    if (push) {
        if (lane == dst) st_sys64(peer + OFF_IFLAG + ibx * 8, ((unsigned long long)seq << 32) | meta);
        return;
    }
    unsigned long long tot;
    if (gather(grp, (unsigned long long)meta << (16 * sub), tot) == (pair ? 1u : 3u) && lane < NR && lane != rank && (dst < 0 || slot == 23 || lane == dst)) {
        char* fp = peer + OFF_XFLAG + (seq & 1u) * XFLAG_SZ + (rank * 8 + kt) * (PPK * 16);
        const bool other = slot == 23 && lane != dst;
        const unsigned ud = (unsigned)(dst - lane - 1) & 7u;
        if (pair) st_sys64(fp + slot * 16 + (sub >> 1) * 8, ((unsigned long long)seq << 32) | (unsigned)tot | (unsigned)(tot >> 32));
        else if (NDUP == 1 && other) st_sys64(fp + 24 * 16 + ud * 8, ((unsigned long long)seq << 32) | (unsigned)(tot >> 32));
        else {
            const u32x4 g = {(unsigned)tot, seq, (unsigned)(tot >> 32), seq};
            asm volatile("global_store_dwordx4 %0, %1, off sc0 sc1\n\ts_nop 1" :: "v"(fp + (other ? 24 + ud : slot) * 16), "v"(g) : "memory");
        }
    }
}

__device__ __forceinline__ unsigned tick_draw(char* ctr, int off) {
    return (unsigned)__builtin_amdgcn_raw_ptr_buffer_atomic_add_i32(1, mk_rsrc(ctr, 4), off, 0, 16);
}

__device__ __forceinline__ void item_decode(int it, int rank, int delta, unsigned par, unsigned& xi, unsigned& at) {
    if (it >= NI_IN) {
        const int o = it - NI_IN;
        xi = at = o + (o / NCB) * NDUP;

        return;
    }
    const int src = (rank + delta) & 7;
    if (it >= NI_DUP) {
        const int o = it - NI_DUP, r = o / NSGT, t = ((r * FSTEP & 15) + r * FSTEP / 16) * 8 + o - r * NSGT;
        xi = 2 * NEXP + t;
        at = NSTG + src * 128 + t;
        return;
    }
    if (it >= NI_MAIN) {
        const int o = it - NI_MAIN, kt = o / (14 * NDUP), r = o - kt * (14 * NDUP), u = r / (2 * NDUP);
        xi = xin_item(kt, 23, 4 - 2 * NDUP + (r & (2 * NDUP - 1)), (rank + 1 + u) & 7) + par;
        at = (src * 8 + kt) * (PPK * 4) + 96 + r;
        return;
    }
    if (it < 0) {
        const int k = -1 - it;
        xi = src * 64 + k;
        at = (src * 8 + (k >> 5)) * (PPK * 4) + 64 + (k & 31);
        return;
    }
    int q = (it >> 2) + 8;
    int kt, slot;
    if (q >= 48 && q < 64) q += q < 56 ? 8 : -8;
    if (q < 56) { kt = (q - 8) >> 4; slot = (q - 8 + 2 * delta) & 15; }
    else if (q < 72) { kt = q < 64 ? 2 : 3; slot = 16 + (q & 7); }
    else if (q < 72 + NFAST) { kt = 3; slot = ((q - 72) * FSTEP + ((2 * delta + FSTEP - 1) & -FSTEP)) & 15; }
    else if ((q -= NFAST) >= 168) { const int e = (q - 168 + 2 * delta - (2 * delta + FSTEP - 1) / FSTEP) % (16 - NFAST); kt = 7; slot = NFAST ? e + e / (FSTEP - 1) + 1 : e; }
    else {
        const int r = q / 24, e = q - r * 24, t3 = e / 3, m3 = e - t3 * 3;
        if (m3 < 2) { slot = (2 * t3 + m3 + 2 * delta) & 15; kt = r + (NFAST && slot % FSTEP == 0 ? 1 : 0); }
        else { kt = r + 1; slot = 16 + t3; }
    }
    xi = xin_item(kt, slot, it & 3, rank) + par;
    at = (src * 8 + kt) * (PPK * 4) + slot * 4 + (it & 3);
}

__device__ __forceinline__ void apply_signs(unsigned (&q)[32], const u32x4& S) {
#pragma unroll
    for (int d = 0; d < 32; ++d) {
        const unsigned sg = (S[d >> 3] >> (d & 7)) & 0x01010101u, s7 = opaque(sg << 7);
        q[d] = ((q[d] ^ (s7 - sg)) + sg) ^ s7;
    }
}
template <int AUX> __device__ __forceinline__ void store_rows(rsrc_t ro, unsigned lane_off, unsigned soff, const unsigned (&q)[32]) {
#pragma unroll
    for (int m = 0; m < 4; ++m)
#pragma unroll
        for (int j = 0; j < 4; ++j) {
            u32x2 v = {q[(m * 4 + j) * 2], q[(m * 4 + j) * 2 + 1]};
            __builtin_amdgcn_raw_buffer_store_b64(v, ro, (int)lane_off, (int)(soff + (m * 16 + j) * NTOT), AUX);
        }
}

__device__ __forceinline__ void puller(const CAS Params& p, char* myws, unsigned scr, unsigned sem, unsigned seq, int link, int g) {
    const int lane = threadIdx.x & 63;
    asm volatile("" : "+s"(link), "+s"(g));
    const int rank = p.rank, delta = link + 1, src = (rank + delta) & 7;
    char* tmo = myws + OFF_TMO;
    const bool once = g < 0;
    const rsrc_t rr = mk_rsrc(once ? myws : p.ws[src], WS_NEED), rl = mk_rsrc(myws, WS_NEED), ro = mk_rsrc(p.out, 4096u * NTOT);
    const unsigned lane_off = (unsigned)(4 * (lane >> 4)) * NTOT + 8 * (lane & 15);
    char* ctr = myws + OFF_TICK + link * 128;
    const unsigned par = (seq & 1u) * NEXP;
    const int win = once ? 1024 : WIN_IN, wout = WIN_OUT;
    const unsigned tbase = (seq - 1u) * (unsigned)NI_ALL - (unsigned)(PULL_WG * 8);
    bool dead = false;
    unsigned nxi = 0, nat = 0, flo = 0, fso = 0, ms = 0;
    unsigned long long mn = 0;
    auto look = [&](int i) __attribute__((always_inline)) {
        item_decode(i, rank, delta, par, nxi, nat);
        const unsigned t = (nat >> 6) * 8 + ((nat >> 1) & 7);
        flo = i >= NI_IN ? OFF_OFLAG + (src * 128 + t) * 8
                         : (i >= NI_DUP ? OFF_GFLAG + (nat - NSTG) * 16 : (once ? OFF_IFLAG + nxi * 8 : OFF_XFLAG + (seq & 1u) * XFLAG_SZ + (nat >> 1) * 8));
        fso = i >= NI_IN ? OFF_SFLAG + (NSTG + src * 128 + t) * 32 : flo + 4;
        mn = ld_sys64(myws + flo);
        ms = ld_sys(myws + fso);
    };
    unsigned pay = 0;
    bool there = false;
#define GRAB() do { SCHED0(); pay = RFL(mn); there = RFL(mn >> 32) == seq && RFL(ms) == seq; SCHED0(); } while (0)
    look(g);
    GRAB();
    for (int it = g; it < NI_ALL;) {
        const unsigned cxi = nxi, cat = nat;
        const bool isout = it >= NI_IN, issg = it >= NI_DUP && !isout;
        const unsigned tile = (cat >> 6) * 8 + ((cat >> 1) & 7);
        const char* fl = myws + flo;
        const char* fs = myws + fso;
        if (!there) {
            unsigned long long t0 = __builtin_amdgcn_s_memrealtime();
            for (unsigned n = 1;; ++n) {
                mn = ld_sys64(fl);
                ms = ld_sys(fs);
                if (RFL(mn >> 32) == seq && RFL(ms) == seq) break;
                if ((n & 31u) == 0u && give_up(tmo, seq, t0)) { mn = 0; dead = true; break; }
                __builtin_amdgcn_s_sleep(1);
            }
            pay = RFL(mn);
        }
        CFENCE();
        const unsigned cmeta = it >= NI_IN ? ((pay >> (((cat >> 4) & 3) * 8 + (cat & 1) * 4)) & 15u) << 8 : (pay >> ((once || issg) ? 0 : (cat & 1) * 16)) & 0xffffu;
        u32x4 c[8];
        unsigned tk;
        {
            const unsigned lines = (cmeta >> 8) & 15u;
            const bool raw = lines == M_RAW;
            const unsigned skip = issg ? cmeta : 0u;
            const unsigned bo = (unsigned)lane < lines * 8u ? lane * 16u : 0x80000000u;
            const unsigned o = raw ? (isout ? OFF_ROUT : (once ? OFF_RIBX : OFF_RIN)) + cxi * 8192
                                   : (isout ? OFF_XOUT + cxi * XO_SZ : (once ? OFF_IBX : OFF_XIN) + cxi * XI_SZ);
            const bool fixed = (unsigned)it < (unsigned)(FIX0 * PULL_WG);
            const unsigned nld = once ? 0u : ((raw && !fixed) ? 8u : 6u);
            unsigned my;
            asm volatile("s_mov_b64 exec, 1\n\tds_add_rtn_u32 %0, %1, %2\n\ts_mov_b64 exec, -1\n\ts_waitcnt lgkmcnt(0)" : "=&v"(my) : "v"(sem), "v"(fixed ? 0u : nld) : "memory");
            my = (fixed ? (unsigned)(it / PULL_WG) * 6u : RFL(my) + FIX0 * 6u) - max((unsigned)(isout ? wout : win), nld);
            unsigned rel = RFL(__hip_atomic_load((LAS unsigned*)(size_t)(sem + 4), __ATOMIC_RELAXED, __HIP_MEMORY_SCOPE_WORKGROUP));
#define GATE(u) while ((int)(my + (u) - rel) >= 0) { rel = RFL(__hip_atomic_load((LAS unsigned*)(size_t)(sem + 4), __ATOMIC_RELAXED, __HIP_MEMORY_SCOPE_WORKGROUP)); CFENCE(); }
#define RELEASE(n, v) asm volatile("s_waitcnt vmcnt(%0)\n\ts_mov_b64 exec, 1\n\tds_add_u32 %1, %2 offset:4\n\ts_mov_b64 exec, -1" :: "n"(n), "v"(sem), "v"(v) : "memory")
            while ((int)(my - 4u - rel) >= 0) { __builtin_amdgcn_s_sleep(2); rel = RFL(__hip_atomic_load((LAS unsigned*)(size_t)(sem + 4), __ATOMIC_RELAXED, __HIP_MEMORY_SCOPE_WORKGROUP)); }
            __builtin_amdgcn_s_setprio(3);
#define OFS(u, v) ((v) | (skip >> (u) << 31))
            GATE(0u); c[4] = bld16(rr, OFS(4, lane * 16u), o + 4096);
            GATE(1u); c[5] = bld16(rr, OFS(5, bo), o + 5120);
            GATE(2u); c[3] = bld16(rr, OFS(3, lane * 16u), o + 3072);
            GATE(3u); c[0] = bld16(rr, OFS(0, lane * 16u), o);
            GATE(4u); c[1] = bld16(rr, OFS(1, lane * 16u), o + 1024);
            GATE(5u); c[2] = bld16(rr, OFS(2, lane * 16u), o + 2048);
            GATE(nld - 1u);
            c[6] = bld16(rr, raw ? OFS(6, lane * 16u) : 0x80000000u, o + 6144);
            c[7] = bld16((isout && !raw) ? rl : rr, (raw || isout) ? OFS(7, lane * 16u) : 0x80000000u,
                         (isout && !raw) ? OFF_STAGE + (NSTG + src * 128 + tile) * 8192 + (((cat >> 4) & 3) * 2 + (cat & 1)) * 1024 : o + 7168);
            tk = tick_draw(ctr, (lane || once) ? 64 : 0);
            const unsigned one = once ? 0u : 1u, last = once ? 0u : nld - 5u;
            RELEASE(8, one); RELEASE(7, one); RELEASE(6, one); RELEASE(5, one); RELEASE(4, one); RELEASE(1, last);
            __builtin_amdgcn_s_setprio(0);
        }
        WAIT_VM0();
        const int nx = once ? NI_ALL : (int)min(RFL(tk) - tbase, (unsigned)NI_ALL);
        if (nx < NI_ALL) look(nx);
        SCHED0();
        {
            const unsigned lines = (cmeta >> 8) & 15u;
            if (it >= NI_IN) {
                unsigned q[32];
                if (dead) {
#pragma unroll
                    for (int d = 0; d < 32; ++d) q[d] = 0x80808080u;
                } else if (lines == M_RAW) {
#pragma unroll
                    for (int d = 0; d < 32; ++d) q[d] = c[d >> 2][d & 3];
                } else {
                    *(LAS u32x4*)(size_t)(scr + SC_BLOB + lane * 16) = c[5];
                    CFENCE();
                    const unsigned P[8] = {c[3].x, c[3].y, c[3].z, c[3].w, c[4].x, c[4].y, c[4].z, c[4].w};
                    decode<8, false>(P, q, scr, 0u, lines * 32u, sem + 1024, lane);
#pragma unroll
                    for (int d = 0; d < 32; ++d) {
                        const unsigned hb = (d & 7) >= 2 ? c[2][d >> 3] >> ((d & 7) - 2) : c[2][d >> 3] << (2 - (d & 7));
                        q[d] |= ((c[d >> 4][(d >> 2) & 3] >> (2 * (d & 3))) & 0x03030303u) | (hb & 0x04040404u);
                    }
                    apply_signs(q, c[7]);
                }
                GRAB();
                store_rows<OUT_AUX>(ro, lane_off, (cat >> 4) * (64u * NTOT) + src * NL + (cat & 15u) * 128u, q);
            } else {
                u32x4 o[8];
                if (lines == M_RAW) {
#pragma unroll
                    for (int u = 0; u < 8; ++u) o[u] = c[u];
                } else {
                    *(LAS u32x4*)(size_t)(scr + SC_BLOB + lane * 16) = c[5];
                    CFENCE();
                    const unsigned P[4] = {c[4].x, c[4].y, c[4].z, c[4].w};
                    const unsigned nex = (cmeta >> 12) & 15u;
                    unsigned ex[16];
                    decode<4, true>(P, ex, scr, nex, lines * 32u, cmeta & 0xffu, lane);
                    for (unsigned k = 0; k < nex; ++k) {
                        const unsigned en = RFL(LDS32(scr + SC_BLOB + k * 4));
                        const unsigned sh = ((en >> 8) & 3u) * 8u, r = (en >> 10) & 15u;
                        const unsigned msk = (unsigned)lane == ((en >> 14) & 63u) ? 0xffu << sh : 0u;
                        const unsigned val = rot8(en & 0xffu) << sh;
#pragma unroll
                        for (int d = 0; d < 16; ++d) { const unsigned mq = r == (unsigned)d ? msk : 0u; ex[d] = (ex[d] & ~mq) | (val & mq); }
                    }
#pragma unroll
                    for (int d = 0; d < 16; ++d) {
                        const unsigned r = c[d >> 2][d & 3];
                        const unsigned H = (r & 0x80808080u) | (ex[d] & 0x7f7f7f7fu), L = (ex[d] & 0x80808080u) | (r & 0x7f7f7f7fu);
                        o[d >> 1][(d & 1) * 2] = PERM(H, L, 0x05010400u);
                        o[d >> 1][(d & 1) * 2 + 1] = PERM(H, L, 0x07030602u);
                    }
                }
#pragma unroll
                for (int u = 0; u < 8; ++u) st16(myws, lane * 16, u & 3, OFF_STAGE + cat * 8192 + (u >> 2) * 4096, o[u]);
                WAIT_VM0();
                GRAB();
                if (lane == 0) st_agent(myws + OFF_SFLAG + cat * 32, seq);
            }
        }
        it = nx;
    }
    if (dead && lane == 0) *(GAS signed char*)(p.out + rank * NL) = -128;
}

__device__ __forceinline__ int step_group(int s, int rank, int d0) {
    if (s < 8) return rank * 8 + s;
    int t = s - 8;
    int kt = t / 7, u = t - kt * 7;
    int v = u < d0 ? u : u + 1;
    int delta = (d0 - v) & 7;
    return ((rank + delta) & 7) * 8 + kt;
}

__device__ __forceinline__ void gemm_role(const CAS Params& p, int gidx, unsigned seq, char* myws, LAS char* lds) {
    int tid = threadIdx.x;
    asm volatile("" : "+v"(tid));
    const int wid = RFL(tid >> 6);
    int lane = tid & 63;
    const int wr = wid >> 1, wc = wid & 1;
    const int rank = p.rank;
    const bool mainw = gidx >= 0;
    int ti, tj;
    if (mainw) { const int xcd = gidx & 7, sl = gidx >> 3; ti = (xcd >> 1) * 4 + (sl >> 2); tj = (xcd & 1) * 4 + (sl & 3); }
    else { const int x = -1 - gidx; ti = (x & 7) * 2 + ((x >> 3) & 1); tj = 8 + (x >> 4); }
    const int d0 = ti >> 1;
    const int myslot = wid < 4 ? ti : 16 + tj, mysub = (NDUP == 1 && tj == 11 && wid >= 4) ? wid & 1 : wid & 3;
    char* tmo = myws + OFF_TMO;
    const char* dsrc = myws + OFF_STAGE + (size_t)myslot * PIECE + mysub * 8192;
    const char* fsrc = myws + OFF_SFLAG + myslot * 128 + mysub * 32;
    const unsigned loff = lane * 16, loff2 = loff + 4096;
    const unsigned ldsw = (unsigned)(size_t)lds + wid * 8192;

    f32x4 acc[4][8];
#pragma unroll
    for (int m = 0; m < 4; ++m)
#pragma unroll
        for (int i = 0; i < 8; ++i) acc[m][i] = (f32x4){0.f, 0.f, 0.f, 0.f};

#define STAGE_ALL(S, D) stage8k(S, loff, loff2, D)
#define PRIO(n) __builtin_amdgcn_s_setprio(n)
    constexpr int T1 = 24;
#define RD1(t) (((t) & 1) == 0 && (t) < 24 ? (t) / 2 : -1)
#define RD2(t) (((t) & 1) == 0 && (t) >= 2 && (t) < 26 ? (t) / 2 - 1 : -1)
#define GL2(t) (((t) & 3) == 1 ? (t) >> 2 : -1)
#define GOFF(st) ((unsigned)__builtin_amdgcn_readlane((int)tab, (st)))
#define GSRC(off) (dsrc + (off))
#define GFLAG(off) (fsrc + ((off) >> 8))
    static_assert(PPK * PIECE / 256 == PPK * 128, "flag offset");
    const unsigned tab = (unsigned)step_group(lane, rank, d0) * (PPK * PIECE);
    const unsigned abase = (unsigned)(size_t)lds + wr * 8192 + (lane & 15) * 128 + (((lane >> 4) ^ ((lane >> 1) & 7)) << 4);
    const unsigned bbase = (unsigned)(size_t)lds + 32768 + wc * 16384 + ((lane >> 3) & 1) * 4096 + (lane >> 4) * 256 + (lane & 7) * 32 + ((lane >> 4) & 1) * 16;
#define FRAG(a) (*(const LAS bf16x8*)(size_t)(a))
    bf16x8 a0[4], b0[8], a1[4], b1[8];

    {
        const unsigned g0 = GOFF(0);
        spin_flag(GFLAG(g0), seq, tmo);
        STAGE_ALL(GSRC(g0), ldsw);
    }
    unsigned gn = GOFF(1);
    unsigned fv = ld_sys(GFLAG(gn));
    WAIT_VM0();
    RAW_BAR();
    bool staged = RFL(fv) >= seq;
    if (staged) { STAGE_ALL(GSRC(gn), ldsw + 65536); gn = GOFF(2); }
    fv = ld_sys(GFLAG(gn));
#pragma unroll
    for (int i = 0; i < 8; ++i) b0[i] = FRAG((bbase ^ ((i >> 2) * 16)) + (i & 3) * 1024);
#pragma unroll
    for (int m = 0; m < 4; ++m) a0[m] = FRAG(abase + m * 2048);

    for (int s = 0; s < 64; ++s) {
        const int cur = (s & 1) * 65536, oth = cur ^ 65536;
        if (!staged) {
            WAIT_VM0();
            if (RFL(fv) >= seq) { STAGE_ALL(GSRC(gn), ldsw + oth); staged = true; gn = GOFF(min(s + 2, 63)); fv = ld_sys(GFLAG(gn)); }
        }
        const unsigned ab2 = abase + oth, bb2 = bbase + oth;
        const unsigned want = s + 2 < 64 ? seq : 0xffffffffu;
        const unsigned g3 = GOFF(min(s + 3, 63));
        const unsigned ld0 = ldsw + cur;
        SCHED0();
        {
            const unsigned ab = (abase + cur) ^ 64u, bb = bbase + cur;
            PRIO(1);
            static_for<T1>([&](auto tc) __attribute__((always_inline)) {
                constexpr int t = decltype(tc)::value;
                acc[t >> 3][t & 7] = __builtin_amdgcn_mfma_f32_16x16x32_bf16(a0[t >> 3], b0[t & 7], acc[t >> 3][t & 7], 0, 0, 0);
                constexpr int rd = RD1(t);
                if constexpr (rd >= 0 && rd < 8) b1[rd] = FRAG((bb ^ ((rd >> 2) * 16)) + (rd & 3) * 1024 + 8192);
                if constexpr (rd >= 8) a1[rd - 8] = FRAG(ab + (rd - 8) * 2048);
                if constexpr (t == 15) PRIO(0);
                SCHED0();
            });
        }
        if (!staged) {
            spin_flag(GFLAG(gn), seq, tmo);
            STAGE_ALL(GSRC(gn), ldsw + oth);
            gn = GOFF(min(s + 2, 63));
            fv = ld_sys(GFLAG(gn));
        }
        const unsigned gcur = GOFF(s);
        WAIT_VM0();
        const unsigned have = RFL(fv);
        const char* gs = GSRC(have >= want ? gn : gcur);
        asm volatile("s_mov_b32 m0, %0" :: "s"(ld0), "s"(gs) : "memory");
        SCHED0();
        static_for<32, T1>([&](auto tc) __attribute__((always_inline)) {
            constexpr int t = decltype(tc)::value;
            acc[t >> 3][t & 7] = __builtin_amdgcn_mfma_f32_16x16x32_bf16(a0[t >> 3], b0[t & 7], acc[t >> 3][t & 7], 0, 0, 0);
            SCHED0();
        });
        __builtin_amdgcn_s_waitcnt(0xC07F);
        PRIO(3);
        SCHED0();
        RAW_BAR();
        SCHED0();
        {
            static_for<32>([&](auto tc) __attribute__((always_inline)) {
                constexpr int t = decltype(tc)::value;
                acc[t >> 3][t & 7] = __builtin_amdgcn_mfma_f32_16x16x32_bf16(a1[t >> 3], b1[t & 7], acc[t >> 3][t & 7], 0, 0, 0);
                constexpr int rd = RD2(t);
                if constexpr (rd >= 0 && rd < 8) b0[rd] = FRAG((bb2 ^ ((rd >> 2) * 16)) + (rd & 3) * 1024);
                if constexpr (rd >= 8) a0[rd - 8] = FRAG(ab2 + (rd - 8) * 2048);
                if constexpr (GL2(t + 1) == 4) asm volatile("s_add_u32 m0, m0, 0x1000" ::: "memory", "scc");
                if constexpr (GL2(t) >= 0)
                    asm volatile("global_load_lds_dwordx4 %0, %1 offset:%2 " GLB :: "v"(GL2(t) < 4 ? loff : loff2), "s"(gs), "n"((GL2(t) & 3) * 1024) : "memory");
                if constexpr (t == 15) PRIO(2);
                if constexpr (t == 30) {
                    const bool go = have >= want;
                    staged = go || s + 2 >= 64;
                    gn = go ? g3 : gn;
                }
                if constexpr (t == 31) fv = ld_sys(GFLAG(gn));
                SCHED0();
            });
        }
    }

    asm volatile("" : "+v"(tid));
    lane = tid & 63;
    char* peer = p.ws[lane & 7];
    u32x4 S;
#pragma unroll
    for (int m = 0; m < 4; ++m) {
        unsigned x = 0;
#pragma unroll
        for (int bb = 0; bb < 8; ++bb) {
            const int j = bb >> 1, h = bb & 1;
            const unsigned t = PERM(PERM(__float_as_uint(acc[m][4 * h + 3][j]), __float_as_uint(acc[m][4 * h + 2][j]), 0x07030703u),
                                    PERM(__float_as_uint(acc[m][4 * h + 1][j]), __float_as_uint(acc[m][4 * h + 0][j]), 0x07030703u), 0x05040100u);
            x |= (t >> (7 - bb)) & (0x01010101u << bb);
        }
        S[m] = x;
    }
    const bool tail = ti * 4 + wr >= 64 - RAW_TAIL;
    if (mainw && tj < NSGT) st16(myws, lane * 16, 0, OFF_RIN + (2 * NEXP + ti * 8 + tj) * 8192 + wid * 1024, S);
    float am = 0.f;
#pragma unroll
    for (int m = 0; m < 4; ++m)
#pragma unroll
        for (int i = 0; i < 8; ++i)
#pragma unroll
            for (int j = 0; j < 4; ++j) am = fmaxf(am, fabsf(acc[m][i][j]));
#pragma unroll
    for (int o = 32; o > 0; o >>= 1) am = fmaxf(am, __shfl_xor(am, o));
    LAS float* red = (LAS float*)(lds + LDS_MISC + 512);
    if (lane == 0) red[wid] = am;
    WAIT_VM0();
    RAW_BAR();
    if (tid < NR && mainw) {
        float a = red[0];
#pragma unroll
        for (int i = 1; i < 8; ++i) a = fmaxf(a, red[i]);
        const unsigned skip = (ti * 4 + 3 >= 64 - RAW_TAIL ? 0xc0u : 0u) | (ti * 4 + 2 >= 64 - RAW_TAIL ? 0x30u : 0u) | (tj * 2 + 1 >= NCB ? 0xaau : 0u);
        const u32x4 g = {(M_RAW << 8) | skip, seq, __float_as_uint(a), seq};
        asm volatile("global_store_dwordx4 %0, %1, off sc0 sc1\n\ts_nop 1" :: "v"(peer + OFF_GFLAG + (rank * 128 + ti * 8 + tj) * 16), "v"(g) : "memory");
    }
    const unsigned scr = (unsigned)(size_t)lds + wid * SCR_SZ;
    zero_scratch(scr, lane);
    lut_init(scr, lane);
    {
        const char* gp = myws + OFF_GFLAG + tid * 16 + 8;
        unsigned long long t0 = __builtin_amdgcn_s_memrealtime();
        unsigned long long a, b;
        for (unsigned n = 1;; ++n) {
            a = ld_sys64(gp);
            b = ld_sys64(gp + 8192);
            bool ok = (unsigned)(a >> 32) == seq && (unsigned)(b >> 32) == seq;
            if (__all(ok)) break;
            if ((n & 31u) == 0u && give_up(tmo, seq, t0)) break;
            __builtin_amdgcn_s_sleep(AM_SLEEP);
        }
        am = fmaxf(__uint_as_float((unsigned)a), __uint_as_float((unsigned)b));
#pragma unroll
        for (int o = 32; o > 0; o >>= 1) am = fmaxf(am, __shfl_xor(am, o));
        if (lane == 0) red[8 + wid] = am;
        WAIT_VM0();
        RAW_BAR();
#pragma unroll
        for (int i = 0; i < 8; ++i) am = fmaxf(am, red[8 + i]);
    }
    const unsigned tmov = ld_agent(tmo);

    const float inv = am > 0.f ? 127.0f / am : 0.f;
    unsigned q[32];
#pragma unroll
    for (int m = 0; m < 4; ++m)
#pragma unroll
        for (int j = 0; j < 4; ++j)
#pragma unroll
            for (int h = 0; h < 2; ++h) {
                const unsigned t0 = __float_as_uint(fmaf(fabsf(acc[m][4 * h + 0][j]), inv, 12582912.f));
                const unsigned t1 = __float_as_uint(fmaf(fabsf(acc[m][4 * h + 1][j]), inv, 12582912.f));
                const unsigned t2 = __float_as_uint(fmaf(fabsf(acc[m][4 * h + 2][j]), inv, 12582912.f));
                const unsigned t3 = __float_as_uint(fmaf(fabsf(acc[m][4 * h + 3][j]), inv, 12582912.f));
                q[(m * 4 + j) * 2 + h] = PERM(t1, t0, 0x0c0c0400u) | PERM(t3, t2, 0x04000c0cu);
            }
    const bool bad = RFL(tmov) == seq;
    const int cb = tj * 2 + wc;
    const unsigned xi = (ti * 4 + wr) * 16 + cb;
    unsigned lines = 0;
    unsigned long long tot;
#define PUBLISH() if (gather((unsigned)(size_t)lds + LDS_MISC + 64, (unsigned long long)(lines << (wid * 4)), tot) == 7u && lane < NR && lane != rank) \
            st_sys64(peer + OFF_OFLAG + (rank * 128 + ti * 8 + tj) * 8, ((unsigned long long)seq << 32) | (unsigned)tot)
    if (mainw && tj < NSGT) {
        if (cb < NCB) {
            unsigned P[8], nb = 1000u;
            if (!tail && !bad) {
                unsigned n[16];
                u32x4 L[2], H;
#pragma unroll
                for (int j = 0; j < 8; ++j) {
                    L[j >> 2][j & 3] = BFI(0x33333333u, BFI(0x0f0f0f0fu, q[4 * j], q[4 * j + 2] << 4), BFI(0x0f0f0f0fu, q[4 * j + 1], q[4 * j + 3] << 4) << 2);
                    n[2 * j] = BFI(0x0f0f0f0fu, q[4 * j] >> 3, q[4 * j + 2] << 1);
                    n[2 * j + 1] = BFI(0x0f0f0f0fu, q[4 * j + 1] >> 3, q[4 * j + 3] << 1);
                }
#pragma unroll
                for (int m = 0; m < 4; ++m) {
                    unsigned x = 0;
#pragma unroll
                    for (int bb = 0; bb < 8; ++bb) x |= (bb >= 2 ? q[8 * m + bb] << (bb - 2) : q[8 * m + bb] >> (2 - bb)) & (0x01010101u << bb);
                    H[m] = x;
                }
                st16(myws, lane * 16, 0, OFF_XOUT + xi * XO_SZ, L[0]);
                st16(myws, lane * 16, 1, OFF_XOUT + xi * XO_SZ, L[1]);
                st16(myws, lane * 16, 2, OFF_XOUT + xi * XO_SZ, H);
                nb = encode<8, false>(n, P, scr, 0u, lane);
            }
            lines = M_RAW;
            if (nb <= 256u) {
                lines = (nb + 31u) >> 5;
                st16(myws, lane * 16, 3, OFF_XOUT + xi * XO_SZ, (u32x4){P[0], P[1], P[2], P[3]});
                st16(myws, lane * 16, 0, OFF_XOUT + xi * XO_SZ + 4096, (u32x4){P[4], P[5], P[6], P[7]});
                if ((unsigned)lane < lines * 8u) st16(myws, lane * 16, 1, OFF_XOUT + xi * XO_SZ + 4096, *(LAS u32x4*)(size_t)(scr + SC_BLOB + lane * 16));
                WAIT_VM0();
            }
        }
        if (lines != M_RAW) { PUBLISH(); }
    }
    apply_signs(q, S);
    if (bad) {
#pragma unroll
        for (int d = 0; d < 32; ++d) q[d] = 0x80808080u;
    }
    if (lines == M_RAW) {
#pragma unroll
        for (int u = 0; u < 8; ++u)
            st16(myws, lane * 16, u & 3, OFF_ROUT + xi * 8192 + (u >> 2) * 4096, (u32x4){q[4 * u], q[4 * u + 1], q[4 * u + 2], q[4 * u + 3]});
        WAIT_VM0();
        PUBLISH();
    }
    const int h = cb - 16;
    if (h < 7 * NDUP)
        store_rows<OUT_AUX>(mk_rsrc(p.out, 4096u * NTOT), (unsigned)(4 * (lane >> 4)) * NTOT + 8 * (lane & 15),
                            (unsigned)(ti * 4 + wr) * (64u * NTOT) + (mainw ? rank * NL + cb * 128 : ((rank + 1 + h / NDUP) & 7) * NL + (NCB + h % NDUP) * 128), q);
}

__global__ __launch_bounds__(NTHR, 2) __attribute__((amdgpu_waves_per_eu(2, 2))) void fused_kernel(Params p_unused) {
    extern __shared__ __attribute__((aligned(1024))) char smem[];
    const CAS Params& p = *(const CAS Params*)__builtin_amdgcn_kernarg_segment_ptr();
    char* myws = p.self;
    const float* px = p.x;
    const float* pw = p.w;
    int rank = p.rank;
    asm volatile("" : "+s"(myws), "+s"(px), "+s"(pw), "+s"(rank));
    LAS char* lds = (LAS char*)smem;
    const int b = blockIdx.x;
    const int wid = RFL(threadIdx.x >> 6), lane = threadIdx.x & 63;
    const unsigned scr = (unsigned)(size_t)lds + wid * SCR_SZ;
    char* cnt = myws + OFF_CNT + (b * 8 + wid) * 4;
    unsigned seq = 0, seqv;
    asm volatile("global_load_dword %0, %1, %2 sc1" : "=v"(seqv) : "v"(0u), "s"(cnt) : "memory");
    if (b < 16) {
        const char* pc = (const char*)__builtin_amdgcn_s_getpc();
        const unsigned o = threadIdx.x * 128u;
        if (o < CODE_BYTES) { unsigned t = *(const volatile GAS unsigned*)(pc + o); asm volatile("" :: "v"(t)); }
    }
    const unsigned sem = (unsigned)(size_t)lds + LDS_MISC;
    if (threadIdx.x < 128) LDS32(sem + threadIdx.x * 4) = 0u;
    if (b < NPWG) unary_init(sem + 1024, threadIdx.x);
    {
        constexpr int NS = NWG - NGEMM - NPWG, NXL = 438;
        static_assert(2 * NPWG >= 96 && EXP_GL * NGEMM + 4 * NPWG + 8 * NS - 448 >= 768 && 8 * NS >= 448 && 4 * NS >= 224 && 4 * NPWG >= 224, "first tasks");
        const bool gw = b >= NWG - NGEMM, spare = !gw && b >= NPWG;
        const int sp = b - NPWG;
        int e = -1, lag = 0, own = 768;
        if (gw) {
            if (wid < EXP_GL) { own = wid * NGEMM + (b - (NWG - NGEMM)); lag = wid ? EXP_LG + wid * EXP_LW : 0; }
        } else if (spare) {
            const int k = wid * NS + sp;
            if (k < 448) { e = 768 + k; lag = k < 224 ? wid * EXP_LPUSH : EXP_L2 + (wid - 224 / NS) * EXP_L3; }
            else { own = EXP_GL * NGEMM + 4 * NPWG + k - 448; lag = 8; }
        } else if (wid >= 4) { const int pr = wid >= 6 ? b : NPWG + b; if (pr < 96) e = ((pr >> 1) + (pr >> 5) * 8) * 4 + (pr & 1) * 2 + (wid & 1); }
        else { own = EXP_GL * NGEMM + (3 - wid) * NPWG + b; lag = EXP_LS; }
        const bool pair = !spare && e >= 0, late_bar = pair && wid >= 6;
        bool push = spare && e >= 0, stage_only = own < 768;
        if (stage_only) e = own;
        if (!late_bar) RAW_BAR();
        if (e >= 0) for (int i = 0; i < lag; ++i) __builtin_amdgcn_s_sleep(4);
        unsigned grp = sem + (wid >> 1) * 16;
        int q = spare ? (wid < 4 ? 0 : NS) + sp - 2 * NS : NXL;
        const int ib = (wid & 3) * NS + sp;
#pragma unroll 1
        for (bool first = true;; first = false) {
            if (e >= 0) export_item(p, myws, px, pw, rank, scr, grp, e, stage_only, push, seq, seqv, first, pair, late_bar);
            else { asm volatile("s_waitcnt vmcnt(0)\n\tv_readfirstlane_b32 %0, %1\n\ts_nop 4" : "=s"(seq) : "v"(seqv) : "memory"); seq += 1u; }
            if (gw) break;
            if (!spare || (first && ib < 224)) {
                puller(p, myws, scr, sem, seq, spare ? (ib >> 2) % 7 : b % 7,
                       spare ? -1 - ((ib / 28) * 4 + (ib & 3) + (wid >= 4 ? 32 : 0)) : (b / 7) + (wid < 4 ? wid : wid ^ 2) * PULL_WG);
            }
            q += 2 * NS;
            if (q >= NXL) break;
            const int qq = q < 56 ? q + 72 : q + 88, r = qq / 72, u = qq - r * 72;
            e = (r == 7 ? (u < 8 ? u * 24 + 23 : (u < 15 ? 233 : 282) + u) : (u < 56 ? 248 + r * 56 + u : (r + 1) * 24 + (u - 56))) * 4 + (wid & 3);
            push = false; stage_only = false;
            grp = sem + 64 + (q / NS) * 16;
        }
        static_assert((NXL / NS + 1) * 16 + 64 <= 512, "gather cells");
    }
    if (lane == 0) st_agent(cnt, seq);
    if (b >= NWG - NGEMM) {
        const int gidx = b - (NWG - NGEMM);
#pragma unroll 1
        for (int g = gidx;; g = -1 - gidx) {
            RAW_BAR();
            gemm_role(p, g, seq, myws, lds);
            if (g < 0 || gidx >= 16 * NXT) break;
        }
    } else if (b < NPWG) {
    }
}

}

extern "C" void kernel_launch(int rank, int nranks, ncclComm_t comm,
                              hipStream_t stream,
                              void* const* d_in, const int* in_sizes, int n_in,
                              void* d_out, int out_size,
                              const int* in_dtypes, int out_dtype,
                              void* const* d_out_all,
                              void* const* d_ws_all, size_t ws_bytes) {
    if (nranks != NR || ws_bytes < WS_NEED || (unsigned)rank >= (unsigned)NR) {
        fprintf(stderr, "unexpected configuration\n");
        return;
    }
    HIP_CHECK(hipFuncSetAttribute((const void*)fused_kernel, hipFuncAttributeMaxDynamicSharedMemorySize, LDS_BYTES));
    Params p;
    p.x = (const float*)d_in[0];
    p.w = (const float*)d_in[1];
    for (int i = 0; i < NR; ++i) p.ws[i] = (char*)d_ws_all[i];
    p.out = (char*)d_out;
    p.self = (char*)d_ws_all[rank];
    p.rank = rank;
    p.pad = 0;
    hipLaunchKernelGGL(fused_kernel, dim3(NWG), dim3(NTHR), LDS_BYTES, stream, p);
    HIP_CHECK(hipGetLastError());
}
```
